```python
import numpy as np

kernel_name = "gpu_gemm_rs_m4096_k4096_n4096_bf16_reluq8"


ABSMAX_RTOL = 0.02

COMPUTE_DTYPE = "bfloat16"
EPILOGUE = "relu_quant_int8"
IN_QUANT = None
OUT_QUANT = "int8"
IN_DTYPES = {"x": "bfloat16", "w_mat": "bfloat16"}
WORLD_SIZE = 8

def _to_bf16(x):
    u = np.ascontiguousarray(x, dtype=np.float32).view(np.uint32)
    return ((u + 0x7FFF + ((u >> 16) & 1)) & 0xFFFF0000).view(np.float32)

def setup_inputs():

    x = _to_bf16(np.random.randn(4096, 4096).astype(np.float32))
    w_mat = _to_bf16((np.random.randn(4096, 4096) / np.sqrt(4096)).astype(np.float32))
    return {"x": x, "w_mat": w_mat}

def _quant_dequant(y, amax):
    scale = amax / 127.0
    q = np.clip(np.round(y / scale), -127.0, 127.0)
    return (q * scale).astype(np.float32)
def _epilogue(y):
    y = np.maximum(y, 0.0)
    return _quant_dequant(y, np.abs(y).max())

def reference(x, w_mat):
    y = (x @ w_mat).astype(np.float32)
    return _epilogue(y)
```

```cpp
#include <hip/hip_runtime.h>
#include <rccl/rccl.h>
#include <cstdio>
#include <cstdint>

#define HIP_CHECK(expr) do { hipError_t _e = (expr); if (_e != hipSuccess) { \
    fprintf(stderr, "HIP error %s at %s:%d\n", hipGetErrorString(_e), __FILE__, __LINE__); \
    return; } } while (0)

namespace {

typedef unsigned u32;
typedef unsigned long long u64;
typedef u32 u32x4 __attribute__((ext_vector_type(4)));
typedef u32 u32x2 __attribute__((ext_vector_type(2)));
typedef float f32x4 __attribute__((ext_vector_type(4)));
typedef short s16x4 __attribute__((ext_vector_type(4)));
typedef short s16x8 __attribute__((ext_vector_type(8)));
typedef __bf16 bf16x8 __attribute__((ext_vector_type(8)));
#define LAS __attribute__((address_space(3)))
#define GAS __attribute__((address_space(1)))

constexpr int NR = 8;
#define NPUSH 24
constexpr int NWORK = 256 - NPUSH;
constexpr int GRID = NPUSH + NWORK;
constexpr int MAXU = 96;
constexpr u64 DEADLINE_TICKS = 100000000ull;

constexpr u32 X_BYTES = 4096u * 512u * 2u, W_BYTES = 512u * 4096u * 2u, OUT_BYTES = 512u * 4096u;
constexpr u32 OFF_CNT = 0;
constexpr u32 OFF_TMO = 16u << 10;
constexpr u32 OFF_TK = (16u << 10) + 128u;
constexpr u32 OFF_OF = 18u << 10;
constexpr u32 OFF_XF = 20u << 10;
constexpr u32 OFF_WF = 24u << 10;
constexpr u32 OFF_AM = 32u << 10;
constexpr u32 OFF_PF = 40u << 10;
constexpr u32 OFF_DBG = 128u << 10;
constexpr u32 OFF_XS = 2u << 20;
constexpr u32 OFF_WS = 8u << 20;
constexpr u32 OFF_IN = 14u << 20;
constexpr u32 OFF_OUT = OFF_IN + 208u * 32768u;
constexpr u32 WS_USED = OFF_OUT + 208u * 32768u;
constexpr u32 PANEL_B = 6u * 65536u;

enum { JOB_NONE = 0, JOB_S = 1, JOB_O = 2, JOB_F = 3 };

struct Params {
  const char* x; const char* w; char* out;
  char* ws[NR];
  int rank; int pad;
  u32 jobs[NWORK];
  u32 ucount[NR];
  u32 units[NR][MAXU / 4];
};

__host__ __device__ inline int tile_dist(int t) { return t < 3 ? 1 : t < 5 ? t - 1 : t < 13 ? 4 + ((t - 5) & 3) : 0; }
__host__ __device__ inline int tile_sub(int t, int a) { return t < 3 ? t : t < 5 ? 3 : t < 13 ? (a & 2) + ((t - 5) >> 2) : t - 13; }
__host__ __device__ inline int box_base(int d) { return d == 1 ? 0 : d < 4 ? 48 + 16 * (d - 2) : 80 + 32 * (d - 4); }
__host__ __device__ inline int box_idx(int t, int p) { return t < 3 ? p * 3 + t : t < 5 ? p : p * 2 + ((t - 5) >> 2); }

__device__ __forceinline__ u32 rfl(u32 v) { return __builtin_amdgcn_readfirstlane(v); }
__device__ __forceinline__ __amdgpu_buffer_rsrc_t mk_rsrc(const void* p, u32 bytes) {
  return __builtin_amdgcn_make_buffer_rsrc((void*)p, 0, bytes, 0x00020000);
}
__device__ __forceinline__ u32x4 mk_rsrc4(const void* p, u32 bytes) {
  u64 a = (u64)p; u32x4 r; r.x = rfl((u32)a); r.y = rfl((u32)(a >> 32)) & 0xffffu; r.z = bytes; r.w = 0x00020000u; return r;
}
__device__ __forceinline__ u32 ld_sys(const char* p) { return __hip_atomic_load((GAS const u32*)p, __ATOMIC_RELAXED, __HIP_MEMORY_SCOPE_SYSTEM); }
__device__ __forceinline__ void st_sys(char* p, u32 v) { __hip_atomic_store((GAS u32*)p, v, __ATOMIC_RELAXED, __HIP_MEMORY_SCOPE_SYSTEM); }
__device__ __forceinline__ void st_agent(char* p, u32 v) { __hip_atomic_store((GAS u32*)p, v, __ATOMIC_RELAXED, __HIP_MEMORY_SCOPE_AGENT); }
#define WAIT_VM(n) asm volatile("s_waitcnt vmcnt(" #n ")" ::: "memory")
#define BARRIER() do { asm volatile("" ::: "memory"); __builtin_amdgcn_s_waitcnt(0xC07F);   __builtin_amdgcn_s_barrier(); asm volatile("" ::: "memory"); } while (0)

#define STAMP_AT(i) do {} while (0)

__device__ __forceinline__ void wave_wait(char* ws, u32 off, bool care, u32 call, int lane) {
  if (lane == 63) { off = OFF_TMO; }
  else if (!care) { off = OFF_TMO; }
  const u64 t0 = __builtin_amdgcn_s_memrealtime();
  for (;;) {
    const u32 v = ld_sys(ws + off);
    const bool ok = (lane == 63 || !care) ? true : (v >= call);
    if (__all(ok)) break;
    const bool dead = __any(lane == 63 && v == call);
    if (dead || __builtin_amdgcn_s_memrealtime() - t0 > DEADLINE_TICKS) {
      if (lane == 0) st_sys(ws + OFF_TMO, call);
      WAIT_VM(0);
      break;
    }
    __builtin_amdgcn_s_sleep(4);
  }
}

__device__ __forceinline__ int wave_poll4(char* ws, __amdgpu_buffer_rsrc_t rws, u32 off, bool care, bool need, bool block, u32 call, int lane) {
  if (!care) off = OFF_TMO;
  const u64 t0 = __builtin_amdgcn_s_memrealtime();
  for (;;) {
    const u32x4 v = __builtin_amdgcn_raw_buffer_load_b128(rws, off, 0, 17);
    asm volatile("" ::: "memory");
    const bool set = !care || (v.x >= call && v.y >= call && v.z >= call && v.w >= call);
    if (__all(!need || set)) return __all(set) ? 2 : 1;
    if (!block) return 0;
    const bool dead = __any(!care && v.x == call);
    if (dead || __builtin_amdgcn_s_memrealtime() - t0 > DEADLINE_TICKS) {
      if (lane == 0) st_sys(ws + OFF_TMO, call);
      WAIT_VM(0);
      return 1;
    }
    __builtin_amdgcn_s_sleep(2);
  }
}
#define DMA_SC " sc0 sc1"

typedef __bf16 bf16x2 __attribute__((ext_vector_type(2)));
__device__ __forceinline__ u32 pack_bf16(float lo, float hi) { bf16x2 r; r.x = (__bf16)lo; r.y = (__bf16)hi; return __builtin_bit_cast(u32, r); }
__device__ __forceinline__ float as_f32(u32 v) { return __builtin_bit_cast(float, v); }
__device__ __forceinline__ float bf_lo(u32 v) { return __builtin_bit_cast(float, v << 16); }
__device__ __forceinline__ float bf_hi(u32 v) { return __builtin_bit_cast(float, v & 0xffff0000u); }
__device__ __forceinline__ float wave_max(float m) {
#pragma unroll
  for (int o = 32; o >= 1; o >>= 1) m = fmaxf(m, __shfl_xor(m, o));
  return m;
}

__device__ __forceinline__ void dma16(u32 lds_dst, u32 voff, u32x4 rs, u32 soff) {
  u32 keep;
  asm volatile("s_mov_b32 %0, m0\n\ts_mov_b32 m0, %1\n\ts_nop 2\n\tbuffer_load_dwordx4 %2, %3, %4 offen sc1 lds\n\ts_mov_b32 m0, %0"
               : "=&s"(keep) : "s"(lds_dst), "v"(voff), "s"(rs), "s"(soff) : "memory");
}
__device__ __forceinline__ void dma4b(u32 lds_dst, u32 voff, u32x4 rs) {
  u32 keep;
  asm volatile("s_mov_b32 %0, m0\n\ts_mov_b32 m0, %1\n\ts_nop 2\n\tbuffer_load_dword %2, %3, 0 offen sc1 lds\n\ts_mov_b32 m0, %0"
               : "=&s"(keep) : "s"(lds_dst), "v"(voff), "s"(rs) : "memory");
}

#define WIN 3
#define WAIT_WIN() do { if (WIN == 2) WAIT_VM(3); else if (WIN == 3) WAIT_VM(5); else if (WIN == 4) WAIT_VM(7); else if (WIN == 5) WAIT_VM(9); else WAIT_VM(11); } while (0)
struct Slice { u32 src, dst, flag, kind, step; int dep; };
__device__ __forceinline__ void pusher(const Params& P, char* lds, char* ws, int lane, int wave) {
  const int d = 7 - wave;
  if (d < 1) return;
  const int a = P.rank, q = a ^ d;
  char* pw = P.ws[d];
  const u32 ns = P.ucount[d] * 4u;
  const u32 codes = P.units[d][lane < MAXU / 4 ? lane : 0];
  auto decode = [&](u32 sidx) -> Slice {
    Slice r;
    const u32 u = sidx >> 2, sl = sidx & 3u;
    const u32 code = (__builtin_amdgcn_readlane(codes, u >> 2) >> ((u & 3u) * 8u)) & 0xffu, h = code & 1u;
    if (code < 32) {
      const int t = code >> 1;
      r.kind = 0; r.dep = -1; r.step = 8192u;
      r.src = ((u32)((q ^ tile_dist(t)) * 512 + tile_sub(t, q) * 128) + (sl & 1u) * 64u) * 1024u + ((u32)(q & 1) * 256u + h * 128u + (sl >> 1) * 64u) * 2u;
      r.dst = OFF_XS + (u32)t * PANEL_B + (u32)(d - 2) * 65536u + h * 32768u;
      r.flag = OFF_XF + (u32)t * 256u + (u32)(d - 2) * 32u + h * 16u;
    } else if (code < 64) {
      const u32 p = (code - 32u) >> 1;
      r.kind = 1; r.dep = -1; r.step = 32768u;
      r.src = ((u32)(q & 1) * 256u + h * 128u + sl * 32u) * 8192u + ((u32)(q >> 2) * 2048u + p * 128u) * 2u;
      r.dst = OFF_WS + p * PANEL_B + (u32)(d - 2) * 65536u + h * 32768u;
      r.flag = OFF_WF + p * 256u + (u32)(d - 2) * 32u + h * 16u;
    } else {
      const u32 slot = (u32)box_base(d) + code - 64u;
      r.kind = 2; r.dep = (int)code - 64; r.step = 1024u;
      r.src = OFF_OUT + slot * 32768u + sl * 8192u;
      r.dst = OFF_IN + slot * 32768u;
      r.flag = OFF_PF + slot * 128u;
    }
    r.dst += sl * 8192u; r.flag += sl * 4u;
    return r;
  };
  const u32 vx0 = (u32)(lane >> 3) * 1024u + (u32)(((lane & 7) ^ (lane >> 4)) * 16), vx1 = vx0 ^ 64u;
  const u32 vw0 = (u32)(lane >> 4) * 8192u + (u32)(((lane & 15) ^ (lane >> 4)) * 16), vw1 = vw0 ^ 128u;
  const u32 vp = (u32)lane * 16u;
  const u32x4 rx = mk_rsrc4(P.x, X_BYTES), rw = mk_rsrc4(P.w, W_BYTES), rl = mk_rsrc4(ws, WS_USED);
  const __amdgpu_buffer_rsrc_t rd = mk_rsrc(pw, WS_USED);
  const u32 ring0 = (u32)(size_t)(LAS char*)lds + (u32)wave * 16384u;
  LAS char* ring = (LAS char*)lds + wave * 16384;
  LAS u32* gat = (LAS u32*)((LAS char*)lds + 131072 + wave * 256);
  const u32 dep_off = OFF_OF + ((u32)box_base(d) + (u32)(lane < 48 ? lane : 0)) * 4u;
  u32x4 rs; u32 va[4];
  auto prep = [&](const Slice& c) {
    if (c.kind == 0) { rs = rx; va[0] = vx0; va[1] = vx1; va[2] = vx0; va[3] = vx1; }
    else if (c.kind == 1) { rs = rw; va[0] = vw0; va[1] = vw0; va[2] = vw1; va[3] = vw1; }
    else { rs = rl; va[0] = va[1] = va[2] = va[3] = vp; }
  };
  auto load_all = [&](const Slice& c, u32 buf) {
    prep(c);
#pragma unroll
    for (int k = 0; k < 8; ++k) dma16(ring0 + buf * 8192u + (u32)k * 1024u, va[k & 3], rs, c.src + (u32)k * c.step);
  };
  u32 cur_i = blockIdx.x, nxt_i = NPUSH + blockIdx.x;
  Slice cur = decode(cur_i);
  u32 buf = 0;
  bool have = cur.dep < 0;
  if (have) load_all(cur, 0);
  gat[lane] = 0;
  u32 call = 0;
  if (lane == 0) call = __hip_atomic_fetch_add((GAS u32*)(ws + OFF_CNT + blockIdx.x * 64 + wave * 4), 1u, __ATOMIC_RELAXED, __HIP_MEMORY_SCOPE_AGENT) + 1u;
  call = rfl(call);
  WAIT_VM(0);
  const u32 tk_base = (call - 1u) * (ns - NPUSH);
  const u32 tk_off = OFF_TK + (u32)d * 128u;
  u32 pend = 0;
  auto publish = [&]() {
    if (lane == 0) {
      st_sys(pw + pend, call);
    }
    pend = 0;
  };
  for (;;) {
    if (!have) {
      WAIT_VM(0);
      if (pend) publish();
      wave_wait(ws, dep_off, lane == cur.dep, call, lane);
      load_all(cur, buf);
      dma4b(ring0 - (u32)wave * 16384u + 131072u + (u32)wave * 256u, dep_off, rl);
      WAIT_VM(0);
    }
    const bool more = nxt_i < ns;
    Slice nxt = cur;
    bool pre = false;
    u32 tk = 1;
    if (more) {
      nxt = decode(nxt_i);
      pre = nxt.dep < 0 || rfl(gat[nxt.dep]) >= call;
      u64 ex;
      asm volatile("s_mov_b64 %1, exec\n\ts_mov_b64 exec, 1\n\tbuffer_atomic_add %0, off, %2, %3 sc0\n\ts_mov_b64 exec, %1"
                   : "+v"(tk), "=&s"(ex) : "s"(rl), "s"(tk_off) : "memory");
    }
    if (pre) {
      prep(nxt);
#pragma unroll
      for (int k = 0; k < 8; ++k) {
        const u32x4 v = *(LAS const u32x4*)(ring + buf * 8192u + k * 1024 + lane * 16);
        __builtin_amdgcn_raw_buffer_store_b128(v, rd, vp, cur.dst + (u32)k * 1024u, 17);
        dma16(ring0 + (buf ^ 1u) * 8192u + (u32)k * 1024u, va[k & 3], rs, nxt.src + (u32)k * nxt.step);
        if (k == 4) dma4b(ring0 - (u32)wave * 16384u + 131072u + (u32)wave * 256u, dep_off, rl);
        WAIT_WIN();
        if (k == WIN - 2 && pend) publish();
      }
    } else {
      WAIT_VM(0);
      if (pend) publish();
#pragma unroll
      for (int k = 0; k < 8; ++k) {
        const u32x4 v = *(LAS const u32x4*)(ring + buf * 8192u + k * 1024 + lane * 16);
        __builtin_amdgcn_raw_buffer_store_b128(v, rd, vp, cur.dst + (u32)k * 1024u, 17);
      }
    }
    pend = cur.flag;
    if (!more) break;
    cur = nxt; have = pre; buf ^= 1u;
    asm volatile("" : "+v"(tk));
    nxt_i = rfl(tk) - tk_base + 2u * NPUSH;
  }
  WAIT_VM(0);
  publish();
}

__device__ __forceinline__ void dma4(u32 lds_dst, u32 v0, u32 v1, u32 v2, u32 v3, u32x4 rs, u32 so, u32 st) {
  u32 keep, t;
  asm volatile(
      "s_mov_b32 %0, m0\n\t"
      "s_mov_b32 m0, %2\n\ts_add_u32 %1, %8, %9\n\ts_nop 1\n\tbuffer_load_dwordx4 %3, %7, %8 offen" DMA_SC " lds\n\t"
      "s_add_u32 m0, %2, 0x400\n\ts_nop 0\n\tbuffer_load_dwordx4 %4, %7, %1 offen" DMA_SC " lds\n\t"
      "s_add_u32 m0, %2, 0x800\n\ts_add_u32 %1, %1, %9\n\tbuffer_load_dwordx4 %5, %7, %1 offen" DMA_SC " lds\n\t"
      "s_add_u32 m0, %2, 0xc00\n\ts_add_u32 %1, %1, %9\n\tbuffer_load_dwordx4 %6, %7, %1 offen" DMA_SC " lds\n\t"
      "s_mov_b32 m0, %0"
      : "=&s"(keep), "=&s"(t)
      : "s"(lds_dst), "v"(v0), "v"(v1), "v"(v2), "v"(v3), "s"(rs), "s"(so), "s"(st)
      : "memory", "scc");
}

constexpr int NST = 5;
__device__ __forceinline__ void worker(const Params& P, char* lds, char* ws, int lane, int wave) {
  const int a = P.rank;
  u32 call = 0;
  if (lane == 0) call = __hip_atomic_fetch_add((GAS u32*)(ws + OFF_CNT + blockIdx.x * 64 + wave * 4), 1u, __ATOMIC_RELAXED, __HIP_MEMORY_SCOPE_AGENT) + 1u;
  call = rfl(call);
  STAMP_AT(0);
  u32 jobs = P.jobs[blockIdx.x - NPUSH];
  const bool loader = wave >= 4;
  const int wr = (wave >> 1) & 1, wc = wave & 1, fr = lane & 15, g = lane >> 4;
  const __amdgpu_buffer_rsrc_t rws = mk_rsrc(ws, WS_USED);
  char* am_dst = P.ws[lane & 7];
  asm volatile("" : "+v"(am_dst));
  int stamp = 1; (void)stamp;

  for (; (jobs & 0x3ffu) != 0; jobs >>= 10) {
    const int kind = (jobs >> 8) & 3, ja = (jobs >> 4) & 15, p = jobs & 15;
    int dO, sub, chalf;
    if (kind == JOB_F) { dO = 0; sub = ja & 3; chalf = (a >> 2) ^ (ja >> 2); }
    else { dO = tile_dist(ja); sub = tile_sub(ja, a); chalf = a >> 2; }
    const u32 col0 = (u32)(chalf * 2048 + p * 128);
    const int nh = kind == JOB_F ? 0 : dO >= 2 ? 10 : 12, nkt = 8 + 2 * nh;
    f32x4 acc[4][4];

    if (loader) {
      const u32 l4 = (u32)(wave - 4) * 4u;
      const u32 row0 = (u32)((a ^ dO) * 512 + sub * 128);
      const int skip = dO >= 2 ? dO - 2 : 6;
      const u32 vx0 = (u32)(lane >> 3) * 1024u + (u32)(((lane & 7) ^ (lane >> 4)) * 16), vx1 = vx0 ^ 64u;
      const u32 vw0 = (u32)(lane >> 4) * 8192u + (u32)(((lane & 15) ^ (lane >> 4)) * 16), vw1 = vw0 ^ 128u;
      const u32 vp = (u32)lane * 16u;
      const u32x4 rx = mk_rsrc4(P.x, X_BYTES), rw = mk_rsrc4(P.w, W_BYTES), rl = mk_rsrc4(ws, WS_USED);
      const u32 lds0 = (u32)(size_t)(LAS char*)lds + l4 * 1024u;
      const int fl = lane & 15;
      const bool care = lane < 32 && fl < 12 && (fl >> 1) != skip;
      const u32 foff = (lane < 16 ? OFF_XF + (u32)ja * 256u : OFF_WF + (u32)p * 256u) + (u32)fl * 16u;
      int issued = 0, islot = 0, open = 8;
      auto issue = [&]() {
        const u32 dst = lds0 + (u32)islot * 32768u;
        if (issued < 8) {
          dma4(dst, vx0, vx1, vx0, vx1, rx, (row0 + l4 * 8u) * 1024u + (u32)issued * 128u, 8192u);
          dma4(dst + 16384u, vw0, vw0, vw1, vw1, rw, ((u32)issued * 64u + l4 * 4u) * 8192u + col0 * 2u, 32768u);
        } else {
          int k = issued - 8; u32 o = l4 * 1024u;
          if (k >= nh) { k -= nh; o += 32768u; }
          int seg = k >> 1; seg += seg >= skip;
          o += (u32)seg * 65536u + (u32)(k & 1) * 16384u;
          dma4(dst, vp, vp, vp, vp, rl, OFF_XS + (u32)ja * PANEL_B + o, 1024u);
          dma4(dst + 16384u, vp, vp, vp, vp, rl, OFF_WS + (u32)p * PANEL_B + o, 1024u);
        }
        ++issued; islot = islot == NST - 1 ? 0 : islot + 1;
      };
      for (int j = 0; j <= nkt; ++j) {
        const int cap = j + NST - (j > 0) < nkt ? j + NST - (j > 0) : nkt;
        if (issued < cap && issued == open) {
          const int hneed = open > 8;
          const int got = wave_poll4(ws, rws, foff, care, (fl & 1) == hneed, issued == j, call, lane);
          if (got) {
            open = got == 2 ? nkt : open + nh;
          }
        }
        while (issued < cap && issued < open) issue();
        const int after = issued - j - 1;
        if (after >= 4) WAIT_VM(32); else if (after == 3) WAIT_VM(24); else if (after == 2) WAIT_VM(16); else if (after == 1) WAIT_VM(8); else WAIT_VM(0);
        __builtin_amdgcn_s_barrier();
      }
    } else {
      const u32 xa0 = (u32)((wr * 64 + fr) * 128 + ((g ^ (fr >> 1)) * 16));
      const int tq = fr >> 2, tp = fr & 3;
      const u32 wa0 = 16384u + (u32)((8 * g + tq) * 256 + (((8 * wc + 2 * tp) ^ (tq | ((g & 1) << 3))) * 16) + (tp & 1) * 8);
#pragma unroll
      for (int m = 0; m < 4; ++m)
#pragma unroll
        for (int n = 0; n < 4; ++n) acc[m][n] = (f32x4){0.f, 0.f, 0.f, 0.f};
      auto readf = [&](bf16x8 (&xf)[4], bf16x8 (&wf)[4], int slot, int ks) {
        LAS char* st = (LAS char*)lds + slot * 32768;
#pragma unroll
        for (int i = 0; i < 4; ++i) {
          const u32 o = (wa0 ^ (u32)((i >> 1) * 16 + (i & 1) * 8)) + (u32)(ks * 8192);
          const s16x4 lo = __builtin_amdgcn_ds_read_tr16_b64_v4i16((LAS s16x4*)(st + o));
          const s16x4 hi = __builtin_amdgcn_ds_read_tr16_b64_v4i16((LAS s16x4*)(st + o + 1024));
          wf[i] = __builtin_bit_cast(bf16x8, __builtin_shufflevector(lo, hi, 0, 1, 2, 3, 4, 5, 6, 7));
          xf[i] = *(LAS const bf16x8*)(st + ((xa0 ^ (u32)(ks * 64)) + (u32)(i * 2048)));
        }
      };
      auto mma = [&](const bf16x8 (&xf)[4], const bf16x8 (&wf)[4]) {
#pragma unroll
        for (int m = 0; m < 4; ++m)
#pragma unroll
          for (int n = 0; n < 4; ++n) acc[m][n] = __builtin_amdgcn_mfma_f32_16x16x32_bf16(wf[n], xf[m], acc[m][n], 0, 0, 0);
      };
#define INTERLEAVE(id) do { _Pragma("unroll") for (int i_ = 0; i_ < 6; ++i_) { __builtin_amdgcn_sched_group_barrier(0x8, 1, id); __builtin_amdgcn_sched_group_barrier(0x100, 2, id); } \
                            __builtin_amdgcn_sched_group_barrier(0x8, 10, id); __builtin_amdgcn_sched_barrier(0); } while (0)
      bf16x8 x0[4], w0[4], x1[4], w1[4];
      int slot = 0;
      BARRIER();
      readf(x0, w0, 0, 0);
      for (int kt = 0; kt < nkt; ++kt) {
        __builtin_amdgcn_s_waitcnt(0xC07F);
        __builtin_amdgcn_sched_barrier(0);
        readf(x1, w1, slot, 1); mma(x0, w0);
        INTERLEAVE(0);
        slot = slot == NST - 1 ? 0 : slot + 1;
        BARRIER();
        __builtin_amdgcn_sched_barrier(0);
        readf(x0, w0, slot, 0); mma(x1, w1);
        INTERLEAVE(1);
      }
      __builtin_amdgcn_s_waitcnt(0xC07F);
      STAMP_AT(stamp++);
    }

    const u32 vo = (u32)((wave & 3) * 8192 + lane * 16);
    if (kind == JOB_S) {
      const u32 slot = (u32)(box_base(dO) + box_idx(ja, p));
      if (!loader) {
#pragma unroll
        for (int jj = 0; jj < 8; ++jj) {
          const f32x4 u0 = acc[jj >> 1][(jj & 1) * 2], u1 = acc[jj >> 1][(jj & 1) * 2 + 1];
          u32x4 v; v.x = pack_bf16(u0[0], u0[1]); v.y = pack_bf16(u0[2], u0[3]); v.z = pack_bf16(u1[0], u1[1]); v.w = pack_bf16(u1[2], u1[3]);
          __builtin_amdgcn_raw_buffer_store_b128(v, rws, vo + (u32)(jj * 1024), OFF_OUT + slot * 32768u, 16);
        }
        WAIT_VM(0);
      }
      BARRIER();
      if (threadIdx.x == 0) st_agent(ws + OFF_OF + slot * 4u, call);
      STAMP_AT(stamp++);
      continue;
    }

    LAS float* red = (LAS float*)lds;
    if (!loader) {
      u32 s0, s1;
      if (kind == JOB_O) { s0 = s1 = (u32)(p * 3 + sub); }
      else if ((ja >> 2) == 0) { s0 = (u32)(48 + p); s1 = (u32)(64 + p); }
      else { const int d0 = a ^ ((chalf << 2) | (sub & 2)); s0 = (u32)(box_base(d0) + p * 2 + (sub & 1)); s1 = (u32)(box_base(d0 ^ 1) + p * 2 + (sub & 1)); }
      wave_wait(ws, OFF_PF + (lane == 0 ? s0 : s1) * 128u + (u32)wave * 4u, lane < 2, call, lane);
      STAMP_AT(stamp++);
      u32x4 v0[8], v1[8];
#pragma unroll
      for (int jj = 0; jj < 8; ++jj) v0[jj] = __builtin_amdgcn_raw_buffer_load_b128(rws, vo + (u32)(jj * 1024), OFF_IN + s0 * 32768u, 17);
      if (kind == JOB_F) {
#pragma unroll
        for (int jj = 0; jj < 8; ++jj) v1[jj] = __builtin_amdgcn_raw_buffer_load_b128(rws, vo + (u32)(jj * 1024), OFF_IN + s1 * 32768u, 17);
      } else {
#pragma unroll
        for (int jj = 0; jj < 8; ++jj) v1[jj] = (u32x4){0u, 0u, 0u, 0u};
      }
      float mx = 0.f;
#pragma unroll
      for (int jj = 0; jj < 8; ++jj) {
        f32x4& u0 = acc[jj >> 1][(jj & 1) * 2]; f32x4& u1 = acc[jj >> 1][(jj & 1) * 2 + 1];
        u0[0] += bf_lo(v0[jj].x) + bf_lo(v1[jj].x); u0[1] += bf_hi(v0[jj].x) + bf_hi(v1[jj].x);
        u0[2] += bf_lo(v0[jj].y) + bf_lo(v1[jj].y); u0[3] += bf_hi(v0[jj].y) + bf_hi(v1[jj].y);
        u1[0] += bf_lo(v0[jj].z) + bf_lo(v1[jj].z); u1[1] += bf_hi(v0[jj].z) + bf_hi(v1[jj].z);
        u1[2] += bf_lo(v0[jj].w) + bf_lo(v1[jj].w); u1[3] += bf_hi(v0[jj].w) + bf_hi(v1[jj].w);
#pragma unroll
        for (int e = 0; e < 4; ++e) mx = fmaxf(mx, fmaxf(u0[e], u1[e]));
      }
      mx = wave_max(mx);
      if (lane == 0) red[wave] = mx;
    }
    BARRIER();
    if (wave == 0 && lane < NR) {
      const float m4 = fmaxf(fmaxf(red[0], red[1]), fmaxf(red[2], red[3]));
      const u32 f = (u32)(p * 8 + (kind == JOB_F ? ja : sub));
      __hip_atomic_store((GAS u64*)(am_dst + OFF_AM + (f * 8u + (u32)a) * 8u), ((u64)call << 32) | __builtin_bit_cast(u32, m4), __ATOMIC_RELAXED, __HIP_MEMORY_SCOPE_SYSTEM);
    }
    STAMP_AT(stamp++);
    float gm = 0.f; u32 tmo_v = 0;
    {
      const u64 t0 = __builtin_amdgcn_s_memrealtime();
      for (;;) {
        const u32x4 v = __builtin_amdgcn_raw_buffer_load_b128(rws, threadIdx.x * 16u, OFF_AM, 17);
        tmo_v = __builtin_amdgcn_raw_buffer_load_b32(rws, 0, OFF_TMO, 17);
        asm volatile("" ::: "memory");
        gm = fmaxf(as_f32(v.x), as_f32(v.z));
        if (__all(v.y >= call && v.w >= call)) break;
        if (rfl(tmo_v) == call || __builtin_amdgcn_s_memrealtime() - t0 > DEADLINE_TICKS) {
          if (lane == 0) st_sys(ws + OFF_TMO, call);
          WAIT_VM(0);
          tmo_v = call;
          break;
        }
        __builtin_amdgcn_s_sleep(2);
      }
    }
    gm = wave_max(gm);
    if (lane == 0) { red[8 + wave] = gm; ((LAS u32*)red)[16 + wave] = tmo_v; }
    BARRIER();
    if (loader) return;
    bool bad = false;
#pragma unroll
    for (int i = 0; i < 8; ++i) { gm = fmaxf(gm, red[8 + i]); bad = bad | (((LAS u32*)red)[16 + i] == call); }
    STAMP_AT(stamp++);
    const float inv = gm > 0.f ? 127.0f / gm : 0.f;
    const __amdgpu_buffer_rsrc_t ro = mk_rsrc(P.out, OUT_BYTES);
#pragma unroll
    for (int m = 0; m < 4; ++m) {
      u32 qd[4];
#pragma unroll
      for (int n = 0; n < 4; ++n) {
        u32 r = 0;
#pragma unroll
        for (int e = 0; e < 4; ++e) r |= (u32)(int)__builtin_rintf(__builtin_amdgcn_fmed3f(acc[m][n][e] * inv, 0.f, 127.f)) << (8 * e);
        qd[n] = bad ? 0x80808080u : r;
      }
      const bool sw = g & 1;
      u32x4 v; v.x = sw ? qd[1] : qd[0]; v.y = sw ? qd[0] : qd[1]; v.z = sw ? qd[3] : qd[2]; v.w = sw ? qd[2] : qd[3];
      __builtin_amdgcn_raw_buffer_store_b128(v, ro, (u32)((wr * 64 + m * 16 + fr) * 4096 + wc * 64 + g * 16), (u32)sub * (128u * 4096u) + col0, 0);
    }
    STAMP_AT(stamp++);
    return;
  }
}

__global__ void __launch_bounds__(512, 2) gemm_rs_q8(const Params P) {
  __shared__ __attribute__((aligned(16))) char lds[5 * 32768];
  const int lane = threadIdx.x & 63, wave = (int)rfl(threadIdx.x >> 6);
  char* ws = P.ws[0];
  if (blockIdx.x < NPUSH) pusher(P, lds, ws, lane, wave);
  else worker(P, lds, ws, lane, wave);
}

#define LAG 4
void build_schedule(Params& P) {
#define ORD 0
  static const int orders[3][32] = {
    {0, 1, 2, 5, 6, 7, 8, 16, 17, 18, 19, 9, 10, 11, 12, 20, 21, 22, 23, 24, 25, 26, 27, 3, 4, 28, 29, 30, 31, 13, 14, 15},
    {0, 1, 2, 16, 5, 6, 17, 7, 8, 18, 19, 9, 10, 11, 12, 20, 21, 22, 23, 24, 25, 4, 3, 26, 27, 28, 29, 30, 31, 13, 14, 15},
    {0, 1, 2, 5, 6, 7, 8, 16, 17, 9, 10, 11, 12, 18, 19, 4, 20, 21, 22, 23, 24, 25, 26, 27, 3, 28, 29, 30, 31, 13, 14, 15}};
  const int* order = orders[ORD];
  int pos[32];
  for (int i = 0; i < 32; ++i) pos[order[i]] = i;
  auto ready = [&](int t, int p) { return pos[t] > pos[16 + p] ? pos[t] : pos[16 + p]; };
  for (int d = 0; d < NR; ++d) {
    unsigned char list[MAXU]; int n = 0;
    for (int i = 0; i < MAXU; ++i) list[i] = 0;
    if (d >= 2)
      for (int i = 0; i < 32 + LAG; ++i)
        for (int h = 0; h < 2; ++h) {
          const int k = i - h * LAG;
          if (k < 0 || k >= 32) continue;
          const int c = order[k];
          if (c < 16 && tile_dist(c) == d) continue;
          list[n++] = (unsigned char)(c * 2 + h);
        }
    if (d >= 1)
      for (int r = 0; r < 32; ++r)
        for (int p = 0; p < 16; ++p)
          for (int t = 0; t < 13; ++t)
            if (tile_dist(t) == d && ready(t, p) == r) list[n++] = (unsigned char)(64 + box_idx(t, p));
    P.ucount[d] = (u32)n;
    for (int i = 0; i < MAXU / 4; ++i) P.units[d][i] = list[4 * i] | list[4 * i + 1] << 8 | list[4 * i + 2] << 16 | (u32)list[4 * i + 3] << 24;
  }
  constexpr int WPX = NWORK / 8, DBL = 32 - WPX;
  for (int x = 0; x < 8; ++x) {
    int ht[32], hp[32], n = 0;
    static const int grp[16] = {0, 1, 0, 2, 1, 1, 0, 1, 0, 1, 0, 1, 0, 2, 2, 2};
    for (int r = 0; r < 32; ++r)
      for (int p = x & 3; p < 16; p += 4)
        for (int t = 0; t < 16; ++t) {
          const int gsel = grp[t] == 2 ? (p >> 2) & 1 : grp[t];
          if (gsel == (x >> 2) && ready(t, p) == r) { ht[n] = t; hp[n] = p; ++n; }
        }
    auto enc = [&](int i) { return (u32)(((ht[i] >= 13 ? JOB_O : JOB_S) << 8) | (ht[i] << 4) | hp[i]); };
    int f = 0;
    for (int s = 0; s < WPX; ++s) {
      u32 jb;
      if (s < DBL) jb = enc(s) | enc(WPX + s) << 10;
      else {
        jb = enc(s);
        if (ht[s] < 13 && f < 10) { jb |= (u32)((JOB_F << 8) | ((3 + f % 5) << 4) | (x + 8 * (f / 5))) << 10; ++f; }
      }
      P.jobs[s * 8 + x] = jb;
    }
  }
}

}

extern "C" void kernel_launch(int rank, int nranks, ncclComm_t comm, hipStream_t stream,
                              void* const* d_in, const int* in_sizes, int n_in,
                              void* d_out, int out_size,
                              const int* in_dtypes, int out_dtype,
                              void* const* d_out_all, void* const* d_ws_all, size_t ws_bytes) {
  (void)comm; (void)in_dtypes; (void)out_dtype; (void)d_out_all;
  if (nranks != NR || (unsigned)rank >= (unsigned)NR || n_in != 2 || ws_bytes < WS_USED || (u32)out_size != OUT_BYTES ||
      (u32)in_sizes[0] * 2u != X_BYTES || (u32)in_sizes[1] * 2u != W_BYTES) {
    fprintf(stderr, "unexpected problem shape\n");
    return;
  }
  Params P;
  P.x = (const char*)d_in[0]; P.w = (const char*)d_in[1]; P.out = (char*)d_out;
  for (int d = 0; d < NR; ++d) P.ws[d] = (char*)d_ws_all[rank ^ d];
  P.rank = rank; P.pad = 0;
  build_schedule(P);
  hipLaunchKernelGGL(gemm_rs_q8, dim3(GRID), dim3(512), 0, stream, P);
  HIP_CHECK(hipGetLastError());
}
```

```cpp
#include <hip/hip_runtime.h>
#include <rccl/rccl.h>
#include <cstdio>
#include <cstdint>

#define HIP_CHECK(expr) do { hipError_t _e = (expr); if (_e != hipSuccess) { \
    fprintf(stderr, "HIP error %s at %s:%d\n", hipGetErrorString(_e), __FILE__, __LINE__); \
    return; } } while (0)

namespace {

typedef unsigned u32;
typedef unsigned long long u64;
typedef u32 u32x4 __attribute__((ext_vector_type(4)));
typedef u32 u32x2 __attribute__((ext_vector_type(2)));
typedef float f32x4 __attribute__((ext_vector_type(4)));
typedef short s16x4 __attribute__((ext_vector_type(4)));
typedef short s16x8 __attribute__((ext_vector_type(8)));
typedef __bf16 bf16x8 __attribute__((ext_vector_type(8)));
#define LAS __attribute__((address_space(3)))
#define GAS __attribute__((address_space(1)))

constexpr int NR = 8;
#define NPUSH 24
constexpr int GRID = 256;
constexpr int MAXU = 64;
constexpr u64 DEADLINE_TICKS = 100000000ull;

constexpr u32 X_BYTES = 4096u * 512u * 2u, W_BYTES = 512u * 4096u * 2u, OUT_BYTES = 512u * 4096u;
constexpr u32 OFF_CNT = 0;
constexpr u32 OFF_TMO = 16u << 10;
constexpr u32 OFF_TK = (16u << 10) + 128u;
constexpr u32 OFF_TP = 18u << 10;
constexpr u32 OFF_XF = 20u << 10;
constexpr u32 OFF_WF = 24u << 10;
constexpr u32 OFF_AM = 32u << 10;
constexpr u32 OFF_PF = 40u << 10;
constexpr u32 OFF_DBG = 512u << 10;
constexpr u32 OFF_XS = 2u << 20;
constexpr u32 OFF_WS = 8u << 20;
constexpr u32 OFF_IN = 14u << 20;
constexpr u32 WS_USED = OFF_IN + 208u * 32768u;
#define TAU 245
#define TAU_C 300
#define TAU_X 230
#define TAU_IN 230
#define T_HEAD 200
constexpr u32 PANEL_B = 6u * 65536u;

enum { JOB_NONE = 0, JOB_S = 1, JOB_O = 2, JOB_F = 3 };

struct Params {
  const char* x; const char* w; char* out;
  char* ws[NR];
  int rank; int pad;
  u32 jobs[GRID];
  u32 ucount[NR];
  u32 units[NR][MAXU / 4];
};

__host__ __device__ inline int tile_dist(int t) { return t < 3 ? 1 : t < 5 ? t - 1 : t < 13 ? 4 + ((t - 5) & 3) : 0; }
__host__ __device__ inline int tile_sub(int t, int a) { return t < 3 ? t : t < 5 ? 3 : t < 13 ? (a & 2) + ((t - 5) >> 2) : t - 13; }
__host__ __device__ inline int box_base(int d) { return d == 1 ? 0 : d < 4 ? 48 + 16 * (d - 2) : 80 + 32 * (d - 4); }
__host__ __device__ inline int box_cnt(int d) { return d == 1 ? 48 : d < 4 ? 16 : 32; }
__host__ __device__ inline int box_idx(int t, int p) { return t < 3 ? p * 3 + t : t < 5 ? p : p * 2 + ((t - 5) >> 2); }

__device__ __forceinline__ u32 rfl(u32 v) { return __builtin_amdgcn_readfirstlane(v); }
__device__ __forceinline__ __amdgpu_buffer_rsrc_t mk_rsrc(const void* p, u32 bytes) {
  return __builtin_amdgcn_make_buffer_rsrc((void*)p, 0, bytes, 0x00020000);
}
__device__ __forceinline__ u32x4 mk_rsrc4(const void* p, u32 bytes) {
  u64 a = (u64)p; u32x4 r; r.x = rfl((u32)a); r.y = rfl((u32)(a >> 32)) & 0xffffu; r.z = bytes; r.w = 0x00020000u; return r;
}
__device__ __forceinline__ u32 ld_sys(const char* p) { return __hip_atomic_load((GAS const u32*)p, __ATOMIC_RELAXED, __HIP_MEMORY_SCOPE_SYSTEM); }
__device__ __forceinline__ void st_sys(char* p, u32 v) { __hip_atomic_store((GAS u32*)p, v, __ATOMIC_RELAXED, __HIP_MEMORY_SCOPE_SYSTEM); }
__device__ __forceinline__ void st_agent(char* p, u32 v) { __hip_atomic_store((GAS u32*)p, v, __ATOMIC_RELAXED, __HIP_MEMORY_SCOPE_AGENT); }
#define WAIT_VM(n) asm volatile("s_waitcnt vmcnt(" #n ")" ::: "memory")
#define BARRIER() do { asm volatile("" ::: "memory"); __builtin_amdgcn_s_waitcnt(0xC07F);   __builtin_amdgcn_s_barrier(); asm volatile("" ::: "memory"); } while (0)

#define STAMP_AT(i) do {} while (0)

__device__ __forceinline__ void wave_wait(char* ws, u32 off, bool care, u32 call, int lane) {
  if (lane == 63) { off = OFF_TMO; }
  else if (!care) { off = OFF_TMO; }
  const u64 t0 = __builtin_amdgcn_s_memrealtime();
  for (;;) {
    const u32 v = ld_sys(ws + off);
    const bool ok = (lane == 63 || !care) ? true : (v >= call);
    if (__all(ok)) break;
    const bool dead = __any(lane == 63 && v == call);
    if (dead || __builtin_amdgcn_s_memrealtime() - t0 > DEADLINE_TICKS) {
      if (lane == 0) st_sys(ws + OFF_TMO, call);
      WAIT_VM(0);
      break;
    }
    __builtin_amdgcn_s_sleep(4);
  }
}

__device__ __forceinline__ int wave_poll4(char* ws, __amdgpu_buffer_rsrc_t rws, u32 off, bool care, int have, bool block, u32 call, int lane) {
  if (!care) off = OFF_TMO;
  const u64 t0 = __builtin_amdgcn_s_memrealtime();
  for (;;) {
    const u32x4 v = __builtin_amdgcn_raw_buffer_load_b128(rws, off, 0, 17);
    asm volatile("" ::: "memory");
    const u32 m = (u32)__ballot(care && v.x >= call && v.y >= call && v.z >= call && v.w >= call);
    const int n = __builtin_ctz(~(m & (m >> 16)));
    if (n > have || !block) return n;
    const bool dead = __any(!care && v.x == call);
    if (dead || __builtin_amdgcn_s_memrealtime() - t0 > DEADLINE_TICKS) {
      if (lane == 0) st_sys(ws + OFF_TMO, call);
      WAIT_VM(0);
      return have + 1;
    }
    __builtin_amdgcn_s_sleep(2);
  }
}
#define DMA_SC " sc0 sc1"

typedef __bf16 bf16x2 __attribute__((ext_vector_type(2)));
__device__ __forceinline__ u32 pack_bf16(float lo, float hi) { bf16x2 r; r.x = (__bf16)lo; r.y = (__bf16)hi; return __builtin_bit_cast(u32, r); }
__device__ __forceinline__ float as_f32(u32 v) { return __builtin_bit_cast(float, v); }
__device__ __forceinline__ float bf_lo(u32 v) { return __builtin_bit_cast(float, v << 16); }
__device__ __forceinline__ float bf_hi(u32 v) { return __builtin_bit_cast(float, v & 0xffff0000u); }
__device__ __forceinline__ float wave_max(float f) {
  int m = __builtin_bit_cast(int, f);
  m = max(m, __builtin_amdgcn_update_dpp(0, m, 0x111, 0xf, 0xf, false));
  m = max(m, __builtin_amdgcn_update_dpp(0, m, 0x112, 0xf, 0xf, false));
  m = max(m, __builtin_amdgcn_update_dpp(0, m, 0x114, 0xf, 0xf, false));
  m = max(m, __builtin_amdgcn_update_dpp(0, m, 0x118, 0xf, 0xf, false));
  const int a = max(__builtin_amdgcn_readlane(m, 15), __builtin_amdgcn_readlane(m, 31));
  const int b = max(__builtin_amdgcn_readlane(m, 47), __builtin_amdgcn_readlane(m, 63));
  return __builtin_bit_cast(float, max(a, b));
}

__device__ __forceinline__ void dma4(u32 lds_dst, u32 v0, u32 v1, u32 v2, u32 v3, u32x4 rs, u32 so, u32 st) {
  u32 keep, t;
  asm volatile(
      "s_mov_b32 %0, m0\n\t"
      "s_mov_b32 m0, %2\n\ts_add_u32 %1, %8, %9\n\ts_nop 1\n\tbuffer_load_dwordx4 %3, %7, %8 offen" DMA_SC " lds\n\t"
      "s_add_u32 m0, %2, 0x400\n\ts_nop 0\n\tbuffer_load_dwordx4 %4, %7, %1 offen" DMA_SC " lds\n\t"
      "s_add_u32 m0, %2, 0x800\n\ts_add_u32 %1, %1, %9\n\tbuffer_load_dwordx4 %5, %7, %1 offen" DMA_SC " lds\n\t"
      "s_add_u32 m0, %2, 0xc00\n\ts_add_u32 %1, %1, %9\n\tbuffer_load_dwordx4 %6, %7, %1 offen" DMA_SC " lds\n\t"
      "s_mov_b32 m0, %0"
      : "=&s"(keep), "=&s"(t)
      : "s"(lds_dst), "v"(v0), "v"(v1), "v"(v2), "v"(v3), "s"(rs), "s"(so), "s"(st)
      : "memory", "scc");
}

#define WIN 3
template <int N> __device__ __forceinline__ void wait_vm() { asm volatile("s_waitcnt vmcnt(%0)" :: "n"(N) : "memory"); }
struct Slice { u32 src, dst, flag, kind; };
__device__ __forceinline__ u32 draw_call(char* ws, int lane, int wave) {
  u32 call = 0;
  if (lane == 0) call = __hip_atomic_fetch_add((GAS u32*)(ws + OFF_CNT + blockIdx.x * 64 + wave * 4), 1u, __ATOMIC_RELAXED, __HIP_MEMORY_SCOPE_AGENT) + 1u;
  return rfl(call);
}
__device__ __forceinline__ void pusher(const Params& P, char* lds, char* ws, int lane, int wave) {
  const int d = 2 + (wave + (int)(blockIdx.x >> 3) * 2 + (int)blockIdx.x) % 6;
  const int a = P.rank, q = a ^ d;
  char* pw = P.ws[d];
  const u32 ns = P.ucount[d] * 4u;
  const u32 codes = P.units[d][lane & (MAXU / 4 - 1)];
  auto decode = [&](u32 sidx) -> Slice {
    Slice r;
    const u32 u = sidx >> 2, sl = sidx & 3u;
    const u32 code = (__builtin_amdgcn_readlane(codes, u >> 2) >> ((u & 3u) * 8u)) & 0xffu, h = code & 1u;
    if (code < 32) {
      const int t = code >> 1;
      r.kind = 0;
      r.src = ((u32)((q ^ tile_dist(t)) * 512 + tile_sub(t, q) * 128) + (sl & 1u) * 64u) * 1024u + ((u32)(q & 1) * 256u + h * 128u + (sl >> 1) * 64u) * 2u;
      r.dst = OFF_XS + (u32)t * PANEL_B;
      r.flag = OFF_XF + (u32)t * 256u;
    } else {
      const u32 p = (code - 32u) >> 1;
      r.kind = 1;
      r.src = ((u32)(q & 1) * 256u + h * 128u + sl * 32u) * 8192u + ((u32)(q >> 2) * 2048u + p * 128u) * 2u;
      r.dst = OFF_WS + p * PANEL_B;
      r.flag = OFF_WF + p * 256u;
    }
    r.dst += (u32)(d - 2) * 65536u + h * 32768u + sl * 8192u; r.flag += (u32)(d - 2) * 32u + h * 16u + sl * 4u;
    return r;
  };
  const u32 vx0 = (u32)(lane >> 3) * 1024u + (u32)(((lane & 7) ^ (lane >> 4)) * 16), vx1 = vx0 ^ 64u;
  const u32 vw0 = (u32)(lane >> 4) * 8192u + (u32)(((lane & 15) ^ (lane >> 4)) * 16), vw1 = vw0 ^ 128u;
  const u32 vp = (u32)lane * 16u;
  const u32x4 rx = mk_rsrc4(P.x, X_BYTES), rw = mk_rsrc4(P.w, W_BYTES), rl = mk_rsrc4(ws, WS_USED);
  const __amdgpu_buffer_rsrc_t rd = mk_rsrc(pw, WS_USED);
  const u32 ring0 = (u32)(size_t)(LAS char*)lds + (u32)wave * 16384u;
  LAS char* ring = (LAS char*)lds + wave * 16384;
  auto load_all = [&](const Slice& c, u32 buf) {
    const u32 dst = ring0 + buf * 8192u;
    if (c.kind == 0) { dma4(dst, vx0, vx1, vx0, vx1, rx, c.src, 8192u); dma4(dst + 4096u, vx0, vx1, vx0, vx1, rx, c.src + 32768u, 8192u); }
    else { dma4(dst, vw0, vw0, vw1, vw1, rw, c.src, 32768u); dma4(dst + 4096u, vw0, vw0, vw1, vw1, rw, c.src + 131072u, 32768u); }
  };
  u32 nxt_i = NPUSH + blockIdx.x;
  Slice cur = decode(blockIdx.x);
  u32 buf = 0;
  load_all(cur, 0);
  const u32 call = draw_call(ws, lane, wave);
  WAIT_VM(0);
  const u32 tk_base = (call - 1u) * (ns - NPUSH);
  const u32 tk_off = OFF_TK + (u32)d * 128u;
  u32 pend = 0;
  auto publish = [&]() {
    if (lane == 0) {
      st_sys(pw + pend, call);
    }
  };
  auto send = [&](int k) {
    const u32x4 v = *(LAS const u32x4*)(ring + buf * 8192u + k * 1024 + lane * 16);
    __builtin_amdgcn_raw_buffer_store_b128(v, rd, vp, cur.dst + (u32)k * 1024u, 17);
  };
  while (nxt_i < ns) {
    const Slice nxt = decode(nxt_i);
    u32 tk = 1;
    send(0);
    load_all(nxt, buf ^ 1u);
    u64 ex;
    asm volatile("s_mov_b64 %1, exec\n\ts_mov_b64 exec, 1\n\tbuffer_atomic_add %0, off, %2, %3 sc0\n\ts_mov_b64 exec, %1"
                 : "+v"(tk), "=&s"(ex) : "s"(rl), "s"(tk_off) : "memory");
#pragma unroll
    for (int k = 0; k < 8; ++k) {
      if (k) send(k);
      if (k < WIN) wait_vm<WIN - 1 + 9>(); else wait_vm<WIN - 1>();
      if (k == WIN - 2 && pend) publish();
    }
    pend = cur.flag;
    cur = nxt; buf ^= 1u;
    asm volatile("" : "+v"(tk));
    nxt_i = rfl(tk) - tk_base + 2u * NPUSH;
  }
#pragma unroll
  for (int k = 0; k < 8; ++k) send(k);
  WAIT_VM(8);
  publish();
  pend = cur.flag;
  WAIT_VM(0);
  publish();
}

constexpr int NST = 5;

struct Job { int kind, ja, p, dO, sub, nh, nkt; u32 col0; };
__device__ __forceinline__ Job decode_job(u32 code, int a) {
  Job j; j.kind = (code >> 8) & 3; j.ja = (code >> 4) & 15; j.p = code & 15;
  int chalf;
  if (j.kind == JOB_F) { j.dO = 0; j.sub = j.ja & 3; chalf = (a >> 2) ^ (j.ja >> 2); }
  else { j.dO = tile_dist(j.ja); j.sub = tile_sub(j.ja, a); chalf = a >> 2; }
  j.col0 = (u32)(chalf * 2048 + j.p * 128);
  j.nh = j.kind == JOB_F ? 0 : j.dO >= 2 ? 10 : 12;
  j.nkt = 8 + 2 * j.nh;
  return j;
}

__device__ __forceinline__ void collect_max(char* lds, char* ws, __amdgpu_buffer_rsrc_t rws, u32 call, int lane, int wave) {
  float gm; u32 tmo_v;
  const u64 t0 = __builtin_amdgcn_s_memrealtime();
  for (;;) {
    const u32x4 v = __builtin_amdgcn_raw_buffer_load_b128(rws, threadIdx.x * 16u, OFF_AM, 17);
    tmo_v = __builtin_amdgcn_raw_buffer_load_b32(rws, 0, OFF_TMO, 17);
    asm volatile("" ::: "memory");
    gm = fmaxf(as_f32(v.x), as_f32(v.z));
    if (__all(v.y >= call && v.w >= call)) break;
    if (rfl(tmo_v) == call || __builtin_amdgcn_s_memrealtime() - t0 > DEADLINE_TICKS) {
      if (lane == 0) st_sys(ws + OFF_TMO, call);
      WAIT_VM(0);
      tmo_v = call;
      break;
    }
    __builtin_amdgcn_s_sleep(2);
  }
  gm = wave_max(gm);
  if (lane == 0) { ((LAS float*)lds)[8 + wave] = gm; ((LAS u32*)lds)[16 + wave] = tmo_v; }
  BARRIER();
}

__device__ __forceinline__ void loader(const Params& P, char* lds, char* ws, int lane, int wave, u32 call, u32 jobs) {
  const int a = P.rank;
  const __amdgpu_buffer_rsrc_t rws = mk_rsrc(ws, WS_USED);
  const u32 l4 = (u32)(wave - 4) * 4u;
  const u32 vx0 = (u32)(lane >> 3) * 1024u + (u32)(((lane & 7) ^ (lane >> 4)) * 16), vx1 = vx0 ^ 64u;
  const u32 vw0 = (u32)(lane >> 4) * 8192u + (u32)(((lane & 15) ^ (lane >> 4)) * 16), vw1 = vw0 ^ 128u;
  const u32 vp = (u32)lane * 16u;
  const u32x4 rx = mk_rsrc4(P.x, X_BYTES), rw = mk_rsrc4(P.w, W_BYTES), rl = mk_rsrc4(ws, WS_USED);
  const u32 lds0 = (u32)(size_t)(LAS char*)lds + l4 * 1024u;
  bool fin = false;
  int stamp = 1; (void)stamp;
  for (; (jobs & 0x3ffu) != 0; jobs >>= 10) {
    const Job J = decode_job(jobs & 0x3ffu, a);
    const int nh = J.nh, nkt = J.nkt, ns = nh >> 1, skip = J.dO >= 2 ? J.dO - 2 : 6;
    const u32 row0 = (u32)((a ^ J.dO) * 512 + J.sub * 128);
    const int j0 = fin && J.kind == JOB_O ? 8 : 0;
    fin = fin || J.kind != JOB_S;
    auto seg_at = [&](int i) { int sg = 5 - i; return sg - (sg <= skip && skip < 6); };
    const int fl = lane & 15;
    const bool care = lane < 32 && fl < nh;
    const u32 foff = (lane < 16 ? OFF_XF + (u32)J.ja * 256u : OFF_WF + (u32)J.p * 256u) + (fl < ns ? (u32)seg_at(fl) * 32u : (u32)seg_at(nh - 1 - fl) * 32u + 16u);
    int issued = j0, islot = 0, open = 8;
    auto issue = [&]() {
      const u32 dst = lds0 + (u32)islot * 32768u;
      if (issued < 8) {
        dma4(dst, vx0, vx1, vx0, vx1, rx, (row0 + l4 * 8u) * 1024u + (u32)issued * 128u, 8192u);
        dma4(dst + 16384u, vw0, vw0, vw1, vw1, rw, ((u32)issued * 64u + l4 * 4u) * 8192u + J.col0 * 2u, 32768u);
      } else {
        const int k = issued - 8, i = k >> 1;
        const u32 o = l4 * 1024u + (u32)(k & 1) * 16384u + (i < ns ? (u32)seg_at(i) * 65536u : (u32)seg_at(nh - 1 - i) * 65536u + 32768u);
        dma4(dst, vp, vp, vp, vp, rl, OFF_XS + (u32)J.ja * PANEL_B + o, 1024u);
        dma4(dst + 16384u, vp, vp, vp, vp, rl, OFF_WS + (u32)J.p * PANEL_B + o, 1024u);
      }
      ++issued; islot = islot == NST - 1 ? 0 : islot + 1;
    };
    for (int j = j0; j <= nkt; ++j) {
      const int cap = j + NST - (j > j0) < nkt ? j + NST - (j > j0) : nkt;
      if (issued < cap && issued == open) {
        const int got = wave_poll4(ws, rws, foff, care, (open - 8) >> 1, issued == j, call, lane);
        open = max(open, 8 + 2 * got);
      }
      while (issued < cap && issued < open) issue();
      const int after = issued - j - 1;
      if (after >= 4) WAIT_VM(32); else if (after == 3) WAIT_VM(24); else if (after == 2) WAIT_VM(16); else if (after == 1) WAIT_VM(8); else WAIT_VM(0);
      __builtin_amdgcn_s_barrier();
    }
    __builtin_amdgcn_s_barrier();
  }
  if (!fin) return;
  __builtin_amdgcn_s_barrier();
  collect_max(lds, ws, rws, call, lane, wave);
}

__device__ __forceinline__ void computer(const Params& P, char* lds, char* ws, int lane, int wave, u32 call, u32 jobs, u64 t_start) {
  const int a = P.rank;
  const int wr = wave >> 1, wc = wave & 1, fr = lane & 15, g = lane >> 4;
  const u32 xa0 = (u32)((wr * 64 + fr) * 128 + ((g ^ (fr >> 1)) * 16));
  const int tq = fr >> 2, tp = fr & 3;
  const u32 wa0 = 16384u + (u32)((8 * g + tq) * 256 + (((8 * wc + 2 * tp) ^ (tq | ((g & 1) << 3))) * 16) + (tp & 1) * 8);
  const u32x4 rl = mk_rsrc4(ws, WS_USED);
  char* am_dst = P.ws[lane & 7];
  asm volatile("" : "+v"(am_dst));
  int stamp = 1; (void)stamp;
  f32x4 fin[4][4];
  u32 fjob = 0;
  const u32 vo = (u32)lane * 16u;

  for (; (jobs & 0x3ffu) != 0; jobs >>= 10) {
    const Job J = decode_job(jobs & 0x3ffu, a);
    const bool cont = fjob != 0 && J.kind == JOB_O;
    f32x4 acc[4][4];
#pragma unroll
    for (int m = 0; m < 4; ++m)
#pragma unroll
      for (int n = 0; n < 4; ++n) acc[m][n] = cont ? fin[m][n] : (f32x4){0.f, 0.f, 0.f, 0.f};
    auto readf = [&](bf16x8 (&xf)[4], bf16x8 (&wf)[4], int slot, int ks) {
      LAS char* st = (LAS char*)lds + slot * 32768;
#pragma unroll
      for (int i = 0; i < 4; ++i) {
        const u32 o = (wa0 ^ (u32)((i >> 1) * 16 + (i & 1) * 8)) + (u32)(ks * 8192);
        const s16x4 lo = __builtin_amdgcn_ds_read_tr16_b64_v4i16((LAS s16x4*)(st + o));
        const s16x4 hi = __builtin_amdgcn_ds_read_tr16_b64_v4i16((LAS s16x4*)(st + o + 1024));
        wf[i] = __builtin_bit_cast(bf16x8, __builtin_shufflevector(lo, hi, 0, 1, 2, 3, 4, 5, 6, 7));
        xf[i] = *(LAS const bf16x8*)(st + ((xa0 ^ (u32)(ks * 64)) + (u32)(i * 2048)));
      }
    };
    auto mma = [&](const bf16x8 (&xf)[4], const bf16x8 (&wf)[4]) {
#pragma unroll
      for (int m = 0; m < 4; ++m)
#pragma unroll
        for (int n = 0; n < 4; ++n) acc[m][n] = __builtin_amdgcn_mfma_f32_16x16x32_bf16(wf[n], xf[m], acc[m][n], 0, 0, 0);
    };
#define INTERLEAVE(id) do { _Pragma("unroll") for (int i_ = 0; i_ < 6; ++i_) { __builtin_amdgcn_sched_group_barrier(0x8, 1, id); __builtin_amdgcn_sched_group_barrier(0x100, 2, id); } \
                            __builtin_amdgcn_sched_group_barrier(0x8, 10, id); __builtin_amdgcn_sched_barrier(0); } while (0)
    bf16x8 x0[4], w0[4], x1[4], w1[4];
    int slot = 0;
    u64 tk = J.dO == 1 ? TAU : TAU_C;
    const int tk_at = J.kind == JOB_S && J.dO < 4 ? J.nkt - 2 : -1;
    const u32 tk_off = OFF_TP + (u32)J.dO * 128u;
    BARRIER();
    readf(x0, w0, 0, 0);
    for (int kt = cont ? 8 : 0; kt < J.nkt; ++kt) {
      if (kt == tk_at) {
        const u64 t0 = (__builtin_amdgcn_s_memrealtime() + 60u) * 16u, t1 = (t_start + T_HEAD) * 16u + P.ucount[J.dO] * (4u * TAU_IN);
        u64 ex, lo = t0 > t1 ? t0 : t1;
        asm volatile("s_mov_b64 %1, exec\n\ts_mov_b64 exec, 1\n\tbuffer_atomic_umax_x2 %2, off, %3, %4\n\tbuffer_atomic_add_x2 %0, off, %3, %4 sc0\n\ts_mov_b64 exec, %1"
                     : "+v"(tk), "=&s"(ex) : "v"(lo), "s"(rl), "s"(tk_off) : "memory");
      }
      __builtin_amdgcn_s_waitcnt(0xC07F);
      __builtin_amdgcn_sched_barrier(0);
      readf(x1, w1, slot, 1); mma(x0, w0);
      INTERLEAVE(0);
      slot = slot == NST - 1 ? 0 : slot + 1;
      BARRIER();
      __builtin_amdgcn_sched_barrier(0);
      readf(x0, w0, slot, 0); mma(x1, w1);
      INTERLEAVE(1);
    }
    BARRIER();
    STAMP_AT(stamp++);

    if (J.kind == JOB_S) {
      const u32 slot = (u32)(box_base(J.dO) + box_idx(J.ja, J.p));
      u32x4 v[8];
#pragma unroll
      for (int jj = 0; jj < 8; ++jj) {
        const f32x4 u0 = acc[jj >> 1][(jj & 1) * 2], u1 = acc[jj >> 1][(jj & 1) * 2 + 1];
        v[jj].x = pack_bf16(u0[0], u0[1]); v[jj].y = pack_bf16(u0[2], u0[3]); v[jj].z = pack_bf16(u1[0], u1[1]); v[jj].w = pack_bf16(u1[2], u1[3]);
      }
      asm volatile("s_waitcnt vmcnt(0)" : "+v"(tk) :: "memory");
      const u32 fixed = (u32)((J.p >> 2) * 32 + (((J.ja - 5) >> 2) * 4 + wave) * 4 + ((J.p + (J.dO >> 1) * 2) & 3));
      const u32 due = J.dO < 4 ? rfl((u32)(tk >> 4)) : (u32)t_start + T_HEAD + ((240u * TAU_IN + fixed * TAU_X) >> 4);
      while (due - (u32)__builtin_amdgcn_s_memrealtime() - 1u < 10000u) __builtin_amdgcn_s_sleep(1);
      STAMP_AT(stamp++);
      char* pw = P.ws[J.dO];
      const __amdgpu_buffer_rsrc_t rd = mk_rsrc(pw, WS_USED);
      const u32 so = OFF_IN + slot * 32768u + (u32)wave * 8192u;
#pragma unroll
      for (int jj = 0; jj < 8; ++jj) __builtin_amdgcn_raw_buffer_store_b128(v[jj], rd, vo + (u32)((jj & 3) * 1024), so + (u32)((jj >> 2) * 4096), 17);
      WAIT_VM(0);
      if (lane == 0) {
        st_sys(pw + OFF_PF + slot * 128u + (u32)wave * 4u, call);
      }
      STAMP_AT(stamp++);
    } else {
      fjob = jobs & 0x3ffu;
#pragma unroll
      for (int m = 0; m < 4; ++m)
#pragma unroll
        for (int n = 0; n < 4; ++n) fin[m][n] = acc[m][n];
    }
  }
  if (!fjob) return;

  const Job J = decode_job(fjob, a);
  const __amdgpu_buffer_rsrc_t rws = mk_rsrc(ws, WS_USED);
  LAS float* red = (LAS float*)lds;
  {
    u32 s0, s1;
    if (J.kind == JOB_O) { s0 = s1 = (u32)(J.p * 3 + J.sub); }
    else if ((J.ja >> 2) == 0) { s0 = (u32)(48 + J.p); s1 = (u32)(64 + J.p); }
    else { const int d0 = 4 | ((a ^ J.sub) & 2) | (a & 1); s0 = (u32)(box_base(d0) + J.p * 2 + (J.sub & 1)); s1 = (u32)(box_base(d0 ^ 1) + J.p * 2 + (J.sub & 1)); }
    wave_wait(ws, OFF_PF + (lane == 0 ? s0 : s1) * 128u + (u32)wave * 4u, lane < 2, call, lane);
    STAMP_AT(stamp++);
    u32x4 v0[8], v1[8];
    const u32 so0 = OFF_IN + s0 * 32768u + (u32)wave * 8192u, so1 = OFF_IN + s1 * 32768u + (u32)wave * 8192u;
#pragma unroll
    for (int jj = 0; jj < 8; ++jj) v0[jj] = __builtin_amdgcn_raw_buffer_load_b128(rws, vo + (u32)((jj & 3) * 1024), so0 + (u32)((jj >> 2) * 4096), 17);
    if (J.kind == JOB_F) {
#pragma unroll
      for (int jj = 0; jj < 8; ++jj) v1[jj] = __builtin_amdgcn_raw_buffer_load_b128(rws, vo + (u32)((jj & 3) * 1024), so1 + (u32)((jj >> 2) * 4096), 17);
    } else {
#pragma unroll
      for (int jj = 0; jj < 8; ++jj) v1[jj] = (u32x4){0u, 0u, 0u, 0u};
    }
    float mx = 0.f;
#pragma unroll
    for (int jj = 0; jj < 8; ++jj) {
      f32x4& u0 = fin[jj >> 1][(jj & 1) * 2]; f32x4& u1 = fin[jj >> 1][(jj & 1) * 2 + 1];
      u0[0] += bf_lo(v0[jj].x) + bf_lo(v1[jj].x); u0[1] += bf_hi(v0[jj].x) + bf_hi(v1[jj].x);
      u0[2] += bf_lo(v0[jj].y) + bf_lo(v1[jj].y); u0[3] += bf_hi(v0[jj].y) + bf_hi(v1[jj].y);
      u1[0] += bf_lo(v0[jj].z) + bf_lo(v1[jj].z); u1[1] += bf_hi(v0[jj].z) + bf_hi(v1[jj].z);
      u1[2] += bf_lo(v0[jj].w) + bf_lo(v1[jj].w); u1[3] += bf_hi(v0[jj].w) + bf_hi(v1[jj].w);
#pragma unroll
      for (int e = 0; e < 4; ++e) mx = fmaxf(mx, fmaxf(u0[e], u1[e]));
    }
    mx = wave_max(mx);
    if (lane == 0) red[wave] = mx;
  }
  BARRIER();
  if (wave == 0 && lane < NR) {
    const float m4 = fmaxf(fmaxf(red[0], red[1]), fmaxf(red[2], red[3]));
    const u32 f = (u32)(J.p * 8 + (J.kind == JOB_F ? J.ja : J.sub));
    __hip_atomic_store((GAS u64*)(am_dst + OFF_AM + (f * 8u + (u32)a) * 8u), ((u64)call << 32) | __builtin_bit_cast(u32, m4), __ATOMIC_RELAXED, __HIP_MEMORY_SCOPE_SYSTEM);
  }
  STAMP_AT(stamp++);
  collect_max(lds, ws, rws, call, lane, wave);
  float gm = 0.f; bool bad = false;
#pragma unroll
  for (int i = 0; i < 8; ++i) { gm = fmaxf(gm, red[8 + i]); bad = bad | (((LAS u32*)red)[16 + i] == call); }
  STAMP_AT(stamp++);
  const float inv = gm > 0.f ? 127.0f / gm : 0.f;
  const __amdgpu_buffer_rsrc_t ro = mk_rsrc(P.out, OUT_BYTES);
#pragma unroll
  for (int m = 0; m < 4; ++m) {
    u32 qd[4];
#pragma unroll
    for (int n = 0; n < 4; ++n) {
      u32 r = 0;
#pragma unroll
      for (int e = 0; e < 4; ++e) r = __builtin_amdgcn_cvt_pk_u8_f32(__builtin_rintf(fin[m][n][e] * inv), e, r);
      qd[n] = bad ? 0x80808080u : r;
    }
    const bool sw = g & 1;
    u32x4 v; v.x = sw ? qd[1] : qd[0]; v.y = sw ? qd[0] : qd[1]; v.z = sw ? qd[3] : qd[2]; v.w = sw ? qd[2] : qd[3];
    __builtin_amdgcn_raw_buffer_store_b128(v, ro, (u32)(fr * 4096 + g * 16), (u32)(J.sub * 128 + wr * 64 + m * 16) * 4096u + J.col0 + (u32)wc * 64u, 0);
  }
  STAMP_AT(stamp++);
}

__global__ void __launch_bounds__(512, 2) gemm_rs_q8(const Params P) {
  __shared__ __attribute__((aligned(16))) char lds[5 * 32768];
  const int lane = threadIdx.x & 63, wave = (int)rfl(threadIdx.x >> 6);
  char* ws = P.ws[0];
  const u64 t_start = __builtin_amdgcn_s_memrealtime();
  if (blockIdx.x < NPUSH) { if (wave < 6) pusher(P, lds, ws, lane, wave); return; }
  const u32 call = draw_call(ws, lane, wave);
  const u32 jobs = P.jobs[blockIdx.x];
  if (wave >= 4) loader(P, lds, ws, lane, wave, call, jobs);
  else computer(P, lds, ws, lane, wave, call, jobs, t_start);
}

#define LAG 12
void build_schedule(Params& P) {
#define SHIFT2 2
#define ORD 2
  static const int orders[4][32] = {
    {0, 1, 2, 5, 6, 7, 8, 16, 17, 18, 19, 9, 10, 11, 12, 20, 21, 22, 23, 24, 25, 26, 27, 3, 4, 28, 29, 30, 31, 13, 14, 15},
    {0, 1, 2, 16, 5, 6, 17, 7, 8, 18, 19, 9, 10, 11, 12, 20, 21, 22, 23, 24, 25, 4, 3, 26, 27, 28, 29, 30, 31, 13, 14, 15},
    {0, 1, 2, 16, 17, 5, 6, 18, 19, 7, 8, 20, 21, 9, 10, 22, 23, 11, 12, 24, 25, 4, 3, 26, 27, 28, 29, 30, 31, 13, 14, 15},
    {0, 1, 2, 16, 17, 5, 18, 6, 19, 7, 20, 8, 21, 22, 23, 9, 10, 11, 12, 24, 25, 4, 3, 26, 27, 28, 29, 30, 31, 13, 14, 15}};
  const int* order = orders[ORD];
  int pos[32];
  for (int i = 0; i < 32; ++i) pos[order[i]] = i;
  auto ready = [&](int t, int p) { return pos[t] > pos[16 + p] ? pos[t] : pos[16 + p]; };
  for (int d = 0; d < NR; ++d) {
    unsigned char list[MAXU]; int n = 0;
    for (int i = 0; i < MAXU; ++i) list[i] = 0;
    if (d >= 2)
      for (int i = 0, lag = ((2 * (d - 2) + 1) * LAG + 10) / 20; i < 32 + lag; ++i)
        for (int h = 0; h < 2; ++h) {
          const int k = i - h * lag;
          if (k < 0 || k >= 32) continue;
          const int c = order[k];
          if (c < 16 && tile_dist(c) == d) continue;
          list[n++] = (unsigned char)(c * 2 + h);
        }
    P.ucount[d] = (u32)n;
    for (int i = 0; i < MAXU / 4; ++i) P.units[d][i] = list[4 * i] | list[4 * i + 1] << 8 | list[4 * i + 2] << 16 | (u32)list[4 * i + 3] << 24;
  }
  static const int grp[16] = {0, 1, 0, 2, 1, 1, 0, 1, 0, 1, 0, 1, 0, 2, 2, 2};
  for (int i = 0; i < GRID; ++i) P.jobs[i] = 0;
  for (int x = 0; x < 8; ++x) {
    int ht[40], hp[40], n = 0;
    for (int r = 0; r < 32; ++r)
      for (int p = 0; p < 16; ++p)
        for (int t = 0; t < 16; ++t) {
          int gsel = grp[t] == 2 ? (p >> 2) & 1 : grp[t];
          if (t == 15 && p < 8) gsel = 0;
          if (gsel == (x >> 2) && ((p + (t == 2 ? SHIFT2 : 0)) & 3) == (x & 3) && ready(t, p) == r) { ht[n] = t; hp[n] = p; ++n; }
        }
    auto enc = [&](int i) { return (u32)(((ht[i] >= 13 ? JOB_O : JOB_S) << 8) | (ht[i] << 4) | hp[i]); };
    const int nw = 32 - NPUSH / 8;
    int f = 0, extra = nw;
    for (int s = 0; s < nw; ++s) {
      u32 jb = enc(s);
      if (ht[s] < 3 && extra < n) { jb = (u32)((JOB_F << 8) | ((ht[extra] - 13) << 4) | hp[extra]) | jb << 10 | enc(extra) << 20; ++extra; }
      else if (ht[s] < 13 && f < 10) { jb = jb << 10 | (u32)((JOB_F << 8) | ((3 + f % 5) << 4) | (x + 8 * (f / 5))); ++f; }
      P.jobs[NPUSH + s * 8 + x] = jb;
    }
  }
}

}

extern "C" void kernel_launch(int rank, int nranks, ncclComm_t comm, hipStream_t stream,
                              void* const* d_in, const int* in_sizes, int n_in,
                              void* d_out, int out_size,
                              const int* in_dtypes, int out_dtype,
                              void* const* d_out_all, void* const* d_ws_all, size_t ws_bytes) {
  (void)comm; (void)in_dtypes; (void)out_dtype; (void)d_out_all;
  if (nranks != NR || (unsigned)rank >= (unsigned)NR || n_in != 2 || ws_bytes < WS_USED || (u32)out_size != OUT_BYTES ||
      (u32)in_sizes[0] * 2u != X_BYTES || (u32)in_sizes[1] * 2u != W_BYTES) {
    fprintf(stderr, "unexpected problem shape\n");
    return;
  }
  Params P;
  P.x = (const char*)d_in[0]; P.w = (const char*)d_in[1]; P.out = (char*)d_out;
  for (int d = 0; d < NR; ++d) P.ws[d] = (char*)d_ws_all[rank ^ d];
  P.rank = rank; P.pad = 0;
  build_schedule(P);
  hipLaunchKernelGGL(gemm_rs_q8, dim3(GRID), dim3(512), 0, stream, P);
  HIP_CHECK(hipGetLastError());
}
```

```cpp
#include <hip/hip_runtime.h>
#include <rccl/rccl.h>
#include <cstdio>
#include <cstdint>
#include <type_traits>

#define HIP_CHECK(expr) do { hipError_t _e = (expr); if (_e != hipSuccess) { \
    fprintf(stderr, "HIP error %s at %s:%d\n", hipGetErrorString(_e), __FILE__, __LINE__); \
    return; } } while (0)

namespace {

typedef unsigned u32;
typedef unsigned long long u64;
typedef u32 u32x4 __attribute__((ext_vector_type(4)));
typedef u32 u32x2 __attribute__((ext_vector_type(2)));
typedef float f32x4 __attribute__((ext_vector_type(4)));
typedef short s16x4 __attribute__((ext_vector_type(4)));
typedef short s16x8 __attribute__((ext_vector_type(8)));
typedef __bf16 bf16x8 __attribute__((ext_vector_type(8)));
#define LAS __attribute__((address_space(3)))
#define GAS __attribute__((address_space(1)))

constexpr int NR = 8;
#define NPUSH 24
constexpr int GRID = 256;
constexpr int MAXU = 64;
constexpr u64 DEADLINE_TICKS = 100000000ull;

constexpr u32 X_BYTES = 4096u * 512u * 2u, W_BYTES = 512u * 4096u * 2u, OUT_BYTES = 512u * 4096u;
constexpr u32 OFF_CNT = 0;
constexpr u32 OFF_TMO = 16u << 10;
constexpr u32 OFF_TK = (16u << 10) + 128u;
constexpr u32 OFF_TP = 18u << 10;
constexpr u32 OFF_T0 = 19u << 10;
constexpr u32 OFF_XF = 20u << 10;
constexpr u32 OFF_WF = 24u << 10;
constexpr u32 OFF_AM = 32u << 10;
constexpr u32 OFF_TOK = 128u << 10;
constexpr u32 OFF_PF = 192u << 10;
constexpr u32 OFF_DBG = 512u << 10;
constexpr u32 OFF_XS = 2u << 20;
constexpr u32 OFF_WS = 8u << 20;
constexpr u32 OFF_IN = 14u << 20;
constexpr u32 WS_USED = OFF_IN + 208u * 32768u;
#define TAU_X 229
#define TAU 245
#define TAU_C 300
#define T_BOOK 70
#define T_REST 800
#define OUT_AUX 0
#define LOOK 3
#define SL_LONG 32
#define SL_GATE 6
#define SL_AM 4
#define SL_HDR 16
#define LPRIO 0
#define CAP 3
constexpr u32 PANEL_B = 6u * 65536u;

enum { JOB_NONE = 0, JOB_S = 1, JOB_O = 2, JOB_F = 3 };

struct Params {
  const char* x; const char* w; char* out;
  char* ws[NR];
  int rank; int pad;
  u32 jobs[GRID];
  u32 first[NPUSH][8];
  u32 ucount[NR];
  u32 units[NR][MAXU / 4];
};

__host__ __device__ inline int tile_dist(int t) { return t < 3 ? 1 : t < 5 ? t - 1 : t < 13 ? 4 + ((t - 5) & 3) : 0; }
__host__ __device__ inline int tile_sub(int t, int a) { return t < 3 ? t : t < 5 ? 3 : t < 13 ? (a & 2) + ((t - 5) >> 2) : t - 13; }
__host__ __device__ inline int box_base(int d) { return d == 1 ? 0 : d < 4 ? 48 + 16 * (d - 2) : 80 + 32 * (d - 4); }
__host__ __device__ inline int box_cnt(int d) { return d == 1 ? 48 : d < 4 ? 16 : 32; }
__host__ __device__ inline int box_idx(int t, int p) { return t < 3 ? p * 3 + t : t < 5 ? p : p * 2 + ((t - 5) >> 2); }

__host__ __device__ inline int push_link(int block, int wave) { return 2 + (wave + (block >> 3) * 2 + block) % 6; }
__host__ __device__ inline u32 slice_src(u32 code, u32 sl, int q) {
  const u32 h = code & 1u;
  if (code < 32) {
    const int t = code >> 1;
    return ((u32)((q ^ tile_dist(t)) * 512 + tile_sub(t, q) * 128) + (sl & 1u) * 64u) * 1024u + ((u32)(q & 1) * 256u + h * 128u + (sl >> 1) * 64u) * 2u;
  }
  return ((u32)(q & 1) * 256u + h * 128u + sl * 32u) * 8192u + ((u32)(q >> 2) * 2048u + ((code - 32u) >> 1) * 128u) * 2u;
}

__device__ __forceinline__ u32 rfl(u32 v) { return __builtin_amdgcn_readfirstlane(v); }
__device__ __forceinline__ __amdgpu_buffer_rsrc_t mk_rsrc(const void* p, u32 bytes) {
  return __builtin_amdgcn_make_buffer_rsrc((void*)p, 0, bytes, 0x00020000);
}
__device__ __forceinline__ u32x4 mk_rsrc4(const void* p, u32 bytes) {
  u64 a = (u64)p; u32x4 r; r.x = rfl((u32)a); r.y = rfl((u32)(a >> 32)) & 0xffffu; r.z = bytes; r.w = 0x00020000u; return r;
}
__device__ __forceinline__ u32 ld_sys(const char* p) { return __hip_atomic_load((GAS const u32*)p, __ATOMIC_RELAXED, __HIP_MEMORY_SCOPE_SYSTEM); }
__device__ __forceinline__ void st_sys(char* p, u32 v) { __hip_atomic_store((GAS u32*)p, v, __ATOMIC_RELAXED, __HIP_MEMORY_SCOPE_SYSTEM); }
__device__ __forceinline__ void st_agent(char* p, u32 v) { __hip_atomic_store((GAS u32*)p, v, __ATOMIC_RELAXED, __HIP_MEMORY_SCOPE_AGENT); }
#define WAIT_VM(n) asm volatile("s_waitcnt vmcnt(" #n ")" ::: "memory")
#define BARRIER() do { asm volatile("" ::: "memory"); __builtin_amdgcn_s_waitcnt(0xC07F);   __builtin_amdgcn_s_barrier(); asm volatile("" ::: "memory"); } while (0)

#define STAMP_AT(i) do {} while (0)

template <int SL> __device__ __forceinline__ void wave_wait(char* ws, u32 off, bool care, u32 call, int lane) {
  if (lane == 63) { off = OFF_TMO; }
  else if (!care) { off = OFF_TMO; }
  const u64 t0 = __builtin_amdgcn_s_memrealtime();
  for (;;) {
    const u32 v = ld_sys(ws + off);
    const bool ok = (lane == 63 || !care) ? true : (v >= call);
    if (__all(ok)) break;
    const bool dead = __any(lane == 63 && v == call);
    if (dead || __builtin_amdgcn_s_memrealtime() - t0 > DEADLINE_TICKS) {
      if (lane == 0) st_sys(ws + OFF_TMO, call);
      WAIT_VM(0);
      break;
    }
    __builtin_amdgcn_s_sleep(SL);
  }
}

__device__ __forceinline__ int wave_poll4(char* ws, __amdgpu_buffer_rsrc_t rws, u32 off, bool care, int have, bool block, u32 call, int lane) {
  if (!care) off = OFF_TMO;
  const u64 t0 = __builtin_amdgcn_s_memrealtime();
  for (;;) {
    const u32x4 v = __builtin_amdgcn_raw_buffer_load_b128(rws, off, 0, 17);
    asm volatile("" ::: "memory");
    const u32 m = (u32)__ballot(care && v.x >= call && v.y >= call && v.z >= call && v.w >= call);
    const int n = __builtin_ctz(~(m & (m >> 16)));
    if (n > have || !block) return n;
    const bool dead = __any(!care && v.x == call);
    if (dead || __builtin_amdgcn_s_memrealtime() - t0 > DEADLINE_TICKS) {
      if (lane == 0) st_sys(ws + OFF_TMO, call);
      WAIT_VM(0);
      return have + 1;
    }
    if (have == 0) __builtin_amdgcn_s_sleep(SL_LONG); else __builtin_amdgcn_s_sleep(SL_GATE);
  }
}
#define DMA_SC " sc0 sc1"

typedef __bf16 bf16x2 __attribute__((ext_vector_type(2)));
__device__ __forceinline__ u32 pack_bf16(float lo, float hi) { bf16x2 r; r.x = (__bf16)lo; r.y = (__bf16)hi; return __builtin_bit_cast(u32, r); }
__device__ __forceinline__ float as_f32(u32 v) { return __builtin_bit_cast(float, v); }
__device__ __forceinline__ float bf_lo(u32 v) { return __builtin_bit_cast(float, v << 16); }
__device__ __forceinline__ float bf_hi(u32 v) { return __builtin_bit_cast(float, v & 0xffff0000u); }
__device__ __forceinline__ float wave_max(float f) {
  int m = __builtin_bit_cast(int, f);
  m = max(m, __builtin_amdgcn_update_dpp(0, m, 0x111, 0xf, 0xf, false));
  m = max(m, __builtin_amdgcn_update_dpp(0, m, 0x112, 0xf, 0xf, false));
  m = max(m, __builtin_amdgcn_update_dpp(0, m, 0x114, 0xf, 0xf, false));
  m = max(m, __builtin_amdgcn_update_dpp(0, m, 0x118, 0xf, 0xf, false));
  const int a = max(__builtin_amdgcn_readlane(m, 15), __builtin_amdgcn_readlane(m, 31));
  const int b = max(__builtin_amdgcn_readlane(m, 47), __builtin_amdgcn_readlane(m, 63));
  return __builtin_bit_cast(float, max(a, b));
}

__device__ __forceinline__ void dma4(u32 lds_dst, u32 v0, u32 v1, u32 v2, u32 v3, u32x4 rs, u32 so, u32 st) {
  u32 keep, t;
  asm volatile(
      "s_mov_b32 %0, m0\n\t"
      "s_mov_b32 m0, %2\n\ts_add_u32 %1, %8, %9\n\ts_nop 1\n\tbuffer_load_dwordx4 %3, %7, %8 offen" DMA_SC " lds\n\t"
      "s_add_u32 m0, %2, 0x400\n\ts_nop 0\n\tbuffer_load_dwordx4 %4, %7, %1 offen" DMA_SC " lds\n\t"
      "s_add_u32 m0, %2, 0x800\n\ts_add_u32 %1, %1, %9\n\tbuffer_load_dwordx4 %5, %7, %1 offen" DMA_SC " lds\n\t"
      "s_add_u32 m0, %2, 0xc00\n\ts_add_u32 %1, %1, %9\n\tbuffer_load_dwordx4 %6, %7, %1 offen" DMA_SC " lds\n\t"
      "s_mov_b32 m0, %0"
      : "=&s"(keep), "=&s"(t)
      : "s"(lds_dst), "v"(v0), "v"(v1), "v"(v2), "v"(v3), "s"(rs), "s"(so), "s"(st)
      : "memory", "scc");
}

__device__ __forceinline__ void dma8_lin(u32 lds_dst, u32 v, u32x4 rs, u32 sx, u32 sw) {
  u32 keep;
  asm volatile(
      "s_mov_b32 %0, m0\n\ts_mov_b32 m0, %1\n\ts_nop 2\n\t"
      "buffer_load_dwordx4 %2, %3, %4 offen" DMA_SC " lds\n\tbuffer_load_dwordx4 %2, %3, %4 offen offset:1024" DMA_SC " lds\n\t"
      "buffer_load_dwordx4 %2, %3, %4 offen offset:2048" DMA_SC " lds\n\tbuffer_load_dwordx4 %2, %3, %4 offen offset:3072" DMA_SC " lds\n\t"
      "s_add_u32 m0, %1, 0x4000\n\ts_nop 0\n\t"
      "buffer_load_dwordx4 %2, %3, %5 offen" DMA_SC " lds\n\tbuffer_load_dwordx4 %2, %3, %5 offen offset:1024" DMA_SC " lds\n\t"
      "buffer_load_dwordx4 %2, %3, %5 offen offset:2048" DMA_SC " lds\n\tbuffer_load_dwordx4 %2, %3, %5 offen offset:3072" DMA_SC " lds\n\t"
      "s_mov_b32 m0, %0"
      : "=&s"(keep) : "s"(lds_dst), "v"(v), "s"(rs), "s"(sx), "s"(sw) : "memory", "scc");
}

#define WIN 3
template <int N> __device__ __forceinline__ void wait_vm() { asm volatile("s_waitcnt vmcnt(%0)" :: "n"(N) : "memory"); }
__device__ __forceinline__ void draw_async(u32& v, u32x4 rl, u32 off) {
  u64 ex; v = 1;
  asm volatile("s_mov_b64 %1, exec\n\ts_mov_b64 exec, 1\n\tbuffer_atomic_add %0, off, %2, %3 sc0\n\ts_mov_b64 exec, %1"
               : "+v"(v), "=&s"(ex) : "s"(rl), "s"(off) : "memory");
}
__device__ __forceinline__ void book(u64& tk, u32x4 rl, u32 off) {
  u64 ex; const u64 lo = (__builtin_amdgcn_s_memrealtime() + T_BOOK) * 16u;
  asm volatile("s_mov_b64 %1, exec\n\ts_mov_b64 exec, 1\n\tbuffer_atomic_umax_x2 %2, off, %3, %4\n\tbuffer_atomic_add_x2 %0, off, %3, %4 sc0\n\ts_mov_b64 exec, %1"
               : "+v"(tk), "=&s"(ex) : "v"(lo), "s"(rl), "s"(off) : "memory");
}
__device__ __forceinline__ void sleep_until(u32 due) {
  for (;;) {
    const u32 left = due - (u32)__builtin_amdgcn_s_memrealtime() - 1u;
    if (left >= 10000u) break;
    if (left > 200u) __builtin_amdgcn_s_sleep(48); else if (left > 40u) __builtin_amdgcn_s_sleep(8); else __builtin_amdgcn_s_sleep(1);
  }
}
struct Slice { u32 src, dst, flag, kind; };
__device__ __forceinline__ u32 draw_call(char* ws, int lane, int wave) {
  u32 call = 0;
  if (lane == 0) call = __hip_atomic_fetch_add((GAS u32*)(ws + OFF_CNT + blockIdx.x * 64 + wave * 4), 1u, __ATOMIC_RELAXED, __HIP_MEMORY_SCOPE_AGENT) + 1u;
  return rfl(call);
}
__device__ __forceinline__ void pusher(const Params& P, char* lds, char* ws, int lane, int wave) {
  const u32 vx0 = (u32)(lane >> 3) * 1024u + (u32)(((lane & 7) ^ (lane >> 4)) * 16), vx1 = vx0 ^ 64u;
  const u32 vw0 = (u32)(lane >> 4) * 8192u + (u32)(((lane & 15) ^ (lane >> 4)) * 16), vw1 = vw0 ^ 128u;
  u32 f0 = P.first[blockIdx.x][wave]; const char* xp = P.x; const char* wp = P.w;
  asm volatile("" : "+s"(f0), "+s"(xp), "+s"(wp));
  const u32x4 rx = mk_rsrc4(xp, X_BYTES), rw = mk_rsrc4(wp, W_BYTES);
  const u32 ring0 = (u32)(size_t)(LAS char*)lds + (u32)wave * 16384u;
  auto load_all = [&](const Slice& c, u32 buf) {
    const u32 dst = ring0 + buf * 8192u;
    if (c.kind == 0) { dma4(dst, vx0, vx1, vx0, vx1, rx, c.src, 8192u); dma4(dst + 4096u, vx0, vx1, vx0, vx1, rx, c.src + 32768u, 8192u); }
    else { dma4(dst, vw0, vw0, vw1, vw1, rw, c.src, 32768u); dma4(dst + 4096u, vw0, vw0, vw1, vw1, rw, c.src + 131072u, 32768u); }
  };
  { Slice c; c.src = f0 & ~1u; c.kind = f0 & 1u; c.dst = c.flag = 0; load_all(c, 0); }
  const u32x4 rl = mk_rsrc4(ws, WS_USED);
  u32 cnt; draw_async(cnt, rl, OFF_CNT + blockIdx.x * 64u + (u32)wave * 4u);
  const int d = push_link(blockIdx.x, wave);
  const int a = P.rank, q = a ^ d;
  char* pw = P.ws[d];
  const u32 ns = P.ucount[d] * 4u;
  auto decode = [&](u32 sidx) -> Slice {
    Slice r;
    const u32 u = sidx >> 2, sl = sidx & 3u;
    const u32 code = (P.units[d][u >> 2] >> ((u & 3u) * 8u)) & 0xffu, h = code & 1u;
    r.kind = code >> 5; r.src = slice_src(code, sl, q);
    r.dst = (code < 32 ? OFF_XS : OFF_WS) + ((code >> 1) & 15u) * PANEL_B;
    r.flag = (code < 32 ? OFF_XF : OFF_WF) + ((code >> 1) & 15u) * 256u;
    r.dst += (u32)(d - 2) * 65536u + h * 32768u + sl * 8192u; r.flag += (u32)(d - 2) * 32u + h * 16u + sl * 4u;
    return r;
  };
  const u32 vp = (u32)lane * 16u;
  const __amdgpu_buffer_rsrc_t rd = mk_rsrc(pw, WS_USED);
  LAS char* ring = (LAS char*)lds + wave * 16384;
  u32 nxt_i = NPUSH + blockIdx.x;
  Slice cur = decode(blockIdx.x), nxt = decode(nxt_i);
  u32 buf = 0;
  asm volatile("s_waitcnt vmcnt(0)" : "+v"(cnt), "+s"(cur.dst), "+s"(nxt.src) :: "memory");
  const u32 call = rfl(cnt) + 1u;
  const u32 tk_base = (call - 1u) * (ns - NPUSH);
  const u32 tk_off = OFF_TK + (u32)d * 128u;
  u32 pend = 0;
  auto publish = [&]() {
    if (lane == 0) {
      st_sys(pw + pend, call);
    }
  };
  auto send = [&](int k) {
    const u32x4 v = *(LAS const u32x4*)(ring + buf * 8192u + k * 1024 + lane * 16);
    __builtin_amdgcn_raw_buffer_store_b128(v, rd, vp, cur.dst + (u32)k * 1024u, 17);
  };
  for (;;) {
    u32 tk;
    send(0);
    load_all(nxt, buf ^ 1u);
    draw_async(tk, rl, tk_off);
    const u32 t_draw = (u32)__builtin_amdgcn_s_memrealtime();
    Slice nn = nxt;
#pragma unroll
    for (int k = 0; k < 8; ++k) {
      if (k) send(k);
      if (k == WIN + 1) {
        asm volatile("" : "+v"(tk));
        nxt_i = rfl(tk) - tk_base;
        if (nxt_i == ns - 40u && lane == 0)
          __hip_atomic_store((GAS u64*)(ws + OFF_T0 + (u32)d * 128u), ((u64)call << 32) | (t_draw + T_REST), __ATOMIC_RELAXED, __HIP_MEMORY_SCOPE_AGENT);
        nxt_i += 2u * NPUSH;
        if (nxt_i < ns) nn = decode(nxt_i);
      }
      if (k < WIN) wait_vm<WIN - 1 + 9>(); else wait_vm<WIN - 1>();
      if (k == WIN - 2 && pend) publish();
    }
    pend = cur.flag;
    cur = nxt; nxt = nn; buf ^= 1u;
    if (nxt_i >= ns) break;
  }
#pragma unroll
  for (int k = 0; k < 8; ++k) send(k);
  WAIT_VM(8);
  publish();
  pend = cur.flag;
  WAIT_VM(0);
  publish();
}

constexpr int NST = 5;

struct Job { int kind, ja, p, dO, sub, nh, nkt; u32 col0; };
__device__ __forceinline__ Job decode_job(u32 code, int a) {
  Job j; j.kind = (code >> 8) & 3; j.ja = (code >> 4) & 15; j.p = code & 15;
  int chalf;
  if (j.kind == JOB_F) { j.dO = 0; j.sub = j.ja & 3; chalf = (a >> 2) ^ (j.ja >> 2); }
  else { j.dO = tile_dist(j.ja); j.sub = tile_sub(j.ja, a); chalf = a >> 2; }
  j.col0 = (u32)(chalf * 2048 + j.p * 128);
  j.nh = j.kind == JOB_F ? 0 : j.dO >= 2 ? 10 : 12;
  j.nkt = 8 + 2 * j.nh;
  return j;
}

__device__ __forceinline__ void collect_max(char* lds, char* ws, __amdgpu_buffer_rsrc_t rws, u32 call, int lane, int wave) {
  float gm = 0.f; u32 tmo_v; bool got = false;
  const u64 t0 = __builtin_amdgcn_s_memrealtime();
  for (;;) {
    const u32x4 v = __builtin_amdgcn_raw_buffer_load_b128(rws, got ? 0x80000000u : threadIdx.x * 16u, OFF_AM, 17);
    tmo_v = __builtin_amdgcn_raw_buffer_load_b32(rws, 0, OFF_TMO, 17);
    asm volatile("" ::: "memory");
    if (v.y >= call && v.w >= call) { got = true; gm = fmaxf(as_f32(v.x), as_f32(v.z)); }
    if (__all(got)) break;
    if (rfl(tmo_v) == call || __builtin_amdgcn_s_memrealtime() - t0 > DEADLINE_TICKS) {
      if (lane == 0) st_sys(ws + OFF_TMO, call);
      WAIT_VM(0);
      tmo_v = call;
      break;
    }
    if (__builtin_popcountll(__ballot(!got)) > 8) __builtin_amdgcn_s_sleep(SL_HDR); else __builtin_amdgcn_s_sleep(SL_AM);
  }
  gm = wave_max(gm);
  if (lane == 0) ((LAS u32*)lds)[8 + wave] = tmo_v == call ? ~0u : __builtin_bit_cast(u32, gm);
  BARRIER();
}

__device__ __forceinline__ void loader(const Params& P, char* lds, char* ws, int lane, int wave, u32 call, u32 jobs) {
  const int a = P.rank;
  const __amdgpu_buffer_rsrc_t rws = mk_rsrc(ws, WS_USED);
  const u32 l4 = (u32)(wave - 4) * 4u;
  const u32 vx0 = (u32)(lane >> 3) * 1024u + (u32)(((lane & 7) ^ (lane >> 4)) * 16), vx1 = vx0 ^ 64u;
  const u32 vw0 = (u32)(lane >> 4) * 8192u + (u32)(((lane & 15) ^ (lane >> 4)) * 16), vw1 = vw0 ^ 128u;
  const u32 vp = (u32)lane * 16u;
  const u32x4 rx = mk_rsrc4(P.x, X_BYTES), rw = mk_rsrc4(P.w, W_BYTES), rl = mk_rsrc4(ws, WS_USED);
  const u32 lds0 = (u32)(size_t)(LAS char*)lds + l4 * 1024u;
  bool fin = false;
  __builtin_amdgcn_s_setprio(LPRIO);
  int stamp = 1; (void)stamp;
  for (; (jobs & 0x3ffu) != 0; jobs >>= 10) {
    const Job J = decode_job(jobs & 0x3ffu, a);
    const int nh = J.nh, nkt = J.nkt, ns = nh >> 1, skip = J.dO >= 2 ? J.dO - 2 : 6;
    const u32 row0 = (u32)((a ^ J.dO) * 512 + J.sub * 128);
    const int j0 = fin && J.kind == JOB_O ? 8 : 0;
    fin = fin || J.kind != JOB_S;
    auto seg_at = [&](int i) { int sg = 5 - i; return sg - (sg <= skip && skip < 6); };
    const int fl = lane & 15;
    const bool care = lane < 32 && fl < nh;
    const u32 foff = (lane < 16 ? OFF_XF + (u32)J.ja * 256u : OFF_WF + (u32)J.p * 256u) + (fl < ns ? (u32)seg_at(fl) * 32u : (u32)seg_at(nh - 1 - fl) * 32u + 16u);
    u32 tab;
    { const int k = lane - 8, i = k >> 1;
      tab = lane < 8 ? (row0 + l4 * 8u) * 1024u + (u32)lane * 128u
                     : OFF_XS + (u32)J.ja * PANEL_B + l4 * 1024u + (u32)(k & 1) * 16384u + (i < ns ? (u32)seg_at(i) * 65536u : (u32)seg_at(nh - 1 - i) * 65536u + 32768u); }
    const u32 dw = OFF_WS - OFF_XS + (u32)(J.p - J.ja) * PANEL_B, w0 = l4 * (4u * 8192u) + J.col0 * 2u;
    int issued = j0, open = 8;
    u32 dst = lds0;
    auto issue = [&]() {
      const u32 ox = __builtin_amdgcn_readlane(tab, issued);
      if (__builtin_expect(issued >= 8, 1)) dma8_lin(dst, vp, rl, ox, ox + dw);
      else {
        dma4(dst, vx0, vx1, vx0, vx1, rx, ox, 8192u);
        dma4(dst + 16384u, vw0, vw0, vw1, vw1, rw, w0 + (u32)issued * (64u * 8192u), 32768u);
      }
      ++issued; dst = dst == lds0 + (NST - 1) * 32768u ? lds0 : dst + 32768u;
    };
    for (int j = j0; j <= nkt; ++j) {
      if (__builtin_expect(issued == j + NST - 2 && issued < open, 1)) {
        issue();
        WAIT_VM(24);
        __builtin_amdgcn_s_barrier();
        continue;
      }
      const int cap = j + NST - (j > j0) < nkt ? j + NST - (j > j0) : nkt;
      if (issued < cap && issued == open) {
        const int got = wave_poll4(ws, rws, foff, care, (open - 8) >> 1, issued == j, call, lane);
        open = max(open, 8 + 2 * got);
      }
      while (issued < cap && issued < open) issue();
      const int after = issued - j - 1;
      if (after >= 4) WAIT_VM(32); else if (after == 3) WAIT_VM(24); else if (after == 2) WAIT_VM(16); else if (after == 1) WAIT_VM(8); else WAIT_VM(0);
      __builtin_amdgcn_s_barrier();
    }
    __builtin_amdgcn_s_barrier();
  }
  if (!fin) return;
  __builtin_amdgcn_s_barrier();
  collect_max(lds, ws, rws, call, lane, wave);
}

__device__ __forceinline__ void computer(const Params& P, char* lds, char* ws, int lane, int wave, u32 call, u32 jobs) {
  const int a = P.rank;
  const int wr = wave >> 1, wc = wave & 1, fr = lane & 15, g = lane >> 4;
  const u32 xa0 = (u32)((wr * 64 + fr) * 128 + ((g ^ (fr >> 1)) * 16));
  const int tq = fr >> 2, tp = fr & 3;
  const u32 wa0 = 16384u + (u32)((8 * g + tq) * 256 + (((8 * wc + 2 * tp) ^ (tq | ((g & 1) << 3))) * 16) + (tp & 1) * 8);
  const u32x4 rl = mk_rsrc4(ws, WS_USED);
  char* am_dst = P.ws[lane & 7];
  asm volatile("" : "+v"(am_dst));
  int stamp = 1; (void)stamp;
  f32x4 fin[4][4];
  u32 fjob = 0;
  const u32 vo = (u32)lane * 16u;

  for (; (jobs & 0x3ffu) != 0; jobs >>= 10) {
    const Job J = decode_job(jobs & 0x3ffu, a);
    const bool cont = fjob != 0 && J.kind == JOB_O;
    f32x4 acc[4][4];
#pragma unroll
    for (int m = 0; m < 4; ++m)
#pragma unroll
      for (int n = 0; n < 4; ++n) acc[m][n] = cont ? fin[m][n] : (f32x4){0.f, 0.f, 0.f, 0.f};
    auto readf = [&](bf16x8 (&xf)[4], bf16x8 (&wf)[4], int slot, int ks) {
      LAS char* st = (LAS char*)lds + slot * 32768;
#pragma unroll
      for (int i = 0; i < 4; ++i) {
        const u32 o = (wa0 ^ (u32)((i >> 1) * 16 + (i & 1) * 8)) + (u32)(ks * 8192);
        const s16x4 lo = __builtin_amdgcn_ds_read_tr16_b64_v4i16((LAS s16x4*)(st + o));
        const s16x4 hi = __builtin_amdgcn_ds_read_tr16_b64_v4i16((LAS s16x4*)(st + o + 1024));
        wf[i] = __builtin_bit_cast(bf16x8, __builtin_shufflevector(lo, hi, 0, 1, 2, 3, 4, 5, 6, 7));
        xf[i] = *(LAS const bf16x8*)(st + ((xa0 ^ (u32)(ks * 64)) + (u32)(i * 2048)));
      }
    };
    auto mma = [&](const bf16x8 (&xf)[4], const bf16x8 (&wf)[4]) {
#pragma unroll
      for (int m = 0; m < 4; ++m)
#pragma unroll
        for (int n = 0; n < 4; ++n) acc[m][n] = __builtin_amdgcn_mfma_f32_16x16x32_bf16(wf[n], xf[m], acc[m][n], 0, 0, 0);
    };
#define INTERLEAVE(id) do { _Pragma("unroll") for (int i_ = 0; i_ < 6; ++i_) { __builtin_amdgcn_sched_group_barrier(0x8, 1, id); __builtin_amdgcn_sched_group_barrier(0x100, 2, id); } \
                            __builtin_amdgcn_sched_group_barrier(0x8, 10, id); __builtin_amdgcn_sched_barrier(0); } while (0)
    bf16x8 x0[4], w0[4], x1[4], w1[4];
    int slot = 0;
    BARRIER();
    readf(x0, w0, 0, 0);
    for (int kt = cont ? 8 : 0; kt < J.nkt; ++kt) {
      __builtin_amdgcn_s_waitcnt(0xC07F);
      __builtin_amdgcn_sched_barrier(0);
      readf(x1, w1, slot, 1); mma(x0, w0);
      INTERLEAVE(0);
      slot = slot == NST - 1 ? 0 : slot + 1;
      BARRIER();
      __builtin_amdgcn_sched_barrier(0);
      readf(x0, w0, slot, 0); mma(x1, w1);
      INTERLEAVE(1);
    }
    BARRIER();
    STAMP_AT(stamp++);

    if (J.kind == JOB_S) {
      const u32 slot = (u32)(box_base(J.dO) + box_idx(J.ja, J.p));
      u32x4 v[8];
#pragma unroll
      for (int jj = 0; jj < 8; ++jj) {
        const f32x4 u0 = acc[jj >> 1][(jj & 1) * 2], u1 = acc[jj >> 1][(jj & 1) * 2 + 1];
        v[jj].x = pack_bf16(u0[0], u0[1]); v[jj].y = pack_bf16(u0[2], u0[3]); v[jj].z = pack_bf16(u1[0], u1[1]); v[jj].w = pack_bf16(u1[2], u1[3]);
      }
      u32x4 hd; hd.x = 0; hd.y = call; hd.z = 0; hd.w = call;
#pragma unroll
      for (int jj = 0; jj < 8; ++jj) {
        const u64 mk = __ballot(v[jj].x & 1u);
        hd.x = lane == jj ? (u32)mk : hd.x; hd.z = lane == jj ? (u32)(mk >> 32) : hd.z;
        v[jj].x = (v[jj].x & ~1u) | (call & 1u);
      }
      u32 t_go = 0, step = 0;
      const u32 idx = (u32)((J.p >> 2) * 32 + (((J.ja - 5) >> 2) * 4 + wave) * 4 + (J.p & 3));
      const u32 tok = OFF_TOK + ((u32)J.dO * 128u + idx) * 64u;
      if (J.dO >= 2) {
        wave_wait<SL_LONG>(ws, OFF_T0 + (u32)J.dO * 128u + 4u, lane == 0, call, lane);
        t_go = rfl(__hip_atomic_load((GAS const u32*)(ws + OFF_T0 + (u32)J.dO * 128u), __ATOMIC_RELAXED, __HIP_MEMORY_SCOPE_AGENT));
      }
      if (J.dO >= 4) {
        t_go += ((idx & ~31u) * TAU_X + (idx & 31u) * (TAU_X / 8)) >> 4;
        step = 4 * TAU_X;
        if (idx >= 32) { sleep_until(t_go - 60u); wave_wait<4>(ws, tok - 32u * 64u, lane == 0, call, lane); }
      } else {
        if (J.dO >= 2) sleep_until(t_go - T_BOOK);
        u64 bk = J.dO == 1 ? TAU : TAU_C;
        book(bk, rl, OFF_TP + (u32)J.dO * 128u);
        asm volatile("s_waitcnt vmcnt(0)" : "+v"(bk) :: "memory");
        t_go = rfl((u32)(bk >> 4));
      }
      STAMP_AT(stamp++);
      const __amdgpu_buffer_rsrc_t rd = mk_rsrc(P.ws[J.dO], WS_USED);
      const u32 so = OFF_IN + slot * 32768u + (u32)wave * 8192u;
#pragma unroll
      for (int jj = 0; jj < 8; ++jj) {
        sleep_until(t_go + (((u32)jj * step) >> 4));
        if (jj == 0 && lane < 8) __builtin_amdgcn_raw_buffer_store_b128(hd, rd, vo, OFF_PF + slot * 512u + (u32)wave * 128u, 17);
        __builtin_amdgcn_raw_buffer_store_b128(v[jj], rd, vo + (u32)((jj & 3) * 1024), so + (u32)((jj >> 2) * 4096), 17);
        if (jj == 6 && J.dO >= 4 && lane == 0) st_agent(ws + tok, call);
        wait_vm<CAP - 1>();
      }
      STAMP_AT(stamp++);
    } else {
      fjob = jobs & 0x3ffu;
#pragma unroll
      for (int m = 0; m < 4; ++m)
#pragma unroll
        for (int n = 0; n < 4; ++n) fin[m][n] = acc[m][n];
    }
  }
  if (!fjob) return;

  const Job J = decode_job(fjob, a);
  const __amdgpu_buffer_rsrc_t rws = mk_rsrc(ws, WS_USED);
  LAS float* red = (LAS float*)lds;
  char* outp = P.out;
  asm volatile("" : "+s"(outp));
  const __amdgpu_buffer_rsrc_t ro = mk_rsrc(outp, OUT_BYTES);
  {
    u32 s0, s1;
    if (J.kind == JOB_O) { s0 = s1 = (u32)(J.p * 3 + J.sub); }
    else if ((J.ja >> 2) == 0) { s0 = (u32)(48 + J.p); s1 = (u32)(64 + J.p); }
    else { const int d0 = 4 | ((a ^ J.sub) & 2) | (a & 1); s0 = (u32)(box_base(d0) + J.p * 2 + (J.sub & 1)); s1 = (u32)(box_base(d0 ^ 1) + J.p * 2 + (J.sub & 1)); }
    const u64 t0 = __builtin_amdgcn_s_memrealtime();
    const u32 hoff = lane < 16 ? OFF_PF + (lane < 8 ? s0 : s1) * 512u + (u32)wave * 128u + (u32)(lane & 7) * 16u : OFF_TMO;
    const u32 so0 = OFF_IN + s0 * 32768u + (u32)wave * 8192u, so1 = OFF_IN + s1 * 32768u + (u32)wave * 8192u;
    float mx = 0.f;
    auto late = [&](bool dead) {
      if (!dead && __builtin_amdgcn_s_memrealtime() - t0 <= DEADLINE_TICKS) return false;
      if (lane == 0) st_sys(ws + OFF_TMO, call);
      WAIT_VM(0);
      return true;
    };
    auto receive = [&](auto two_c) {
      constexpr bool TWO = decltype(two_c)::value;
      u32x4 v0[8], v1[8];
      auto fetch = [&](int jj) {
        v0[jj] = __builtin_amdgcn_raw_buffer_load_b128(rws, vo + (u32)((jj & 3) * 1024), so0 + (u32)((jj >> 2) * 4096), 17);
        if (TWO) v1[jj] = __builtin_amdgcn_raw_buffer_load_b128(rws, vo + (u32)((jj & 3) * 1024), so1 + (u32)((jj >> 2) * 4096), 17);
      };
      u32x4 hd = __builtin_amdgcn_raw_buffer_load_b128(rws, hoff, 0, 17);
#pragma unroll
      for (int jj = 0; jj < 8; ++jj) fetch(jj);
      while (!__all(lane >= 16 || (hd.y >= call && hd.w >= call))) {
        if (late((u32)__builtin_amdgcn_readlane(hd.x, 63) == call)) break;
        __builtin_amdgcn_s_sleep(SL_HDR);
        asm volatile("" ::: "memory");
        hd = __builtin_amdgcn_raw_buffer_load_b128(rws, hoff, 0, 17);
      }
#pragma unroll
      for (int jj = 0; jj < 8; ++jj) {
        while (__builtin_expect(!__all((((v0[jj].x ^ call) | (TWO ? v1[jj].x ^ call : 0u)) & 1u) == 0), 0)) {
          asm volatile("" ::: "memory");
#pragma unroll
          for (int k = jj; k < jj + LOOK && k < 8; ++k) fetch(k);
          if (late(false)) break;
        }
        f32x4& u0 = fin[jj >> 1][(jj & 1) * 2]; f32x4& u1 = fin[jj >> 1][(jj & 1) * 2 + 1];
        auto add = [&](u32x4 c, int hl) {
          const u64 m = (u64)(u32)__builtin_amdgcn_readlane(hd.z, hl) << 32 | (u32)__builtin_amdgcn_readlane(hd.x, hl);
          const u32 lo = (c.x << 16 & ~0x10000u) | (__builtin_amdgcn_inverse_ballot_w64(m) ? 0x10000u : 0u);
          u0 += (f32x4){as_f32(lo), bf_hi(c.x), bf_lo(c.y), bf_hi(c.y)};
          u1 += (f32x4){bf_lo(c.z), bf_hi(c.z), bf_lo(c.w), bf_hi(c.w)};
        };
        add(v0[jj], jj);
        if (TWO) add(v1[jj], 8 + jj);
#pragma unroll
        for (int e = 0; e < 4; ++e) mx = fmaxf(mx, fmaxf(u0[e], u1[e]));
        asm volatile("" : "+v"(u0), "+v"(u1), "+v"(mx));
      }
    };
    if (J.kind == JOB_F) receive(std::true_type{}); else receive(std::false_type{});
    STAMP_AT(stamp++);
    mx = wave_max(mx);
    if (lane == 0) red[wave] = mx;
  }
  BARRIER();
  if (wave == 0 && lane < NR) {
    const float m4 = fmaxf(fmaxf(red[0], red[1]), fmaxf(red[2], red[3]));
    const u32 f = (u32)(J.p * 8 + (J.kind == JOB_F ? J.ja : J.sub));
    __hip_atomic_store((GAS u64*)(am_dst + OFF_AM + (f * 8u + (u32)a) * 8u), ((u64)call << 32) | __builtin_bit_cast(u32, m4), __ATOMIC_RELAXED, __HIP_MEMORY_SCOPE_SYSTEM);
  }
  STAMP_AT(stamp++);
  if (g & 1) {
#pragma unroll
    for (int m = 0; m < 4; ++m) { f32x4 t = fin[m][0]; fin[m][0] = fin[m][1]; fin[m][1] = t; t = fin[m][2]; fin[m][2] = fin[m][3]; fin[m][3] = t; }
  }
  collect_max(lds, ws, rws, call, lane, wave);
  const u32x4 r0 = *(LAS const u32x4*)(red + 8), r1 = *(LAS const u32x4*)(red + 12);
  const u32 gmax = rfl(max(max(max(r0.x, r0.y), max(r0.z, r0.w)), max(max(r1.x, r1.y), max(r1.z, r1.w))));
  STAMP_AT(stamp++);
  const float inv = gmax ? 127.0f / as_f32(gmax) : 0.f;
#pragma unroll
  for (int m = 0; m < 4; ++m) {
    u32x4 v;
#pragma unroll
    for (int n = 0; n < 4; ++n) {
      const f32x4 y = fin[m][n] * inv;
      u32 r = 0;
#pragma unroll
      for (int e = 0; e < 4; ++e) r = __builtin_amdgcn_cvt_pk_u8_f32(y[e], e, r);
      v[n] = r;
    }
    if (gmax == ~0u) v = (u32x4){0x80808080u, 0x80808080u, 0x80808080u, 0x80808080u};
    __builtin_amdgcn_raw_buffer_store_b128(v, ro, (u32)(fr * 4096 + g * 16), (u32)(J.sub * 128 + wr * 64 + m * 16) * 4096u + J.col0 + (u32)wc * 64u, OUT_AUX);
  }
  STAMP_AT(stamp++);
}

__global__ void __launch_bounds__(512, 2) gemm_rs_q8(const Params P) {
  __shared__ __attribute__((aligned(16))) char lds[5 * 32768];
  const int lane = threadIdx.x & 63, wave = (int)rfl(threadIdx.x >> 6);
  char* ws = P.ws[0];
  if (__builtin_expect(blockIdx.x < NPUSH, 1)) { if (wave < 6) pusher(P, lds, ws, lane, wave); return; }
  const u32 call = draw_call(ws, lane, wave);
  const u32 jobs = P.jobs[blockIdx.x];
  if (wave >= 4) loader(P, lds, ws, lane, wave, call, jobs);
  else computer(P, lds, ws, lane, wave, call, jobs);
}

#define LAG 12
void build_schedule(Params& P) {
#define RAMP 4
#define SHIFT2 2
#define ORD 3
  static const int orders[4][32] = {
    {0, 1, 2, 5, 6, 7, 8, 16, 17, 18, 19, 9, 10, 11, 12, 20, 21, 22, 23, 24, 25, 26, 27, 3, 4, 28, 29, 30, 31, 13, 14, 15},
    {0, 1, 2, 16, 5, 6, 17, 7, 8, 18, 19, 9, 10, 11, 12, 20, 21, 22, 23, 24, 25, 4, 3, 26, 27, 28, 29, 30, 31, 13, 14, 15},
    {0, 1, 2, 16, 17, 5, 6, 18, 19, 7, 8, 20, 21, 9, 10, 22, 23, 11, 12, 24, 25, 4, 3, 26, 27, 28, 29, 30, 31, 13, 14, 15},
    {0, 1, 2, 16, 17, 5, 18, 6, 19, 7, 20, 8, 21, 22, 23, 9, 10, 11, 12, 24, 25, 4, 3, 26, 27, 28, 29, 30, 31, 13, 14, 15}};
  const int* order = orders[ORD];
  int pos[32];
  for (int i = 0; i < 32; ++i) pos[order[i]] = i;
  auto ready = [&](int t, int p) { return pos[t] > pos[16 + p] ? pos[t] : pos[16 + p]; };
  for (int d = 0; d < NR; ++d) {
    unsigned char list[MAXU]; int n = 0;
    for (int i = 0; i < MAXU; ++i) list[i] = 0;
    if (d >= 2)
      for (int i = 0, nh1 = 0, lag = ((2 * (d - 2) + 1) * LAG + 10) / 20; i < 64; ++i) {
        auto put = [&](int k, int h) { const int c = order[k]; if (!(c < 16 && tile_dist(c) == d)) list[n++] = (unsigned char)(c * 2 + h); };
        if (i < 32) put(i, 0);
        for (; nh1 < 32 && nh1 + (nh1 * RAMP / 8 < lag ? nh1 * RAMP / 8 : lag) <= i; ++nh1) put(nh1, 1);
      }
    P.ucount[d] = (u32)n;
    for (int i = 0; i < MAXU / 4; ++i) P.units[d][i] = list[4 * i] | list[4 * i + 1] << 8 | list[4 * i + 2] << 16 | (u32)list[4 * i + 3] << 24;
  }
  for (int b = 0; b < NPUSH; ++b)
    for (int w = 0; w < 8; ++w) {
      const int d = push_link(b, w);
      const u32 code = (P.units[d][b >> 4] >> (((b >> 2) & 3) * 8)) & 0xffu;
      P.first[b][w] = slice_src(code, (u32)b & 3u, P.rank ^ d) | code >> 5;
    }
  static const int grp[16] = {0, 1, 0, 2, 1, 1, 0, 1, 0, 1, 0, 1, 0, 2, 2, 2};
  for (int i = 0; i < GRID; ++i) P.jobs[i] = 0;
  for (int x = 0; x < 8; ++x) {
    int ht[40], hp[40], n = 0;
    for (int r = 0; r < 32; ++r)
      for (int p = 0; p < 16; ++p)
        for (int t = 0; t < 16; ++t) {
          int gsel = grp[t] == 2 ? (p >> 2) & 1 : grp[t];
          if (t == 15 && p < 8) gsel = 0;
          if (gsel == (x >> 2) && ((p + (t == 2 ? SHIFT2 : 0)) & 3) == (x & 3) && ready(t, p) == r) { ht[n] = t; hp[n] = p; ++n; }
        }
    auto enc = [&](int i) { return (u32)(((ht[i] >= 13 ? JOB_O : JOB_S) << 8) | (ht[i] << 4) | hp[i]); };
    const int nw = 32 - NPUSH / 8;
    int f = 0, extra = nw;
    for (int s = 0; s < nw; ++s) {
      u32 jb = enc(s);
      if (ht[s] < 3 && extra < n) { jb = (u32)((JOB_F << 8) | ((ht[extra] - 13) << 4) | hp[extra]) | jb << 10 | enc(extra) << 20; ++extra; }
      else if (ht[s] < 13 && f < 10) { jb = jb << 10 | (u32)((JOB_F << 8) | ((3 + f % 5) << 4) | (x + 8 * (f / 5))); ++f; }
      P.jobs[NPUSH + s * 8 + x] = jb;
    }
  }
}

}

extern "C" void kernel_launch(int rank, int nranks, ncclComm_t comm, hipStream_t stream,
                              void* const* d_in, const int* in_sizes, int n_in,
                              void* d_out, int out_size,
                              const int* in_dtypes, int out_dtype,
                              void* const* d_out_all, void* const* d_ws_all, size_t ws_bytes) {
  (void)comm; (void)in_dtypes; (void)out_dtype; (void)d_out_all;
  if (nranks != NR || (unsigned)rank >= (unsigned)NR || n_in != 2 || ws_bytes < WS_USED || (u32)out_size != OUT_BYTES ||
      (u32)in_sizes[0] * 2u != X_BYTES || (u32)in_sizes[1] * 2u != W_BYTES) {
    fprintf(stderr, "unexpected problem shape\n");
    return;
  }
  Params P;
  P.x = (const char*)d_in[0]; P.w = (const char*)d_in[1]; P.out = (char*)d_out;
  for (int d = 0; d < NR; ++d) P.ws[d] = (char*)d_ws_all[rank ^ d];
  P.rank = rank; P.pad = 0;
  build_schedule(P);
  hipLaunchKernelGGL(gemm_rs_q8, dim3(GRID), dim3(512), 0, stream, P);
  HIP_CHECK(hipGetLastError());
}
```

```cpp
#include <hip/hip_runtime.h>
#include <rccl/rccl.h>
#include <cstdio>
#include <cstdint>

#define HIP_CHECK(expr) do { hipError_t _e = (expr); if (_e != hipSuccess) { \
    fprintf(stderr, "HIP error %s at %s:%d\n", hipGetErrorString(_e), __FILE__, __LINE__); \
    return; } } while (0)

namespace {

typedef unsigned u32;
typedef unsigned long long u64;
typedef u32 u32x4 __attribute__((ext_vector_type(4)));
typedef u32 u32x2 __attribute__((ext_vector_type(2)));
typedef float f32x4 __attribute__((ext_vector_type(4)));
typedef short s16x4 __attribute__((ext_vector_type(4)));
typedef short s16x8 __attribute__((ext_vector_type(8)));
typedef __bf16 bf16x8 __attribute__((ext_vector_type(8)));
#define LAS __attribute__((address_space(3)))
#define GAS __attribute__((address_space(1)))

constexpr int NR = 8;
#define NPUSH 24
constexpr int GRID = 256;
constexpr int MAXU = 64;
constexpr u64 DEADLINE_TICKS = 100000000ull;

constexpr u32 X_BYTES = 4096u * 512u * 2u, W_BYTES = 512u * 4096u * 2u, OUT_BYTES = 512u * 4096u;
constexpr u32 OFF_CNT = 0;
constexpr u32 OFF_TMO = 16u << 10;
constexpr u32 OFF_TK = (16u << 10) + 128u;
constexpr u32 OFF_TP = 18u << 10;
constexpr u32 OFF_T0 = 19u << 10;
constexpr u32 OFF_XF = 20u << 10;
constexpr u32 OFF_WF = 24u << 10;
constexpr u32 OFF_TR = 28u << 10;
constexpr u32 OFF_GO = 29u << 10;
constexpr u32 OFF_AM = 32u << 10;
constexpr u32 OFF_LM = 64u << 10;
constexpr u32 OFF_TOK = 128u << 10;
constexpr u32 OFF_PF = 192u << 10;
constexpr u32 OFF_DBG = 512u << 10;
constexpr u32 OFF_XS = 2u << 20;
constexpr u32 OFF_WS = 8u << 20;
constexpr u32 OFF_IN = 14u << 20;
constexpr u32 WS_USED = OFF_IN + 208u * 32768u;
#define TAU_X 224
#define TAU 245
#define TAU_C 300
#define PD 2
#define T_BOOK 70
#define T_REST 775
#define OUT_AUX 0
#define LOOK 3
#define SL_LONG 32
#define SL_GATE 6
#define SL_AM 3
#define SL_HDR 16
#define WDELAY 150
#define SL_AML 16
#define T_FLY 80
#define SL_GO 16
#define LASTW 3
#define CAP 3
constexpr u32 PANEL_B = 6u * 65536u;

enum { JOB_NONE = 0, JOB_S = 1, JOB_O = 2, JOB_F = 3 };

struct Params {
  const char* x; const char* w; char* out;
  char* ws[NR];
  int rank; int pad;
  u32 jobs[GRID];
  u32 first[NPUSH][16];
  u32 ucount[NR];
  u32 units[NR][MAXU / 4];
};

__host__ __device__ inline int tile_dist(int t) { return t < 3 ? 1 : t < 5 ? t - 1 : t < 13 ? 4 + ((t - 5) & 3) : 0; }
__host__ __device__ inline int tile_sub(int t, int a) { return t < 3 ? t : t < 5 ? 3 : t < 13 ? (a & 2) + ((t - 5) >> 2) : t - 13; }
__host__ __device__ inline int box_base(int d) { return d == 1 ? 0 : d < 4 ? 48 + 16 * (d - 2) : 80 + 32 * (d - 4); }
__host__ __device__ inline int box_cnt(int d) { return d == 1 ? 48 : d < 4 ? 16 : 32; }
__host__ __device__ inline int box_idx(int t, int p) { return t < 3 ? p * 3 + t : t < 5 ? p : p * 2 + ((t - 5) >> 2); }

__host__ __device__ inline u32 link_units(int d) { return d < 4 ? 62u : 60u; }
__host__ __device__ inline int push_link(int block, int wave) { return 2 + (wave + (block >> 3) * 2 + block) % 6; }
__host__ __device__ inline u32 slice_src(u32 code, u32 sl, int q) {
  const u32 h = code & 1u;
  if (code < 32) {
    const int t = code >> 1;
    return ((u32)((q ^ tile_dist(t)) * 512 + tile_sub(t, q) * 128) + (sl & 1u) * 64u) * 1024u + ((u32)(q & 1) * 256u + h * 128u + (sl >> 1) * 64u) * 2u;
  }
  return ((u32)(q & 1) * 256u + h * 128u + sl * 32u) * 8192u + ((u32)(q >> 2) * 2048u + ((code - 32u) >> 1) * 128u) * 2u;
}

__device__ __forceinline__ u32 rfl(u32 v) { return __builtin_amdgcn_readfirstlane(v); }
__device__ __forceinline__ __amdgpu_buffer_rsrc_t mk_rsrc(const void* p, u32 bytes) {
  return __builtin_amdgcn_make_buffer_rsrc((void*)p, 0, bytes, 0x00020000);
}
__device__ __forceinline__ u32x4 mk_rsrc4(const void* p, u32 bytes) {
  u64 a = (u64)p; u32x4 r; r.x = rfl((u32)a); r.y = rfl((u32)(a >> 32)) & 0xffffu; r.z = bytes; r.w = 0x00020000u; return r;
}
__device__ __forceinline__ u32 ld_sys(const char* p) { return __hip_atomic_load((GAS const u32*)p, __ATOMIC_RELAXED, __HIP_MEMORY_SCOPE_SYSTEM); }
__device__ __forceinline__ void st_sys(char* p, u32 v) { __hip_atomic_store((GAS u32*)p, v, __ATOMIC_RELAXED, __HIP_MEMORY_SCOPE_SYSTEM); }
__device__ __forceinline__ void st_agent(char* p, u32 v) { __hip_atomic_store((GAS u32*)p, v, __ATOMIC_RELAXED, __HIP_MEMORY_SCOPE_AGENT); }
#define WAIT_VM(n) asm volatile("s_waitcnt vmcnt(" #n ")" ::: "memory")
#define BARRIER() do { asm volatile("" ::: "memory"); __builtin_amdgcn_s_waitcnt(0xC07F);   __builtin_amdgcn_s_barrier(); asm volatile("" ::: "memory"); } while (0)

#define TSTAMP(i) do {} while (0)
#define STAMP_AT(i) do {} while (0)

template <int SL> __device__ __forceinline__ void wave_wait(char* ws, u32 off, bool care, u32 call, int lane) {
  if (lane == 63) { off = OFF_TMO; }
  else if (!care) { off = OFF_TMO; }
  const u64 t0 = __builtin_amdgcn_s_memrealtime();
  for (;;) {
    const u32 v = ld_sys(ws + off);
    const bool ok = (lane == 63 || !care) ? true : (v >= call);
    if (__all(ok)) break;
    const bool dead = __any(lane == 63 && v == call);
    if (dead || __builtin_amdgcn_s_memrealtime() - t0 > DEADLINE_TICKS) {
      if (lane == 0) st_sys(ws + OFF_TMO, call);
      WAIT_VM(0);
      break;
    }
    __builtin_amdgcn_s_sleep(SL);
  }
}

__device__ __forceinline__ int wave_poll4(char* ws, __amdgpu_buffer_rsrc_t rws, u32 off, bool care, int have, bool block, u32 call, int lane) {
  if (!care) off = OFF_TMO;
  const u64 t0 = __builtin_amdgcn_s_memrealtime();
  for (;;) {
    const u32x4 v = __builtin_amdgcn_raw_buffer_load_b128(rws, off, 0, 17);
    asm volatile("" ::: "memory");
    const u32 m = (u32)__ballot(care && v.x >= call && v.y >= call && v.z >= call && v.w >= call);
    const int n = __builtin_ctz(~(m & (m >> 16)));
    if (n > have || !block) return n;
    const bool dead = __any(!care && v.x == call);
    if (dead || __builtin_amdgcn_s_memrealtime() - t0 > DEADLINE_TICKS) {
      if (lane == 0) st_sys(ws + OFF_TMO, call);
      WAIT_VM(0);
      return have + 1;
    }
    if (have == 0) __builtin_amdgcn_s_sleep(SL_LONG); else __builtin_amdgcn_s_sleep(SL_GATE);
  }
}
#define DMA_SC " sc0 sc1"

typedef __bf16 bf16x2 __attribute__((ext_vector_type(2)));
__device__ __forceinline__ u32 pack_bf16(float lo, float hi) { bf16x2 r; r.x = (__bf16)lo; r.y = (__bf16)hi; return __builtin_bit_cast(u32, r); }
__device__ __forceinline__ float as_f32(u32 v) { return __builtin_bit_cast(float, v); }
__device__ __forceinline__ float bf_lo(u32 v) { return __builtin_bit_cast(float, v << 16); }
__device__ __forceinline__ float bf_hi(u32 v) { return __builtin_bit_cast(float, v & 0xffff0000u); }
__device__ __forceinline__ u32 wave_max(u32 m) {
  m = max(m, (u32)__builtin_amdgcn_update_dpp(0, (int)m, 0x111, 0xf, 0xf, false));
  m = max(m, (u32)__builtin_amdgcn_update_dpp(0, (int)m, 0x112, 0xf, 0xf, false));
  m = max(m, (u32)__builtin_amdgcn_update_dpp(0, (int)m, 0x114, 0xf, 0xf, false));
  m = max(m, (u32)__builtin_amdgcn_update_dpp(0, (int)m, 0x118, 0xf, 0xf, false));
  const u32 a = max((u32)__builtin_amdgcn_readlane(m, 15), (u32)__builtin_amdgcn_readlane(m, 31));
  const u32 b = max((u32)__builtin_amdgcn_readlane(m, 47), (u32)__builtin_amdgcn_readlane(m, 63));
  return max(a, b);
}

__device__ __forceinline__ void dma4(u32 lds_dst, u32 v0, u32 v1, u32 v2, u32 v3, u32x4 rs, u32 so, u32 st) {
  u32 keep, t;
  asm volatile(
      "s_mov_b32 %0, m0\n\t"
      "s_mov_b32 m0, %2\n\ts_add_u32 %1, %8, %9\n\ts_nop 1\n\tbuffer_load_dwordx4 %3, %7, %8 offen" DMA_SC " lds\n\t"
      "s_add_u32 m0, %2, 0x400\n\ts_nop 0\n\tbuffer_load_dwordx4 %4, %7, %1 offen" DMA_SC " lds\n\t"
      "s_add_u32 m0, %2, 0x800\n\ts_add_u32 %1, %1, %9\n\tbuffer_load_dwordx4 %5, %7, %1 offen" DMA_SC " lds\n\t"
      "s_add_u32 m0, %2, 0xc00\n\ts_add_u32 %1, %1, %9\n\tbuffer_load_dwordx4 %6, %7, %1 offen" DMA_SC " lds\n\t"
      "s_mov_b32 m0, %0"
      : "=&s"(keep), "=&s"(t)
      : "s"(lds_dst), "v"(v0), "v"(v1), "v"(v2), "v"(v3), "s"(rs), "s"(so), "s"(st)
      : "memory", "scc");
}

__device__ __forceinline__ void dma8_lin(u32 lds_dst, u32 v, u32x4 rs, u32 sx, u32 sw) {
  u32 keep;
  asm volatile(
      "s_mov_b32 %0, m0\n\ts_mov_b32 m0, %1\n\ts_nop 2\n\t"
      "buffer_load_dwordx4 %2, %3, %4 offen" DMA_SC " lds\n\tbuffer_load_dwordx4 %2, %3, %4 offen offset:1024" DMA_SC " lds\n\t"
      "buffer_load_dwordx4 %2, %3, %4 offen offset:2048" DMA_SC " lds\n\tbuffer_load_dwordx4 %2, %3, %4 offen offset:3072" DMA_SC " lds\n\t"
      "s_add_u32 m0, %1, 0x4000\n\ts_nop 0\n\t"
      "buffer_load_dwordx4 %2, %3, %5 offen" DMA_SC " lds\n\tbuffer_load_dwordx4 %2, %3, %5 offen offset:1024" DMA_SC " lds\n\t"
      "buffer_load_dwordx4 %2, %3, %5 offen offset:2048" DMA_SC " lds\n\tbuffer_load_dwordx4 %2, %3, %5 offen offset:3072" DMA_SC " lds\n\t"
      "s_mov_b32 m0, %0"
      : "=&s"(keep) : "s"(lds_dst), "v"(v), "s"(rs), "s"(sx), "s"(sw) : "memory", "scc");
}

#define WIN 3
template <int N> __device__ __forceinline__ void wait_vm() { asm volatile("s_waitcnt vmcnt(%0)" :: "n"(N) : "memory"); }
template <int IMM> __device__ __forceinline__ void st16(u32x4 v, u32x4 rs, u32 voff, u32 soff) {
  asm volatile("buffer_store_dwordx4 %0, %1, %2, %3 offen offset:%4 sc0 sc1\n\ts_nop 1" :: "v"(v), "v"(voff), "s"(rs), "s"(soff), "n"(IMM) : "memory");
}
__device__ __forceinline__ void draw_async(u32& v, u32x4 rl, u32 off) {
  u64 ex; v = 1;
  asm volatile("s_mov_b64 %1, exec\n\ts_mov_b64 exec, 1\n\tbuffer_atomic_add %0, off, %2, %3 sc0\n\ts_mov_b64 exec, %1"
               : "+v"(v), "=&s"(ex) : "s"(rl), "s"(off) : "memory");
}
__device__ __forceinline__ void book(u64& tk, u32x4 rl, u32 off) {
  u64 ex; const u64 lo = (__builtin_amdgcn_s_memrealtime() + T_BOOK) * 16u;
  asm volatile("s_mov_b64 %1, exec\n\ts_mov_b64 exec, 1\n\tbuffer_atomic_umax_x2 %2, off, %3, %4\n\tbuffer_atomic_add_x2 %0, off, %3, %4 sc0\n\ts_mov_b64 exec, %1"
               : "+v"(tk), "=&s"(ex) : "v"(lo), "s"(rl), "s"(off) : "memory");
}
__device__ __forceinline__ void sleep_until(u32 due) {
  for (;;) {
    const u32 left = due - (u32)__builtin_amdgcn_s_memrealtime() - 1u;
    if (left >= 10000u) break;
    if (left > 200u) __builtin_amdgcn_s_sleep(48); else if (left > 40u) __builtin_amdgcn_s_sleep(8); else __builtin_amdgcn_s_sleep(1);
  }
}
struct Slice { u32 src, dst, flag, kind; };
__device__ __forceinline__ u32 draw_call(char* ws, int lane, int wave) {
  u32 call = 0;
  if (lane == 0) call = __hip_atomic_fetch_add((GAS u32*)(ws + OFF_CNT + blockIdx.x * 64 + wave * 4), 1u, __ATOMIC_RELAXED, __HIP_MEMORY_SCOPE_AGENT) + 1u;
  return rfl(call);
}
__device__ __forceinline__ void pusher(const Params& P, char* lds, char* ws, int lane, int wave) {
  const u32 vx0 = (u32)(lane >> 3) * 1024u + (u32)(((lane & 7) ^ (lane >> 4)) * 16), vx1 = vx0 ^ 64u;
  const u32 vw0 = (u32)(lane >> 4) * 8192u + (u32)(((lane & 15) ^ (lane >> 4)) * 16), vw1 = vw0 ^ 128u;
  const int d = push_link(blockIdx.x, wave);
  u32 f0 = P.first[blockIdx.x][2 * wave], f1 = P.first[blockIdx.x][2 * wave + 1]; const char* xp = P.x; const char* wp = P.w;
  char* pw = P.ws[d];
  asm volatile("" : "+s"(f0), "+s"(f1), "+s"(xp), "+s"(wp), "+s"(pw), "+s"(ws));
  const u32x4 rx = mk_rsrc4(xp, X_BYTES), rw = mk_rsrc4(wp, W_BYTES);
  const u32 ring0 = (u32)(size_t)(LAS char*)lds + (u32)wave * 16384u;
  auto load_all = [&](const Slice& c, u32 buf) {
    const u32 dst = ring0 + buf * 8192u;
    if (c.kind == 0) { dma4(dst, vx0, vx1, vx0, vx1, rx, c.src, 8192u); dma4(dst + 4096u, vx0, vx1, vx0, vx1, rx, c.src + 32768u, 8192u); }
    else { dma4(dst, vw0, vw0, vw1, vw1, rw, c.src, 32768u); dma4(dst + 4096u, vw0, vw0, vw1, vw1, rw, c.src + 131072u, 32768u); }
  };
  { Slice c; c.src = f0 & ~1u; c.kind = f0 & 1u; c.dst = c.flag = 0; load_all(c, 0); }
  const u32x4 rl = mk_rsrc4(ws, WS_USED);
  u32 cnt; draw_async(cnt, rl, OFF_CNT + blockIdx.x * 64u + (u32)wave * 4u);
  const int q = (int)(f1 >> 16);
  const u32 ns = link_units(d) * 4u;
  auto decode = [&](u32 sidx, u32 code) -> Slice {
    Slice r;
    const u32 sl = sidx & 3u, h = code & 1u;
    r.kind = code >> 5; r.src = slice_src(code, sl, q);
    r.dst = (code < 32 ? OFF_XS : OFF_WS) + ((code >> 1) & 15u) * PANEL_B;
    r.flag = (code < 32 ? OFF_XF : OFF_WF) + ((code >> 1) & 15u) * 256u;
    r.dst += (u32)(d - 2) * 65536u + h * 32768u + sl * 8192u; r.flag += (u32)(d - 2) * 32u + h * 16u + sl * 4u;
    return r;
  };
  const u32 vp = (u32)lane * 16u;
  const u32x4 rd = mk_rsrc4(pw, WS_USED);
  LAS char* ring = (LAS char*)lds + wave * 16384;
  u32 nxt_i = NPUSH + blockIdx.x;
  Slice cur = decode(blockIdx.x, f1 & 0xffu), nxt = decode(nxt_i, (f1 >> 8) & 0xffu);
  u32 buf = 0;
  asm volatile("s_waitcnt vmcnt(0)" : "+v"(cnt), "+s"(cur.dst), "+s"(nxt.src) :: "memory");
  const u32 call = rfl(cnt) + 1u;
  const u32 tk_base = (call - 1u) * (ns - NPUSH);
  const u32 tk_off = OFF_TK + (u32)d * 128u;
  u32 pend = 0;
  auto publish = [&]() {
    if (lane == 0) {
      st_sys(pw + pend, call);
    }
  };
  auto send = [&](int k) {
    const u32x4 v = *(LAS const u32x4*)(ring + buf * 8192u + k * 1024 + lane * 16);
    st16<0>(v, rd, vp, cur.dst + (u32)k * 1024u);
  };
  for (;;) {
    u32 tk;
    send(0);
    load_all(nxt, buf ^ 1u);
    draw_async(tk, rl, tk_off);
    const u32 t_draw = (u32)__builtin_amdgcn_s_memrealtime();
    Slice nn = nxt;
#pragma unroll
    for (int k = 0; k < 8; ++k) {
      if (k) send(k);
      if (k == WIN + 1) {
        asm volatile("" : "+v"(tk));
        nxt_i = rfl(tk) - tk_base;
        if (nxt_i == ns - 40u && lane == 0)
          __hip_atomic_store((GAS u64*)(ws + OFF_T0 + (u32)d * 128u), ((u64)call << 32) | (t_draw + T_REST), __ATOMIC_RELAXED, __HIP_MEMORY_SCOPE_AGENT);
        nxt_i += 2u * NPUSH;
        if (nxt_i < ns) nn = decode(nxt_i, (P.units[d][nxt_i >> 4] >> ((nxt_i & 12u) * 2u)) & 0xffu);
      }
      if (k < WIN) wait_vm<WIN - 1 + 9>(); else wait_vm<WIN - 1>();
      if (k == WIN - 2 && pend) publish();
    }
    pend = cur.flag;
    cur = nxt; nxt = nn; buf ^= 1u;
    if (nxt_i >= ns) break;
  }
#pragma unroll
  for (int k = 0; k < 8; ++k) {
    send(k);
    wait_vm<LASTW - 1>();
    if (k == LASTW - 2) publish();
  }
  pend = cur.flag;
  WAIT_VM(0);
  publish();
}

__device__ __forceinline__ void coordinator(const Params& P, char* ws, int lane, u32 call) {
  const u32 lo = (u32)(lane & 7) * 128u, off = (lane & 8 ? OFF_TR : OFF_T0) + lo;
  char* peer = P.ws[lane & 7];
  asm volatile("" : "+v"(peer));
  const u64 t0 = __builtin_amdgcn_s_memrealtime();
  if (lane >= 2 && lane < 8) st_sys(peer + OFF_TR + lo + 4u, call);
  auto rl = [&](u32 v, int i) { return (u32)__builtin_amdgcn_readlane(v, i); };
  bool seen = (lane & 7) < 2 || lane >= 16;
  u32 t = 0;
  for (;;) {
    u64 m = 0;
    if (!seen) m = __hip_atomic_load((GAS const u64*)(ws + off), __ATOMIC_RELAXED, __HIP_MEMORY_SCOPE_SYSTEM);
    asm volatile("" : "+v"(m));
    const u64 now = __builtin_amdgcn_s_memrealtime();
    if (!seen && ((u32)(m >> 32) >= call || now - t0 > DEADLINE_TICKS)) {
      t = lane & 8 ? (u32)max((int)(u32)(now - t0) - T_FLY, 0) : (u32)m;
      seen = true;
    }
    if (__all(seen)) break;
    __builtin_amdgcn_s_sleep(1);
  }
  const u32 tf = rl(t, 4) + (u32)((int)(rl(t, 5) + rl(t, 6) + rl(t, 7) - 3u * rl(t, 4)) >> 2), tn = rl(t, 2) + (u32)((int)(rl(t, 3) - rl(t, 2)) >> 1);
  const u32 lag = max(t, (u32)__builtin_amdgcn_update_dpp(0, (int)t, 0xb1, 0xf, 0xf, false));
  if (lane == 10 || lane == 12 || lane == 14)
    __hip_atomic_store((GAS u64*)(ws + OFF_GO + lo / 2u), ((u64)call << 32) | ((lane >= 12 ? tf : tn) + lag), __ATOMIC_RELAXED, __HIP_MEMORY_SCOPE_AGENT);
}

constexpr int NST = 5;

struct Job { int kind, ja, p, dO, sub, nh, nkt; u32 col0; };
__device__ __forceinline__ Job decode_job(u32 code, int a) {
  Job j; j.kind = (code >> 8) & 3; j.ja = (code >> 4) & 15; j.p = code & 15;
  int chalf;
  if (j.kind == JOB_F) { j.dO = 0; j.sub = j.ja & 3; chalf = (a >> 2) ^ (j.ja >> 2); }
  else { j.dO = tile_dist(j.ja); j.sub = tile_sub(j.ja, a); chalf = a >> 2; }
  j.col0 = (u32)(chalf * 2048 + j.p * 128);
  j.nh = j.kind == JOB_F ? 0 : j.dO >= 2 ? 10 : 12;
  j.nkt = 8 + 2 * j.nh;
  return j;
}

struct Polls { u32x2 va, vb; u32 ta, tb;
  __device__ __forceinline__ void retire() { asm volatile("" :: "v"(va), "v"(vb), "v"(ta), "v"(tb)); } };
__device__ __forceinline__ u32 gather_max(char* ws, u32 off, u32 call, int lane, int lazy, Polls& q) {
  u32x2& va = q.va; u32x2& vb = q.vb; u32& ta = q.ta; u32& tb = q.tb;
  const __amdgpu_buffer_rsrc_t rws = mk_rsrc(ws, WS_USED);
  u32 gm = 0, tmo_v;
  const u32 toff = lane ? 0x80000000u : OFF_TMO;
  const u64 t0 = __builtin_amdgcn_s_memrealtime();
  va = __builtin_amdgcn_raw_buffer_load_b64(rws, off, 0, 17);
  ta = __builtin_amdgcn_raw_buffer_load_b32(rws, toff, 0, 17);
  __builtin_amdgcn_s_sleep(SL_AM);
  for (;;) {
    asm volatile("" ::: "memory");
    vb = __builtin_amdgcn_raw_buffer_load_b64(rws, off, 0, 17); tb = __builtin_amdgcn_raw_buffer_load_b32(rws, toff, 0, 17);
    asm volatile("" ::: "memory");
    if (va.y >= call) { gm = va.x; off = 0x80000000u; }
    tmo_v = rfl(ta);
    if (__all(off == 0x80000000u)) break;
    va = __builtin_amdgcn_raw_buffer_load_b64(rws, off, 0, 17); ta = __builtin_amdgcn_raw_buffer_load_b32(rws, toff, 0, 17);
    asm volatile("" ::: "memory");
    if (vb.y >= call) { gm = vb.x; off = 0x80000000u; }
    tmo_v = rfl(tb);
    const u64 missing = __ballot(off != 0x80000000u);
    if (!missing) break;
    if (tmo_v == call || __builtin_amdgcn_s_memrealtime() - t0 > DEADLINE_TICKS) {
      if (lane == 0) st_sys(ws + OFF_TMO, call);
      WAIT_VM(0);
      tmo_v = call;
      break;
    }
    if (__builtin_popcountll(missing) > lazy) __builtin_amdgcn_s_sleep(SL_AML);
  }
  gm = wave_max(gm);
  return tmo_v == call ? ~0u : gm;
}

__device__ __forceinline__ void loader(const Params& P, char* lds, char* ws, int lane, int wave, u32 call, u32 jobs) {
  const int a = P.rank;
  const __amdgpu_buffer_rsrc_t rws = mk_rsrc(ws, WS_USED);
  const u32 l4 = (u32)(wave - 4) * 4u;
  const u32 vx0 = (u32)(lane >> 3) * 1024u + (u32)(((lane & 7) ^ (lane >> 4)) * 16), vx1 = vx0 ^ 64u;
  const u32 vw0 = (u32)(lane >> 4) * 8192u + (u32)(((lane & 15) ^ (lane >> 4)) * 16), vw1 = vw0 ^ 128u;
  const u32 vp = (u32)lane * 16u;
  const u32x4 rx = mk_rsrc4(P.x, X_BYTES), rw = mk_rsrc4(P.w, W_BYTES), rl = mk_rsrc4(ws, WS_USED);
  const u32 lds0 = (u32)(size_t)(LAS char*)lds + l4 * 1024u;
  bool fin = false;
  if (WDELAY) sleep_until((u32)__builtin_amdgcn_s_memrealtime() + WDELAY);
  int stamp = 1; (void)stamp;
  for (; (jobs & 0x3ffu) != 0; jobs >>= 10) {
    const Job J = decode_job(jobs & 0x3ffu, a);
    const int nh = J.nh, nkt = J.nkt, ns = nh >> 1, skip = J.dO >= 2 ? J.dO - 2 : 6;
    const u32 row0 = (u32)((a ^ J.dO) * 512 + J.sub * 128);
    const int j0 = fin && J.kind == JOB_O ? 8 : 0;
    fin = fin || J.kind != JOB_S;
    auto seg_at = [&](int i) { int sg = 5 - i; return sg - (sg <= skip && skip < 6); };
    const int fl = lane & 15;
    const bool care = lane < 32 && fl < nh;
    const u32 foff = (lane < 16 ? OFF_XF + (u32)J.ja * 256u : OFF_WF + (u32)J.p * 256u) + (fl < ns ? (u32)seg_at(fl) * 32u : (u32)seg_at(nh - 1 - fl) * 32u + 16u);
    u32 tab;
    { const int k = lane - 8, i = k >> 1;
      tab = lane < 8 ? (row0 + l4 * 8u) * 1024u + (u32)lane * 128u
                     : OFF_XS + (u32)J.ja * PANEL_B + l4 * 1024u + (u32)(k & 1) * 16384u + (i < ns ? (u32)seg_at(i) * 65536u : (u32)seg_at(nh - 1 - i) * 65536u + 32768u); }
    const u32 dw = OFF_WS - OFF_XS + (u32)(J.p - J.ja) * PANEL_B, w0 = l4 * (4u * 8192u) + J.col0 * 2u;
    int issued = j0, open = 8;
    u32 dst = lds0;
    auto issue = [&]() {
      const u32 ox = __builtin_amdgcn_readlane(tab, issued);
      if (__builtin_expect(issued >= 8, 1)) dma8_lin(dst, vp, rl, ox, ox + dw);
      else {
        dma4(dst, vx0, vx1, vx0, vx1, rx, ox, 8192u);
        dma4(dst + 16384u, vw0, vw0, vw1, vw1, rw, w0 + (u32)issued * (64u * 8192u), 32768u);
      }
      ++issued; dst = dst == lds0 + (NST - 1) * 32768u ? lds0 : dst + 32768u;
    };
    for (int j = j0; j <= nkt; ++j) {
      if (__builtin_expect(issued == j + NST - 2 && issued < open, 1)) {
        issue();
        WAIT_VM(24);
        __builtin_amdgcn_s_barrier();
        continue;
      }
      const int cap = j + NST - (j > j0) < nkt ? j + NST - (j > j0) : nkt;
      if (issued < cap && issued == open) {
        const int got = wave_poll4(ws, rws, foff, care, (open - 8) >> 1, issued == j, call, lane);
        open = max(open, 8 + 2 * got);
      }
      while (issued < cap && issued < open) issue();
      const int after = issued - j - 1;
      if (after >= 4) WAIT_VM(32); else if (after == 3) WAIT_VM(24); else if (after == 2) WAIT_VM(16); else if (after == 1) WAIT_VM(8); else WAIT_VM(0);
      __builtin_amdgcn_s_barrier();
    }
    __builtin_amdgcn_s_barrier();
  }
}

__device__ __forceinline__ void computer(const Params& P, char* lds, char* ws, int lane, int wave, u32 call, u32 jobs) {
  const int a = P.rank;
  const int wr = wave >> 1, wc = wave & 1, fr = lane & 15, g = lane >> 4;
  const u32 xa0 = (u32)((wr * 64 + fr) * 128 + ((g ^ (fr >> 1)) * 16));
  const int tq = fr >> 2, tp = fr & 3;
  const u32 wa0 = 16384u + (u32)((8 * g + tq) * 256 + (((8 * wc + 2 * tp) ^ (tq | ((g & 1) << 3))) * 16) + (tp & 1) * 8);
  const u32x4 rl = mk_rsrc4(ws, WS_USED);
  int stamp = 1; (void)stamp;
  f32x4 fin[4][4];
  u32 fjob = 0;
  const u32 vo = (u32)lane * 16u;

  for (; (jobs & 0x3ffu) != 0; jobs >>= 10) {
    const Job J = decode_job(jobs & 0x3ffu, a);
    const bool cont = fjob != 0 && J.kind == JOB_O;
    f32x4 acc[4][4];
#pragma unroll
    for (int m = 0; m < 4; ++m)
#pragma unroll
      for (int n = 0; n < 4; ++n) acc[m][n] = cont ? fin[m][n] : (f32x4){0.f, 0.f, 0.f, 0.f};
    auto readf = [&](bf16x8 (&xf)[4], bf16x8 (&wf)[4], int slot, int ks) {
      LAS char* st = (LAS char*)lds + slot * 32768;
#pragma unroll
      for (int i = 0; i < 4; ++i) {
        const u32 o = (wa0 ^ (u32)((i >> 1) * 16 + (i & 1) * 8)) + (u32)(ks * 8192);
        const s16x4 lo = __builtin_amdgcn_ds_read_tr16_b64_v4i16((LAS s16x4*)(st + o));
        const s16x4 hi = __builtin_amdgcn_ds_read_tr16_b64_v4i16((LAS s16x4*)(st + o + 1024));
        wf[i] = __builtin_bit_cast(bf16x8, __builtin_shufflevector(lo, hi, 0, 1, 2, 3, 4, 5, 6, 7));
        xf[i] = *(LAS const bf16x8*)(st + ((xa0 ^ (u32)(ks * 64)) + (u32)(i * 2048)));
      }
    };
    auto mma = [&](const bf16x8 (&xf)[4], const bf16x8 (&wf)[4]) {
#pragma unroll
      for (int m = 0; m < 4; ++m)
#pragma unroll
        for (int n = 0; n < 4; ++n) acc[m][n] = __builtin_amdgcn_mfma_f32_16x16x32_bf16(wf[n], xf[m], acc[m][n], 0, 0, 0);
    };
#define INTERLEAVE(id) do { _Pragma("unroll") for (int i_ = 0; i_ < 6; ++i_) { __builtin_amdgcn_sched_group_barrier(0x8, 1, id); __builtin_amdgcn_sched_group_barrier(0x100, 2, id); } \
                            __builtin_amdgcn_sched_group_barrier(0x8, 10, id); __builtin_amdgcn_sched_barrier(0); } while (0)
    bf16x8 x0[4], w0[4], x1[4], w1[4];
    int slot = 0;
    BARRIER();
    readf(x0, w0, 0, 0);
    for (int kt = cont ? 8 : 0; kt < J.nkt; ++kt) {
      __builtin_amdgcn_s_waitcnt(0xC07F);
      __builtin_amdgcn_sched_barrier(0);
      readf(x1, w1, slot, 1); mma(x0, w0);
      INTERLEAVE(0);
      slot = slot == NST - 1 ? 0 : slot + 1;
      BARRIER();
      __builtin_amdgcn_sched_barrier(0);
      readf(x0, w0, slot, 0); mma(x1, w1);
      INTERLEAVE(1);
    }
    BARRIER();
    STAMP_AT(stamp++);

    if (J.kind == JOB_S) {
      const u32 slot = (u32)(box_base(J.dO) + box_idx(J.ja, J.p));
      u32x4 v[8];
#pragma unroll
      for (int jj = 0; jj < 8; ++jj) {
        const f32x4 u0 = acc[jj >> 1][(jj & 1) * 2], u1 = acc[jj >> 1][(jj & 1) * 2 + 1];
        v[jj].x = pack_bf16(u0[0], u0[1]); v[jj].y = pack_bf16(u0[2], u0[3]); v[jj].z = pack_bf16(u1[0], u1[1]); v[jj].w = pack_bf16(u1[2], u1[3]);
      }
      const u32 so = OFF_IN + slot * 32768u + (u32)wave * 8192u, fo = OFF_PF + slot * 512u + (u32)wave * 128u;
      if (J.dO >= 2) {
#pragma unroll
        for (int jj = 0; jj < 8; ++jj) st16<0>(v[jj], rl, vo, so + (u32)(jj * 1024));
        WAIT_VM(0);
        if (lane == 0) st_sys(P.ws[J.dO] + fo, call);
        STAMP_AT(stamp++);
      } else {
        u64 bk = TAU;
        book(bk, rl, OFF_TP);
        u32x4 hd; hd.x = 0; hd.y = call; hd.z = 0; hd.w = call;
#pragma unroll
        for (int jj = 0; jj < 8; ++jj) {
          const u64 mk = __ballot(v[jj].x & 1u);
          hd.x = lane == jj ? (u32)mk : hd.x; hd.z = lane == jj ? (u32)(mk >> 32) : hd.z;
          v[jj].x = (v[jj].x & ~1u) | (call & 1u);
        }
        asm volatile("s_waitcnt vmcnt(0)" : "+v"(bk) :: "memory");
        STAMP_AT(stamp++);
        const u32x4 rd = mk_rsrc4(P.ws[1], WS_USED);
        sleep_until(rfl((u32)(bk >> 4)));
        if (lane < 8) st16<0>(hd, rd, vo, fo);
#pragma unroll
        for (int jj = 0; jj < 8; ++jj) {
          st16<0>(v[jj], rd, vo, so + (u32)(jj * 1024));
          wait_vm<CAP - 1>();
        }
      }
      STAMP_AT(stamp++);
    } else {
      fjob = jobs & 0x3ffu;
#pragma unroll
      for (int m = 0; m < 4; ++m)
#pragma unroll
        for (int n = 0; n < 4; ++n) fin[m][n] = acc[m][n];
    }
  }
  if (!fjob) return;

  const Job J = decode_job(fjob, a);
  const __amdgpu_buffer_rsrc_t rws = mk_rsrc(ws, WS_USED);
  char* outp = P.out;
  asm volatile("" : "+s"(outp));
  const __amdgpu_buffer_rsrc_t ro = mk_rsrc(outp, OUT_BYTES);
  {
    float mx = 0.f;
    auto add = [&](int jj, u32 x_lo, u32x4 c) {
      f32x4& u0 = fin[jj >> 1][(jj & 1) * 2]; f32x4& u1 = fin[jj >> 1][(jj & 1) * 2 + 1];
      u0 += (f32x4){as_f32(x_lo), bf_hi(c.x), bf_lo(c.y), bf_hi(c.y)};
      u1 += (f32x4){bf_lo(c.z), bf_hi(c.z), bf_lo(c.w), bf_hi(c.w)};
    };
    auto done = [&](int jj) {
      f32x4& u0 = fin[jj >> 1][(jj & 1) * 2]; f32x4& u1 = fin[jj >> 1][(jj & 1) * 2 + 1];
#pragma unroll
      for (int e = 0; e < 4; ++e) mx = fmaxf(mx, fmaxf(u0[e], u1[e]));
      asm volatile("" : "+v"(u0), "+v"(u1), "+v"(mx));
    };
    if (J.kind == JOB_F) {
      const bool far = (J.ja >> 2) != 0;
      const int d0 = far ? 4 | ((a ^ J.sub) & 2) : 2;
      const u32 bi = far ? (u32)(J.p * 2 + (J.sub & 1)) : (u32)J.p;
      const u32 s0 = (u32)box_base(d0) + bi, s1 = (u32)box_base(d0 + 1) + bi;
      const __amdgpu_buffer_rsrc_t r0 = mk_rsrc(P.ws[d0], WS_USED), r1 = mk_rsrc(P.ws[d0 + 1], WS_USED);
      const u32 o = OFF_GO + (u32)d0 * 64u;
      wave_wait<SL_GO>(ws, o + 4u, lane == 0, call, lane);
      u32 t_go = rfl(__hip_atomic_load((GAS const u32*)(ws + o), __ATOMIC_RELAXED, __HIP_MEMORY_SCOPE_AGENT));
      const u32 tok = OFF_TOK + (u32)((J.p * 8 + J.ja) * 4 + wave) * 64u;
      const u32 step = far ? 4 * TAU_X : 2 * TAU_C;
      t_go += ((u32)(J.p >> 2) * 8u * step + (u32)(((J.sub & 1) * 4 + wave) * 4 + (J.p & 3)) * (step >> 5)) >> 4;
      sleep_until(t_go - 50u);
      wave_wait<4>(ws, lane == 2 ? tok - 128u * 64u : OFF_PF + (lane == 0 ? s0 : s1) * 512u + (u32)wave * 128u, lane < (J.p >= 4 ? 3 : 2), call, lane);
      const u32 so0 = OFF_IN + s0 * 32768u + (u32)wave * 8192u, so1 = OFF_IN + s1 * 32768u + (u32)wave * 8192u;
      u32x4 v0[8], v1[8];
#pragma unroll
      for (int jj = 0; jj < 8 + PD; ++jj) {
        if (jj < 8) {
          sleep_until(t_go + (((u32)jj * step) >> 4));
          v0[jj] = __builtin_amdgcn_raw_buffer_load_b128(r0, vo + (u32)((jj & 3) * 1024), so0 + (u32)((jj >> 2) * 4096), 17);
          v1[jj] = __builtin_amdgcn_raw_buffer_load_b128(r1, vo + (u32)((jj & 3) * 1024), so1 + (u32)((jj >> 2) * 4096), 17);
          if (jj == 6 && lane == 0) st_agent(ws + tok, call);
        }
        if (jj >= PD) { add(jj - PD, v0[jj - PD].x << 16, v0[jj - PD]); add(jj - PD, v1[jj - PD].x << 16, v1[jj - PD]); done(jj - PD); }
      }
    } else {
      const u32 s0 = (u32)(J.p * 3 + J.sub);
      const u64 t0 = __builtin_amdgcn_s_memrealtime();
      const u32 hoff = lane < 8 ? OFF_PF + s0 * 512u + (u32)wave * 128u + (u32)lane * 16u : OFF_TMO;
      const u32 so0 = OFF_IN + s0 * 32768u + (u32)wave * 8192u;
      auto late = [&](bool dead) {
        if (!dead && __builtin_amdgcn_s_memrealtime() - t0 <= DEADLINE_TICKS) return false;
        if (lane == 0) st_sys(ws + OFF_TMO, call);
        WAIT_VM(0);
        return true;
      };
      u32x4 v0[8];
      auto fetch = [&](int jj) { v0[jj] = __builtin_amdgcn_raw_buffer_load_b128(rws, vo + (u32)((jj & 3) * 1024), so0 + (u32)((jj >> 2) * 4096), 17); };
      u32x4 hd = __builtin_amdgcn_raw_buffer_load_b128(rws, hoff, 0, 17);
#pragma unroll
      for (int jj = 0; jj < 8; ++jj) fetch(jj);
      while (!__all(lane >= 8 || (hd.y >= call && hd.w >= call))) {
        if (late((u32)__builtin_amdgcn_readlane(hd.x, 63) == call)) break;
        __builtin_amdgcn_s_sleep(SL_HDR);
        asm volatile("" ::: "memory");
        hd = __builtin_amdgcn_raw_buffer_load_b128(rws, hoff, 0, 17);
      }
#pragma unroll
      for (int jj = 0; jj < 8; ++jj) {
        while (__builtin_expect(!__all(((v0[jj].x ^ call) & 1u) == 0), 0)) {
          asm volatile("" ::: "memory");
#pragma unroll
          for (int k = jj; k < jj + LOOK && k < 8; ++k) fetch(k);
          if (late(false)) break;
        }
        const u64 m = (u64)(u32)__builtin_amdgcn_readlane(hd.z, jj) << 32 | (u32)__builtin_amdgcn_readlane(hd.x, jj);
        add(jj, (v0[jj].x << 16 & ~0x10000u) | (__builtin_amdgcn_inverse_ballot_w64(m) ? 0x10000u : 0u), v0[jj]);
        done(jj);
      }
    }
    TSTAMP(10);
    const u32 wm = wave_max(__builtin_bit_cast(u32, mx));
    if (lane == 0)
      __hip_atomic_store((GAS u64*)(ws + OFF_LM + (u32)((J.p * 8 + (J.kind == JOB_F ? J.ja : J.sub)) * 4 + wave) * 64u), ((u64)call << 32) | wm, __ATOMIC_RELAXED, __HIP_MEMORY_SCOPE_AGENT);
  }
  TSTAMP(11);
  if (g & 1) {
#pragma unroll
    for (int m = 0; m < 4; ++m) { f32x4 t = fin[m][0]; fin[m][0] = fin[m][1]; fin[m][1] = t; t = fin[m][2]; fin[m][2] = fin[m][3]; fin[m][3] = t; }
  }
  u32 oo[4];
#pragma unroll
  for (int m = 0; m < 4; ++m) { oo[m] = (u32)(J.sub * 128 + wr * 64 + m * 16 + fr) * 4096u + J.col0 + (u32)(wc * 64 + g * 16); asm volatile("" : "+v"(oo[m])); }
  Polls pq;
  const u32 gmax = gather_max(ws, OFF_AM + (u32)lane * 128u, call, lane, 16, pq);
  TSTAMP(12);
  const float inv = gmax ? 127.0f * __builtin_amdgcn_rcpf(as_f32(gmax)) : 0.f;
#pragma unroll
  for (int m = 0; m < 4; ++m) {
    u32x4 v;
#pragma unroll
    for (int n = 0; n < 4; ++n) {
      f32x4 y = fin[m][n] * inv;
      asm volatile("" : "+v"(y));
      u32 r = 0;
#pragma unroll
      for (int e = 0; e < 4; ++e) r = __builtin_amdgcn_cvt_pk_u8_f32(y[e], e, r);
      v[n] = r;
    }
    if (gmax == ~0u) v = (u32x4){0x80808080u, 0x80808080u, 0x80808080u, 0x80808080u};
    __builtin_amdgcn_raw_buffer_store_b128(v, ro, oo[m], 0, OUT_AUX);
  }
  pq.retire();
  TSTAMP(13);
}

__global__ void __launch_bounds__(512, 2) gemm_rs_q8(const Params P) {
  __shared__ __attribute__((aligned(16))) char lds[5 * 32768];
  const int lane = threadIdx.x & 63, wave = (int)rfl(threadIdx.x >> 6);
  char* ws = P.ws[0];
  if (__builtin_expect(blockIdx.x < NPUSH, 1)) {
    if (wave < 6) pusher(P, lds, ws, lane, wave);
    return;
  }
  const u32 call = draw_call(ws, lane, wave);
  if (blockIdx.x == GRID - 1) {
    if (wave == 0) coordinator(P, ws, lane, call);
    char* dst = P.ws[lane & 7] + OFF_AM + (u32)(wave * 8 + P.rank) * 128u;
    asm volatile("" : "+v"(dst));
    Polls pq;
    const u32 m = gather_max(ws, OFF_LM + (u32)(wave * 64 + lane) * 64u, call, lane, 40, pq);
    if (lane < NR) __hip_atomic_store((GAS u64*)dst, ((u64)call << 32) | m, __ATOMIC_RELAXED, __HIP_MEMORY_SCOPE_SYSTEM);
    pq.retire();
    return;
  }
  const u32 jobs = P.jobs[blockIdx.x];
  if (wave >= 4) loader(P, lds, ws, lane, wave, call, jobs);
  else computer(P, lds, ws, lane, wave, call, jobs);
}

#define LAG 12
void build_schedule(Params& P) {
#define RAMP 4
#define SHIFT2 2
#define ORD 3
  static const int orders[4][32] = {
    {0, 1, 2, 5, 6, 7, 8, 16, 17, 18, 19, 9, 10, 11, 12, 20, 21, 22, 23, 24, 25, 26, 27, 3, 4, 28, 29, 30, 31, 13, 14, 15},
    {0, 1, 2, 16, 5, 6, 17, 7, 8, 18, 19, 9, 10, 11, 12, 20, 21, 22, 23, 24, 25, 4, 3, 26, 27, 28, 29, 30, 31, 13, 14, 15},
    {0, 1, 2, 16, 17, 5, 6, 18, 19, 7, 8, 20, 21, 9, 10, 22, 23, 11, 12, 24, 25, 4, 3, 26, 27, 28, 29, 30, 31, 13, 14, 15},
    {0, 1, 2, 16, 17, 5, 18, 6, 19, 7, 20, 8, 21, 22, 23, 9, 10, 11, 12, 24, 25, 4, 3, 26, 27, 28, 29, 30, 31, 13, 14, 15}};
  const int* order = orders[ORD];
  int pos[32];
  for (int i = 0; i < 32; ++i) pos[order[i]] = i;
  auto ready = [&](int t, int p) { return pos[t] > pos[16 + p] ? pos[t] : pos[16 + p]; };
  for (int d = 0; d < NR; ++d) {
    unsigned char list[MAXU]; int n = 0;
    for (int i = 0; i < MAXU; ++i) list[i] = 0;
    if (d >= 2)
      for (int i = 0, nh1 = 0, lag = ((2 * (d - 2) + 1) * LAG + 10) / 20; i < 64; ++i) {
        auto put = [&](int k, int h) { const int c = order[k]; if (!(c < 16 && tile_dist(c) == d)) list[n++] = (unsigned char)(c * 2 + h); };
        if (i < 32) put(i, 0);
        for (; nh1 < 32 && nh1 + (nh1 * RAMP / 8 < lag ? nh1 * RAMP / 8 : lag) <= i; ++nh1) put(nh1, 1);
      }
    P.ucount[d] = (u32)n;
    for (int i = 0; i < MAXU / 4; ++i) P.units[d][i] = list[4 * i] | list[4 * i + 1] << 8 | list[4 * i + 2] << 16 | (u32)list[4 * i + 3] << 24;
  }
  for (int b = 0; b < NPUSH; ++b)
    for (int w = 0; w < 8; ++w) {
      const int d = push_link(b, w);
      auto code = [&](int i) { return (P.units[d][i >> 4] >> ((i & 12) * 2)) & 0xffu; };
      P.first[b][2 * w] = slice_src(code(b), (u32)b & 3u, P.rank ^ d) | code(b) >> 5;
      P.first[b][2 * w + 1] = code(b) | code(b + NPUSH) << 8 | (u32)(P.rank ^ d) << 16;
    }
  static const int grp[16] = {0, 1, 0, 2, 1, 1, 0, 1, 0, 1, 0, 1, 0, 2, 2, 2};
  for (int i = 0; i < GRID; ++i) P.jobs[i] = 0;
  for (int x = 0; x < 8; ++x) {
    int ht[40], hp[40], n = 0;
    for (int r = 0; r < 32; ++r)
      for (int p = 0; p < 16; ++p)
        for (int t = 0; t < 16; ++t) {
          int gsel = grp[t] == 2 ? (p >> 2) & 1 : grp[t];
          if (t == 15 && p < 8) gsel = 0;
          if (gsel == (x >> 2) && ((p + (t == 2 ? SHIFT2 : 0)) & 3) == (x & 3) && ready(t, p) == r) { ht[n] = t; hp[n] = p; ++n; }
        }
    auto enc = [&](int i) { return (u32)(((ht[i] >= 13 ? JOB_O : JOB_S) << 8) | (ht[i] << 4) | hp[i]); };
    const int nw = 32 - NPUSH / 8 - (x == 7);
    static const int fsub[10] = {4, 6, 4, 6, 3, 4, 6, 4, 6, 3}, fpan[10] = {0, 0, 4, 4, 0, 8, 8, 12, 12, 8};
    int f = 0, extra = nw;
    for (int s = 0; s < nw; ++s) {
      u32 jb = enc(s);
      if (ht[s] >= 3 && ht[s] < 13 && f < 10) {
        jb = jb << 10 | (u32)((JOB_F << 8) | ((fsub[f] == 3 ? 3 : fsub[f] + (x >> 2)) << 4) | (fpan[f] + (fsub[f] == 3 ? x : x & 3))); ++f;
      } else if (ht[s] < 13 && extra < n) { jb = (u32)((JOB_F << 8) | ((ht[extra] - 13) << 4) | hp[extra]) | jb << 10 | enc(extra) << 20; ++extra; }
      P.jobs[NPUSH + s * 8 + x] = jb;
    }
  }
}

}

extern "C" void kernel_launch(int rank, int nranks, ncclComm_t comm, hipStream_t stream,
                              void* const* d_in, const int* in_sizes, int n_in,
                              void* d_out, int out_size,
                              const int* in_dtypes, int out_dtype,
                              void* const* d_out_all, void* const* d_ws_all, size_t ws_bytes) {
  (void)comm; (void)in_dtypes; (void)out_dtype; (void)d_out_all;
  if (nranks != NR || (unsigned)rank >= (unsigned)NR || n_in != 2 || ws_bytes < WS_USED || (u32)out_size != OUT_BYTES ||
      (u32)in_sizes[0] * 2u != X_BYTES || (u32)in_sizes[1] * 2u != W_BYTES) {
    fprintf(stderr, "unexpected problem shape\n");
    return;
  }
  Params P;
  P.x = (const char*)d_in[0]; P.w = (const char*)d_in[1]; P.out = (char*)d_out;
  for (int d = 0; d < NR; ++d) P.ws[d] = (char*)d_ws_all[rank ^ d];
  P.rank = rank; P.pad = 0;
  build_schedule(P);
  hipLaunchKernelGGL(gemm_rs_q8, dim3(GRID), dim3(512), 0, stream, P);
  HIP_CHECK(hipGetLastError());
}
```

```cpp
#include <hip/hip_runtime.h>
#include <rccl/rccl.h>
#include <cstdio>
#include <cstdint>

#define HIP_CHECK(expr) do { hipError_t _e = (expr); if (_e != hipSuccess) { \
    fprintf(stderr, "HIP error %s at %s:%d\n", hipGetErrorString(_e), __FILE__, __LINE__); \
    return; } } while (0)

namespace {

typedef unsigned u32;
typedef unsigned long long u64;
typedef u32 u32x4 __attribute__((ext_vector_type(4)));
typedef u32 u32x2 __attribute__((ext_vector_type(2)));
typedef float f32x4 __attribute__((ext_vector_type(4)));
typedef short s16x4 __attribute__((ext_vector_type(4)));
typedef short s16x8 __attribute__((ext_vector_type(8)));
typedef __bf16 bf16x8 __attribute__((ext_vector_type(8)));
#define LAS __attribute__((address_space(3)))
#define GAS __attribute__((address_space(1)))

constexpr int NR = 8;
#define NPUSH 24
constexpr int GRID = 256;
#define RELAY 4
constexpr int MAXU = 72;
constexpr u64 DEADLINE_TICKS = 100000000ull;

constexpr u32 X_BYTES = 4096u * 512u * 2u, W_BYTES = 512u * 4096u * 2u, OUT_BYTES = 512u * 4096u;
constexpr u32 OFF_CNT = 0;
constexpr u32 OFF_TMO = 16u << 10;
constexpr u32 OFF_TK = (16u << 10) + 128u;
constexpr u32 OFF_TP = 18u << 10;
constexpr u32 OFF_T0 = 19u << 10;
constexpr u32 OFF_XF = 20u << 10;
constexpr u32 OFF_WF = 24u << 10;
constexpr u32 OFF_TR = 28u << 10;
constexpr u32 OFF_GO = 29u << 10;
constexpr u32 OFF_AM = 32u << 10;
constexpr u32 OFF_LM = 64u << 10;
constexpr u32 OFF_TOK = 128u << 10;
constexpr u32 OFF_PF = 192u << 10;
constexpr u32 OFF_DBG = 512u << 10;
constexpr u32 OFF_XS = 2u << 20;
constexpr u32 OFF_WS = 8u << 20;
constexpr u32 OFF_IN = 14u << 20;
constexpr u32 WS_USED = OFF_IN + 208u * 32768u;
#define TAU_X 224
#define TAU 232
#define TAU_C 232
#define PD 2
#define T_BOOK 70
#define T_REST 760
#define OUT_AUX 0
#define LOOK 3
#define SL_LONG 32
#define SL_GATE 6
#define SL_AM 3
#define SL_HDR 16
#define WDELAY 150
#define SL_AML 16
#define T_FLY 80
#define SL_GO 16
#define LASTW 3
#define Q_STEP 712
#define Q_CRED 700
#define Q_PAUSE 900
#define Q_FROM 0
#define Q_ALL 0
#define CATCH 13
#define R_END 16
#define PURE 4
#define CAP 3
constexpr u32 PANEL_B = 6u * 65536u;

enum { JOB_NONE = 0, JOB_S = 1, JOB_O = 2, JOB_F = 3 };

struct Params {
  const char* x; const char* w; char* out;
  char* ws[NR];
  int rank; int pad;
  u32 jobs[GRID];
  u32 first[NPUSH][16];
  u32 ucount[NR];
  u32 units[NR][MAXU / 4];
  u64 korder[2][8];
  u32 relayed;
};

__host__ __device__ inline int tile_dist(int t) { return t < 3 ? 1 : t < 5 ? t - 1 : t < 13 ? 4 + ((t - 5) & 3) : 0; }
__host__ __device__ inline int tile_sub(int t, int a) { return t < 3 ? t : t < 5 ? 3 : t < 13 ? (a & 2) + ((t - 5) >> 2) : t - 13; }
__host__ __device__ inline int box_base(int d) { return d == 1 ? 0 : d < 4 ? 48 + 16 * (d - 2) : 80 + 32 * (d - 4); }
__host__ __device__ inline int box_cnt(int d) { return d == 1 ? 48 : d < 4 ? 16 : 32; }
__host__ __device__ inline int box_idx(int t, int p) { return t < 3 ? p * 3 + t : t < 5 ? p : p * 2 + ((t - 5) >> 2); }

__host__ __device__ inline u32 link_units(int d) { return d < 4 ? 62u + 2u * RELAY : 60u - RELAY; }
__host__ __device__ inline int push_link(int block, int wave) { return 2 + (wave + (block >> 3) * 2 + block) % 6; }
__host__ __device__ inline u32 k_half(int q, int d) { return (u32)(q ^ (d >= 4 ? (q >> 1) & (q ^ d) : 0)) & 1u; }
__host__ __device__ inline u32 slice_src(u32 code, u32 sl, int q, int d) {
  const u32 h = code & 1u, kh = k_half(q, d);
  if (code < 32) {
    const int t = code >> 1;
    return ((u32)((q ^ tile_dist(t)) * 512 + tile_sub(t, q) * 128) + (sl & 1u) * 64u) * 1024u + (kh * 256u + h * 128u + (sl >> 1) * 64u) * 2u;
  }
  return (kh * 256u + h * 128u + sl * 32u) * 8192u + ((u32)(q >> 2) * 2048u + ((code - 32u) >> 1) * 128u) * 2u;
}

__device__ __forceinline__ u32 rfl(u32 v) { return __builtin_amdgcn_readfirstlane(v); }
__device__ __forceinline__ __amdgpu_buffer_rsrc_t mk_rsrc(const void* p, u32 bytes) {
  return __builtin_amdgcn_make_buffer_rsrc((void*)p, 0, bytes, 0x00020000);
}
__device__ __forceinline__ u32x4 mk_rsrc4(const void* p, u32 bytes) {
  u64 a = (u64)p; u32x4 r; r.x = rfl((u32)a); r.y = rfl((u32)(a >> 32)) & 0xffffu; r.z = bytes; r.w = 0x00020000u; return r;
}
__device__ __forceinline__ u32 ld_sys(const char* p) { return __hip_atomic_load((GAS const u32*)p, __ATOMIC_RELAXED, __HIP_MEMORY_SCOPE_SYSTEM); }
__device__ __forceinline__ void st_sys(char* p, u32 v) { __hip_atomic_store((GAS u32*)p, v, __ATOMIC_RELAXED, __HIP_MEMORY_SCOPE_SYSTEM); }
__device__ __forceinline__ void st_agent(char* p, u32 v) { __hip_atomic_store((GAS u32*)p, v, __ATOMIC_RELAXED, __HIP_MEMORY_SCOPE_AGENT); }
#define WAIT_VM(n) asm volatile("s_waitcnt vmcnt(" #n ")" ::: "memory")
#define BARRIER() do { asm volatile("" ::: "memory"); __builtin_amdgcn_s_waitcnt(0xC07F);   __builtin_amdgcn_s_barrier(); asm volatile("" ::: "memory"); } while (0)

#define TSTAMP(i) do {} while (0)
#define STAMP_AT(i) do {} while (0)

template <int SL> __device__ __forceinline__ void wave_wait(char* ws, u32 off, bool care, u32 call, int lane) {
  if (lane == 63) { off = OFF_TMO; }
  else if (!care) { off = OFF_TMO; }
  const u64 t0 = __builtin_amdgcn_s_memrealtime();
  for (;;) {
    const u32 v = ld_sys(ws + off);
    const bool ok = (lane == 63 || !care) ? true : (v >= call);
    if (__all(ok)) break;
    const bool dead = __any(lane == 63 && v == call);
    if (dead || __builtin_amdgcn_s_memrealtime() - t0 > DEADLINE_TICKS) {
      if (lane == 0) st_sys(ws + OFF_TMO, call);
      WAIT_VM(0);
      break;
    }
    __builtin_amdgcn_s_sleep(SL);
  }
}

__device__ __forceinline__ int wave_poll4(char* ws, __amdgpu_buffer_rsrc_t rws, u32 off, bool care, int have, bool block, u32 call, int lane) {
  if (!care) off = OFF_TMO;
  const u64 t0 = __builtin_amdgcn_s_memrealtime();
  for (;;) {
    const u32x4 v = __builtin_amdgcn_raw_buffer_load_b128(rws, off, 0, 17);
    asm volatile("" ::: "memory");
    const u32 m = (u32)__ballot(care && v.x >= call && v.y >= call && v.z >= call && v.w >= call);
    const int n = __builtin_ctz(~(m & (m >> 16)));
    if (n > have || !block) return n;
    const bool dead = __any(!care && v.x == call);
    if (dead || __builtin_amdgcn_s_memrealtime() - t0 > DEADLINE_TICKS) {
      if (lane == 0) st_sys(ws + OFF_TMO, call);
      WAIT_VM(0);
      return have + 1;
    }
    if (have == 0) __builtin_amdgcn_s_sleep(SL_LONG); else __builtin_amdgcn_s_sleep(SL_GATE);
  }
}
#define DMA_SC " sc0 sc1"

typedef __bf16 bf16x2 __attribute__((ext_vector_type(2)));
__device__ __forceinline__ u32 pack_bf16(float lo, float hi) { bf16x2 r; r.x = (__bf16)lo; r.y = (__bf16)hi; return __builtin_bit_cast(u32, r); }
__device__ __forceinline__ float as_f32(u32 v) { return __builtin_bit_cast(float, v); }
__device__ __forceinline__ float bf_lo(u32 v) { return __builtin_bit_cast(float, v << 16); }
__device__ __forceinline__ float bf_hi(u32 v) { return __builtin_bit_cast(float, v & 0xffff0000u); }
__device__ __forceinline__ u32 wave_max(u32 m) {
  m = max(m, (u32)__builtin_amdgcn_update_dpp(0, (int)m, 0x111, 0xf, 0xf, false));
  m = max(m, (u32)__builtin_amdgcn_update_dpp(0, (int)m, 0x112, 0xf, 0xf, false));
  m = max(m, (u32)__builtin_amdgcn_update_dpp(0, (int)m, 0x114, 0xf, 0xf, false));
  m = max(m, (u32)__builtin_amdgcn_update_dpp(0, (int)m, 0x118, 0xf, 0xf, false));
  const u32 a = max((u32)__builtin_amdgcn_readlane(m, 15), (u32)__builtin_amdgcn_readlane(m, 31));
  const u32 b = max((u32)__builtin_amdgcn_readlane(m, 47), (u32)__builtin_amdgcn_readlane(m, 63));
  return max(a, b);
}

__device__ __forceinline__ void dma4(u32 lds_dst, u32 v0, u32 v1, u32 v2, u32 v3, u32x4 rs, u32 so, u32 st) {
  u32 keep, t;
  asm volatile(
      "s_mov_b32 %0, m0\n\t"
      "s_mov_b32 m0, %2\n\ts_add_u32 %1, %8, %9\n\ts_nop 1\n\tbuffer_load_dwordx4 %3, %7, %8 offen" DMA_SC " lds\n\t"
      "s_add_u32 m0, %2, 0x400\n\ts_nop 0\n\tbuffer_load_dwordx4 %4, %7, %1 offen" DMA_SC " lds\n\t"
      "s_add_u32 m0, %2, 0x800\n\ts_add_u32 %1, %1, %9\n\tbuffer_load_dwordx4 %5, %7, %1 offen" DMA_SC " lds\n\t"
      "s_add_u32 m0, %2, 0xc00\n\ts_add_u32 %1, %1, %9\n\tbuffer_load_dwordx4 %6, %7, %1 offen" DMA_SC " lds\n\t"
      "s_mov_b32 m0, %0"
      : "=&s"(keep), "=&s"(t)
      : "s"(lds_dst), "v"(v0), "v"(v1), "v"(v2), "v"(v3), "s"(rs), "s"(so), "s"(st)
      : "memory", "scc");
}

__device__ __forceinline__ void dma8_lin(u32 lds_dst, u32 v, u32x4 rs, u32 sx, u32 sw) {
  u32 keep;
  asm volatile(
      "s_mov_b32 %0, m0\n\ts_mov_b32 m0, %1\n\ts_nop 2\n\t"
      "buffer_load_dwordx4 %2, %3, %4 offen" DMA_SC " lds\n\tbuffer_load_dwordx4 %2, %3, %4 offen offset:1024" DMA_SC " lds\n\t"
      "buffer_load_dwordx4 %2, %3, %4 offen offset:2048" DMA_SC " lds\n\tbuffer_load_dwordx4 %2, %3, %4 offen offset:3072" DMA_SC " lds\n\t"
      "s_add_u32 m0, %1, 0x4000\n\ts_nop 0\n\t"
      "buffer_load_dwordx4 %2, %3, %5 offen" DMA_SC " lds\n\tbuffer_load_dwordx4 %2, %3, %5 offen offset:1024" DMA_SC " lds\n\t"
      "buffer_load_dwordx4 %2, %3, %5 offen offset:2048" DMA_SC " lds\n\tbuffer_load_dwordx4 %2, %3, %5 offen offset:3072" DMA_SC " lds\n\t"
      "s_mov_b32 m0, %0"
      : "=&s"(keep) : "s"(lds_dst), "v"(v), "s"(rs), "s"(sx), "s"(sw) : "memory", "scc");
}

#define WIN 3
template <int N> __device__ __forceinline__ void wait_vm() { asm volatile("s_waitcnt vmcnt(%0)" :: "n"(N) : "memory"); }
template <int IMM> __device__ __forceinline__ void st16(u32x4 v, u32x4 rs, u32 voff, u32 soff) {
  asm volatile("buffer_store_dwordx4 %0, %1, %2, %3 offen offset:%4 sc0 sc1\n\ts_nop 1" :: "v"(v), "v"(voff), "s"(rs), "s"(soff), "n"(IMM) : "memory");
}
__device__ __forceinline__ void draw_async(u32& v, u32x4 rl, u32 off) {
  u64 ex; v = 1;
  asm volatile("s_mov_b64 %1, exec\n\ts_mov_b64 exec, 1\n\tbuffer_atomic_add %0, off, %2, %3 sc0\n\ts_mov_b64 exec, %1"
               : "+v"(v), "=&s"(ex) : "s"(rl), "s"(off) : "memory");
}
__device__ __forceinline__ void book(u64& tk, u32x4 rl, u32 off) {
  u64 ex; const u64 lo = (__builtin_amdgcn_s_memrealtime() + T_BOOK) * 16u;
  asm volatile("s_mov_b64 %1, exec\n\ts_mov_b64 exec, 1\n\tbuffer_atomic_umax_x2 %2, off, %3, %4\n\tbuffer_atomic_add_x2 %0, off, %3, %4 sc0\n\ts_mov_b64 exec, %1"
               : "+v"(tk), "=&s"(ex) : "v"(lo), "s"(rl), "s"(off) : "memory");
}
__device__ __forceinline__ void sleep_until(u32 due) {
  for (;;) {
    const u32 left = due - (u32)__builtin_amdgcn_s_memrealtime() - 1u;
    if (left >= 10000u) break;
    if (left > 200u) __builtin_amdgcn_s_sleep(48); else if (left > 40u) __builtin_amdgcn_s_sleep(8); else __builtin_amdgcn_s_sleep(1);
  }
}
struct Slice { u32 src, dst, flag, kind, dep; };
__device__ __forceinline__ u32 draw_call(char* ws, int lane, int wave) {
  u32 call = 0;
  if (lane == 0) call = __hip_atomic_fetch_add((GAS u32*)(ws + OFF_CNT + blockIdx.x * 64 + wave * 4), 1u, __ATOMIC_RELAXED, __HIP_MEMORY_SCOPE_AGENT) + 1u;
  return rfl(call);
}
__device__ __forceinline__ void pusher(const Params& P, char* lds, char* ws, int lane, int wave) {
  const u32 vx0 = (u32)(lane >> 3) * 1024u + (u32)(((lane & 7) ^ (lane >> 4)) * 16), vx1 = vx0 ^ 64u;
  const u32 vw0 = (u32)(lane >> 4) * 8192u + (u32)(((lane & 15) ^ (lane >> 4)) * 16), vw1 = vw0 ^ 128u;
  const int d = push_link(blockIdx.x, wave);
  u32 f0 = P.first[blockIdx.x][2 * wave], f1 = P.first[blockIdx.x][2 * wave + 1]; const char* xp = P.x; const char* wp = P.w;
  char* pw = P.ws[d];
  asm volatile("" : "+s"(f0), "+s"(f1), "+s"(xp), "+s"(wp), "+s"(pw), "+s"(ws));
  const u32x4 rx = mk_rsrc4(xp, X_BYTES), rw = mk_rsrc4(wp, W_BYTES), rl = mk_rsrc4(ws, WS_USED);
  const u32 vp = (u32)lane * 16u;
  const u32 ring0 = (u32)(size_t)(LAS char*)lds + (u32)wave * 16384u;
  auto load_all = [&](const Slice& c, u32 buf) {
    const u32 dst = ring0 + buf * 8192u;
    if (c.kind == 0) { dma4(dst, vx0, vx1, vx0, vx1, rx, c.src, 8192u); dma4(dst + 4096u, vx0, vx1, vx0, vx1, rx, c.src + 32768u, 8192u); }
    else if (c.kind == 1) { dma4(dst, vw0, vw0, vw1, vw1, rw, c.src, 32768u); dma4(dst + 4096u, vw0, vw0, vw1, vw1, rw, c.src + 131072u, 32768u); }
    else { dma4(dst, vp, vp, vp, vp, rl, c.src, 1024u); dma4(dst + 4096u, vp, vp, vp, vp, rl, c.src + 4096u, 1024u); }
  };
  { Slice c; c.src = f0 & ~1u; c.kind = f0 & 1u; c.dst = c.flag = c.dep = 0; load_all(c, 0); }
  u32 cnt; draw_async(cnt, rl, OFF_CNT + blockIdx.x * 64u + (u32)wave * 4u);
  const int q = (int)(f1 >> 16);
  const u32 ns = link_units(d) * 4u;
  auto decode = [&](u32 sidx, u32 code) -> Slice {
    Slice r;
    const u32 sl = sidx & 3u, h = code & 1u;
    if (code >= 64) {
      const u32 ds = code >> 4, pp = code & 15u;
      r.kind = 2;
      r.src = OFF_WS + pp * PANEL_B + (ds - 2u) * 65536u + 32768u + sl * 8192u;
      r.dep = OFF_WF + pp * 256u + (ds - 2u) * 32u + 16u + sl * 4u;
      r.dst = OFF_WS + pp * PANEL_B + ((ds ^ (u32)d) - 2u) * 65536u + 32768u + sl * 8192u;
      r.flag = OFF_WF + pp * 256u + ((ds ^ (u32)d) - 2u) * 32u + 16u + sl * 4u;
      return r;
    }
    r.kind = code >> 5; r.src = slice_src(code, sl, q, d); r.dep = OFF_TMO;
    r.dst = (code < 32 ? OFF_XS : OFF_WS) + ((code >> 1) & 15u) * PANEL_B + (u32)(d - 2) * 65536u + h * 32768u + sl * 8192u;
    r.flag = (code < 32 ? OFF_XF : OFF_WF) + ((code >> 1) & 15u) * 256u + (u32)(d - 2) * 32u + h * 16u + sl * 4u;
    return r;
  };
  const u32x4 rd = mk_rsrc4(pw, WS_USED);
  LAS char* ring = (LAS char*)lds + wave * 16384;
  u32 nxt_i = NPUSH + blockIdx.x;
  Slice cur = decode(blockIdx.x, f1 & 0xffu), nxt = decode(nxt_i, (f1 >> 8) & 0xffu);
  u32 buf = 0;
  asm volatile("s_waitcnt vmcnt(0)" : "+v"(cnt), "+s"(cur.dst), "+s"(nxt.src) :: "memory");
  const u32 call = rfl(cnt) + 1u;
  const u32 tk_base = (call - 1u) * (ns - NPUSH);
  const u32 tk_off = OFF_TK + (u32)d * 128u;
  u32 pend = 0;
  auto publish = [&]() {
    if (lane == 0) {
      st_sys(pw + pend, call);
    }
  };
  bool quiet = false;
  u32 due = 0;
  auto send = [&](int k) {
    const u32x4 v = *(LAS const u32x4*)(ring + buf * 8192u + k * 1024 + lane * 16);
    if (quiet) {
      u32 now = (u32)__builtin_amdgcn_s_memrealtime() * 16u;
      while ((int)(due - now) > 0) { __builtin_amdgcn_s_sleep(1); now = (u32)__builtin_amdgcn_s_memrealtime() * 16u; }
      due = ((int)(now - Q_CRED - due) > 0 ? now - Q_CRED : due) + Q_STEP;
    }
    st16<0>(v, rd, vp, cur.dst + (u32)k * 1024u);
  };
  for (;;) {
    u32 tk;
    send(0);
    load_all(nxt, buf ^ 1u);
    draw_async(tk, rl, tk_off);
    const u32 t_draw = (u32)__builtin_amdgcn_s_memrealtime();
    Slice nn = nxt;
    u32 dv;
#pragma unroll
    for (int k = 0; k < 8; ++k) {
      if (k) send(k);
      if (k == WIN + 1) {
        asm volatile("" : "+v"(tk));
        nxt_i = rfl(tk) - tk_base;
        if (nxt_i == ns - 40u && lane == 0)
          __hip_atomic_store((GAS u64*)(ws + OFF_T0 + (u32)d * 128u), ((u64)call << 32) | (t_draw + T_REST), __ATOMIC_RELAXED, __HIP_MEMORY_SCOPE_AGENT);
        nxt_i += 2u * NPUSH;
        if (nxt_i < ns) nn = decode(nxt_i, (P.units[d][nxt_i >> 4] >> ((nxt_i & 12u) * 2u)) & 0xffu);
        asm volatile("buffer_load_dword %0, off, %1, %2 sc0 sc1" : "=v"(dv) : "s"(rl), "s"(nn.dep) : "memory");
      }
      if (k < WIN) wait_vm<WIN - 1 + 9>(); else wait_vm<WIN - 1>();
      if (k == WIN - 2 && pend) publish();
    }
    pend = cur.flag;
    cur = nxt; nxt = nn; buf ^= 1u;
    asm volatile("" : "+v"(dv));
    if (!quiet && nxt_i >= link_units(4) * 4u + Q_FROM && (d < 4 || Q_ALL)) { quiet = true; due = (u32)__builtin_amdgcn_s_memrealtime() * 16u + Q_PAUSE; }
    if (nxt_i >= ns) break;
    if (nxt.kind == 2 && rfl(dv) < call) wave_wait<2>(ws, nxt.dep, lane == 0, call, lane);
  }
#pragma unroll
  for (int k = 0; k < 8; ++k) {
    send(k);
    wait_vm<LASTW - 1>();
    if (k == LASTW - 2) publish();
  }
  pend = cur.flag;
  WAIT_VM(0);
  publish();
}

__device__ __forceinline__ void coordinator(const Params& P, char* ws, int lane, u32 call) {
  const u32 lo = (u32)(lane & 7) * 128u, off = (lane & 8 ? OFF_TR : OFF_T0) + lo;
  char* peer = P.ws[lane & 7];
  asm volatile("" : "+v"(peer));
  const u64 t0 = __builtin_amdgcn_s_memrealtime();
  if (lane >= 2 && lane < 8) st_sys(peer + OFF_TR + lo + 4u, call);
  auto rl = [&](u32 v, int i) { return (u32)__builtin_amdgcn_readlane(v, i); };
  bool seen = (lane & 7) < 2 || lane >= 16;
  u32 t = 0, sent = 0;
  for (;;) {
    u64 m = 0;
    if (!seen) m = __hip_atomic_load((GAS const u64*)(ws + off), __ATOMIC_RELAXED, __HIP_MEMORY_SCOPE_SYSTEM);
    asm volatile("" : "+v"(m));
    const u64 now = __builtin_amdgcn_s_memrealtime();
    if (!seen && ((u32)(m >> 32) >= call || now - t0 > DEADLINE_TICKS)) {
      t = lane & 8 ? (u32)max((int)(u32)(now - t0) - T_FLY, 0) : (u32)m;
      seen = true;
    }
    const u32 sm = (u32)__ballot(seen);
    const u32 rdy = ((sm & 0xf0f0u) == 0xf0f0u ? 1u : 0u) | ((sm & 0x0c0cu) == 0x0c0cu ? 2u : 0u);
    if (rdy & ~sent) {
      const u32 tf = rl(t, 4) + (u32)((int)(rl(t, 5) + rl(t, 6) + rl(t, 7) - 3u * rl(t, 4)) >> 2), tn = rl(t, 2) + (u32)((int)(rl(t, 3) - rl(t, 2)) >> 1);
      const u32 lag = max(t, (u32)__builtin_amdgcn_update_dpp(0, (int)t, 0xb1, 0xf, 0xf, false));
      if (lane == 10 ? rdy & 2u : (lane == 12 || lane == 14) && (rdy & 1u))
        __hip_atomic_store((GAS u64*)(ws + OFF_GO + lo / 2u), ((u64)call << 32) | ((lane >= 12 ? tf : tn) + lag), __ATOMIC_RELAXED, __HIP_MEMORY_SCOPE_AGENT);
      sent = rdy;
      if (sent == 3u) return;
    }
    __builtin_amdgcn_s_sleep(1);
  }
}

constexpr int NST = 5;

struct Job { int kind, ja, p, dO, sub, nh, nkt; u32 col0; };
__device__ __forceinline__ Job decode_job(u32 code, int a) {
  Job j; j.kind = (code >> 8) & 3; j.ja = (code >> 4) & 15; j.p = code & 15;
  int chalf;
  if (j.kind == JOB_F) { j.dO = 0; j.sub = j.ja & 3; chalf = (a >> 2) ^ (j.ja >> 2); }
  else { j.dO = tile_dist(j.ja); j.sub = tile_sub(j.ja, a); chalf = a >> 2; }
  j.col0 = (u32)(chalf * 2048 + j.p * 128);
  j.nh = j.kind == JOB_F ? 0 : j.dO >= 2 ? 10 : 12;
  j.nkt = 8 + 2 * j.nh;
  return j;
}

struct Polls { u32x2 va, vb; u32 ta, tb;
  __device__ __forceinline__ void retire() { asm volatile("" :: "v"(va), "v"(vb), "v"(ta), "v"(tb)); } };
__device__ __forceinline__ u32 gather_max(char* ws, u32 off, u32 call, int lane, int lazy, Polls& q) {
  u32x2& va = q.va; u32x2& vb = q.vb; u32& ta = q.ta; u32& tb = q.tb;
  const __amdgpu_buffer_rsrc_t rws = mk_rsrc(ws, WS_USED);
  u32 gm = 0, tmo_v;
  const u32 toff = lane ? 0x80000000u : OFF_TMO;
  const u64 t0 = __builtin_amdgcn_s_memrealtime();
  va = __builtin_amdgcn_raw_buffer_load_b64(rws, off, 0, 17);
  ta = __builtin_amdgcn_raw_buffer_load_b32(rws, toff, 0, 17);
  __builtin_amdgcn_s_sleep(SL_AM);
  for (;;) {
    asm volatile("" ::: "memory");
    vb = __builtin_amdgcn_raw_buffer_load_b64(rws, off, 0, 17); tb = __builtin_amdgcn_raw_buffer_load_b32(rws, toff, 0, 17);
    asm volatile("" ::: "memory");
    if (va.y >= call) { gm = va.x; off = 0x80000000u; }
    tmo_v = rfl(ta);
    if (__all(off == 0x80000000u)) break;
    va = __builtin_amdgcn_raw_buffer_load_b64(rws, off, 0, 17); ta = __builtin_amdgcn_raw_buffer_load_b32(rws, toff, 0, 17);
    asm volatile("" ::: "memory");
    if (vb.y >= call) { gm = vb.x; off = 0x80000000u; }
    tmo_v = rfl(tb);
    const u64 missing = __ballot(off != 0x80000000u);
    if (!missing) break;
    if (tmo_v == call || __builtin_amdgcn_s_memrealtime() - t0 > DEADLINE_TICKS) {
      if (lane == 0) st_sys(ws + OFF_TMO, call);
      WAIT_VM(0);
      tmo_v = call;
      break;
    }
    if (__builtin_popcountll(missing) > lazy) __builtin_amdgcn_s_sleep(SL_AML);
  }
  gm = wave_max(gm);
  return tmo_v == call ? ~0u : gm;
}

__device__ __forceinline__ void loader(const Params& P, char* lds, char* ws, int lane, int wave, u32 call, u32 jobs) {
  const int a = P.rank;
  const __amdgpu_buffer_rsrc_t rws = mk_rsrc(ws, WS_USED);
  const u32 l4 = (u32)(wave - 4) * 4u;
  const u32 vx0 = (u32)(lane >> 3) * 1024u + (u32)(((lane & 7) ^ (lane >> 4)) * 16), vx1 = vx0 ^ 64u;
  const u32 vw0 = (u32)(lane >> 4) * 8192u + (u32)(((lane & 15) ^ (lane >> 4)) * 16), vw1 = vw0 ^ 128u;
  const u32 vp = (u32)lane * 16u;
  const u32x4 rx = mk_rsrc4(P.x, X_BYTES), rw = mk_rsrc4(P.w, W_BYTES), rl = mk_rsrc4(ws, WS_USED);
  const u32 lds0 = (u32)(size_t)(LAS char*)lds + l4 * 1024u;
  bool fin = false;
  if (WDELAY && (jobs >> 4) - (JOB_S << 4) >= 3u) sleep_until((u32)__builtin_amdgcn_s_memrealtime() + WDELAY);
  int stamp = 1; (void)stamp;
  for (; (jobs & 0x3ffu) != 0; jobs >>= 10) {
    const Job J = decode_job(jobs & 0x3ffu, a);
    const int nh = J.nh, nkt = J.nkt;
    const u32 row0 = (u32)((a ^ J.dO) * 512 + J.sub * 128);
    const int j0 = fin && J.kind == JOB_O ? 8 : 0;
    fin = fin || J.kind != JOB_S;
    const u64 ko = P.korder[(P.relayed >> J.p) & 1u][J.dO >= 2 ? J.dO - 2 : 6];
    const int fl = lane & 15;
    const bool care = lane < 32 && fl < nh;
    const u32 foff = (lane < 16 ? OFF_XF + (u32)J.ja * 256u : OFF_WF + (u32)J.p * 256u) + ((u32)(ko >> (fl * 4)) & 15u) * 16u;
    const u32 tab = lane < 8 ? (row0 + l4 * 8u) * 1024u + (u32)lane * 128u
                             : OFF_XS + (u32)J.ja * PANEL_B + l4 * 1024u + (u32)(lane & 1) * 16384u + ((u32)(ko >> ((((lane - 8) >> 1) & 15) * 4)) & 15u) * 32768u;
    const u32 dw = OFF_WS - OFF_XS + (u32)(J.p - J.ja) * PANEL_B, w0 = l4 * (4u * 8192u) + J.col0 * 2u;
    int issued = j0, open = 8;
    u32 dst = lds0;
    auto issue = [&]() {
      const u32 ox = __builtin_amdgcn_readlane(tab, issued);
      if (__builtin_expect(issued >= 8, 1)) dma8_lin(dst, vp, rl, ox, ox + dw);
      else {
        dma4(dst, vx0, vx1, vx0, vx1, rx, ox, 8192u);
        dma4(dst + 16384u, vw0, vw0, vw1, vw1, rw, w0 + (u32)issued * (64u * 8192u), 32768u);
      }
      ++issued; dst = dst == lds0 + (NST - 1) * 32768u ? lds0 : dst + 32768u;
    };
    for (int j = j0; j <= nkt; ++j) {
      if (__builtin_expect(issued == j + NST - 2 && issued < open, 1)) {
        issue();
        WAIT_VM(24);
        __builtin_amdgcn_s_barrier();
        continue;
      }
      const int cap = j + NST - (j > j0) < nkt ? j + NST - (j > j0) : nkt;
      if (issued < cap && issued == open) {
        const int got = wave_poll4(ws, rws, foff, care, (open - 8) >> 1, issued == j, call, lane);
        open = max(open, 8 + 2 * got);
      }
      while (issued < cap && issued < open) issue();
      const int after = issued - j - 1;
      if (after >= 4) WAIT_VM(32); else if (after == 3) WAIT_VM(24); else if (after == 2) WAIT_VM(16); else if (after == 1) WAIT_VM(8); else WAIT_VM(0);
      __builtin_amdgcn_s_barrier();
    }
    __builtin_amdgcn_s_barrier();
  }
}

__device__ __forceinline__ void computer(const Params& P, char* lds, char* ws, int lane, int wave, u32 call, u32 jobs) {
  const int a = P.rank;
  const int wr = wave >> 1, wc = wave & 1, fr = lane & 15, g = lane >> 4;
  const u32 xa0 = (u32)((wr * 64 + fr) * 128 + ((g ^ (fr >> 1)) * 16));
  const int tq = fr >> 2, tp = fr & 3;
  const u32 wa0 = 16384u + (u32)((8 * g + tq) * 256 + (((8 * wc + 2 * tp) ^ (tq | ((g & 1) << 3))) * 16) + (tp & 1) * 8);
  const u32x4 rl = mk_rsrc4(ws, WS_USED);
  int stamp = 1; (void)stamp;
  f32x4 fin[4][4];
  u32 fjob = 0;
  const u32 vo = (u32)lane * 16u;

  for (; (jobs & 0x3ffu) != 0; jobs >>= 10) {
    const Job J = decode_job(jobs & 0x3ffu, a);
    const bool cont = fjob != 0 && J.kind == JOB_O;
    f32x4 acc[4][4];
#pragma unroll
    for (int m = 0; m < 4; ++m)
#pragma unroll
      for (int n = 0; n < 4; ++n) acc[m][n] = cont ? fin[m][n] : (f32x4){0.f, 0.f, 0.f, 0.f};
    auto readf = [&](bf16x8 (&xf)[4], bf16x8 (&wf)[4], int slot, int ks) {
      LAS char* st = (LAS char*)lds + slot * 32768;
#pragma unroll
      for (int i = 0; i < 4; ++i) {
        const u32 o = (wa0 ^ (u32)((i >> 1) * 16 + (i & 1) * 8)) + (u32)(ks * 8192);
        const s16x4 lo = __builtin_amdgcn_ds_read_tr16_b64_v4i16((LAS s16x4*)(st + o));
        const s16x4 hi = __builtin_amdgcn_ds_read_tr16_b64_v4i16((LAS s16x4*)(st + o + 1024));
        wf[i] = __builtin_bit_cast(bf16x8, __builtin_shufflevector(lo, hi, 0, 1, 2, 3, 4, 5, 6, 7));
        xf[i] = *(LAS const bf16x8*)(st + ((xa0 ^ (u32)(ks * 64)) + (u32)(i * 2048)));
      }
    };
    auto mma = [&](const bf16x8 (&xf)[4], const bf16x8 (&wf)[4]) {
#pragma unroll
      for (int m = 0; m < 4; ++m)
#pragma unroll
        for (int n = 0; n < 4; ++n) acc[m][n] = __builtin_amdgcn_mfma_f32_16x16x32_bf16(wf[n], xf[m], acc[m][n], 0, 0, 0);
    };
#define INTERLEAVE(id) do { _Pragma("unroll") for (int i_ = 0; i_ < 6; ++i_) { __builtin_amdgcn_sched_group_barrier(0x8, 1, id); __builtin_amdgcn_sched_group_barrier(0x100, 2, id); } \
                            __builtin_amdgcn_sched_group_barrier(0x8, 10, id); __builtin_amdgcn_sched_barrier(0); } while (0)
    bf16x8 x0[4], w0[4], x1[4], w1[4];
    int slot = 0;
    BARRIER();
    readf(x0, w0, 0, 0);
    for (int kt = cont ? 8 : 0; kt < J.nkt; ++kt) {
      __builtin_amdgcn_s_waitcnt(0xC07F);
      __builtin_amdgcn_sched_barrier(0);
      readf(x1, w1, slot, 1); mma(x0, w0);
      INTERLEAVE(0);
      slot = slot == NST - 1 ? 0 : slot + 1;
      BARRIER();
      __builtin_amdgcn_sched_barrier(0);
      readf(x0, w0, slot, 0); mma(x1, w1);
      INTERLEAVE(1);
    }
    BARRIER();
    STAMP_AT(stamp++);

    if (J.kind == JOB_S) {
      const u32 slot = (u32)(box_base(J.dO) + box_idx(J.ja, J.p));
      u32x4 v[8];
#pragma unroll
      for (int jj = 0; jj < 8; ++jj) {
        const f32x4 u0 = acc[jj >> 1][(jj & 1) * 2], u1 = acc[jj >> 1][(jj & 1) * 2 + 1];
        v[jj].x = pack_bf16(u0[0], u0[1]); v[jj].y = pack_bf16(u0[2], u0[3]); v[jj].z = pack_bf16(u1[0], u1[1]); v[jj].w = pack_bf16(u1[2], u1[3]);
      }
      const u32 so = OFF_IN + slot * 32768u + (u32)wave * 8192u, fo = OFF_PF + slot * 512u + (u32)wave * 128u;
      if (J.dO >= 2) {
#pragma unroll
        for (int jj = 0; jj < 8; ++jj) st16<0>(v[jj], rl, vo, so + (u32)(jj * 1024));
        WAIT_VM(0);
        if (lane == 0) st_sys(P.ws[J.dO] + fo, call);
        STAMP_AT(stamp++);
      } else {
        u64 bk = TAU;
        book(bk, rl, OFF_TP);
        u32x4 hd; hd.x = 0; hd.y = call; hd.z = 0; hd.w = call;
#pragma unroll
        for (int jj = 0; jj < 8; ++jj) {
          const u64 mk = __ballot(v[jj].x & 1u);
          hd.x = lane == jj ? (u32)mk : hd.x; hd.z = lane == jj ? (u32)(mk >> 32) : hd.z;
          v[jj].x = (v[jj].x & ~1u) | (call & 1u);
        }
        asm volatile("s_waitcnt vmcnt(0)" : "+v"(bk) :: "memory");
        STAMP_AT(stamp++);
        const u32x4 rd = mk_rsrc4(P.ws[1], WS_USED);
        sleep_until(rfl((u32)(bk >> 4)));
        if (lane < 8) st16<0>(hd, rd, vo, fo);
#pragma unroll
        for (int jj = 0; jj < 8; ++jj) {
          st16<0>(v[jj], rd, vo, so + (u32)(jj * 1024));
          wait_vm<CAP - 1>();
        }
      }
      STAMP_AT(stamp++);
    } else {
      fjob = jobs & 0x3ffu;
#pragma unroll
      for (int m = 0; m < 4; ++m)
#pragma unroll
        for (int n = 0; n < 4; ++n) fin[m][n] = acc[m][n];
    }
  }
  if (!fjob) return;

  const Job J = decode_job(fjob, a);
  const __amdgpu_buffer_rsrc_t rws = mk_rsrc(ws, WS_USED);
  char* outp = P.out;
  asm volatile("" : "+s"(outp));
  const __amdgpu_buffer_rsrc_t ro = mk_rsrc(outp, OUT_BYTES);
  {
    float mx = 0.f;
    auto add = [&](int jj, u32 x_lo, u32x4 c) {
      f32x4& u0 = fin[jj >> 1][(jj & 1) * 2]; f32x4& u1 = fin[jj >> 1][(jj & 1) * 2 + 1];
      u0 += (f32x4){as_f32(x_lo), bf_hi(c.x), bf_lo(c.y), bf_hi(c.y)};
      u1 += (f32x4){bf_lo(c.z), bf_hi(c.z), bf_lo(c.w), bf_hi(c.w)};
    };
    auto done = [&](int jj) {
      f32x4& u0 = fin[jj >> 1][(jj & 1) * 2]; f32x4& u1 = fin[jj >> 1][(jj & 1) * 2 + 1];
#pragma unroll
      for (int e = 0; e < 4; ++e) mx = fmaxf(mx, fmaxf(u0[e], u1[e]));
      asm volatile("" : "+v"(u0), "+v"(u1), "+v"(mx));
    };
    if (J.kind == JOB_F) {
      const bool far = (J.ja >> 2) != 0;
      const int d0 = far ? 4 | ((a ^ J.sub) & 2) : 2;
      const u32 bi = far ? (u32)(J.p * 2 + (J.sub & 1)) : (u32)J.p;
      const u32 s0 = (u32)box_base(d0) + bi, s1 = (u32)box_base(d0 + 1) + bi;
      const __amdgpu_buffer_rsrc_t r0 = mk_rsrc(P.ws[d0], WS_USED), r1 = mk_rsrc(P.ws[d0 + 1], WS_USED);
      const u32 o = OFF_GO + (u32)d0 * 64u;
      wave_wait<SL_GO>(ws, o + 4u, lane == 0, call, lane);
      u32 t_go = rfl(__hip_atomic_load((GAS const u32*)(ws + o), __ATOMIC_RELAXED, __HIP_MEMORY_SCOPE_AGENT));
      const u32 round = (u32)(!far ? J.p >> 3 : J.p < 8 ? J.sub & 1 : J.p >> 2);
      const u32 seat = (u32)((!far || J.p < 8 ? J.p & 7 : (J.p & 3) | (J.sub & 1) << 2) + wave * 8);
      const u32 tok = OFF_TOK + (((u32)d0 * 2u + round) * 32u + seat) * 64u;
      const u32 step = far ? 4 * TAU_X : 4 * TAU_C;
      t_go += (round * 8u * step + seat * (step >> 5)) >> 4;
      sleep_until(t_go - 50u);
      wave_wait<4>(ws, lane == 2 ? tok - 32u * 64u : OFF_PF + (lane == 0 ? s0 : s1) * 512u + (u32)wave * 128u, lane < (round ? 3 : 2), call, lane);
      const u32 so0 = OFF_IN + s0 * 32768u + (u32)wave * 8192u, so1 = OFF_IN + s1 * 32768u + (u32)wave * 8192u;
      u32x4 v0[8], v1[8];
      u32 t_min = t_go;
#pragma unroll
      for (int jj = 0; jj < 8 + PD; ++jj) {
        if (jj < 8) {
          const u32 ti = t_go + (((u32)jj * step) >> 4);
          sleep_until((int)(ti - t_min) > 0 ? ti : t_min);
          t_min = (u32)__builtin_amdgcn_s_memrealtime() + ((step * CATCH) >> 8);
          v0[jj] = __builtin_amdgcn_raw_buffer_load_b128(r0, vo + (u32)((jj & 3) * 1024), so0 + (u32)((jj >> 2) * 4096), 17);
          v1[jj] = __builtin_amdgcn_raw_buffer_load_b128(r1, vo + (u32)((jj & 3) * 1024), so1 + (u32)((jj >> 2) * 4096), 17);
          if (jj == 6 && lane == 0) st_agent(ws + tok, call);
        }
        if (jj >= PD) { add(jj - PD, v0[jj - PD].x << 16, v0[jj - PD]); add(jj - PD, v1[jj - PD].x << 16, v1[jj - PD]); done(jj - PD); }
      }
    } else {
      const u32 s0 = (u32)(J.p * 3 + J.sub);
      const u64 t0 = __builtin_amdgcn_s_memrealtime();
      const u32 hoff = lane < 8 ? OFF_PF + s0 * 512u + (u32)wave * 128u + (u32)lane * 16u : OFF_TMO;
      const u32 so0 = OFF_IN + s0 * 32768u + (u32)wave * 8192u;
      auto late = [&](bool dead) {
        if (!dead && __builtin_amdgcn_s_memrealtime() - t0 <= DEADLINE_TICKS) return false;
        if (lane == 0) st_sys(ws + OFF_TMO, call);
        WAIT_VM(0);
        return true;
      };
      u32x4 v0[8];
      auto fetch = [&](int jj) { v0[jj] = __builtin_amdgcn_raw_buffer_load_b128(rws, vo + (u32)((jj & 3) * 1024), so0 + (u32)((jj >> 2) * 4096), 17); };
      u32x4 hd = __builtin_amdgcn_raw_buffer_load_b128(rws, hoff, 0, 17);
#pragma unroll
      for (int jj = 0; jj < 8; ++jj) fetch(jj);
      while (!__all(lane >= 8 || (hd.y >= call && hd.w >= call))) {
        if (late((u32)__builtin_amdgcn_readlane(hd.x, 63) == call)) break;
        __builtin_amdgcn_s_sleep(SL_HDR);
        asm volatile("" ::: "memory");
        hd = __builtin_amdgcn_raw_buffer_load_b128(rws, hoff, 0, 17);
      }
#pragma unroll
      for (int jj = 0; jj < 8; ++jj) {
        while (__builtin_expect(!__all(((v0[jj].x ^ call) & 1u) == 0), 0)) {
          asm volatile("" ::: "memory");
#pragma unroll
          for (int k = jj; k < jj + LOOK && k < 8; ++k) fetch(k);
          if (late(false)) break;
        }
        const u64 m = (u64)(u32)__builtin_amdgcn_readlane(hd.z, jj) << 32 | (u32)__builtin_amdgcn_readlane(hd.x, jj);
        add(jj, (v0[jj].x << 16 & ~0x10000u) | (__builtin_amdgcn_inverse_ballot_w64(m) ? 0x10000u : 0u), v0[jj]);
        done(jj);
      }
    }
    TSTAMP(10);
    const u32 wm = wave_max(__builtin_bit_cast(u32, mx));
    if (lane == 0)
      __hip_atomic_store((GAS u64*)(ws + OFF_LM + (u32)((J.p * 8 + (J.kind == JOB_F ? J.ja : J.sub)) * 4 + wave) * 64u), ((u64)call << 32) | wm, __ATOMIC_RELAXED, __HIP_MEMORY_SCOPE_AGENT);
  }
  TSTAMP(11);
  if (g & 1) {
#pragma unroll
    for (int m = 0; m < 4; ++m) { f32x4 t = fin[m][0]; fin[m][0] = fin[m][1]; fin[m][1] = t; t = fin[m][2]; fin[m][2] = fin[m][3]; fin[m][3] = t; }
  }
  u32 oo[4];
#pragma unroll
  for (int m = 0; m < 4; ++m) { oo[m] = (u32)(J.sub * 128 + wr * 64 + m * 16 + fr) * 4096u + J.col0 + (u32)(wc * 64 + g * 16); asm volatile("" : "+v"(oo[m])); }
  Polls pq;
  const u32 gmax = gather_max(ws, OFF_AM + (u32)lane * 128u, call, lane, 16, pq);
  TSTAMP(12);
  const float inv = gmax ? 127.0f * __builtin_amdgcn_rcpf(as_f32(gmax)) : 0.f;
#pragma unroll
  for (int m = 0; m < 4; ++m) {
    u32x4 v;
#pragma unroll
    for (int n = 0; n < 4; ++n) {
      f32x4 y = fin[m][n] * inv;
      asm volatile("" : "+v"(y));
      u32 r = 0;
#pragma unroll
      for (int e = 0; e < 4; ++e) r = __builtin_amdgcn_cvt_pk_u8_f32(y[e], e, r);
      v[n] = r;
    }
    if (gmax == ~0u) v = (u32x4){0x80808080u, 0x80808080u, 0x80808080u, 0x80808080u};
    __builtin_amdgcn_raw_buffer_store_b128(v, ro, oo[m], 0, OUT_AUX);
  }
  pq.retire();
  TSTAMP(13);
}

__global__ void __launch_bounds__(512, 2) gemm_rs_q8(const Params P) {
  __shared__ __attribute__((aligned(16))) char lds[5 * 32768];
  const int lane = threadIdx.x & 63, wave = (int)rfl(threadIdx.x >> 6);
  char* ws = P.ws[0];
  if (__builtin_expect(blockIdx.x < NPUSH, 1)) {
    if (wave < 6) pusher(P, lds, ws, lane, wave);
    return;
  }
  const u32 call = draw_call(ws, lane, wave);
  if (blockIdx.x == GRID - 1) {
    if (wave == 0) coordinator(P, ws, lane, call);
    char* dst = P.ws[lane & 7] + OFF_AM + (u32)(wave * 8 + P.rank) * 128u;
    asm volatile("" : "+v"(dst));
    Polls pq;
    const u32 m = gather_max(ws, OFF_LM + (u32)(wave * 64 + lane) * 64u, call, lane, 40, pq);
    if (lane < NR) __hip_atomic_store((GAS u64*)dst, ((u64)call << 32) | m, __ATOMIC_RELAXED, __HIP_MEMORY_SCOPE_SYSTEM);
    pq.retire();
    return;
  }
  const u32 jobs = P.jobs[blockIdx.x];
  if (wave >= 4) loader(P, lds, ws, lane, wave, call, jobs);
  else computer(P, lds, ws, lane, wave, call, jobs);
}

#define LAG 12
#define R_FWD 2
#define R_EARLY 6
void build_schedule(Params& P) {
#define RAMP 4
#define SHIFT2 2
#define ORD 0
  static const int orders[5][32] = {
    {0, 1, 2, 16, 17, 5, 18, 6, 19, 7, 20, 8, 21, 22, 23, 9, 10, 11, 12, 24, 25, 26, 27, 4, 3, 28, 29, 30, 31, 13, 14, 15},
    {0, 1, 2, 16, 17, 5, 18, 6, 19, 7, 20, 8, 21, 22, 23, 24, 9, 25, 10, 11, 26, 12, 27, 4, 3, 28, 29, 30, 31, 13, 14, 15},
    {0, 1, 2, 16, 17, 5, 18, 6, 19, 7, 20, 8, 21, 22, 23, 24, 25, 9, 10, 26, 11, 12, 27, 4, 3, 28, 29, 30, 31, 13, 14, 15},
    {0, 1, 2, 16, 17, 18, 5, 19, 6, 20, 7, 21, 8, 22, 23, 24, 9, 25, 10, 26, 11, 12, 27, 4, 3, 28, 29, 30, 31, 13, 14, 15},
    {0, 1, 2, 16, 17, 5, 18, 6, 19, 7, 20, 8, 21, 22, 23, 24, 9, 25, 10, 11, 26, 12, 4, 27, 3, 28, 29, 30, 31, 13, 14, 15}};
  const int* order = orders[ORD];
  int pos[32];
  for (int i = 0; i < 32; ++i) pos[order[i]] = i;
  auto ready = [&](int t, int p) { return pos[t] > pos[16 + p] ? pos[t] : pos[16 + p]; };
  for (int d = 0; d < NR; ++d) {
    unsigned char list[MAXU]; int n = 0;
    for (int i = 0; i < MAXU; ++i) list[i] = 0;
    const int q = P.rank ^ d;
    auto relay = [&](int c) { return c >= 16 + R_END - 2 * RELAY && c < 16 + R_END; };
    if (d >= 2)
      for (int i = 0, nh1 = 0, lag = ((2 * (d - (d >= 4 ? 4 : 2)) + 1) * LAG + 10) / 20; i < 64; ++i) {
        auto put = [&](int k, int h) { const int c = order[k]; if (!(c < 16 && tile_dist(c) == d)) list[n++] = (unsigned char)(c * 2 + h); };
        if (i < 32) put(i, 0);
        for (; nh1 < 32 && nh1 + (nh1 * RAMP / 8 < lag ? nh1 * RAMP / 8 : lag) <= i; ++nh1)
          if (d < 4 || !relay(order[nh1])) put(nh1, 1);
        if (d >= 4 && i + R_EARLY < 32 && relay(order[i + R_EARLY]) && ((q >> 1) & 1) == (order[i + R_EARLY] & 1)) put(i + R_EARLY, 1);
        if (d < 4 && i >= R_FWD && i - R_FWD < 32 && relay(order[i - R_FWD]) && ((P.rank >> 1) & 1) == (order[i - R_FWD] & 1))
          for (int ds = 4; ds < 8; ++ds)
            if (((P.rank ^ ds) & 1) == (d & 1)) list[n++] = (unsigned char)(ds * 16 + order[i - R_FWD] - 16);
      }
    P.ucount[d] = (u32)n;
    for (int i = 0; i < MAXU / 4; ++i) P.units[d][i] = list[4 * i] | list[4 * i + 1] << 8 | list[4 * i + 2] << 16 | (u32)list[4 * i + 3] << 24;
  }
  P.relayed = 0;
  for (int p = R_END - 2 * RELAY; p < R_END; ++p) if (((P.rank >> 1) & 1) != (p & 1)) P.relayed |= 1u << p;
  static const int kord[2][12] = {{10, 8, 6, 4, 5, 7, 9, 11, 2, 0, 1, 3}, {10, 8, 6, 4, 2, 0, 1, 3, 5, 7, 9, 11}};
  for (int c = 0; c < 2; ++c)
    for (int sk = 0; sk < 8; ++sk) {
      u64 v = 0;
      for (int i = 0, n = 0; i < 12; ++i) if ((kord[c][i] >> 1) != sk) v |= (u64)kord[c][i] << (4 * n++);
      P.korder[c][sk] = v;
    }
  for (int b = 0; b < NPUSH; ++b)
    for (int w = 0; w < 8; ++w) {
      const int d = push_link(b, w);
      auto code = [&](int i) { return (P.units[d][i >> 4] >> ((i & 12) * 2)) & 0xffu; };
      P.first[b][2 * w] = slice_src(code(b), (u32)b & 3u, P.rank ^ d, d) | code(b) >> 5;
      P.first[b][2 * w + 1] = code(b) | code(b + NPUSH) << 8 | (u32)(P.rank ^ d) << 16;
    }
  static const int grp[16] = {0, 1, 0, 2, 1, 1, 0, 1, 0, 1, 0, 1, 0, 2, 2, 2};
  for (int i = 0; i < GRID; ++i) P.jobs[i] = 0;
  for (int x = 0; x < 8; ++x) {
    int ht[40], hp[40], n = 0;
    for (int r = 0; r < 32; ++r)
      for (int p = 0; p < 16; ++p)
        for (int t = 0; t < 16; ++t) {
          const int gsel = grp[t] == 2 ? (p >> 2) & 1 : grp[t];
          if (gsel == (x >> 2) && ((p + (t == 2 ? SHIFT2 : 0)) & 3) == (x & 3) && ready(t, p) == r) { ht[n] = t; hp[n] = p; ++n; }
        }
    auto enc = [&](int i) { return (u32)(((ht[i] >= 13 ? JOB_O : JOB_S) << 8) | (ht[i] << 4) | hp[i]); };
    const int nw = 32 - NPUSH / 8 - (x == 7);
    const int hi = x >> 2, lo = x & 3;
    const int fsub[10] = {4, 6, 5, 7, 3, 4 + hi, 6 + hi, 3, 5 - hi, 7 - hi};
    const int fpan[10] = {4 * hi + lo, 4 * hi + lo, 4 - 4 * hi + lo, 4 - 4 * hi + lo, x, 8 + lo, 8 + lo, 8 + x, 12 + lo, 12 + lo};
    int f = 0, extra = nw;
    for (int s = 0; s < nw; ++s) {
      u32 jb = enc(s);
      const bool host = extra < n && (ht[s] < 3 ? hp[s] >= PURE && hp[s] < 8 : f >= 4 && ht[s] < 13);
      if (host) { jb = (u32)((JOB_F << 8) | ((ht[extra] - 13) << 4) | hp[extra]) | jb << 10 | enc(extra) << 20; ++extra; }
      else if (ht[s] >= 3 && ht[s] < 13 && f < 10) {
        jb = jb << 10 | (u32)((JOB_F << 8) | (fsub[f] << 4) | fpan[f]); ++f;
      }
      P.jobs[NPUSH + s * 8 + x] = jb;
    }
  }
}

}

extern "C" void kernel_launch(int rank, int nranks, ncclComm_t comm, hipStream_t stream,
                              void* const* d_in, const int* in_sizes, int n_in,
                              void* d_out, int out_size,
                              const int* in_dtypes, int out_dtype,
                              void* const* d_out_all, void* const* d_ws_all, size_t ws_bytes) {
  (void)comm; (void)in_dtypes; (void)out_dtype; (void)d_out_all;
  if (nranks != NR || (unsigned)rank >= (unsigned)NR || n_in != 2 || ws_bytes < WS_USED || (u32)out_size != OUT_BYTES ||
      (u32)in_sizes[0] * 2u != X_BYTES || (u32)in_sizes[1] * 2u != W_BYTES) {
    fprintf(stderr, "unexpected problem shape\n");
    return;
  }
  Params P;
  P.x = (const char*)d_in[0]; P.w = (const char*)d_in[1]; P.out = (char*)d_out;
  for (int d = 0; d < NR; ++d) P.ws[d] = (char*)d_ws_all[rank ^ d];
  P.rank = rank; P.pad = 0;
  build_schedule(P);
  hipLaunchKernelGGL(gemm_rs_q8, dim3(GRID), dim3(512), 0, stream, P);
  HIP_CHECK(hipGetLastError());
}
```

```cpp
#include <hip/hip_runtime.h>
#include <rccl/rccl.h>
#include <cstdio>
#include <cstdint>

#define HIP_CHECK(expr) do { hipError_t _e = (expr); if (_e != hipSuccess) { \
    fprintf(stderr, "HIP error %s at %s:%d\n", hipGetErrorString(_e), __FILE__, __LINE__); \
    return; } } while (0)

namespace {

typedef unsigned u32;
typedef unsigned long long u64;
typedef u32 u32x4 __attribute__((ext_vector_type(4)));
typedef u32 u32x2 __attribute__((ext_vector_type(2)));
typedef float f32x4 __attribute__((ext_vector_type(4)));
typedef short s16x4 __attribute__((ext_vector_type(4)));
typedef short s16x8 __attribute__((ext_vector_type(8)));
typedef __bf16 bf16x8 __attribute__((ext_vector_type(8)));
#define LAS __attribute__((address_space(3)))
#define GAS __attribute__((address_space(1)))

constexpr int NR = 8;
#define NPUSH 32
#define PW 3
constexpr int NW = NPUSH * PW / 6;
constexpr int GRID = 256;
#define RELAY 4
constexpr int MAXU = 72;
constexpr u64 DEADLINE_TICKS = 100000000ull;

constexpr u32 X_BYTES = 4096u * 512u * 2u, W_BYTES = 512u * 4096u * 2u, OUT_BYTES = 512u * 4096u;
constexpr u32 OFF_CNT = 0;
constexpr u32 OFF_TMO = 16u << 10;
constexpr u32 OFF_TK = (16u << 10) + 128u;
constexpr u32 OFF_TP = 18u << 10;
constexpr u32 OFF_T0 = 19u << 10;
constexpr u32 OFF_XF = 20u << 10;
constexpr u32 OFF_WF = 24u << 10;
constexpr u32 OFF_TR = 28u << 10;
constexpr u32 OFF_GO = 29u << 10;
constexpr u32 OFF_AM = 32u << 10;
constexpr u32 OFF_LM = 64u << 10;
constexpr u32 OFF_TOK = 128u << 10;
constexpr u32 OFF_PF = 192u << 10;
constexpr u32 OFF_DBG = 512u << 10;
constexpr u32 OFF_XS = 2u << 20;
constexpr u32 OFF_WS = 8u << 20;
constexpr u32 OFF_IN = 14u << 20;
constexpr u32 WS_USED = OFF_IN + 208u * 32768u;
#define TAU_X 221
#define TAU 232
#define TAU_C 224
#define PD 3
#define T_BOOK 70
#define T_REST 700
#define OUT_AUX 0
#define LOOK 3
#define SL_LONG 32
#define SL_GATE 6
#define SL_AM 3
#define SL_HDR 16
#define WDELAY 150
#define SL_AML 16
#define T_FLY 80
#define SL_GO 16
#define LASTW WIN
#define CATCH 10
#define R_END 16
#define PURE 4
#define OWNLAST 1
#define EARLYBOOK 1
#define LZ_F 7
#define LZ_C 40
#define T_RESTC T_REST
#define CAP 3
constexpr u32 PANEL_B = 6u * 65536u;

enum { JOB_NONE = 0, JOB_S = 1, JOB_O = 2, JOB_F = 3 };

struct Params {
  const char* x; const char* w; char* out;
  char* ws[NR];
  int rank; int pad;
  u32 jobs[GRID];
  u32 first[NPUSH][2 * PW];
  u32 ucount[NR];
  u32 units[NR][MAXU / 4];
  u64 korder[2][8];
  u32 relayed[16];
};

__host__ __device__ inline int tile_dist(int t) { return t < 3 ? 1 : t < 5 ? t - 1 : t < 13 ? 4 + ((t - 5) & 3) : 0; }
__host__ __device__ inline int tile_sub(int t, int a) { return t < 3 ? t : t < 5 ? 3 : t < 13 ? (a & 2) + ((t - 5) >> 2) : t - 13; }
__host__ __device__ inline int box_base(int d) { return d == 1 ? 0 : d < 4 ? 48 + 16 * (d - 2) : 80 + 32 * (d - 4); }
__host__ __device__ inline int box_cnt(int d) { return d == 1 ? 48 : d < 4 ? 16 : 32; }
__host__ __device__ inline int box_idx(int t, int p) { return t < 3 ? p * 3 + t : t < 5 ? p : p * 2 + ((t - 5) >> 2); }

__host__ __device__ inline u32 link_units(int d) { return d < 4 ? 62u + 2u * RELAY : 60u - RELAY; }
__host__ __device__ inline int push_link(int block, int wave) { return 2 + ((block >> 3) * PW + wave + (block & 7)) % 6; }
__host__ __device__ inline int push_seat(int block, int wave) { return ((block >> 3) * PW + wave) / 6 * 8 + (block & 7); }
__host__ __device__ inline u32 k_half(int q, int d) { return (u32)(q ^ (d >= 4 ? (q >> 1) & (q ^ d) : 0)) & 1u; }
__host__ __device__ inline u32 slice_src(u32 code, u32 sl, int q, int d) {
  const u32 h = code & 1u, kh = k_half(q, d);
  if (code < 32) {
    const int t = code >> 1;
    return ((u32)((q ^ tile_dist(t)) * 512 + tile_sub(t, q) * 128) + (sl & 1u) * 64u) * 1024u + (kh * 256u + h * 128u + (sl >> 1) * 64u) * 2u;
  }
  return (kh * 256u + h * 128u + sl * 32u) * 8192u + ((u32)(q >> 2) * 2048u + ((code - 32u) >> 1) * 128u) * 2u;
}

__device__ __forceinline__ u32 rfl(u32 v) { return __builtin_amdgcn_readfirstlane(v); }
__device__ __forceinline__ __amdgpu_buffer_rsrc_t mk_rsrc(const void* p, u32 bytes) {
  return __builtin_amdgcn_make_buffer_rsrc((void*)p, 0, bytes, 0x00020000);
}
__device__ __forceinline__ u32x4 mk_rsrc4(const void* p, u32 bytes) {
  u64 a = (u64)p; u32x4 r; r.x = rfl((u32)a); r.y = rfl((u32)(a >> 32)) & 0xffffu; r.z = bytes; r.w = 0x00020000u; return r;
}
__device__ __forceinline__ u32 ld_sys(const char* p) { return __hip_atomic_load((GAS const u32*)p, __ATOMIC_RELAXED, __HIP_MEMORY_SCOPE_SYSTEM); }
__device__ __forceinline__ void st_sys(char* p, u32 v) { __hip_atomic_store((GAS u32*)p, v, __ATOMIC_RELAXED, __HIP_MEMORY_SCOPE_SYSTEM); }
__device__ __forceinline__ void st_agent(char* p, u32 v) { __hip_atomic_store((GAS u32*)p, v, __ATOMIC_RELAXED, __HIP_MEMORY_SCOPE_AGENT); }
#define WAIT_VM(n) asm volatile("s_waitcnt vmcnt(" #n ")" ::: "memory")
#define BARRIER() do { asm volatile("" ::: "memory"); __builtin_amdgcn_s_waitcnt(0xC07F);   __builtin_amdgcn_s_barrier(); asm volatile("" ::: "memory"); } while (0)

#define TSTAMP(i) do {} while (0)
#define STAMP_AT(i) do {} while (0)

template <int SL> __device__ __forceinline__ void wave_wait(char* ws, u32 off, bool care, u32 call, int lane) {
  if (lane == 63) { off = OFF_TMO; }
  else if (!care) { off = OFF_TMO; }
  const u64 t0 = __builtin_amdgcn_s_memrealtime();
  for (;;) {
    const u32 v = ld_sys(ws + off);
    const bool ok = (lane == 63 || !care) ? true : (v >= call);
    if (__all(ok)) break;
    const bool dead = __any(lane == 63 && v == call);
    if (dead || __builtin_amdgcn_s_memrealtime() - t0 > DEADLINE_TICKS) {
      if (lane == 0) st_sys(ws + OFF_TMO, call);
      WAIT_VM(0);
      break;
    }
    __builtin_amdgcn_s_sleep(SL);
  }
}

__device__ __forceinline__ int wave_poll4(char* ws, __amdgpu_buffer_rsrc_t rws, u32 off, bool care, int have, bool block, u32 call, int lane) {
  if (!care) off = OFF_TMO;
  const u64 t0 = __builtin_amdgcn_s_memrealtime();
  for (;;) {
    const u32x4 v = __builtin_amdgcn_raw_buffer_load_b128(rws, off, 0, 17);
    asm volatile("" ::: "memory");
    const u32 m = (u32)__ballot(care && v.x >= call && v.y >= call && v.z >= call && v.w >= call);
    const int n = __builtin_ctz(~(m & (m >> 16)));
    if (n > have || !block) return n;
    const bool dead = __any(!care && v.x == call);
    if (dead || __builtin_amdgcn_s_memrealtime() - t0 > DEADLINE_TICKS) {
      if (lane == 0) st_sys(ws + OFF_TMO, call);
      WAIT_VM(0);
      return have + 1;
    }
    if (have == 0) __builtin_amdgcn_s_sleep(SL_LONG); else __builtin_amdgcn_s_sleep(SL_GATE);
  }
}
#define DMA_SC " sc0 sc1"

typedef __bf16 bf16x2 __attribute__((ext_vector_type(2)));
__device__ __forceinline__ u32 pack_bf16(float lo, float hi) { bf16x2 r; r.x = (__bf16)lo; r.y = (__bf16)hi; return __builtin_bit_cast(u32, r); }
__device__ __forceinline__ float as_f32(u32 v) { return __builtin_bit_cast(float, v); }
__device__ __forceinline__ float bf_lo(u32 v) { return __builtin_bit_cast(float, v << 16); }
__device__ __forceinline__ float bf_hi(u32 v) { return __builtin_bit_cast(float, v & 0xffff0000u); }
__device__ __forceinline__ u32 wave_max(u32 m) {
  m = max(m, (u32)__builtin_amdgcn_update_dpp(0, (int)m, 0x111, 0xf, 0xf, false));
  m = max(m, (u32)__builtin_amdgcn_update_dpp(0, (int)m, 0x112, 0xf, 0xf, false));
  m = max(m, (u32)__builtin_amdgcn_update_dpp(0, (int)m, 0x114, 0xf, 0xf, false));
  m = max(m, (u32)__builtin_amdgcn_update_dpp(0, (int)m, 0x118, 0xf, 0xf, false));
  const u32 a = max((u32)__builtin_amdgcn_readlane(m, 15), (u32)__builtin_amdgcn_readlane(m, 31));
  const u32 b = max((u32)__builtin_amdgcn_readlane(m, 47), (u32)__builtin_amdgcn_readlane(m, 63));
  return max(a, b);
}

__device__ __forceinline__ void dma4(u32 lds_dst, u32 v0, u32 v1, u32 v2, u32 v3, u32x4 rs, u32 so, u32 st) {
  u32 keep, t;
  asm volatile(
      "s_mov_b32 %0, m0\n\t"
      "s_mov_b32 m0, %2\n\ts_add_u32 %1, %8, %9\n\ts_nop 1\n\tbuffer_load_dwordx4 %3, %7, %8 offen" DMA_SC " lds\n\t"
      "s_add_u32 m0, %2, 0x400\n\ts_nop 0\n\tbuffer_load_dwordx4 %4, %7, %1 offen" DMA_SC " lds\n\t"
      "s_add_u32 m0, %2, 0x800\n\ts_add_u32 %1, %1, %9\n\tbuffer_load_dwordx4 %5, %7, %1 offen" DMA_SC " lds\n\t"
      "s_add_u32 m0, %2, 0xc00\n\ts_add_u32 %1, %1, %9\n\tbuffer_load_dwordx4 %6, %7, %1 offen" DMA_SC " lds\n\t"
      "s_mov_b32 m0, %0"
      : "=&s"(keep), "=&s"(t)
      : "s"(lds_dst), "v"(v0), "v"(v1), "v"(v2), "v"(v3), "s"(rs), "s"(so), "s"(st)
      : "memory", "scc");
}

__device__ __forceinline__ void dma8_lin(u32 lds_dst, u32 v, u32x4 rs, u32 sx, u32 sw) {
  u32 keep;
  asm volatile(
      "s_mov_b32 %0, m0\n\ts_mov_b32 m0, %1\n\ts_nop 2\n\t"
      "buffer_load_dwordx4 %2, %3, %4 offen" DMA_SC " lds\n\tbuffer_load_dwordx4 %2, %3, %4 offen offset:1024" DMA_SC " lds\n\t"
      "buffer_load_dwordx4 %2, %3, %4 offen offset:2048" DMA_SC " lds\n\tbuffer_load_dwordx4 %2, %3, %4 offen offset:3072" DMA_SC " lds\n\t"
      "s_add_u32 m0, %1, 0x4000\n\ts_nop 0\n\t"
      "buffer_load_dwordx4 %2, %3, %5 offen" DMA_SC " lds\n\tbuffer_load_dwordx4 %2, %3, %5 offen offset:1024" DMA_SC " lds\n\t"
      "buffer_load_dwordx4 %2, %3, %5 offen offset:2048" DMA_SC " lds\n\tbuffer_load_dwordx4 %2, %3, %5 offen offset:3072" DMA_SC " lds\n\t"
      "s_mov_b32 m0, %0"
      : "=&s"(keep) : "s"(lds_dst), "v"(v), "s"(rs), "s"(sx), "s"(sw) : "memory", "scc");
}

#define WIN 4
template <int N> __device__ __forceinline__ void wait_vm() { asm volatile("s_waitcnt vmcnt(%0)" :: "n"(N) : "memory"); }
template <int IMM> __device__ __forceinline__ void st16(u32x4 v, u32x4 rs, u32 voff, u32 soff) {
  asm volatile("buffer_store_dwordx4 %0, %1, %2, %3 offen offset:%4 sc0 sc1\n\ts_nop 1" :: "v"(v), "v"(voff), "s"(rs), "s"(soff), "n"(IMM) : "memory");
}
__device__ __forceinline__ void draw_async(u32& v, u32x4 rl, u32 off) {
  u64 ex; v = 1;
  asm volatile("s_mov_b64 %1, exec\n\ts_mov_b64 exec, 1\n\tbuffer_atomic_add %0, off, %2, %3 sc0\n\ts_mov_b64 exec, %1"
               : "+v"(v), "=&s"(ex) : "s"(rl), "s"(off) : "memory");
}
__device__ __forceinline__ void book(u64& tk, u32x4 rl, u32 off) {
  u64 ex; const u64 lo = (__builtin_amdgcn_s_memrealtime() + T_BOOK) * 16u;
  asm volatile("s_mov_b64 %1, exec\n\ts_mov_b64 exec, 1\n\tbuffer_atomic_umax_x2 %2, off, %3, %4\n\tbuffer_atomic_add_x2 %0, off, %3, %4 sc0\n\ts_mov_b64 exec, %1"
               : "+v"(tk), "=&s"(ex) : "v"(lo), "s"(rl), "s"(off) : "memory");
}
__device__ __forceinline__ void sleep_until(u32 due) {
  for (;;) {
    const u32 left = due - (u32)__builtin_amdgcn_s_memrealtime() - 1u;
    if (left >= 10000u) break;
    if (left > 200u) __builtin_amdgcn_s_sleep(48); else if (left > 40u) __builtin_amdgcn_s_sleep(8); else __builtin_amdgcn_s_sleep(1);
  }
}
struct Slice { u32 src, dst, flag, kind, dep; };
__device__ __forceinline__ u32 draw_call(char* ws, int lane, int wave) {
  u32 call = 0;
  if (lane == 0) call = __hip_atomic_fetch_add((GAS u32*)(ws + OFF_CNT + blockIdx.x * 64 + wave * 4), 1u, __ATOMIC_RELAXED, __HIP_MEMORY_SCOPE_AGENT) + 1u;
  return rfl(call);
}
__device__ __forceinline__ void pusher(const Params& P, char* lds, char* ws, int lane, int wave) {
  const u32 vx0 = (u32)(lane >> 3) * 1024u + (u32)(((lane & 7) ^ (lane >> 4)) * 16), vx1 = vx0 ^ 64u;
  const u32 vw0 = (u32)(lane >> 4) * 8192u + (u32)(((lane & 15) ^ (lane >> 4)) * 16), vw1 = vw0 ^ 128u;
  const int d = push_link(blockIdx.x, wave);
  u32 f0 = P.first[blockIdx.x][2 * wave], f1 = P.first[blockIdx.x][2 * wave + 1]; const char* xp = P.x; const char* wp = P.w;
  char* pw = P.ws[d];
  asm volatile("" : "+s"(f0), "+s"(f1), "+s"(xp), "+s"(wp), "+s"(pw), "+s"(ws));
  const u32x4 rx = mk_rsrc4(xp, X_BYTES), rw = mk_rsrc4(wp, W_BYTES), rl = mk_rsrc4(ws, WS_USED);
  const u32 vp = (u32)lane * 16u;
  const u32 ring0 = (u32)(size_t)(LAS char*)lds + (u32)wave * 16384u;
  auto load_all = [&](const Slice& c, u32 buf) {
    const u32 dst = ring0 + buf * 8192u;
    if (c.kind == 0) { dma4(dst, vx0, vx1, vx0, vx1, rx, c.src, 8192u); dma4(dst + 4096u, vx0, vx1, vx0, vx1, rx, c.src + 32768u, 8192u); }
    else if (c.kind == 1) { dma4(dst, vw0, vw0, vw1, vw1, rw, c.src, 32768u); dma4(dst + 4096u, vw0, vw0, vw1, vw1, rw, c.src + 131072u, 32768u); }
    else { dma4(dst, vp, vp, vp, vp, rl, c.src, 1024u); dma4(dst + 4096u, vp, vp, vp, vp, rl, c.src + 4096u, 1024u); }
  };
  { Slice c; c.src = f0 & ~1u; c.kind = f0 & 1u; c.dst = c.flag = c.dep = 0; load_all(c, 0); }
  u32 cnt; draw_async(cnt, rl, OFF_CNT + blockIdx.x * 64u + (u32)wave * 4u);
  const int q = (int)(f1 >> 16);
  const u32 ns = link_units(d) * 4u;
  auto decode = [&](u32 sidx, u32 code) -> Slice {
    Slice r;
    const u32 sl = sidx & 3u, h = code & 1u;
    if (code >= 64) {
      const u32 ds = code >> 4, pp = code & 15u;
      r.kind = 2;
      r.src = OFF_WS + pp * PANEL_B + (ds - 2u) * 65536u + 32768u + sl * 8192u;
      r.dep = OFF_WF + pp * 256u + (ds - 2u) * 32u + 16u + sl * 4u;
      r.dst = OFF_WS + pp * PANEL_B + ((ds ^ (u32)d) - 2u) * 65536u + 32768u + sl * 8192u;
      r.flag = OFF_WF + pp * 256u + ((ds ^ (u32)d) - 2u) * 32u + 16u + sl * 4u;
      return r;
    }
    r.kind = code >> 5; r.src = slice_src(code, sl, q, d); r.dep = OFF_TMO;
    r.dst = (code < 32 ? OFF_XS : OFF_WS) + ((code >> 1) & 15u) * PANEL_B + (u32)(d - 2) * 65536u + h * 32768u + sl * 8192u;
    r.flag = (code < 32 ? OFF_XF : OFF_WF) + ((code >> 1) & 15u) * 256u + (u32)(d - 2) * 32u + h * 16u + sl * 4u;
    return r;
  };
  const u32x4 rd = mk_rsrc4(pw, WS_USED);
  LAS char* ring = (LAS char*)lds + wave * 16384;
  u32 nxt_i = blockIdx.x;
  Slice cur = decode(blockIdx.x, f1 & 0xffu), nxt = decode(nxt_i, (f1 >> 8) & 0xffu);
  u32 buf = 0;
  asm volatile("s_waitcnt vmcnt(0)" : "+v"(cnt), "+s"(cur.dst), "+s"(nxt.src) :: "memory");
  const u32 call = rfl(cnt) + 1u;
  const u32 tk_base = (call - 1u) * (ns - NW);
  const u32 tk_off = OFF_TK + (u32)d * 128u;
  u32 pend = 0;
  auto publish = [&]() {
    if (lane == 0) {
      st_sys(pw + pend, call);
    }
  };
  auto send = [&](int k) {
    const u32x4 v = *(LAS const u32x4*)(ring + buf * 8192u + k * 1024 + lane * 16);
    st16<0>(v, rd, vp, cur.dst + (u32)k * 1024u);
  };
  for (;;) {
    u32 tk;
    send(0);
    load_all(nxt, buf ^ 1u);
    draw_async(tk, rl, tk_off);
    const u32 t_draw = (u32)__builtin_amdgcn_s_memrealtime();
    Slice nn = nxt;
    u32 dv = 0;
#pragma unroll
    for (int k = 0; k < 8; ++k) {
      if (k) send(k);
      if (k == WIN + 1) {
        asm volatile("" : "+v"(tk));
        nxt_i = rfl(tk) - tk_base;
        if (nxt_i == ns - 40u && lane == 0)
          __hip_atomic_store((GAS u64*)(ws + OFF_T0 + (u32)d * 128u), ((u64)call << 32) | (t_draw + (d < 4 ? T_RESTC : T_REST)), __ATOMIC_RELAXED, __HIP_MEMORY_SCOPE_AGENT);
        nxt_i += 2u * NW;
        if (nxt_i < ns) nn = decode(nxt_i, (P.units[d][nxt_i >> 4] >> ((nxt_i & 12u) * 2u)) & 0xffu);
        if (nxt_i < ns && nn.kind == 2) asm volatile("buffer_load_dword %0, off, %1, %2 sc0 sc1" : "+v"(dv) : "s"(rl), "s"(nn.dep) : "memory");
      }
      if (k < WIN) wait_vm<WIN - 1 + 9>(); else wait_vm<WIN - 1>();
      if (k == WIN - 2 && pend) publish();
    }
    pend = cur.flag;
    cur = nxt; nxt = nn; buf ^= 1u;
    if (nxt_i >= ns) break;
    if (nxt.kind == 2) {
      asm volatile("s_waitcnt vmcnt(%1)" : "+v"(dv) : "n"(6 - WIN) : "memory");
      if (rfl(dv) < call) wave_wait<2>(ws, nxt.dep, lane == 0, call, lane);
    }
  }
#pragma unroll
  for (int k = 0; k < 8; ++k) {
    send(k);
    wait_vm<LASTW - 1>();
    if (k == LASTW - 2) publish();
  }
  pend = cur.flag;
  WAIT_VM(0);
  publish();
}

__device__ __forceinline__ void coordinator(const Params& P, char* ws, int lane, u32 call) {
  const u32 lo = (u32)(lane & 7) * 128u, off = (lane & 8 ? OFF_TR : OFF_T0) + lo;
  char* peer = P.ws[lane & 7];
  asm volatile("" : "+v"(peer));
  const u64 t0 = __builtin_amdgcn_s_memrealtime();
  if (lane >= 1 && lane < 8) st_sys(peer + OFF_TR + lo + 4u, call);
  auto rl = [&](u32 v, int i) { return (u32)__builtin_amdgcn_readlane(v, i); };
  bool seen = lane < 2 || lane == 8 || lane >= 16;
  u32 t = 0, sent = 0;
  for (;;) {
    u64 m = 0;
    if (!seen) m = __hip_atomic_load((GAS const u64*)(ws + off), __ATOMIC_RELAXED, __HIP_MEMORY_SCOPE_SYSTEM);
    asm volatile("" : "+v"(m));
    const u64 now = __builtin_amdgcn_s_memrealtime();
    if (!seen && ((u32)(m >> 32) >= call || now - t0 > DEADLINE_TICKS)) {
      t = lane & 8 ? (u32)max((int)(u32)(now - t0) - T_FLY, 0) : (u32)m;
      seen = true;
    }
    const u32 sm = (u32)__ballot(seen);
    const u32 rdy = ((sm & 0xfff0u) == 0xfff0u ? 1u : 0u) | ((sm & 0xff0cu) == 0xff0cu ? 2u : 0u);
    if (rdy & ~sent) {
      const u32 tf = rl(t, 4) + (u32)((int)(rl(t, 5) + rl(t, 6) + rl(t, 7) - 3u * rl(t, 4)) >> 2), tn = rl(t, 2) + (u32)((int)(rl(t, 3) - rl(t, 2)) >> 1);
      const u32 lag = wave_max(lane >= 8 ? t : 0u);
      if (lane < 2 && ((rdy >> lane) & 1u))
        __hip_atomic_store((GAS u64*)(ws + OFF_GO + lo), ((u64)call << 32) | ((lane ? tn : tf) + lag), __ATOMIC_RELAXED, __HIP_MEMORY_SCOPE_AGENT);
      sent = rdy;
      if (sent == 3u) return;
    }
    __builtin_amdgcn_s_sleep(1);
  }
}

constexpr int NST = 5;

struct Job { int kind, ja, p, dO, sub, nh, nkt; u32 col0; };
__device__ __forceinline__ Job decode_job(u32 code, int a) {
  Job j; j.kind = (code >> 8) & 3; j.ja = (code >> 4) & 15; j.p = code & 15;
  int chalf;
  if (j.kind == JOB_F) { j.dO = 0; j.sub = j.ja & 3; chalf = (a >> 2) ^ (j.ja >> 2); }
  else { j.dO = tile_dist(j.ja); j.sub = tile_sub(j.ja, a); chalf = a >> 2; }
  j.col0 = (u32)(chalf * 2048 + j.p * 128);
  j.nh = j.kind == JOB_F ? 0 : j.dO >= 2 ? 10 : 12;
  j.nkt = 8 + 2 * j.nh;
  return j;
}

struct Polls { u32x2 va, vb; u32 ta, tb;
  __device__ __forceinline__ void retire() { asm volatile("" :: "v"(va), "v"(vb), "v"(ta), "v"(tb)); } };
__device__ __forceinline__ u32 gather_max(char* ws, u32 off, u32 call, int lane, int lazy, Polls& q) {
  u32x2& va = q.va; u32x2& vb = q.vb; u32& ta = q.ta; u32& tb = q.tb;
  const __amdgpu_buffer_rsrc_t rws = mk_rsrc(ws, WS_USED);
  u32 gm = 0, tmo_v;
  const u32 toff = lane ? 0x80000000u : OFF_TMO;
  const u64 t0 = __builtin_amdgcn_s_memrealtime();
  va = __builtin_amdgcn_raw_buffer_load_b64(rws, off, 0, 17);
  ta = __builtin_amdgcn_raw_buffer_load_b32(rws, toff, 0, 17);
  __builtin_amdgcn_s_sleep(SL_AM);
  for (;;) {
    asm volatile("" ::: "memory");
    vb = __builtin_amdgcn_raw_buffer_load_b64(rws, off, 0, 17); tb = __builtin_amdgcn_raw_buffer_load_b32(rws, toff, 0, 17);
    asm volatile("" ::: "memory");
    if (va.y >= call) { gm = va.x; off = 0x80000000u; }
    tmo_v = rfl(ta);
    if (__all(off == 0x80000000u)) break;
    va = __builtin_amdgcn_raw_buffer_load_b64(rws, off, 0, 17); ta = __builtin_amdgcn_raw_buffer_load_b32(rws, toff, 0, 17);
    asm volatile("" ::: "memory");
    if (vb.y >= call) { gm = vb.x; off = 0x80000000u; }
    tmo_v = rfl(tb);
    const u64 missing = __ballot(off != 0x80000000u);
    if (!missing) break;
    if (tmo_v == call || __builtin_amdgcn_s_memrealtime() - t0 > DEADLINE_TICKS) {
      if (lane == 0) st_sys(ws + OFF_TMO, call);
      WAIT_VM(0);
      tmo_v = call;
      break;
    }
    if (__builtin_popcountll(missing) > lazy) __builtin_amdgcn_s_sleep(SL_AML);
  }
  gm = wave_max(gm);
  return tmo_v == call ? ~0u : gm;
}

__device__ __forceinline__ void loader(const Params& P, char* lds, char* ws, int lane, int wave, u32 call, u32 jobs) {
  const int a = P.rank;
  const __amdgpu_buffer_rsrc_t rws = mk_rsrc(ws, WS_USED);
  const u32 l4 = (u32)(wave - 4) * 4u;
  const u32 vx0 = (u32)(lane >> 3) * 1024u + (u32)(((lane & 7) ^ (lane >> 4)) * 16), vx1 = vx0 ^ 64u;
  const u32 vw0 = (u32)(lane >> 4) * 8192u + (u32)(((lane & 15) ^ (lane >> 4)) * 16), vw1 = vw0 ^ 128u;
  const u32 vp = (u32)lane * 16u;
  const u32x4 rx = mk_rsrc4(P.x, X_BYTES), rw = mk_rsrc4(P.w, W_BYTES), rl = mk_rsrc4(ws, WS_USED);
  const u32 lds0 = (u32)(size_t)(LAS char*)lds + l4 * 1024u;
  bool fin = false;
  if (WDELAY && (jobs >> 4) - (JOB_S << 4) >= 3u) sleep_until((u32)__builtin_amdgcn_s_memrealtime() + WDELAY);
  int stamp = 1; (void)stamp;
  for (; (jobs & 0x3ffu) != 0; jobs >>= 10) {
    const Job J = decode_job(jobs & 0x3ffu, a);
    const int nh = J.nh, nkt = J.nkt;
    const u32 row0 = (u32)((a ^ J.dO) * 512 + J.sub * 128);
    const int j0 = fin && J.kind == JOB_O ? 8 : 0;
    fin = fin || J.kind != JOB_S;
    const u64 ko = P.korder[(P.relayed[J.ja] >> J.p) & 1u][J.dO >= 2 ? J.dO - 2 : 6];
    const int fl = lane & 15;
    const bool care = lane < 32 && fl < nh;
    const u32 foff = (lane < 16 ? OFF_XF + (u32)J.ja * 256u : OFF_WF + (u32)J.p * 256u) + ((u32)(ko >> (fl * 4)) & 15u) * 16u;
    const u32 tab = lane < 8 ? (row0 + l4 * 8u) * 1024u + (u32)lane * 128u
                             : OFF_XS + (u32)J.ja * PANEL_B + l4 * 1024u + (u32)(lane & 1) * 16384u + ((u32)(ko >> ((((lane - 8) >> 1) & 15) * 4)) & 15u) * 32768u;
    const u32 dw = OFF_WS - OFF_XS + (u32)(J.p - J.ja) * PANEL_B, w0 = l4 * (4u * 8192u) + J.col0 * 2u;
    int issued = j0, open = 8;
    u32 dst = lds0;
    auto issue = [&]() {
      const u32 ox = __builtin_amdgcn_readlane(tab, issued);
      if (__builtin_expect(issued >= 8, 1)) dma8_lin(dst, vp, rl, ox, ox + dw);
      else {
        dma4(dst, vx0, vx1, vx0, vx1, rx, ox, 8192u);
        dma4(dst + 16384u, vw0, vw0, vw1, vw1, rw, w0 + (u32)issued * (64u * 8192u), 32768u);
      }
      ++issued; dst = dst == lds0 + (NST - 1) * 32768u ? lds0 : dst + 32768u;
    };
    for (int j = j0; j <= nkt; ++j) {
      if (__builtin_expect(issued == j + NST - 2 && issued < open, 1)) {
        issue();
        WAIT_VM(24);
        __builtin_amdgcn_s_barrier();
        continue;
      }
      const int cap = j + NST - (j > j0) < nkt ? j + NST - (j > j0) : nkt;
      if (issued < cap && issued == open) {
        const int got = wave_poll4(ws, rws, foff, care, (open - 8) >> 1, issued == j, call, lane);
        open = max(open, 8 + 2 * got);
      }
      while (issued < cap && issued < open) issue();
      const int after = issued - j - 1;
      if (after >= 4) WAIT_VM(32); else if (after == 3) WAIT_VM(24); else if (after == 2) WAIT_VM(16); else if (after == 1) WAIT_VM(8); else WAIT_VM(0);
      __builtin_amdgcn_s_barrier();
    }
    __builtin_amdgcn_s_barrier();
  }
}

__device__ __forceinline__ void computer(const Params& P, char* lds, char* ws, int lane, int wave, u32 call, u32 jobs) {
  const int a = P.rank;
  const int wr = wave >> 1, wc = wave & 1, fr = lane & 15, g = lane >> 4;
  const u32 xa0 = (u32)((wr * 64 + fr) * 128 + ((g ^ (fr >> 1)) * 16));
  const int tq = fr >> 2, tp = fr & 3;
  const u32 wa0 = 16384u + (u32)((8 * g + tq) * 256 + (((8 * wc + 2 * tp) ^ (tq | ((g & 1) << 3))) * 16) + (tp & 1) * 8);
  const u32x4 rl = mk_rsrc4(ws, WS_USED);
  int stamp = 1; (void)stamp;
  f32x4 fin[4][4];
  u32 fjob = 0;
  const u32 vo = (u32)lane * 16u;

  for (; (jobs & 0x3ffu) != 0; jobs >>= 10) {
    const Job J = decode_job(jobs & 0x3ffu, a);
    const bool cont = fjob != 0 && J.kind == JOB_O;
    f32x4 acc[4][4];
#pragma unroll
    for (int m = 0; m < 4; ++m)
#pragma unroll
      for (int n = 0; n < 4; ++n) acc[m][n] = cont ? fin[m][n] : (f32x4){0.f, 0.f, 0.f, 0.f};
    auto readf = [&](bf16x8 (&xf)[4], bf16x8 (&wf)[4], int slot, int ks) {
      LAS char* st = (LAS char*)lds + slot * 32768;
#pragma unroll
      for (int i = 0; i < 4; ++i) {
        const u32 o = (wa0 ^ (u32)((i >> 1) * 16 + (i & 1) * 8)) + (u32)(ks * 8192);
        const s16x4 lo = __builtin_amdgcn_ds_read_tr16_b64_v4i16((LAS s16x4*)(st + o));
        const s16x4 hi = __builtin_amdgcn_ds_read_tr16_b64_v4i16((LAS s16x4*)(st + o + 1024));
        wf[i] = __builtin_bit_cast(bf16x8, __builtin_shufflevector(lo, hi, 0, 1, 2, 3, 4, 5, 6, 7));
        xf[i] = *(LAS const bf16x8*)(st + ((xa0 ^ (u32)(ks * 64)) + (u32)(i * 2048)));
      }
    };
    auto mma = [&](const bf16x8 (&xf)[4], const bf16x8 (&wf)[4]) {
#pragma unroll
      for (int m = 0; m < 4; ++m)
#pragma unroll
        for (int n = 0; n < 4; ++n) acc[m][n] = __builtin_amdgcn_mfma_f32_16x16x32_bf16(wf[n], xf[m], acc[m][n], 0, 0, 0);
    };
#define INTERLEAVE(id) do { _Pragma("unroll") for (int i_ = 0; i_ < 6; ++i_) { __builtin_amdgcn_sched_group_barrier(0x8, 1, id); __builtin_amdgcn_sched_group_barrier(0x100, 2, id); } \
                            __builtin_amdgcn_sched_group_barrier(0x8, 10, id); __builtin_amdgcn_sched_barrier(0); } while (0)
    bf16x8 x0[4], w0[4], x1[4], w1[4];
    int slot = 0;
    BARRIER();
    readf(x0, w0, 0, 0);
    u64 bk = TAU;
    const int bk_at = EARLYBOOK && J.kind == JOB_S && J.dO == 1 ? J.nkt - 2 : -1;
    for (int kt = cont ? 8 : 0; kt < J.nkt; ++kt) {
      if (kt == bk_at) book(bk, rl, OFF_TP);
      __builtin_amdgcn_s_waitcnt(0xC07F);
      __builtin_amdgcn_sched_barrier(0);
      readf(x1, w1, slot, 1); mma(x0, w0);
      INTERLEAVE(0);
      slot = slot == NST - 1 ? 0 : slot + 1;
      BARRIER();
      __builtin_amdgcn_sched_barrier(0);
      readf(x0, w0, slot, 0); mma(x1, w1);
      INTERLEAVE(1);
    }
    BARRIER();
    STAMP_AT(stamp++);

    if (J.kind == JOB_S) {
      const u32 slot = (u32)(box_base(J.dO) + box_idx(J.ja, J.p));
      u32x4 v[8];
#pragma unroll
      for (int jj = 0; jj < 8; ++jj) {
        const f32x4 u0 = acc[jj >> 1][(jj & 1) * 2], u1 = acc[jj >> 1][(jj & 1) * 2 + 1];
        v[jj].x = pack_bf16(u0[0], u0[1]); v[jj].y = pack_bf16(u0[2], u0[3]); v[jj].z = pack_bf16(u1[0], u1[1]); v[jj].w = pack_bf16(u1[2], u1[3]);
      }
      const u32 so = OFF_IN + slot * 32768u + (u32)wave * 8192u, fo = OFF_PF + slot * 512u + (u32)wave * 128u;
      if (J.dO >= 2) {
#pragma unroll
        for (int jj = 0; jj < 8; ++jj) st16<0>(v[jj], rl, vo, so + (u32)(jj * 1024));
        WAIT_VM(0);
        if (lane == 0) st_sys(P.ws[J.dO] + fo, call);
        STAMP_AT(stamp++);
      } else {
        if (!EARLYBOOK) book(bk, rl, OFF_TP);
        u32x4 hd; hd.x = 0; hd.y = call; hd.z = 0; hd.w = call;
#pragma unroll
        for (int jj = 0; jj < 8; ++jj) {
          const u64 mk = __ballot(v[jj].x & 1u);
          hd.x = lane == jj ? (u32)mk : hd.x; hd.z = lane == jj ? (u32)(mk >> 32) : hd.z;
          v[jj].x = (v[jj].x & ~1u) | (call & 1u);
        }
        asm volatile("s_waitcnt vmcnt(0)" : "+v"(bk) :: "memory");
        STAMP_AT(stamp++);
        const u32x4 rd = mk_rsrc4(P.ws[1], WS_USED);
        sleep_until(rfl((u32)(bk >> 4)));
        if (lane < 8) st16<0>(hd, rd, vo, fo);
#pragma unroll
        for (int jj = 0; jj < 8; ++jj) {
          st16<0>(v[jj], rd, vo, so + (u32)(jj * 1024));
          wait_vm<CAP - 1>();
        }
      }
      STAMP_AT(stamp++);
    } else {
      fjob = jobs & 0x3ffu;
#pragma unroll
      for (int m = 0; m < 4; ++m)
#pragma unroll
        for (int n = 0; n < 4; ++n) fin[m][n] = acc[m][n];
    }
  }
  if (!fjob) return;

  if (threadIdx.x < 2) ((LAS u32*)lds)[threadIdx.x] = 0;
  BARRIER();
  const Job J = decode_job(fjob, a);
  const __amdgpu_buffer_rsrc_t rws = mk_rsrc(ws, WS_USED);
  u32 n = 0;
  char* outp = P.out;
  asm volatile("" : "+s"(outp));
  const __amdgpu_buffer_rsrc_t ro = mk_rsrc(outp, OUT_BYTES);
  {
    float mx = 0.f;
    auto add = [&](int jj, u32 x_lo, u32x4 c) {
      f32x4& u0 = fin[jj >> 1][(jj & 1) * 2]; f32x4& u1 = fin[jj >> 1][(jj & 1) * 2 + 1];
      u0 += (f32x4){as_f32(x_lo), bf_hi(c.x), bf_lo(c.y), bf_hi(c.y)};
      u1 += (f32x4){bf_lo(c.z), bf_hi(c.z), bf_lo(c.w), bf_hi(c.w)};
    };
    auto done = [&](int jj) {
      f32x4& u0 = fin[jj >> 1][(jj & 1) * 2]; f32x4& u1 = fin[jj >> 1][(jj & 1) * 2 + 1];
#pragma unroll
      for (int e = 0; e < 4; ++e) mx = fmaxf(mx, fmaxf(u0[e], u1[e]));
      asm volatile("" : "+v"(u0), "+v"(u1), "+v"(mx));
    };
    if (J.kind == JOB_F) {
      const bool far = (J.ja >> 2) != 0;
      const int d0 = far ? 4 | ((a ^ J.sub) & 2) : 2;
      const u32 bi = far ? (u32)(J.p * 2 + (J.sub & 1)) : (u32)J.p;
      const u32 s0 = (u32)box_base(d0) + bi, s1 = (u32)box_base(d0 + 1) + bi;
      const __amdgpu_buffer_rsrc_t r0 = mk_rsrc(P.ws[d0], WS_USED), r1 = mk_rsrc(P.ws[d0 + 1], WS_USED);
      const u32 o = OFF_GO + (far ? 0u : 128u);
      wave_wait<SL_GO>(ws, o + 4u, lane == 0, call, lane);
      u32 t_go = rfl(__hip_atomic_load((GAS const u32*)(ws + o), __ATOMIC_RELAXED, __HIP_MEMORY_SCOPE_AGENT));
      const u32 round = (u32)(!far ? J.p >> 3 : J.p < 8 ? J.sub & 1 : J.p >> 2);
      const u32 seat = (u32)((!far || J.p < 8 ? J.p & 7 : (J.p & 3) | (J.sub & 1) << 2) + wave * 8);
      const u32 tok = OFF_TOK + (((u32)d0 * 2u + round) * 32u + seat) * 64u;
      const u32 step = far ? 4 * TAU_X : 4 * TAU_C;
      t_go += (round * 8u * step + seat * (step >> 5)) >> 4;
      sleep_until(t_go - 50u);
      wave_wait<4>(ws, lane == 2 ? tok - 32u * 64u : OFF_PF + (lane == 0 ? s0 : s1) * 512u + (u32)wave * 128u, lane < (round ? 3 : 2), call, lane);
      const u32 so0 = OFF_IN + s0 * 32768u + (u32)wave * 8192u, so1 = OFF_IN + s1 * 32768u + (u32)wave * 8192u;
      u32x4 v0[8], v1[8];
      u32 t_min = t_go;
#pragma unroll
      for (int jj = 0; jj < 8 + PD; ++jj) {
        if (jj < 8) {
          const u32 ti = t_go + (((u32)jj * step) >> 4);
          sleep_until((int)(ti - t_min) > 0 ? ti : t_min);
          t_min = (u32)__builtin_amdgcn_s_memrealtime() + ((step * CATCH) >> 8);
          v0[jj] = __builtin_amdgcn_raw_buffer_load_b128(r0, vo + (u32)((jj & 3) * 1024), so0 + (u32)((jj >> 2) * 4096), 17);
          v1[jj] = __builtin_amdgcn_raw_buffer_load_b128(r1, vo + (u32)((jj & 3) * 1024), so1 + (u32)((jj >> 2) * 4096), 17);
          if (jj == 6 && lane == 0) st_agent(ws + tok, call);
        }
        if (jj >= PD) { add(jj - PD, v0[jj - PD].x << 16, v0[jj - PD]); add(jj - PD, v1[jj - PD].x << 16, v1[jj - PD]); done(jj - PD); }
      }
    } else {
      const u32 s0 = (u32)(J.p * 3 + J.sub);
      const u64 t0 = __builtin_amdgcn_s_memrealtime();
      const u32 hoff = lane < 8 ? OFF_PF + s0 * 512u + (u32)wave * 128u + (u32)lane * 16u : OFF_TMO;
      const u32 so0 = OFF_IN + s0 * 32768u + (u32)wave * 8192u;
      auto late = [&](bool dead) {
        if (!dead && __builtin_amdgcn_s_memrealtime() - t0 <= DEADLINE_TICKS) return false;
        if (lane == 0) st_sys(ws + OFF_TMO, call);
        WAIT_VM(0);
        return true;
      };
      u32x4 v0[8];
      auto fetch = [&](int jj) { v0[jj] = __builtin_amdgcn_raw_buffer_load_b128(rws, vo + (u32)((jj & 3) * 1024), so0 + (u32)((jj >> 2) * 4096), 17); };
      u32x4 hd = __builtin_amdgcn_raw_buffer_load_b128(rws, hoff, 0, 17);
#pragma unroll
      for (int jj = 0; jj < 8; ++jj) fetch(jj);
      while (!__all(lane >= 8 || (hd.y >= call && hd.w >= call))) {
        if (late((u32)__builtin_amdgcn_readlane(hd.x, 63) == call)) break;
        __builtin_amdgcn_s_sleep(SL_HDR);
        asm volatile("" ::: "memory");
        hd = __builtin_amdgcn_raw_buffer_load_b128(rws, hoff, 0, 17);
      }
#pragma unroll
      for (int jj = 0; jj < 8; ++jj) {
        while (__builtin_expect(!__all(((v0[jj].x ^ call) & 1u) == 0), 0)) {
          asm volatile("" ::: "memory");
#pragma unroll
          for (int k = jj; k < jj + LOOK && k < 8; ++k) fetch(k);
          if (late(false)) break;
        }
        const u64 m = (u64)(u32)__builtin_amdgcn_readlane(hd.z, jj) << 32 | (u32)__builtin_amdgcn_readlane(hd.x, jj);
        add(jj, (v0[jj].x << 16 & ~0x10000u) | (__builtin_amdgcn_inverse_ballot_w64(m) ? 0x10000u : 0u), v0[jj]);
        done(jj);
      }
    }
    TSTAMP(10);
    const u32 wm = wave_max(__builtin_bit_cast(u32, mx));
    if (lane == 0) {
      __hip_atomic_fetch_max((LAS u32*)lds, wm, __ATOMIC_RELAXED, __HIP_MEMORY_SCOPE_WORKGROUP);
      n = __hip_atomic_fetch_add((LAS u32*)lds + 1, 1u, __ATOMIC_RELAXED, __HIP_MEMORY_SCOPE_WORKGROUP);
      const u32 tm = __hip_atomic_load((LAS u32*)lds, __ATOMIC_RELAXED, __HIP_MEMORY_SCOPE_WORKGROUP);
      if (n == 3u)
        __hip_atomic_store((GAS u64*)(ws + OFF_LM + (u32)(J.p * 8 + (J.kind == JOB_F ? J.ja : J.sub)) * 128u), ((u64)call << 32) | tm, __ATOMIC_RELAXED, __HIP_MEMORY_SCOPE_AGENT);
    }
  }
  TSTAMP(11);
  if (g & 1) {
#pragma unroll
    for (int m = 0; m < 4; ++m) { f32x4 t = fin[m][0]; fin[m][0] = fin[m][1]; fin[m][1] = t; t = fin[m][2]; fin[m][2] = fin[m][3]; fin[m][3] = t; }
  }
  u32 oo[4];
#pragma unroll
  for (int m = 0; m < 4; ++m) { oo[m] = (u32)(J.sub * 128 + wr * 64 + m * 16 + fr) * 4096u + J.col0 + (u32)(wc * 64 + g * 16); asm volatile("" : "+v"(oo[m])); }
  Polls pq = {};
  if (rfl(n) == 3u) {
    const u32 gm = gather_max(ws, lane < NR ? OFF_AM + (u32)lane * 128u : 0x80000000u, call, lane, LZ_F, pq);
    if (lane == 0) ((LAS u32*)lds)[2] = gm;
  }
  BARRIER();
  const u32 gmax = rfl(((LAS u32*)lds)[2]);
  TSTAMP(12);
  const float inv = gmax ? 127.0f * __builtin_amdgcn_rcpf(as_f32(gmax)) : 0.f;
#pragma unroll
  for (int m = 0; m < 4; ++m) {
    u32x4 v;
#pragma unroll
    for (int n = 0; n < 4; ++n) {
      f32x4 y = fin[m][n] * inv;
      asm volatile("" : "+v"(y));
      u32 r = 0;
#pragma unroll
      for (int e = 0; e < 4; ++e) r = __builtin_amdgcn_cvt_pk_u8_f32(y[e], e, r);
      v[n] = r;
    }
    if (gmax == ~0u) v = (u32x4){0x80808080u, 0x80808080u, 0x80808080u, 0x80808080u};
    __builtin_amdgcn_raw_buffer_store_b128(v, ro, oo[m], 0, OUT_AUX);
  }
  pq.retire();
  TSTAMP(13);
}

__global__ void __launch_bounds__(512, 2) gemm_rs_q8(const Params P) {
  __shared__ __attribute__((aligned(16))) char lds[5 * 32768];
  const int lane = threadIdx.x & 63, wave = (int)rfl(threadIdx.x >> 6);
  char* ws = P.ws[0];
  if (__builtin_expect(blockIdx.x < NPUSH, 1)) {
    if (wave < PW) pusher(P, lds, ws, lane, wave);
    return;
  }
  const u32 call = draw_call(ws, lane, wave);
  if (blockIdx.x == GRID - 1) {
    if (wave) return;
    coordinator(P, ws, lane, call);
    char* dst = P.ws[lane & 7] + OFF_AM + (u32)P.rank * 128u;
    asm volatile("" : "+v"(dst));
    Polls pq;
    u32 m = 0;
    for (u32 half = 0; half < 2; ++half) m = max(m, gather_max(ws, OFF_LM + (half * 64u + (u32)lane) * 128u, call, lane, LZ_C, pq));
    if (lane < NR) __hip_atomic_store((GAS u64*)dst, ((u64)call << 32) | m, __ATOMIC_RELAXED, __HIP_MEMORY_SCOPE_SYSTEM);
    pq.retire();
    return;
  }
  const u32 jobs = P.jobs[blockIdx.x];
  if (wave >= 4) loader(P, lds, ws, lane, wave, call, jobs);
  else computer(P, lds, ws, lane, wave, call, jobs);
}

#define LAG 12
#define R_FWD 2
#define R_EARLY 6
void build_schedule(Params& P) {
#define RAMP 4
#define SHIFT2 2
#define ORD 0
  static const int orders[5][32] = {
    {0, 1, 2, 16, 17, 5, 18, 6, 19, 7, 20, 8, 21, 22, 23, 9, 10, 11, 12, 24, 25, 26, 27, 4, 3, 28, 29, 30, 31, 13, 14, 15},
    {0, 1, 2, 16, 17, 5, 18, 6, 19, 7, 20, 8, 21, 22, 23, 24, 9, 25, 10, 11, 26, 12, 27, 4, 3, 28, 29, 30, 31, 13, 14, 15},
    {0, 1, 2, 16, 17, 5, 18, 6, 19, 7, 20, 8, 21, 22, 23, 24, 25, 9, 10, 26, 11, 12, 27, 4, 3, 28, 29, 30, 31, 13, 14, 15},
    {0, 1, 2, 16, 17, 18, 5, 19, 6, 20, 7, 21, 8, 22, 23, 24, 9, 25, 10, 26, 11, 12, 27, 4, 3, 28, 29, 30, 31, 13, 14, 15},
    {0, 1, 2, 16, 17, 5, 18, 6, 19, 7, 20, 8, 21, 22, 23, 24, 9, 25, 10, 11, 26, 12, 4, 27, 3, 28, 29, 30, 31, 13, 14, 15}};
  const int* order = orders[ORD];
  int pos[32];
  for (int i = 0; i < 32; ++i) pos[order[i]] = i;
  auto ready = [&](int t, int p) { return pos[t] > pos[16 + p] ? pos[t] : pos[16 + p]; };
  for (int d = 0; d < NR; ++d) {
    unsigned char list[MAXU]; int n = 0;
    for (int i = 0; i < MAXU; ++i) list[i] = 0;
    const int q = P.rank ^ d;
    auto relay = [&](int c) { return c >= 16 + R_END - 2 * RELAY && c < 16 + R_END; };
    if (d >= 2)
      for (int i = 0, nh1 = 0, lag = ((2 * (d - (d >= 4 ? 4 : 2)) + 1) * LAG + 10) / 20; i < 64; ++i) {
        auto put = [&](int k, int h) { const int c = order[k]; if (!(c < 16 && tile_dist(c) == d)) list[n++] = (unsigned char)(c * 2 + h); };
        if (i < 32) put(i, 0);
        for (; nh1 < 32 && nh1 + (nh1 * RAMP / 8 < lag ? nh1 * RAMP / 8 : lag) <= i; ++nh1)
          if (d < 4 || !relay(order[nh1])) put(nh1, 1);
        if (d >= 4 && i + R_EARLY < 32 && relay(order[i + R_EARLY]) && ((q >> 1) & 1) == (order[i + R_EARLY] & 1)) put(i + R_EARLY, 1);
        if (d < 4 && i >= R_FWD && i - R_FWD < 32 && relay(order[i - R_FWD]) && ((P.rank >> 1) & 1) == (order[i - R_FWD] & 1))
          for (int ds = 4; ds < 8; ++ds)
            if (((P.rank ^ ds) & 1) == (d & 1)) list[n++] = (unsigned char)(ds * 16 + order[i - R_FWD] - 16);
      }
    if (OWNLAST && d < 4) {
      unsigned char tmp[MAXU]; int m = 0;
      for (int pass = 0; pass < 2; ++pass)
        for (int i = 0; i < n; ++i) if ((list[i] >= 26 && list[i] < 32) == (pass == 1)) tmp[m++] = list[i];
      for (int i = 0; i < n; ++i) list[i] = tmp[i];
    }
    P.ucount[d] = (u32)n;
    for (int i = 0; i < MAXU / 4; ++i) P.units[d][i] = list[4 * i] | list[4 * i + 1] << 8 | list[4 * i + 2] << 16 | (u32)list[4 * i + 3] << 24;
  }
  for (int t = 0; t < 16; ++t) {
    P.relayed[t] = 0;
    for (int p = R_END - 2 * RELAY; p < R_END; ++p)
      if (((P.rank >> 1) & 1) != (p & 1) && t < 13 && pos[16 + p] + R_FWD > pos[t]) P.relayed[t] |= 1u << p;
  }
  static const int kord[2][12] = {{10, 8, 6, 4, 5, 7, 9, 11, 2, 0, 1, 3}, {10, 8, 6, 4, 2, 0, 1, 3, 5, 7, 9, 11}};
  for (int c = 0; c < 2; ++c)
    for (int sk = 0; sk < 8; ++sk) {
      u64 v = 0;
      for (int i = 0, n = 0; i < 12; ++i) if ((kord[c][i] >> 1) != sk) v |= (u64)kord[c][i] << (4 * n++);
      P.korder[c][sk] = v;
    }
  for (int b = 0; b < NPUSH; ++b)
    for (int w = 0; w < PW; ++w) {
      const int d = push_link(b, w);
      auto code = [&](int i) { return (P.units[d][i >> 4] >> ((i & 12) * 2)) & 0xffu; };
      const int i = push_seat(b, w);
      P.first[b][2 * w] = slice_src(code(i), (u32)i & 3u, P.rank ^ d, d) | code(i) >> 5;
      P.first[b][2 * w + 1] = code(i) | code(i + NW) << 8 | (u32)(P.rank ^ d) << 16;
    }
  static const int grp[16] = {0, 1, 0, 2, 1, 1, 0, 1, 0, 1, 0, 1, 0, 2, 2, 2};
  for (int i = 0; i < GRID; ++i) P.jobs[i] = 0;
  for (int x = 0; x < 8; ++x) {
    int ht[40], hp[40], n = 0;
    for (int r = 0; r < 32; ++r)
      for (int p = 0; p < 16; ++p)
        for (int t = 0; t < 16; ++t) {
          const int gsel = grp[t] == 2 ? (p >> 2) & 1 : grp[t];
          if (gsel == (x >> 2) && ((p + (t == 2 ? SHIFT2 : 0)) & 3) == (x & 3) && ready(t, p) == r) { ht[n] = t; hp[n] = p; ++n; }
        }
    auto enc = [&](int i) { return (u32)(((ht[i] >= 13 ? JOB_O : JOB_S) << 8) | (ht[i] << 4) | hp[i]); };
    const int nw = 32 - NPUSH / 8 - (x == 7);
    const int hi = x >> 2, lo = x & 3;
    const int fsub[10] = {4, 6, 5, 7, 3, 4 + hi, 6 + hi, 3, 5 - hi, 7 - hi};
    const int fpan[10] = {4 * hi + lo, 4 * hi + lo, 4 - 4 * hi + lo, 4 - 4 * hi + lo, x, 8 + lo, 8 + lo, 8 + x, 12 + lo, 12 + lo};
    int f = 0, extra = nw;
    for (int s = 0; s < nw; ++s) {
      u32 jb = enc(s);
      const bool host = extra < n && (ht[s] < 3 ? hp[s] >= PURE && hp[s] < 8 : f >= 4 && ht[s] < 13);
      if (host) { jb = (u32)((JOB_F << 8) | ((ht[extra] - 13) << 4) | hp[extra]) | jb << 10 | enc(extra) << 20; ++extra; }
      else if (ht[s] >= 3 && ht[s] < 13 && f < 10) {
        jb = jb << 10 | (u32)((JOB_F << 8) | (fsub[f] << 4) | fpan[f]); ++f;
      }
      P.jobs[NPUSH + s * 8 + x] = jb;
    }
  }
}

}

extern "C" void kernel_launch(int rank, int nranks, ncclComm_t comm, hipStream_t stream,
                              void* const* d_in, const int* in_sizes, int n_in,
                              void* d_out, int out_size,
                              const int* in_dtypes, int out_dtype,
                              void* const* d_out_all, void* const* d_ws_all, size_t ws_bytes) {
  (void)comm; (void)in_dtypes; (void)out_dtype; (void)d_out_all;
  if (nranks != NR || (unsigned)rank >= (unsigned)NR || n_in != 2 || ws_bytes < WS_USED || (u32)out_size != OUT_BYTES ||
      (u32)in_sizes[0] * 2u != X_BYTES || (u32)in_sizes[1] * 2u != W_BYTES) {
    fprintf(stderr, "unexpected problem shape\n");
    return;
  }
  Params P;
  P.x = (const char*)d_in[0]; P.w = (const char*)d_in[1]; P.out = (char*)d_out;
  for (int d = 0; d < NR; ++d) P.ws[d] = (char*)d_ws_all[rank ^ d];
  P.rank = rank; P.pad = 0;
  build_schedule(P);
  hipLaunchKernelGGL(gemm_rs_q8, dim3(GRID), dim3(512), 0, stream, P);
  HIP_CHECK(hipGetLastError());
}
```

```cpp
#include <hip/hip_runtime.h>
#include <rccl/rccl.h>
#include <cstdio>
#include <cstdint>

#define HIP_CHECK(expr) do { hipError_t _e = (expr); if (_e != hipSuccess) { \
    fprintf(stderr, "HIP error %s at %s:%d\n", hipGetErrorString(_e), __FILE__, __LINE__); \
    return; } } while (0)

namespace {

typedef unsigned u32;
typedef unsigned long long u64;
typedef u32 u32x4 __attribute__((ext_vector_type(4)));
typedef u32 u32x2 __attribute__((ext_vector_type(2)));
typedef float f32x4 __attribute__((ext_vector_type(4)));
typedef short s16x4 __attribute__((ext_vector_type(4)));
typedef short s16x8 __attribute__((ext_vector_type(8)));
typedef __bf16 bf16x8 __attribute__((ext_vector_type(8)));
#define LAS __attribute__((address_space(3)))
#define GAS __attribute__((address_space(1)))

constexpr int NR = 8;
#define NPUSH 32
#define PW 3
constexpr int NW = NPUSH * PW / 6;
constexpr int GRID = 256;
#define RELAY 4
constexpr int MAXU = 72;
constexpr u64 DEADLINE_TICKS = 100000000ull;

constexpr u32 X_BYTES = 4096u * 512u * 2u, W_BYTES = 512u * 4096u * 2u, OUT_BYTES = 512u * 4096u;
constexpr u32 OFF_CNT = 0;
constexpr u32 OFF_TMO = 16u << 10;
constexpr u32 OFF_TK = (16u << 10) + 128u;
constexpr u32 OFF_TP = 18u << 10;
constexpr u32 OFF_T0 = 19u << 10;
constexpr u32 OFF_XF = 20u << 10;
constexpr u32 OFF_WF = 24u << 10;
constexpr u32 OFF_TR = 28u << 10;
constexpr u32 OFF_GO = 29u << 10;
constexpr u32 OFF_AM = 32u << 10;
constexpr u32 OFF_LM = 64u << 10;
constexpr u32 OFF_TOK = 128u << 10;
constexpr u32 OFF_PF = 192u << 10;
constexpr u32 OFF_DBG = 512u << 10;
constexpr u32 OFF_XS = 2u << 20;
constexpr u32 OFF_WS = 8u << 20;
constexpr u32 OFF_IN = 14u << 20;
constexpr u32 WS_USED = OFF_IN + 208u * 32768u;
#define TAU_X 221
#define TAU 232
#define TAU_C 224
#define PD 3
#define T_BOOK 70
#define T_REST 700
#define OUT_AUX 0
#define SL_LONG 32
#define SL_GATE 6
#define SL_AM 3
#define SL_HDR 4
#define WDELAY 150
#define SL_AML 16
#define T_FLY 80
#define SL_GO 16
#define LASTW WIN
#define CATCH 10
#define R_END 16
#define PURE 4
#define OWNLAST 1
#define LZ_F 7
#define LZ_C 40
#define T_RESTC T_REST
#define TOUCH 1
#define TDELAY 50
#define CAP 6
constexpr u32 PANEL_B = 6u * 65536u;

enum { JOB_NONE = 0, JOB_S = 1, JOB_O = 2, JOB_F = 3 };

struct Params {
  const char* x; const char* w; char* out;
  char* ws[NR];
  int rank; int pad;
  u32 jobs[GRID];
  u32 first[NPUSH][2 * PW];
  u32 ucount[NR];
  u32 units[NR][MAXU / 4];
  u64 korder[2][8];
  u32 relayed[16];
};

__host__ __device__ inline int tile_dist(int t) { return t < 3 ? 1 : t < 5 ? t - 1 : t < 13 ? 4 + ((t - 5) & 3) : 0; }
__host__ __device__ inline int tile_sub(int t, int a) { return t < 3 ? t : t < 5 ? 3 : t < 13 ? (a & 2) + ((t - 5) >> 2) : t - 13; }
__host__ __device__ inline int box_base(int d) { return d == 1 ? 0 : d < 4 ? 48 + 16 * (d - 2) : 80 + 32 * (d - 4); }
__host__ __device__ inline int box_cnt(int d) { return d == 1 ? 48 : d < 4 ? 16 : 32; }
__host__ __device__ inline int box_idx(int t, int p) { return t < 3 ? p * 3 + t : t < 5 ? p : p * 2 + ((t - 5) >> 2); }

__host__ __device__ inline u32 link_units(int d) { return d < 4 ? 62u + 2u * RELAY : 60u - RELAY; }
__host__ __device__ inline int push_link(int block, int wave) { return 2 + ((block >> 3) * PW + wave + (block & 7)) % 6; }
__host__ __device__ inline int push_seat(int block, int wave) { return ((block >> 3) * PW + wave) / 6 * 8 + (block & 7); }
__host__ __device__ inline u32 k_half(int q, int d) { return (u32)(q ^ (d >= 4 ? (q >> 1) & (q ^ d) : 0)) & 1u; }
__host__ __device__ inline u32 slice_src(u32 code, u32 sl, int q, int d) {
  const u32 h = code & 1u, kh = k_half(q, d);
  if (code < 32) {
    const int t = code >> 1;
    return ((u32)((q ^ tile_dist(t)) * 512 + tile_sub(t, q) * 128) + (sl & 1u) * 64u) * 1024u + (kh * 256u + h * 128u + (sl >> 1) * 64u) * 2u;
  }
  return (kh * 256u + h * 128u + sl * 32u) * 8192u + ((u32)(q >> 2) * 2048u + ((code - 32u) >> 1) * 128u) * 2u;
}

__device__ __forceinline__ u32 rfl(u32 v) { return __builtin_amdgcn_readfirstlane(v); }
__device__ __forceinline__ __amdgpu_buffer_rsrc_t mk_rsrc(const void* p, u32 bytes) {
  return __builtin_amdgcn_make_buffer_rsrc((void*)p, 0, bytes, 0x00020000);
}
__device__ __forceinline__ u32x4 mk_rsrc4(const void* p, u32 bytes) {
  u64 a = (u64)p; u32x4 r; r.x = rfl((u32)a); r.y = rfl((u32)(a >> 32)) & 0xffffu; r.z = bytes; r.w = 0x00020000u; return r;
}
__device__ __forceinline__ u32 ld_sys(const char* p) { return __hip_atomic_load((GAS const u32*)p, __ATOMIC_RELAXED, __HIP_MEMORY_SCOPE_SYSTEM); }
__device__ __forceinline__ void st_sys(char* p, u32 v) { __hip_atomic_store((GAS u32*)p, v, __ATOMIC_RELAXED, __HIP_MEMORY_SCOPE_SYSTEM); }
__device__ __forceinline__ void st_agent(char* p, u32 v) { __hip_atomic_store((GAS u32*)p, v, __ATOMIC_RELAXED, __HIP_MEMORY_SCOPE_AGENT); }
#define WAIT_VM(n) asm volatile("s_waitcnt vmcnt(" #n ")" ::: "memory")
#define BARRIER() do { asm volatile("" ::: "memory"); __builtin_amdgcn_s_waitcnt(0xC07F);   __builtin_amdgcn_s_barrier(); asm volatile("" ::: "memory"); } while (0)

#define TSTAMP(i) do {} while (0)
#define STAMP_AT(i) do {} while (0)

template <int SL> __device__ __forceinline__ void wave_wait(char* ws, u32 off, bool care, u32 call, int lane) {
  if (lane == 63) { off = OFF_TMO; }
  else if (!care) { off = OFF_TMO; }
  const u64 t0 = __builtin_amdgcn_s_memrealtime();
  for (;;) {
    const u32 v = ld_sys(ws + off);
    const bool ok = (lane == 63 || !care) ? true : (v >= call);
    if (__all(ok)) break;
    const bool dead = __any(lane == 63 && v == call);
    if (dead || __builtin_amdgcn_s_memrealtime() - t0 > DEADLINE_TICKS) {
      if (lane == 0) st_sys(ws + OFF_TMO, call);
      WAIT_VM(0);
      break;
    }
    __builtin_amdgcn_s_sleep(SL);
  }
}

__device__ __forceinline__ int wave_poll4(char* ws, __amdgpu_buffer_rsrc_t rws, u32 off, bool care, int have, bool block, u32 call, int lane) {
  if (!care) off = OFF_TMO;
  const u64 t0 = __builtin_amdgcn_s_memrealtime();
  for (;;) {
    const u32x4 v = __builtin_amdgcn_raw_buffer_load_b128(rws, off, 0, 17);
    asm volatile("" ::: "memory");
    const u32 m = (u32)__ballot(care && v.x >= call && v.y >= call && v.z >= call && v.w >= call);
    const int n = __builtin_ctz(~(m & (m >> 16)));
    if (n > have || !block) return n;
    const bool dead = __any(!care && v.x == call);
    if (dead || __builtin_amdgcn_s_memrealtime() - t0 > DEADLINE_TICKS) {
      if (lane == 0) st_sys(ws + OFF_TMO, call);
      WAIT_VM(0);
      return have + 1;
    }
    if (have == 0) __builtin_amdgcn_s_sleep(SL_LONG); else __builtin_amdgcn_s_sleep(SL_GATE);
  }
}
#define DMA_SC " sc0 sc1"

typedef __bf16 bf16x2 __attribute__((ext_vector_type(2)));
__device__ __forceinline__ u32 pack_bf16(float lo, float hi) { bf16x2 r; r.x = (__bf16)lo; r.y = (__bf16)hi; return __builtin_bit_cast(u32, r); }
__device__ __forceinline__ float as_f32(u32 v) { return __builtin_bit_cast(float, v); }
__device__ __forceinline__ float bf_lo(u32 v) { return __builtin_bit_cast(float, v << 16); }
__device__ __forceinline__ float bf_hi(u32 v) { return __builtin_bit_cast(float, v & 0xffff0000u); }
__device__ __forceinline__ u32 wave_max(u32 m) {
  m = max(m, (u32)__builtin_amdgcn_update_dpp(0, (int)m, 0x111, 0xf, 0xf, false));
  m = max(m, (u32)__builtin_amdgcn_update_dpp(0, (int)m, 0x112, 0xf, 0xf, false));
  m = max(m, (u32)__builtin_amdgcn_update_dpp(0, (int)m, 0x114, 0xf, 0xf, false));
  m = max(m, (u32)__builtin_amdgcn_update_dpp(0, (int)m, 0x118, 0xf, 0xf, false));
  const u32 a = max((u32)__builtin_amdgcn_readlane(m, 15), (u32)__builtin_amdgcn_readlane(m, 31));
  const u32 b = max((u32)__builtin_amdgcn_readlane(m, 47), (u32)__builtin_amdgcn_readlane(m, 63));
  return max(a, b);
}

__device__ __forceinline__ void dma4(u32 lds_dst, u32 v0, u32 v1, u32 v2, u32 v3, u32x4 rs, u32 so, u32 st) {
  u32 keep, t;
  asm volatile(
      "s_mov_b32 %0, m0\n\t"
      "s_mov_b32 m0, %2\n\ts_add_u32 %1, %8, %9\n\ts_nop 1\n\tbuffer_load_dwordx4 %3, %7, %8 offen" DMA_SC " lds\n\t"
      "s_add_u32 m0, %2, 0x400\n\ts_nop 0\n\tbuffer_load_dwordx4 %4, %7, %1 offen" DMA_SC " lds\n\t"
      "s_add_u32 m0, %2, 0x800\n\ts_add_u32 %1, %1, %9\n\tbuffer_load_dwordx4 %5, %7, %1 offen" DMA_SC " lds\n\t"
      "s_add_u32 m0, %2, 0xc00\n\ts_add_u32 %1, %1, %9\n\tbuffer_load_dwordx4 %6, %7, %1 offen" DMA_SC " lds\n\t"
      "s_mov_b32 m0, %0"
      : "=&s"(keep), "=&s"(t)
      : "s"(lds_dst), "v"(v0), "v"(v1), "v"(v2), "v"(v3), "s"(rs), "s"(so), "s"(st)
      : "memory", "scc");
}

__device__ __forceinline__ void dma8_lin(u32 lds_dst, u32 v, u32x4 rs, u32 sx, u32 sw) {
  u32 keep;
  asm volatile(
      "s_mov_b32 %0, m0\n\ts_mov_b32 m0, %1\n\ts_nop 2\n\t"
      "buffer_load_dwordx4 %2, %3, %4 offen" DMA_SC " lds\n\tbuffer_load_dwordx4 %2, %3, %4 offen offset:1024" DMA_SC " lds\n\t"
      "buffer_load_dwordx4 %2, %3, %4 offen offset:2048" DMA_SC " lds\n\tbuffer_load_dwordx4 %2, %3, %4 offen offset:3072" DMA_SC " lds\n\t"
      "s_add_u32 m0, %1, 0x4000\n\ts_nop 0\n\t"
      "buffer_load_dwordx4 %2, %3, %5 offen" DMA_SC " lds\n\tbuffer_load_dwordx4 %2, %3, %5 offen offset:1024" DMA_SC " lds\n\t"
      "buffer_load_dwordx4 %2, %3, %5 offen offset:2048" DMA_SC " lds\n\tbuffer_load_dwordx4 %2, %3, %5 offen offset:3072" DMA_SC " lds\n\t"
      "s_mov_b32 m0, %0"
      : "=&s"(keep) : "s"(lds_dst), "v"(v), "s"(rs), "s"(sx), "s"(sw) : "memory", "scc");
}

#define WIN 4
template <int N> __device__ __forceinline__ void wait_vm() { asm volatile("s_waitcnt vmcnt(%0)" :: "n"(N) : "memory"); }
template <int IMM> __device__ __forceinline__ void st16(u32x4 v, u32x4 rs, u32 voff, u32 soff) {
  asm volatile("buffer_store_dwordx4 %0, %1, %2, %3 offen offset:%4 sc0 sc1\n\ts_nop 1" :: "v"(v), "v"(voff), "s"(rs), "s"(soff), "n"(IMM) : "memory");
}
__device__ __forceinline__ void draw_async(u32& v, u32x4 rl, u32 off) {
  u64 ex; v = 1;
  asm volatile("s_mov_b64 %1, exec\n\ts_mov_b64 exec, 1\n\tbuffer_atomic_add %0, off, %2, %3 sc0\n\ts_mov_b64 exec, %1"
               : "+v"(v), "=&s"(ex) : "s"(rl), "s"(off) : "memory");
}
__device__ __forceinline__ void book(u64& tk, u32x4 rl, u32 off) {
  u64 ex; const u64 lo = (__builtin_amdgcn_s_memrealtime() + T_BOOK) * 16u;
  asm volatile("s_mov_b64 %1, exec\n\ts_mov_b64 exec, 1\n\tbuffer_atomic_umax_x2 %2, off, %3, %4\n\tbuffer_atomic_add_x2 %0, off, %3, %4 sc0\n\ts_mov_b64 exec, %1"
               : "+v"(tk), "=&s"(ex) : "v"(lo), "s"(rl), "s"(off) : "memory");
}
__device__ __forceinline__ void sleep_until(u32 due) {
  for (;;) {
    const u32 left = due - (u32)__builtin_amdgcn_s_memrealtime() - 1u;
    if (left >= 10000u) break;
    if (left > 200u) __builtin_amdgcn_s_sleep(48); else if (left > 40u) __builtin_amdgcn_s_sleep(8); else __builtin_amdgcn_s_sleep(1);
  }
}
struct Slice { u32 src, dst, flag, kind, dep; };
__device__ __forceinline__ u32 draw_call(char* ws, int lane, int wave) {
  u32 call = 0;
  if (lane == 0) call = __hip_atomic_fetch_add((GAS u32*)(ws + OFF_CNT + blockIdx.x * 64 + wave * 4), 1u, __ATOMIC_RELAXED, __HIP_MEMORY_SCOPE_AGENT) + 1u;
  return rfl(call);
}
__device__ __forceinline__ void pusher(const Params& P, char* lds, char* ws, int lane, int wave) {
  const u32 vx0 = (u32)(lane >> 3) * 1024u + (u32)(((lane & 7) ^ (lane >> 4)) * 16), vx1 = vx0 ^ 64u;
  const u32 vw0 = (u32)(lane >> 4) * 8192u + (u32)(((lane & 15) ^ (lane >> 4)) * 16), vw1 = vw0 ^ 128u;
  const int d = push_link(blockIdx.x, wave);
  u32 f0 = P.first[blockIdx.x][2 * wave], f1 = P.first[blockIdx.x][2 * wave + 1]; const char* xp = P.x; const char* wp = P.w;
  char* pw = P.ws[d];
  asm volatile("" : "+s"(f0), "+s"(f1), "+s"(xp), "+s"(wp), "+s"(pw), "+s"(ws));
  const u32x4 rx = mk_rsrc4(xp, X_BYTES), rw = mk_rsrc4(wp, W_BYTES), rl = mk_rsrc4(ws, WS_USED);
  const u32 vp = (u32)lane * 16u;
  const u32 ring0 = (u32)(size_t)(LAS char*)lds + (u32)wave * 16384u;
  auto load_all = [&](const Slice& c, u32 buf) {
    const u32 dst = ring0 + buf * 8192u;
    if (c.kind == 0) { dma4(dst, vx0, vx1, vx0, vx1, rx, c.src, 8192u); dma4(dst + 4096u, vx0, vx1, vx0, vx1, rx, c.src + 32768u, 8192u); }
    else if (c.kind == 1) { dma4(dst, vw0, vw0, vw1, vw1, rw, c.src, 32768u); dma4(dst + 4096u, vw0, vw0, vw1, vw1, rw, c.src + 131072u, 32768u); }
    else { dma4(dst, vp, vp, vp, vp, rl, c.src, 1024u); dma4(dst + 4096u, vp, vp, vp, vp, rl, c.src + 4096u, 1024u); }
  };
  { Slice c; c.src = f0 & ~1u; c.kind = f0 & 1u; c.dst = c.flag = c.dep = 0; load_all(c, 0); }
  u32 cnt; draw_async(cnt, rl, OFF_CNT + blockIdx.x * 64u + (u32)wave * 4u);
  const int q = (int)(f1 >> 16);
  const u32 ns = link_units(d) * 4u;
  auto decode = [&](u32 sidx, u32 code) -> Slice {
    Slice r;
    const u32 sl = sidx & 3u, h = code & 1u;
    if (code >= 64) {
      const u32 ds = code >> 4, pp = code & 15u;
      r.kind = 2;
      r.src = OFF_WS + pp * PANEL_B + (ds - 2u) * 65536u + 32768u + sl * 8192u;
      r.dep = OFF_WF + pp * 256u + (ds - 2u) * 32u + 16u + sl * 4u;
      r.dst = OFF_WS + pp * PANEL_B + ((ds ^ (u32)d) - 2u) * 65536u + 32768u + sl * 8192u;
      r.flag = OFF_WF + pp * 256u + ((ds ^ (u32)d) - 2u) * 32u + 16u + sl * 4u;
      return r;
    }
    r.kind = code >> 5; r.src = slice_src(code, sl, q, d); r.dep = OFF_TMO;
    r.dst = (code < 32 ? OFF_XS : OFF_WS) + ((code >> 1) & 15u) * PANEL_B + (u32)(d - 2) * 65536u + h * 32768u + sl * 8192u;
    r.flag = (code < 32 ? OFF_XF : OFF_WF) + ((code >> 1) & 15u) * 256u + (u32)(d - 2) * 32u + h * 16u + sl * 4u;
    return r;
  };
  const u32x4 rd = mk_rsrc4(pw, WS_USED);
  LAS char* ring = (LAS char*)lds + wave * 16384;
  u32 nxt_i = blockIdx.x;
  Slice cur = decode(blockIdx.x, f1 & 0xffu), nxt = decode(nxt_i, (f1 >> 8) & 0xffu);
  u32 buf = 0;
  asm volatile("s_waitcnt vmcnt(0)" : "+v"(cnt), "+s"(cur.dst), "+s"(nxt.src) :: "memory");
  const u32 call = rfl(cnt) + 1u;
  const u32 tk_base = (call - 1u) * (ns - NW);
  const u32 tk_off = OFF_TK + (u32)d * 128u;
  u32 pend = 0;
  auto publish = [&]() {
    if (lane == 0) {
      st_sys(pw + pend, call);
    }
  };
  auto send = [&](int k) {
    const u32x4 v = *(LAS const u32x4*)(ring + buf * 8192u + k * 1024 + lane * 16);
    st16<0>(v, rd, vp, cur.dst + (u32)k * 1024u);
  };
  for (;;) {
    u32 tk;
    send(0);
    load_all(nxt, buf ^ 1u);
    draw_async(tk, rl, tk_off);
    const u32 t_draw = (u32)__builtin_amdgcn_s_memrealtime();
    Slice nn = nxt;
    u32 dv = 0;
#pragma unroll
    for (int k = 0; k < 8; ++k) {
      if (k) send(k);
      if (k == WIN + 1) {
        asm volatile("" : "+v"(tk));
        nxt_i = rfl(tk) - tk_base;
        if (nxt_i == ns - 40u && lane == 0)
          __hip_atomic_store((GAS u64*)(ws + OFF_T0 + (u32)d * 128u), ((u64)call << 32) | (t_draw + (d < 4 ? T_RESTC : T_REST)), __ATOMIC_RELAXED, __HIP_MEMORY_SCOPE_AGENT);
        nxt_i += 2u * NW;
        if (nxt_i < ns) nn = decode(nxt_i, (P.units[d][nxt_i >> 4] >> ((nxt_i & 12u) * 2u)) & 0xffu);
        if (nxt_i < ns && nn.kind == 2) asm volatile("buffer_load_dword %0, off, %1, %2 sc0 sc1" : "+v"(dv) : "s"(rl), "s"(nn.dep) : "memory");
      }
      if (k < WIN) wait_vm<WIN - 1 + 9>(); else wait_vm<WIN - 1>();
      if (k == WIN - 2 && pend) publish();
    }
    pend = cur.flag;
    cur = nxt; nxt = nn; buf ^= 1u;
    if (nxt_i >= ns) break;
    if (nxt.kind == 2) {
      asm volatile("s_waitcnt vmcnt(%1)" : "+v"(dv) : "n"(6 - WIN) : "memory");
      if (rfl(dv) < call) wave_wait<2>(ws, nxt.dep, lane == 0, call, lane);
    }
  }
#pragma unroll
  for (int k = 0; k < 8; ++k) {
    send(k);
    wait_vm<LASTW - 1>();
    if (k == LASTW - 2) publish();
  }
  pend = cur.flag;
  WAIT_VM(0);
  publish();
}

__device__ __forceinline__ void coordinator(const Params& P, char* ws, int lane, u32 call) {
  const u32 lo = (u32)(lane & 7) * 128u, off = (lane & 8 ? OFF_TR : OFF_T0) + lo;
  char* peer = P.ws[lane & 7];
  asm volatile("" : "+v"(peer));
  const u64 t0 = __builtin_amdgcn_s_memrealtime();
  if (lane >= 1 && lane < 8) st_sys(peer + OFF_TR + lo + 4u, call);
  auto rl = [&](u32 v, int i) { return (u32)__builtin_amdgcn_readlane(v, i); };
  bool seen = lane < 2 || lane == 8 || lane >= 16;
  u32 t = 0, sent = 0;
  for (;;) {
    u64 m = 0;
    if (!seen) m = __hip_atomic_load((GAS const u64*)(ws + off), __ATOMIC_RELAXED, __HIP_MEMORY_SCOPE_SYSTEM);
    asm volatile("" : "+v"(m));
    const u64 now = __builtin_amdgcn_s_memrealtime();
    if (!seen && ((u32)(m >> 32) >= call || now - t0 > DEADLINE_TICKS)) {
      t = lane & 8 ? (u32)max((int)(u32)(now - t0) - T_FLY, 0) : (u32)m;
      seen = true;
    }
    const u32 sm = (u32)__ballot(seen);
    const u32 rdy = ((sm & 0xfff0u) == 0xfff0u ? 1u : 0u) | ((sm & 0xff0cu) == 0xff0cu ? 2u : 0u);
    if (rdy & ~sent) {
      const u32 tf = rl(t, 4) + (u32)((int)(rl(t, 5) + rl(t, 6) + rl(t, 7) - 3u * rl(t, 4)) >> 2), tn = rl(t, 2) + (u32)((int)(rl(t, 3) - rl(t, 2)) >> 1);
      const u32 lag = wave_max(lane >= 8 ? t : 0u);
      if (lane < 2 && ((rdy >> lane) & 1u))
        __hip_atomic_store((GAS u64*)(ws + OFF_GO + lo), ((u64)call << 32) | ((lane ? tn : tf) + lag), __ATOMIC_RELAXED, __HIP_MEMORY_SCOPE_AGENT);
      sent = rdy;
      if (sent == 3u) return;
    }
    __builtin_amdgcn_s_sleep(1);
  }
}

constexpr int NST = 5;

struct Job { int kind, ja, p, dO, sub, nh, nkt; u32 col0; };
__device__ __forceinline__ Job decode_job(u32 code, int a) {
  Job j; j.kind = (code >> 8) & 3; j.ja = (code >> 4) & 15; j.p = code & 15;
  int chalf;
  if (j.kind == JOB_F) { j.dO = 0; j.sub = j.ja & 3; chalf = (a >> 2) ^ (j.ja >> 2); }
  else { j.dO = tile_dist(j.ja); j.sub = tile_sub(j.ja, a); chalf = a >> 2; }
  j.col0 = (u32)(chalf * 2048 + j.p * 128);
  j.nh = j.kind == JOB_F ? 0 : j.dO >= 2 ? 10 : 12;
  j.nkt = 8 + 2 * j.nh;
  return j;
}

struct Polls { u32x2 va, vb; u32 ta, tb;
  __device__ __forceinline__ void retire() { asm volatile("" :: "v"(va), "v"(vb), "v"(ta), "v"(tb)); } };
__device__ __forceinline__ u32 gather_max(char* ws, u32 off, u32 call, int lane, int lazy, Polls& q) {
  u32x2& va = q.va; u32x2& vb = q.vb; u32& ta = q.ta; u32& tb = q.tb;
  const __amdgpu_buffer_rsrc_t rws = mk_rsrc(ws, WS_USED);
  u32 gm = 0, tmo_v;
  const u32 toff = lane ? 0x80000000u : OFF_TMO;
  const u64 t0 = __builtin_amdgcn_s_memrealtime();
  va = __builtin_amdgcn_raw_buffer_load_b64(rws, off, 0, 17);
  ta = __builtin_amdgcn_raw_buffer_load_b32(rws, toff, 0, 17);
  __builtin_amdgcn_s_sleep(SL_AM);
  for (;;) {
    asm volatile("" ::: "memory");
    vb = __builtin_amdgcn_raw_buffer_load_b64(rws, off, 0, 17); tb = __builtin_amdgcn_raw_buffer_load_b32(rws, toff, 0, 17);
    asm volatile("" ::: "memory");
    if (va.y >= call) { gm = va.x; off = 0x80000000u; }
    tmo_v = rfl(ta);
    if (__all(off == 0x80000000u)) break;
    va = __builtin_amdgcn_raw_buffer_load_b64(rws, off, 0, 17); ta = __builtin_amdgcn_raw_buffer_load_b32(rws, toff, 0, 17);
    asm volatile("" ::: "memory");
    if (vb.y >= call) { gm = vb.x; off = 0x80000000u; }
    tmo_v = rfl(tb);
    const u64 missing = __ballot(off != 0x80000000u);
    if (!missing) break;
    if (tmo_v == call || __builtin_amdgcn_s_memrealtime() - t0 > DEADLINE_TICKS) {
      if (lane == 0) st_sys(ws + OFF_TMO, call);
      WAIT_VM(0);
      tmo_v = call;
      break;
    }
    if (__builtin_popcountll(missing) > lazy) __builtin_amdgcn_s_sleep(SL_AML);
  }
  gm = wave_max(gm);
  return tmo_v == call ? ~0u : gm;
}

__device__ __forceinline__ void loader(const Params& P, char* lds, char* ws, int lane, int wave, u32 call, u32 jobs) {
  const int a = P.rank;
  const __amdgpu_buffer_rsrc_t rws = mk_rsrc(ws, WS_USED);
  const u32 l4 = (u32)(wave - 4) * 4u;
  const u32 vx0 = (u32)(lane >> 3) * 1024u + (u32)(((lane & 7) ^ (lane >> 4)) * 16), vx1 = vx0 ^ 64u;
  const u32 vw0 = (u32)(lane >> 4) * 8192u + (u32)(((lane & 15) ^ (lane >> 4)) * 16), vw1 = vw0 ^ 128u;
  const u32 vp = (u32)lane * 16u;
  const u32x4 rx = mk_rsrc4(P.x, X_BYTES), rw = mk_rsrc4(P.w, W_BYTES), rl = mk_rsrc4(ws, WS_USED);
  const u32 lds0 = (u32)(size_t)(LAS char*)lds + l4 * 1024u;
  bool fin = false;
  if (WDELAY && (jobs >> 4) - (JOB_S << 4) >= 3u) sleep_until((u32)__builtin_amdgcn_s_memrealtime() + WDELAY);
  int stamp = 1; (void)stamp;
  for (; (jobs & 0x3ffu) != 0; jobs >>= 10) {
    const Job J = decode_job(jobs & 0x3ffu, a);
    const int nh = J.nh, nkt = J.nkt;
    const u32 row0 = (u32)((a ^ J.dO) * 512 + J.sub * 128);
    const int j0 = fin && J.kind == JOB_O ? 8 : 0;
    fin = fin || J.kind != JOB_S;
    const u64 ko = P.korder[(P.relayed[J.ja] >> J.p) & 1u][J.dO >= 2 ? J.dO - 2 : 6];
    const int fl = lane & 15;
    const bool care = lane < 32 && fl < nh;
    const u32 foff = (lane < 16 ? OFF_XF + (u32)J.ja * 256u : OFF_WF + (u32)J.p * 256u) + ((u32)(ko >> (fl * 4)) & 15u) * 16u;
    const u32 tab = lane < 8 ? (row0 + l4 * 8u) * 1024u + (u32)lane * 128u
                             : OFF_XS + (u32)J.ja * PANEL_B + l4 * 1024u + (u32)(lane & 1) * 16384u + ((u32)(ko >> ((((lane - 8) >> 1) & 15) * 4)) & 15u) * 32768u;
    const u32 dw = OFF_WS - OFF_XS + (u32)(J.p - J.ja) * PANEL_B, w0 = l4 * (4u * 8192u) + J.col0 * 2u;
    int issued = j0, open = 8;
    u32 dst = lds0;
    auto issue = [&]() {
      const u32 ox = __builtin_amdgcn_readlane(tab, issued);
      if (__builtin_expect(issued >= 8, 1)) dma8_lin(dst, vp, rl, ox, ox + dw);
      else {
        dma4(dst, vx0, vx1, vx0, vx1, rx, ox, 8192u);
        dma4(dst + 16384u, vw0, vw0, vw1, vw1, rw, w0 + (u32)issued * (64u * 8192u), 32768u);
      }
      ++issued; dst = dst == lds0 + (NST - 1) * 32768u ? lds0 : dst + 32768u;
    };
    for (int j = j0; j <= nkt; ++j) {
      if (__builtin_expect(issued == j + NST - 2 && issued < open, 1)) {
        issue();
        WAIT_VM(24);
        __builtin_amdgcn_s_barrier();
        continue;
      }
      const int cap = j + NST - (j > j0) < nkt ? j + NST - (j > j0) : nkt;
      if (issued < cap && issued == open) {
        const int got = wave_poll4(ws, rws, foff, care, (open - 8) >> 1, issued == j, call, lane);
        open = max(open, 8 + 2 * got);
      }
      while (issued < cap && issued < open) issue();
      const int after = issued - j - 1;
      if (after >= 4) WAIT_VM(32); else if (after == 3) WAIT_VM(24); else if (after == 2) WAIT_VM(16); else if (after == 1) WAIT_VM(8); else WAIT_VM(0);
      __builtin_amdgcn_s_barrier();
    }
    __builtin_amdgcn_s_barrier();
  }
}

__device__ __forceinline__ void computer(const Params& P, char* lds, char* ws, int lane, int wave, u32 call, u32 jobs) {
  const int a = P.rank;
  const int wr = wave >> 1, wc = wave & 1, fr = lane & 15, g = lane >> 4;
  const u32 xa0 = (u32)((wr * 64 + fr) * 128 + ((g ^ (fr >> 1)) * 16));
  const int tq = fr >> 2, tp = fr & 3;
  const u32 wa0 = 16384u + (u32)((8 * g + tq) * 256 + (((8 * wc + 2 * tp) ^ (tq | ((g & 1) << 3))) * 16) + (tp & 1) * 8);
  const u32x4 rl = mk_rsrc4(ws, WS_USED);
  int stamp = 1; (void)stamp;
  f32x4 fin[4][4];
  u32 fjob = 0;
  const u32 vo = (u32)lane * 16u;

  for (; (jobs & 0x3ffu) != 0; jobs >>= 10) {
    const Job J = decode_job(jobs & 0x3ffu, a);
    const bool cont = fjob != 0 && J.kind == JOB_O;
    f32x4 acc[4][4];
#pragma unroll
    for (int m = 0; m < 4; ++m)
#pragma unroll
      for (int n = 0; n < 4; ++n) acc[m][n] = cont ? fin[m][n] : (f32x4){0.f, 0.f, 0.f, 0.f};
    auto readf = [&](bf16x8 (&xf)[4], bf16x8 (&wf)[4], int slot, int ks) {
      LAS char* st = (LAS char*)lds + slot * 32768;
#pragma unroll
      for (int i = 0; i < 4; ++i) {
        const u32 o = (wa0 ^ (u32)((i >> 1) * 16 + (i & 1) * 8)) + (u32)(ks * 8192);
        const s16x4 lo = __builtin_amdgcn_ds_read_tr16_b64_v4i16((LAS s16x4*)(st + o));
        const s16x4 hi = __builtin_amdgcn_ds_read_tr16_b64_v4i16((LAS s16x4*)(st + o + 1024));
        wf[i] = __builtin_bit_cast(bf16x8, __builtin_shufflevector(lo, hi, 0, 1, 2, 3, 4, 5, 6, 7));
        xf[i] = *(LAS const bf16x8*)(st + ((xa0 ^ (u32)(ks * 64)) + (u32)(i * 2048)));
      }
    };
    auto mma = [&](const bf16x8 (&xf)[4], const bf16x8 (&wf)[4]) {
#pragma unroll
      for (int m = 0; m < 4; ++m)
#pragma unroll
        for (int n = 0; n < 4; ++n) acc[m][n] = __builtin_amdgcn_mfma_f32_16x16x32_bf16(wf[n], xf[m], acc[m][n], 0, 0, 0);
    };
#define INTERLEAVE(id) do { _Pragma("unroll") for (int i_ = 0; i_ < 6; ++i_) { __builtin_amdgcn_sched_group_barrier(0x8, 1, id); __builtin_amdgcn_sched_group_barrier(0x100, 2, id); } \
                            __builtin_amdgcn_sched_group_barrier(0x8, 10, id); __builtin_amdgcn_sched_barrier(0); } while (0)
    bf16x8 x0[4], w0[4], x1[4], w1[4];
    int slot = 0;
    BARRIER();
    readf(x0, w0, 0, 0);
    u64 bk = TAU;
    const int bk_at = J.kind == JOB_S && J.dO == 1 ? J.nkt - 2 : -1;
    for (int kt = cont ? 8 : 0; kt < J.nkt; ++kt) {
      if (kt == bk_at) book(bk, rl, OFF_TP);
      __builtin_amdgcn_s_waitcnt(0xC07F);
      __builtin_amdgcn_sched_barrier(0);
      readf(x1, w1, slot, 1); mma(x0, w0);
      INTERLEAVE(0);
      slot = slot == NST - 1 ? 0 : slot + 1;
      BARRIER();
      __builtin_amdgcn_sched_barrier(0);
      readf(x0, w0, slot, 0); mma(x1, w1);
      INTERLEAVE(1);
    }
    BARRIER();
    STAMP_AT(stamp++);

    if (J.kind == JOB_S) {
      const u32 slot = (u32)(box_base(J.dO) + box_idx(J.ja, J.p));
      u32x4 v[8];
#pragma unroll
      for (int jj = 0; jj < 8; ++jj) {
        const f32x4 u0 = acc[jj >> 1][(jj & 1) * 2], u1 = acc[jj >> 1][(jj & 1) * 2 + 1];
        v[jj].x = pack_bf16(u0[0], u0[1]); v[jj].y = pack_bf16(u0[2], u0[3]); v[jj].z = pack_bf16(u1[0], u1[1]); v[jj].w = pack_bf16(u1[2], u1[3]);
      }
      const u32 so = OFF_IN + slot * 32768u + (u32)wave * 8192u, fo = OFF_PF + slot * 512u + (u32)wave * 128u;
      if (J.dO >= 2) {
#pragma unroll
        for (int jj = 0; jj < 8; ++jj) st16<0>(v[jj], rl, vo, so + (u32)(jj * 1024));
        WAIT_VM(0);
        if (lane == 0) st_sys(P.ws[J.dO] + fo, call);
        STAMP_AT(stamp++);
      } else {
        asm volatile("s_waitcnt vmcnt(0)" : "+v"(bk) :: "memory");
        STAMP_AT(stamp++);
        const u32x4 rd = mk_rsrc4(P.ws[1], WS_USED);
        sleep_until(rfl((u32)(bk >> 4)));
#pragma unroll
        for (int jj = 0; jj < 8; ++jj) {
          st16<0>(v[jj], rd, vo, so + (u32)(jj * 1024));
          if (jj < 7) wait_vm<CAP - 1>();
        }
        WAIT_VM(4);
        if (lane == 0) st_sys(P.ws[1] + fo, call);
        WAIT_VM(1);
        if (lane == 0) st_sys(P.ws[1] + fo + 4u, call);
      }
      STAMP_AT(stamp++);
    } else {
      fjob = jobs & 0x3ffu;
#pragma unroll
      for (int m = 0; m < 4; ++m)
#pragma unroll
        for (int n = 0; n < 4; ++n) fin[m][n] = acc[m][n];
    }
  }
  if (!fjob) return;

  if (threadIdx.x < 2) ((LAS u32*)lds)[threadIdx.x] = 0;
  BARRIER();
  const Job J = decode_job(fjob, a);
  const __amdgpu_buffer_rsrc_t rws = mk_rsrc(ws, WS_USED);
  u32 n = 0;
  char* outp = P.out;
  asm volatile("" : "+s"(outp));
  const __amdgpu_buffer_rsrc_t ro = mk_rsrc(outp, OUT_BYTES);
  {
    float mx = 0.f;
    auto add = [&](int jj, u32x4 c) {
      f32x4& u0 = fin[jj >> 1][(jj & 1) * 2]; f32x4& u1 = fin[jj >> 1][(jj & 1) * 2 + 1];
      u0 += (f32x4){bf_lo(c.x), bf_hi(c.x), bf_lo(c.y), bf_hi(c.y)};
      u1 += (f32x4){bf_lo(c.z), bf_hi(c.z), bf_lo(c.w), bf_hi(c.w)};
    };
    auto done = [&](int jj) {
      f32x4& u0 = fin[jj >> 1][(jj & 1) * 2]; f32x4& u1 = fin[jj >> 1][(jj & 1) * 2 + 1];
#pragma unroll
      for (int e = 0; e < 4; ++e) mx = fmaxf(mx, fmaxf(u0[e], u1[e]));
      asm volatile("" : "+v"(u0), "+v"(u1), "+v"(mx));
    };
    if (J.kind == JOB_F) {
      const bool far = (J.ja >> 2) != 0;
      const int d0 = far ? 4 | ((a ^ J.sub) & 2) : 2;
      const u32 bi = far ? (u32)(J.p * 2 + (J.sub & 1)) : (u32)J.p;
      const u32 s0 = (u32)box_base(d0) + bi, s1 = (u32)box_base(d0 + 1) + bi;
      const __amdgpu_buffer_rsrc_t r0 = mk_rsrc(P.ws[d0], WS_USED), r1 = mk_rsrc(P.ws[d0 + 1], WS_USED);
      const u32 o = OFF_GO + (far ? 0u : 128u);
      wave_wait<SL_GO>(ws, o + 4u, lane == 0, call, lane);
      u32 t_go = rfl(__hip_atomic_load((GAS const u32*)(ws + o), __ATOMIC_RELAXED, __HIP_MEMORY_SCOPE_AGENT));
      const u32 round = (u32)(!far ? J.p >> 3 : J.p < 8 ? J.sub & 1 : J.p >> 2);
      const u32 seat = (u32)((!far || J.p < 8 ? J.p & 7 : (J.p & 3) | (J.sub & 1) << 2) + wave * 8);
      const u32 tok = OFF_TOK + (((u32)d0 * 2u + round) * 32u + seat) * 64u;
      const u32 step = far ? 4 * TAU_X : 4 * TAU_C;
      t_go += (round * 8u * step + seat * (step >> 5)) >> 4;
      sleep_until(t_go - 50u);
      wave_wait<4>(ws, lane == 2 ? tok - 32u * 64u : OFF_PF + (lane == 0 ? s0 : s1) * 512u + (u32)wave * 128u, lane < (round ? 3 : 2), call, lane);
      const u32 so0 = OFF_IN + s0 * 32768u + (u32)wave * 8192u, so1 = OFF_IN + s1 * 32768u + (u32)wave * 8192u;
      u32x4 v0[8], v1[8];
      u32 t_min = t_go;
#pragma unroll
      for (int jj = 0; jj < 8 + PD; ++jj) {
        if (jj < 8) {
          const u32 ti = t_go + (((u32)jj * step) >> 4);
          sleep_until((int)(ti - t_min) > 0 ? ti : t_min);
          t_min = (u32)__builtin_amdgcn_s_memrealtime() + ((step * CATCH) >> 8);
          v0[jj] = __builtin_amdgcn_raw_buffer_load_b128(r0, vo + (u32)((jj & 3) * 1024), so0 + (u32)((jj >> 2) * 4096), 17);
          v1[jj] = __builtin_amdgcn_raw_buffer_load_b128(r1, vo + (u32)((jj & 3) * 1024), so1 + (u32)((jj >> 2) * 4096), 17);
          if (jj == 6 && lane == 0) st_agent(ws + tok, call);
        }
        if (jj >= PD) { add(jj - PD, v0[jj - PD]); add(jj - PD, v1[jj - PD]); done(jj - PD); }
      }
    } else {
      const u32 s0 = (u32)(J.p * 3 + J.sub);
      const u32 so0 = OFF_IN + s0 * 32768u + (u32)wave * 8192u, fo = OFF_PF + s0 * 512u + (u32)wave * 128u;
#pragma unroll
      for (int hf = 0; hf < 2; ++hf) {
        wave_wait<SL_HDR>(ws, fo + (u32)hf * 4u, lane == 0, call, lane);
        u32x4 v0[4];
#pragma unroll
        for (int jj = 0; jj < 4; ++jj) v0[jj] = __builtin_amdgcn_raw_buffer_load_b128(rws, vo + (u32)(jj * 1024), so0 + (u32)(hf * 4096), 17);
#pragma unroll
        for (int jj = 0; jj < 4; ++jj) { add(hf * 4 + jj, v0[jj]); done(hf * 4 + jj); }
      }
    }
    TSTAMP(10);
    const u32 wm = wave_max(__builtin_bit_cast(u32, mx));
    if (lane == 0) {
      __hip_atomic_fetch_max((LAS u32*)lds, wm, __ATOMIC_RELAXED, __HIP_MEMORY_SCOPE_WORKGROUP);
      n = __hip_atomic_fetch_add((LAS u32*)lds + 1, 1u, __ATOMIC_RELAXED, __HIP_MEMORY_SCOPE_WORKGROUP);
      const u32 tm = __hip_atomic_load((LAS u32*)lds, __ATOMIC_RELAXED, __HIP_MEMORY_SCOPE_WORKGROUP);
      if (n == 3u)
        __hip_atomic_store((GAS u64*)(ws + OFF_LM + (u32)(J.p * 8 + (J.kind == JOB_F ? J.ja : J.sub)) * 128u), ((u64)call << 32) | tm, __ATOMIC_RELAXED, __HIP_MEMORY_SCOPE_AGENT);
    }
  }
  TSTAMP(11);
  if (g & 1) {
#pragma unroll
    for (int m = 0; m < 4; ++m) { f32x4 t = fin[m][0]; fin[m][0] = fin[m][1]; fin[m][1] = t; t = fin[m][2]; fin[m][2] = fin[m][3]; fin[m][3] = t; }
  }
  u32 oo[4];
#pragma unroll
  for (int m = 0; m < 4; ++m) { oo[m] = (u32)(J.sub * 128 + wr * 64 + m * 16 + fr) * 4096u + J.col0 + (u32)(wc * 64 + g * 16); asm volatile("" : "+v"(oo[m])); }
  Polls pq = {};
  if (rfl(n) == 3u) {
    const u32 gm = gather_max(ws, lane < NR ? OFF_AM + (u32)lane * 128u : 0x80000000u, call, lane, LZ_F, pq);
    if (lane == 0) ((LAS u32*)lds)[2] = gm;
  }
  BARRIER();
  const u32 gmax = rfl(((LAS u32*)lds)[2]);
  TSTAMP(12);
  const float inv = gmax ? 127.0f * __builtin_amdgcn_rcpf(as_f32(gmax)) : 0.f;
#pragma unroll
  for (int m = 0; m < 4; ++m) {
    u32x4 v;
#pragma unroll
    for (int n = 0; n < 4; ++n) {
      f32x4 y = fin[m][n] * inv;
      asm volatile("" : "+v"(y));
      u32 r = 0;
#pragma unroll
      for (int e = 0; e < 4; ++e) r = __builtin_amdgcn_cvt_pk_u8_f32(y[e], e, r);
      v[n] = r;
    }
    if (gmax == ~0u) v = (u32x4){0x80808080u, 0x80808080u, 0x80808080u, 0x80808080u};
    __builtin_amdgcn_raw_buffer_store_b128(v, ro, oo[m], 0, OUT_AUX);
  }
  pq.retire();
  TSTAMP(13);
}

__global__ void __launch_bounds__(512, 2) gemm_rs_q8(const Params P) {
  __shared__ __attribute__((aligned(16))) char lds[5 * 32768];
  const int lane = threadIdx.x & 63, wave = (int)rfl(threadIdx.x >> 6);
  char* ws = P.ws[0];
  if (__builtin_expect(blockIdx.x < NPUSH, 1)) {
    if (wave < PW) pusher(P, lds, ws, lane, wave);
    return;
  }
  if (blockIdx.x >= NPUSH + 8 && blockIdx.x < NPUSH + 24) {
    const bool second = blockIdx.x >= NPUSH + 16;
    const u32 o = (u32)(P.rank * 8 + (blockIdx.x & 7)) * 256u;
    const char* q = P.ws[wave] + (u32)lane * (2u << 20) + o + (320u << 10);
    bool on = lane < 12 && ((0x13u >> lane) & 1u) != second;
    if (second) {
      sleep_until((u32)__builtin_amdgcn_s_memrealtime() + TDELAY);
      if (wave == 0 && lane >= 12 && lane < 20) { on = true; q = (lane < 14 ? P.out : lane < 17 ? P.x : P.w) + (lane < 14 ? (u32)(lane - 12) * (OUT_BYTES - 32768u) : (u32)((lane - 14) % 3) * (X_BYTES / 2u - 16384u)) + o; }
    }
    if (on) { const u32 t = *(GAS const u32*)q; asm volatile("" :: "v"(t)); }
  }
  const u32 call = draw_call(ws, lane, wave);
  if (blockIdx.x == GRID - 1) {
    if (wave) return;
    coordinator(P, ws, lane, call);
    char* dst = P.ws[lane & 7] + OFF_AM + (u32)P.rank * 128u;
    asm volatile("" : "+v"(dst));
    Polls pq;
    u32 m = 0;
    for (u32 half = 0; half < 2; ++half) m = max(m, gather_max(ws, OFF_LM + (half * 64u + (u32)lane) * 128u, call, lane, LZ_C, pq));
    if (lane < NR) __hip_atomic_store((GAS u64*)dst, ((u64)call << 32) | m, __ATOMIC_RELAXED, __HIP_MEMORY_SCOPE_SYSTEM);
    pq.retire();
    return;
  }
  const u32 jobs = P.jobs[blockIdx.x];
  if (wave >= 4) loader(P, lds, ws, lane, wave, call, jobs);
  else computer(P, lds, ws, lane, wave, call, jobs);
}

#define LAG 12
#define R_FWD 2
#define R_EARLY 6
void build_schedule(Params& P) {
#define RAMP 4
#define SHIFT2 2
#define ORD 0
  static const int orders[5][32] = {
    {0, 1, 2, 16, 17, 5, 18, 6, 19, 7, 20, 8, 21, 22, 23, 9, 10, 11, 12, 24, 25, 26, 27, 4, 3, 28, 29, 30, 31, 13, 14, 15},
    {0, 1, 2, 16, 17, 5, 18, 6, 19, 7, 20, 8, 21, 22, 23, 24, 9, 25, 10, 11, 26, 12, 27, 4, 3, 28, 29, 30, 31, 13, 14, 15},
    {0, 1, 2, 16, 17, 5, 18, 6, 19, 7, 20, 8, 21, 22, 23, 24, 25, 9, 10, 26, 11, 12, 27, 4, 3, 28, 29, 30, 31, 13, 14, 15},
    {0, 1, 2, 16, 17, 18, 5, 19, 6, 20, 7, 21, 8, 22, 23, 24, 9, 25, 10, 26, 11, 12, 27, 4, 3, 28, 29, 30, 31, 13, 14, 15},
    {0, 1, 2, 16, 17, 5, 18, 6, 19, 7, 20, 8, 21, 22, 23, 24, 9, 25, 10, 11, 26, 12, 4, 27, 3, 28, 29, 30, 31, 13, 14, 15}};
  const int* order = orders[ORD];
  int pos[32];
  for (int i = 0; i < 32; ++i) pos[order[i]] = i;
  auto ready = [&](int t, int p) { return pos[t] > pos[16 + p] ? pos[t] : pos[16 + p]; };
  for (int d = 0; d < NR; ++d) {
    unsigned char list[MAXU]; int n = 0;
    for (int i = 0; i < MAXU; ++i) list[i] = 0;
    const int q = P.rank ^ d;
    auto relay = [&](int c) { return c >= 16 + R_END - 2 * RELAY && c < 16 + R_END; };
    if (d >= 2)
      for (int i = 0, nh1 = 0, lag = ((2 * (d - (d >= 4 ? 4 : 2)) + 1) * LAG + 10) / 20; i < 64; ++i) {
        auto put = [&](int k, int h) { const int c = order[k]; if (!(c < 16 && tile_dist(c) == d)) list[n++] = (unsigned char)(c * 2 + h); };
        if (i < 32) put(i, 0);
        for (; nh1 < 32 && nh1 + (nh1 * RAMP / 8 < lag ? nh1 * RAMP / 8 : lag) <= i; ++nh1)
          if (d < 4 || !relay(order[nh1])) put(nh1, 1);
        if (d >= 4 && i + R_EARLY < 32 && relay(order[i + R_EARLY]) && ((q >> 1) & 1) == (order[i + R_EARLY] & 1)) put(i + R_EARLY, 1);
        if (d < 4 && i >= R_FWD && i - R_FWD < 32 && relay(order[i - R_FWD]) && ((P.rank >> 1) & 1) == (order[i - R_FWD] & 1))
          for (int ds = 4; ds < 8; ++ds)
            if (((P.rank ^ ds) & 1) == (d & 1)) list[n++] = (unsigned char)(ds * 16 + order[i - R_FWD] - 16);
      }
    if (OWNLAST && d < 4) {
      unsigned char tmp[MAXU]; int m = 0;
      for (int pass = 0; pass < 2; ++pass)
        for (int i = 0; i < n; ++i) if ((list[i] >= 26 && list[i] < 32) == (pass == 1)) tmp[m++] = list[i];
      for (int i = 0; i < n; ++i) list[i] = tmp[i];
    }
    P.ucount[d] = (u32)n;
    for (int i = 0; i < MAXU / 4; ++i) P.units[d][i] = list[4 * i] | list[4 * i + 1] << 8 | list[4 * i + 2] << 16 | (u32)list[4 * i + 3] << 24;
  }
  for (int t = 0; t < 16; ++t) {
    P.relayed[t] = 0;
    for (int p = R_END - 2 * RELAY; p < R_END; ++p)
      if (((P.rank >> 1) & 1) != (p & 1) && t < 13 && pos[16 + p] + R_FWD > pos[t]) P.relayed[t] |= 1u << p;
  }
  static const int kord[2][12] = {{10, 8, 6, 4, 5, 7, 9, 11, 2, 0, 1, 3}, {10, 8, 6, 4, 2, 0, 1, 3, 5, 7, 9, 11}};
  for (int c = 0; c < 2; ++c)
    for (int sk = 0; sk < 8; ++sk) {
      u64 v = 0;
      for (int i = 0, n = 0; i < 12; ++i) if ((kord[c][i] >> 1) != sk) v |= (u64)kord[c][i] << (4 * n++);
      P.korder[c][sk] = v;
    }
  for (int b = 0; b < NPUSH; ++b)
    for (int w = 0; w < PW; ++w) {
      const int d = push_link(b, w);
      auto code = [&](int i) { return (P.units[d][i >> 4] >> ((i & 12) * 2)) & 0xffu; };
      const int i = push_seat(b, w);
      P.first[b][2 * w] = slice_src(code(i), (u32)i & 3u, P.rank ^ d, d) | code(i) >> 5;
      P.first[b][2 * w + 1] = code(i) | code(i + NW) << 8 | (u32)(P.rank ^ d) << 16;
    }
  static const int grp[16] = {0, 1, 0, 2, 1, 1, 0, 1, 0, 1, 0, 1, 0, 2, 2, 2};
  for (int i = 0; i < GRID; ++i) P.jobs[i] = 0;
  for (int x = 0; x < 8; ++x) {
    int ht[40], hp[40], n = 0;
    for (int r = 0; r < 32; ++r)
      for (int p = 0; p < 16; ++p)
        for (int t = 0; t < 16; ++t) {
          const int gsel = grp[t] == 2 ? (p >> 2) & 1 : grp[t];
          if (gsel == (x >> 2) && ((p + (t == 2 ? SHIFT2 : 0)) & 3) == (x & 3) && ready(t, p) == r) { ht[n] = t; hp[n] = p; ++n; }
        }
    auto enc = [&](int i) { return (u32)(((ht[i] >= 13 ? JOB_O : JOB_S) << 8) | (ht[i] << 4) | hp[i]); };
    const int nw = 32 - NPUSH / 8 - (x == 7);
    const int hi = x >> 2, lo = x & 3;
    const int fsub[10] = {4, 6, 5, 7, 3, 4 + hi, 6 + hi, 3, 5 - hi, 7 - hi};
    const int fpan[10] = {4 * hi + lo, 4 * hi + lo, 4 - 4 * hi + lo, 4 - 4 * hi + lo, x, 8 + lo, 8 + lo, 8 + x, 12 + lo, 12 + lo};
    int f = 0, extra = nw;
    for (int s = 0; s < nw; ++s) {
      u32 jb = enc(s);
      const bool host = extra < n && (ht[s] < 3 ? hp[s] >= PURE && hp[s] < 8 : f >= 4 && ht[s] < 13);
      if (host) { jb = (u32)((JOB_F << 8) | ((ht[extra] - 13) << 4) | hp[extra]) | jb << 10 | enc(extra) << 20; ++extra; }
      else if (ht[s] >= 3 && ht[s] < 13 && f < 10) {
        jb = jb << 10 | (u32)((JOB_F << 8) | (fsub[f] << 4) | fpan[f]); ++f;
      }
      P.jobs[NPUSH + s * 8 + x] = jb;
    }
  }
}

}

extern "C" void kernel_launch(int rank, int nranks, ncclComm_t comm, hipStream_t stream,
                              void* const* d_in, const int* in_sizes, int n_in,
                              void* d_out, int out_size,
                              const int* in_dtypes, int out_dtype,
                              void* const* d_out_all, void* const* d_ws_all, size_t ws_bytes) {
  (void)comm; (void)in_dtypes; (void)out_dtype; (void)d_out_all;
  if (nranks != NR || (unsigned)rank >= (unsigned)NR || n_in != 2 || ws_bytes < (24u << 20) || (u32)out_size != OUT_BYTES ||
      (u32)in_sizes[0] * 2u != X_BYTES || (u32)in_sizes[1] * 2u != W_BYTES) {
    fprintf(stderr, "unexpected problem shape\n");
    return;
  }
  Params P;
  P.x = (const char*)d_in[0]; P.w = (const char*)d_in[1]; P.out = (char*)d_out;
  for (int d = 0; d < NR; ++d) P.ws[d] = (char*)d_ws_all[rank ^ d];
  P.rank = rank; P.pad = 0;
  build_schedule(P);
  hipLaunchKernelGGL(gemm_rs_q8, dim3(GRID), dim3(512), 0, stream, P);
  HIP_CHECK(hipGetLastError());
}
```

```cpp
#include <hip/hip_runtime.h>
#include <rccl/rccl.h>
#include <cstdio>
#include <cstdint>

#define HIP_CHECK(expr) do { hipError_t _e = (expr); if (_e != hipSuccess) { \
    fprintf(stderr, "HIP error %s at %s:%d\n", hipGetErrorString(_e), __FILE__, __LINE__); \
    return; } } while (0)

namespace {

typedef unsigned u32;
typedef unsigned long long u64;
typedef u32 u32x4 __attribute__((ext_vector_type(4)));
typedef u32 u32x2 __attribute__((ext_vector_type(2)));
typedef float f32x4 __attribute__((ext_vector_type(4)));
typedef short s16x4 __attribute__((ext_vector_type(4)));
typedef short s16x8 __attribute__((ext_vector_type(8)));
typedef __bf16 bf16x8 __attribute__((ext_vector_type(8)));
#define LAS __attribute__((address_space(3)))
#define GAS __attribute__((address_space(1)))

constexpr int NR = 8;
#define NPUSH 32
#define PW 3
constexpr int NW = NPUSH * PW / 6;
constexpr int GRID = 256;
#define RELAY 4
constexpr int MAXU = 72;
constexpr u64 DEADLINE_TICKS = 100000000ull;

constexpr u32 X_BYTES = 4096u * 512u * 2u, W_BYTES = 512u * 4096u * 2u, OUT_BYTES = 512u * 4096u;
constexpr u32 OFF_CNT = 0;
constexpr u32 OFF_TMO = 16u << 10;
constexpr u32 OFF_TK = (16u << 10) + 128u;
constexpr u32 OFF_TP = 18u << 10;
constexpr u32 OFF_T0 = 19u << 10;
constexpr u32 OFF_XF = 20u << 10;
constexpr u32 OFF_WF = 24u << 10;
constexpr u32 OFF_TR = 28u << 10;
constexpr u32 OFF_GO = 29u << 10;
constexpr u32 OFF_AM = 32u << 10;
constexpr u32 OFF_LM = 64u << 10;
constexpr u32 OFF_TOK = 128u << 10;
constexpr u32 OFF_PF = 192u << 10;
constexpr u32 OFF_DBG = 512u << 10;
constexpr u32 OFF_XS = 2u << 20;
constexpr u32 OFF_WS = 8u << 20;
constexpr u32 OFF_IN = 14u << 20;
constexpr u32 WS_USED = OFF_IN + 208u * 32768u;
#define TAU_X 223
#define TAU 232
#define TAU_C 227
#define PD 3
#define T_BOOK 70
#define ANCH 28
#ifndef M_FAR
#define M_FAR 360
#endif
#ifndef M_NEAR
#define M_NEAR 590
#endif
#define T_REST (700 - (40 - ANCH) * 230 / 16)
#define OUT_AUX 0
#define SL_LONG 32
#define SL_GATE 6
#define SL_AM 3
#define SL_HDR 4
#define WDELAY 150
#define SL_AML 16
#define T_FLY 100
#define SL_GO 16
#define LASTW WIN
#define CATCH 10
#define R_END 16
#define PURE 4
#define OWNLAST 1
#define LZ_F 7
#define LZ_C 40
#define T_RESTC T_REST
#define TOUCH 1
#define TDELAY 50
#define CAP 6
constexpr u32 PANEL_B = 6u * 65536u;

enum { JOB_NONE = 0, JOB_S = 1, JOB_O = 2, JOB_F = 3 };

struct Params {
  const char* x; const char* w; char* out;
  char* ws[NR];
  int rank; int pad;
  u32 jobs[GRID];
  u32 first[NPUSH][2 * PW];
  u32 ucount[NR];
  u32 units[NR][MAXU / 4];
  u64 korder[2][8];
  u32 relayed[16];
};

__host__ __device__ inline int tile_dist(int t) { return t < 3 ? 1 : t < 5 ? t - 1 : t < 13 ? 4 + ((t - 5) & 3) : 0; }
__host__ __device__ inline int tile_sub(int t, int a) { return t < 3 ? t : t < 5 ? 3 : t < 13 ? (a & 2) + ((t - 5) >> 2) : t - 13; }
__host__ __device__ inline int box_base(int d) { return d == 1 ? 0 : d < 4 ? 48 + 16 * (d - 2) : 80 + 32 * (d - 4); }
__host__ __device__ inline int box_cnt(int d) { return d == 1 ? 48 : d < 4 ? 16 : 32; }
__host__ __device__ inline int box_idx(int t, int p) { return t < 3 ? p * 3 + t : t < 5 ? p : p * 2 + ((t - 5) >> 2); }

__host__ __device__ inline u32 link_units(int d) { return d < 4 ? 62u + 2u * RELAY : 60u - RELAY; }
__host__ __device__ inline int push_link(int block, int wave) { return 2 + ((block >> 3) * PW + wave + (block & 7)) % 6; }
__host__ __device__ inline int push_seat(int block, int wave) { return ((block >> 3) * PW + wave) / 6 * 8 + (block & 7); }
__host__ __device__ inline u32 k_half(int q, int d) { return (u32)(q ^ (d >= 4 ? (q >> 1) & (q ^ d) : 0)) & 1u; }
__host__ __device__ inline u32 slice_src(u32 code, u32 sl, int q, int d) {
  const u32 h = code & 1u, kh = k_half(q, d);
  if (code < 32) {
    const int t = code >> 1;
    return ((u32)((q ^ tile_dist(t)) * 512 + tile_sub(t, q) * 128) + (sl & 1u) * 64u) * 1024u + (kh * 256u + h * 128u + (sl >> 1) * 64u) * 2u;
  }
  return (kh * 256u + h * 128u + sl * 32u) * 8192u + ((u32)(q >> 2) * 2048u + ((code - 32u) >> 1) * 128u) * 2u;
}

__device__ __forceinline__ u32 rfl(u32 v) { return __builtin_amdgcn_readfirstlane(v); }
__device__ __forceinline__ __amdgpu_buffer_rsrc_t mk_rsrc(const void* p, u32 bytes) {
  return __builtin_amdgcn_make_buffer_rsrc((void*)p, 0, bytes, 0x00020000);
}
__device__ __forceinline__ u32x4 mk_rsrc4(const void* p, u32 bytes) {
  u64 a = (u64)p; u32x4 r; r.x = rfl((u32)a); r.y = rfl((u32)(a >> 32)) & 0xffffu; r.z = bytes; r.w = 0x00020000u; return r;
}
__device__ __forceinline__ u32 ld_sys(const char* p) { return __hip_atomic_load((GAS const u32*)p, __ATOMIC_RELAXED, __HIP_MEMORY_SCOPE_SYSTEM); }
__device__ __forceinline__ void st_sys(char* p, u32 v) { __hip_atomic_store((GAS u32*)p, v, __ATOMIC_RELAXED, __HIP_MEMORY_SCOPE_SYSTEM); }
__device__ __forceinline__ void st_agent(char* p, u32 v) { __hip_atomic_store((GAS u32*)p, v, __ATOMIC_RELAXED, __HIP_MEMORY_SCOPE_AGENT); }
#define WAIT_VM(n) asm volatile("s_waitcnt vmcnt(" #n ")" ::: "memory")
#define BARRIER() do { asm volatile("" ::: "memory"); __builtin_amdgcn_s_waitcnt(0xC07F);   __builtin_amdgcn_s_barrier(); asm volatile("" ::: "memory"); } while (0)

#define TSTAMP(i) do {} while (0)
#define STAMP_AT(i) do {} while (0)

template <int SL> __device__ __forceinline__ void wave_wait(char* ws, u32 off, bool care, u32 call, int lane) {
  if (lane == 63) { off = OFF_TMO; }
  else if (!care) { off = OFF_TMO; }
  const u64 t0 = __builtin_amdgcn_s_memrealtime();
  for (;;) {
    const u32 v = ld_sys(ws + off);
    const bool ok = (lane == 63 || !care) ? true : (v >= call);
    if (__all(ok)) break;
    const bool dead = __any(lane == 63 && v == call);
    if (dead || __builtin_amdgcn_s_memrealtime() - t0 > DEADLINE_TICKS) {
      if (lane == 0) st_sys(ws + OFF_TMO, call);
      WAIT_VM(0);
      break;
    }
    __builtin_amdgcn_s_sleep(SL);
  }
}

__device__ __forceinline__ int wave_poll4(char* ws, __amdgpu_buffer_rsrc_t rws, u32 off, bool care, int have, bool block, u32 call, int lane) {
  if (!care) off = OFF_TMO;
  const u64 t0 = __builtin_amdgcn_s_memrealtime();
  for (;;) {
    const u32x4 v = __builtin_amdgcn_raw_buffer_load_b128(rws, off, 0, 17);
    asm volatile("" ::: "memory");
    const u32 m = (u32)__ballot(care && v.x >= call && v.y >= call && v.z >= call && v.w >= call);
    const int n = __builtin_ctz(~(m & (m >> 16)));
    if (n > have || !block) return n;
    const bool dead = __any(!care && v.x == call);
    if (dead || __builtin_amdgcn_s_memrealtime() - t0 > DEADLINE_TICKS) {
      if (lane == 0) st_sys(ws + OFF_TMO, call);
      WAIT_VM(0);
      return have + 1;
    }
    if (have == 0) __builtin_amdgcn_s_sleep(SL_LONG); else __builtin_amdgcn_s_sleep(SL_GATE);
  }
}
#define DMA_SC " sc0 sc1"

typedef __bf16 bf16x2 __attribute__((ext_vector_type(2)));
__device__ __forceinline__ u32 pack_bf16(float lo, float hi) { bf16x2 r; r.x = (__bf16)lo; r.y = (__bf16)hi; return __builtin_bit_cast(u32, r); }
__device__ __forceinline__ float as_f32(u32 v) { return __builtin_bit_cast(float, v); }
__device__ __forceinline__ float bf_lo(u32 v) { return __builtin_bit_cast(float, v << 16); }
__device__ __forceinline__ float bf_hi(u32 v) { return __builtin_bit_cast(float, v & 0xffff0000u); }
__device__ __forceinline__ u32 wave_max(u32 m) {
  m = max(m, (u32)__builtin_amdgcn_update_dpp(0, (int)m, 0x111, 0xf, 0xf, false));
  m = max(m, (u32)__builtin_amdgcn_update_dpp(0, (int)m, 0x112, 0xf, 0xf, false));
  m = max(m, (u32)__builtin_amdgcn_update_dpp(0, (int)m, 0x114, 0xf, 0xf, false));
  m = max(m, (u32)__builtin_amdgcn_update_dpp(0, (int)m, 0x118, 0xf, 0xf, false));
  const u32 a = max((u32)__builtin_amdgcn_readlane(m, 15), (u32)__builtin_amdgcn_readlane(m, 31));
  const u32 b = max((u32)__builtin_amdgcn_readlane(m, 47), (u32)__builtin_amdgcn_readlane(m, 63));
  return max(a, b);
}

__device__ __forceinline__ void dma4(u32 lds_dst, u32 v0, u32 v1, u32 v2, u32 v3, u32x4 rs, u32 so, u32 st) {
  u32 keep, t;
  asm volatile(
      "s_mov_b32 %0, m0\n\t"
      "s_mov_b32 m0, %2\n\ts_add_u32 %1, %8, %9\n\ts_nop 1\n\tbuffer_load_dwordx4 %3, %7, %8 offen" DMA_SC " lds\n\t"
      "s_add_u32 m0, %2, 0x400\n\ts_nop 0\n\tbuffer_load_dwordx4 %4, %7, %1 offen" DMA_SC " lds\n\t"
      "s_add_u32 m0, %2, 0x800\n\ts_add_u32 %1, %1, %9\n\tbuffer_load_dwordx4 %5, %7, %1 offen" DMA_SC " lds\n\t"
      "s_add_u32 m0, %2, 0xc00\n\ts_add_u32 %1, %1, %9\n\tbuffer_load_dwordx4 %6, %7, %1 offen" DMA_SC " lds\n\t"
      "s_mov_b32 m0, %0"
      : "=&s"(keep), "=&s"(t)
      : "s"(lds_dst), "v"(v0), "v"(v1), "v"(v2), "v"(v3), "s"(rs), "s"(so), "s"(st)
      : "memory", "scc");
}

__device__ __forceinline__ void dma8_lin(u32 lds_dst, u32 v, u32x4 rs, u32 sx, u32 sw) {
  u32 keep;
  asm volatile(
      "s_mov_b32 %0, m0\n\ts_mov_b32 m0, %1\n\ts_nop 2\n\t"
      "buffer_load_dwordx4 %2, %3, %4 offen" DMA_SC " lds\n\tbuffer_load_dwordx4 %2, %3, %4 offen offset:1024" DMA_SC " lds\n\t"
      "buffer_load_dwordx4 %2, %3, %4 offen offset:2048" DMA_SC " lds\n\tbuffer_load_dwordx4 %2, %3, %4 offen offset:3072" DMA_SC " lds\n\t"
      "s_add_u32 m0, %1, 0x4000\n\ts_nop 0\n\t"
      "buffer_load_dwordx4 %2, %3, %5 offen" DMA_SC " lds\n\tbuffer_load_dwordx4 %2, %3, %5 offen offset:1024" DMA_SC " lds\n\t"
      "buffer_load_dwordx4 %2, %3, %5 offen offset:2048" DMA_SC " lds\n\tbuffer_load_dwordx4 %2, %3, %5 offen offset:3072" DMA_SC " lds\n\t"
      "s_mov_b32 m0, %0"
      : "=&s"(keep) : "s"(lds_dst), "v"(v), "s"(rs), "s"(sx), "s"(sw) : "memory", "scc");
}

#define WIN 4
template <int N> __device__ __forceinline__ void wait_vm() { asm volatile("s_waitcnt vmcnt(%0)" :: "n"(N) : "memory"); }
template <int IMM> __device__ __forceinline__ void st16(u32x4 v, u32x4 rs, u32 voff, u32 soff) {
  asm volatile("buffer_store_dwordx4 %0, %1, %2, %3 offen offset:%4 sc0 sc1\n\ts_nop 1" :: "v"(v), "v"(voff), "s"(rs), "s"(soff), "n"(IMM) : "memory");
}
__device__ __forceinline__ void draw_async(u32& v, u32x4 rl, u32 off) {
  u64 ex; v = 1;
  asm volatile("s_mov_b64 %1, exec\n\ts_mov_b64 exec, 1\n\tbuffer_atomic_add %0, off, %2, %3 sc0\n\ts_mov_b64 exec, %1"
               : "+v"(v), "=&s"(ex) : "s"(rl), "s"(off) : "memory");
}
__device__ __forceinline__ void book(u64& tk, u32x4 rl, u32 off) {
  u64 ex; const u64 lo = (__builtin_amdgcn_s_memrealtime() + T_BOOK) * 16u;
  asm volatile("s_mov_b64 %1, exec\n\ts_mov_b64 exec, 1\n\tbuffer_atomic_umax_x2 %2, off, %3, %4\n\tbuffer_atomic_add_x2 %0, off, %3, %4 sc0\n\ts_mov_b64 exec, %1"
               : "+v"(tk), "=&s"(ex) : "v"(lo), "s"(rl), "s"(off) : "memory");
}
__device__ __forceinline__ void sleep_until(u32 due) {
  for (;;) {
    const u32 left = due - (u32)__builtin_amdgcn_s_memrealtime() - 1u;
    if (left >= 10000u) break;
    if (left > 200u) __builtin_amdgcn_s_sleep(48); else if (left > 40u) __builtin_amdgcn_s_sleep(8); else __builtin_amdgcn_s_sleep(1);
  }
}
struct Slice { u32 src, dst, flag, kind, dep; };
__device__ __forceinline__ u32 draw_call(char* ws, int lane, int wave) {
  u32 call = 0;
  if (lane == 0) call = __hip_atomic_fetch_add((GAS u32*)(ws + OFF_CNT + blockIdx.x * 64 + wave * 4), 1u, __ATOMIC_RELAXED, __HIP_MEMORY_SCOPE_AGENT) + 1u;
  return rfl(call);
}
__device__ __forceinline__ void pusher(const Params& P, char* lds, char* ws, int lane, int wave) {
  const u32 vx0 = (u32)(lane >> 3) * 1024u + (u32)(((lane & 7) ^ (lane >> 4)) * 16), vx1 = vx0 ^ 64u;
  const u32 vw0 = (u32)(lane >> 4) * 8192u + (u32)(((lane & 15) ^ (lane >> 4)) * 16), vw1 = vw0 ^ 128u;
  const int d = push_link(blockIdx.x, wave);
  u32 f0 = P.first[blockIdx.x][2 * wave], f1 = P.first[blockIdx.x][2 * wave + 1]; const char* xp = P.x; const char* wp = P.w;
  char* pw = P.ws[d];
  asm volatile("" : "+s"(f0), "+s"(f1), "+s"(xp), "+s"(wp), "+s"(pw), "+s"(ws));
  const u32x4 rx = mk_rsrc4(xp, X_BYTES), rw = mk_rsrc4(wp, W_BYTES), rl = mk_rsrc4(ws, WS_USED);
  const u32 vp = (u32)lane * 16u;
  const u32 ring0 = (u32)(size_t)(LAS char*)lds + (u32)wave * 16384u;
  auto load_all = [&](const Slice& c, u32 buf) {
    const u32 dst = ring0 + buf * 8192u;
    if (c.kind == 0) { dma4(dst, vx0, vx1, vx0, vx1, rx, c.src, 8192u); dma4(dst + 4096u, vx0, vx1, vx0, vx1, rx, c.src + 32768u, 8192u); }
    else if (c.kind == 1) { dma4(dst, vw0, vw0, vw1, vw1, rw, c.src, 32768u); dma4(dst + 4096u, vw0, vw0, vw1, vw1, rw, c.src + 131072u, 32768u); }
    else { dma4(dst, vp, vp, vp, vp, rl, c.src, 1024u); dma4(dst + 4096u, vp, vp, vp, vp, rl, c.src + 4096u, 1024u); }
  };
  { Slice c; c.src = f0 & ~1u; c.kind = f0 & 1u; c.dst = c.flag = c.dep = 0; load_all(c, 0); }
  u32 cnt; draw_async(cnt, rl, OFF_CNT + blockIdx.x * 64u + (u32)wave * 4u);
  const int q = (int)(f1 >> 16);
  const u32 ns = link_units(d) * 4u;
  auto decode = [&](u32 sidx, u32 code) -> Slice {
    Slice r;
    const u32 sl = sidx & 3u, h = code & 1u;
    if (code >= 64) {
      const u32 ds = code >> 4, pp = code & 15u;
      r.kind = 2;
      r.src = OFF_WS + pp * PANEL_B + (ds - 2u) * 65536u + 32768u + sl * 8192u;
      r.dep = OFF_WF + pp * 256u + (ds - 2u) * 32u + 16u + sl * 4u;
      r.dst = OFF_WS + pp * PANEL_B + ((ds ^ (u32)d) - 2u) * 65536u + 32768u + sl * 8192u;
      r.flag = OFF_WF + pp * 256u + ((ds ^ (u32)d) - 2u) * 32u + 16u + sl * 4u;
      return r;
    }
    r.kind = code >> 5; r.src = slice_src(code, sl, q, d); r.dep = OFF_TMO;
    r.dst = (code < 32 ? OFF_XS : OFF_WS) + ((code >> 1) & 15u) * PANEL_B + (u32)(d - 2) * 65536u + h * 32768u + sl * 8192u;
    r.flag = (code < 32 ? OFF_XF : OFF_WF) + ((code >> 1) & 15u) * 256u + (u32)(d - 2) * 32u + h * 16u + sl * 4u;
    return r;
  };
  const u32x4 rd = mk_rsrc4(pw, WS_USED);
  LAS char* ring = (LAS char*)lds + wave * 16384;
  u32 nxt_i = blockIdx.x;
  Slice cur = decode(blockIdx.x, f1 & 0xffu), nxt = decode(nxt_i, (f1 >> 8) & 0xffu);
  u32 buf = 0;
  asm volatile("s_waitcnt vmcnt(0)" : "+v"(cnt), "+s"(cur.dst), "+s"(nxt.src) :: "memory");
  const u32 call = rfl(cnt) + 1u;
  const u32 tk_base = (call - 1u) * (ns - NW);
  const u32 tk_off = OFF_TK + (u32)d * 128u;
  u32 pend = 0;
  auto publish = [&]() {
    if (lane == 0) {
      st_sys(pw + pend, call);
    }
  };
  auto send = [&](int k) {
    const u32x4 v = *(LAS const u32x4*)(ring + buf * 8192u + k * 1024 + lane * 16);
    st16<0>(v, rd, vp, cur.dst + (u32)k * 1024u);
  };
  for (;;) {
    u32 tk;
    send(0);
    load_all(nxt, buf ^ 1u);
    draw_async(tk, rl, tk_off);
    const u32 t_draw = (u32)__builtin_amdgcn_s_memrealtime();
    Slice nn = nxt;
    u32 dv = 0;
#pragma unroll
    for (int k = 0; k < 8; ++k) {
      if (k) send(k);
      if (k == WIN + 1) {
        asm volatile("" : "+v"(tk));
        nxt_i = rfl(tk) - tk_base;
        if (nxt_i == ns - ANCH && lane == 0)
          __hip_atomic_store((GAS u64*)(ws + OFF_T0 + (u32)d * 128u), ((u64)call << 32) | (t_draw + (d < 4 ? T_RESTC : T_REST)), __ATOMIC_RELAXED, __HIP_MEMORY_SCOPE_AGENT);
        nxt_i += 2u * NW;
        if (nxt_i < ns) nn = decode(nxt_i, (P.units[d][nxt_i >> 4] >> ((nxt_i & 12u) * 2u)) & 0xffu);
        if (nxt_i < ns && nn.kind == 2) asm volatile("buffer_load_dword %0, off, %1, %2 sc0 sc1" : "+v"(dv) : "s"(rl), "s"(nn.dep) : "memory");
      }
      if (k < WIN) wait_vm<WIN - 1 + 9>(); else wait_vm<WIN - 1>();
      if (k == WIN - 2 && pend) publish();
    }
    pend = cur.flag;
    cur = nxt; nxt = nn; buf ^= 1u;
    if (nxt_i >= ns) break;
    if (nxt.kind == 2) {
      asm volatile("s_waitcnt vmcnt(%1)" : "+v"(dv) : "n"(6 - WIN) : "memory");
      if (rfl(dv) < call) wave_wait<2>(ws, nxt.dep, lane == 0, call, lane);
    }
  }
#pragma unroll
  for (int k = 0; k < 8; ++k) {
    send(k);
    wait_vm<LASTW - 1>();
    if (k == LASTW - 2) publish();
  }
  pend = cur.flag;
  WAIT_VM(0);
  publish();
}

__device__ __forceinline__ void coordinator(const Params& P, char* ws, int lane, u32 call) {
  const u32 lo = (u32)(lane & 7) * 128u;
  const u32 off = lane < 16 ? (lane & 8 ? OFF_TR : OFF_T0) + lo : OFF_WF + (lane < 32 ? 14u : 12u) * 256u + (u32)((lane >> 2) - 6) * 32u + (u32)(lane & 3) * 4u - 4u;
  char* peer = P.ws[lane & 7];
  asm volatile("" : "+v"(peer));
  const u64 t0 = __builtin_amdgcn_s_memrealtime();
  if (lane >= 1 && lane < 8) st_sys(peer + OFF_TR + lo + 4u, call);
  auto rl = [&](u32 v, int i) { return (u32)__builtin_amdgcn_readlane(v, i); };
  bool seen = lane < 2 || lane == 8 || (lane >= 16 && lane < 24) || lane >= 48;
  u32 t = 0, sent = 0;
  for (;;) {
    u32 m1 = 0;
    if (!seen) m1 = ld_sys(ws + off + 4u);
    asm volatile("" : "+v"(m1));
    const u64 now = __builtin_amdgcn_s_memrealtime() - t0;
    if (!seen && (m1 >= call || now > DEADLINE_TICKS)) {
      t = lane < 8 ? ld_sys(ws + off) - (u32)t0 : lane < 16 ? (u32)max((int)(u32)now - T_FLY, 0) : (u32)now + (lane < 32 ? M_NEAR : M_FAR) - ((lane & 28) == 24 || (lane & 60) == 32 ? 57u : 0u);
      seen = true;
    }
    const u64 sm = __ballot(seen);
    const u32 rdy = ((sm & 0xffff0000fff0ull) == 0xffff0000fff0ull ? 1u : 0u) | ((sm & 0xff00ff0cull) == 0xff00ff0cull ? 2u : 0u);
    if (rdy & ~sent) {
      const u32 lag = wave_max(lane >> 3 == 1 ? t : 0u);
      u32 tq = t + (u32)__builtin_amdgcn_update_dpp(0, (int)t, 0xb1, 0xf, 0xf, false);
      tq = (tq + (u32)__builtin_amdgcn_update_dpp(0, (int)tq, 0x4e, 0xf, 0xf, false)) >> 2;
      const u32 tf = max(((rl(t, 4) + rl(t, 5) + rl(t, 6) + rl(t, 7)) >> 2) + lag, wave_max(lane >= 32 ? tq : 0u));
      const u32 tn = max(((rl(t, 2) + rl(t, 3)) >> 1) + lag, wave_max(lane >> 4 == 1 ? tq : 0u));
      if (lane < 2 && ((rdy >> lane) & 1u))
        __hip_atomic_store((GAS u64*)(ws + OFF_GO + lo), ((u64)call << 32) | ((lane ? tn : tf) + (u32)t0), __ATOMIC_RELAXED, __HIP_MEMORY_SCOPE_AGENT);
      sent = rdy;
      if (sent == 3u) return;
    }
    __builtin_amdgcn_s_sleep(1);
  }
}

constexpr int NST = 5;

struct Job { int kind, ja, p, dO, sub, nh, nkt; u32 col0; };
__device__ __forceinline__ Job decode_job(u32 code, int a) {
  Job j; j.kind = (code >> 8) & 3; j.ja = (code >> 4) & 15; j.p = code & 15;
  int chalf;
  if (j.kind == JOB_F) { j.dO = 0; j.sub = j.ja & 3; chalf = (a >> 2) ^ (j.ja >> 2); }
  else { j.dO = tile_dist(j.ja); j.sub = tile_sub(j.ja, a); chalf = a >> 2; }
  j.col0 = (u32)(chalf * 2048 + j.p * 128);
  j.nh = j.kind == JOB_F ? 0 : j.dO >= 2 ? 10 : 12;
  j.nkt = 8 + 2 * j.nh;
  return j;
}

struct Polls { u32x2 va, vb; u32 ta, tb;
  __device__ __forceinline__ void retire() { asm volatile("" :: "v"(va), "v"(vb), "v"(ta), "v"(tb)); } };
__device__ __forceinline__ u32 gather_max(char* ws, u32 off, u32 call, int lane, int lazy, Polls& q) {
  u32x2& va = q.va; u32x2& vb = q.vb; u32& ta = q.ta; u32& tb = q.tb;
  const __amdgpu_buffer_rsrc_t rws = mk_rsrc(ws, WS_USED);
  u32 gm = 0, tmo_v;
  const u32 toff = lane ? 0x80000000u : OFF_TMO;
  const u64 t0 = __builtin_amdgcn_s_memrealtime();
  va = __builtin_amdgcn_raw_buffer_load_b64(rws, off, 0, 17);
  ta = __builtin_amdgcn_raw_buffer_load_b32(rws, toff, 0, 17);
  __builtin_amdgcn_s_sleep(SL_AM);
  for (;;) {
    asm volatile("" ::: "memory");
    vb = __builtin_amdgcn_raw_buffer_load_b64(rws, off, 0, 17); tb = __builtin_amdgcn_raw_buffer_load_b32(rws, toff, 0, 17);
    asm volatile("" ::: "memory");
    if (va.y >= call) { gm = va.x; off = 0x80000000u; }
    tmo_v = rfl(ta);
    if (__all(off == 0x80000000u)) break;
    va = __builtin_amdgcn_raw_buffer_load_b64(rws, off, 0, 17); ta = __builtin_amdgcn_raw_buffer_load_b32(rws, toff, 0, 17);
    asm volatile("" ::: "memory");
    if (vb.y >= call) { gm = vb.x; off = 0x80000000u; }
    tmo_v = rfl(tb);
    const u64 missing = __ballot(off != 0x80000000u);
    if (!missing) break;
    if (tmo_v == call || __builtin_amdgcn_s_memrealtime() - t0 > DEADLINE_TICKS) {
      if (lane == 0) st_sys(ws + OFF_TMO, call);
      WAIT_VM(0);
      tmo_v = call;
      break;
    }
    if (__builtin_popcountll(missing) > lazy) __builtin_amdgcn_s_sleep(SL_AML);
  }
  gm = wave_max(gm);
  return tmo_v == call ? ~0u : gm;
}

__device__ __forceinline__ void loader(const Params& P, char* lds, char* ws, int lane, int wave, u32 call, u32 jobs) {
  const int a = P.rank;
  const __amdgpu_buffer_rsrc_t rws = mk_rsrc(ws, WS_USED);
  const u32 l4 = (u32)(wave - 4) * 4u;
  const u32 vx0 = (u32)(lane >> 3) * 1024u + (u32)(((lane & 7) ^ (lane >> 4)) * 16), vx1 = vx0 ^ 64u;
  const u32 vw0 = (u32)(lane >> 4) * 8192u + (u32)(((lane & 15) ^ (lane >> 4)) * 16), vw1 = vw0 ^ 128u;
  const u32 vp = (u32)lane * 16u;
  const u32x4 rx = mk_rsrc4(P.x, X_BYTES), rw = mk_rsrc4(P.w, W_BYTES), rl = mk_rsrc4(ws, WS_USED);
  const u32 lds0 = (u32)(size_t)(LAS char*)lds + l4 * 1024u;
  bool fin = false;
  if (WDELAY && (jobs >> 4) - (JOB_S << 4) >= 3u) sleep_until((u32)__builtin_amdgcn_s_memrealtime() + WDELAY);
  int stamp = 1; (void)stamp;
  for (; (jobs & 0x3ffu) != 0; jobs >>= 10) {
    const Job J = decode_job(jobs & 0x3ffu, a);
    const int nh = J.nh, nkt = J.nkt;
    const u32 row0 = (u32)((a ^ J.dO) * 512 + J.sub * 128);
    const int j0 = fin && J.kind == JOB_O ? 8 : 0;
    fin = fin || J.kind != JOB_S;
    const u64 ko = P.korder[(P.relayed[J.ja] >> J.p) & 1u][J.dO >= 2 ? J.dO - 2 : 6];
    const int fl = lane & 15;
    const bool care = lane < 32 && fl < nh;
    const u32 foff = (lane < 16 ? OFF_XF + (u32)J.ja * 256u : OFF_WF + (u32)J.p * 256u) + ((u32)(ko >> (fl * 4)) & 15u) * 16u;
    const u32 tab = lane < 8 ? (row0 + l4 * 8u) * 1024u + (u32)lane * 128u
                             : OFF_XS + (u32)J.ja * PANEL_B + l4 * 1024u + (u32)(lane & 1) * 16384u + ((u32)(ko >> ((((lane - 8) >> 1) & 15) * 4)) & 15u) * 32768u;
    const u32 dw = OFF_WS - OFF_XS + (u32)(J.p - J.ja) * PANEL_B, w0 = l4 * (4u * 8192u) + J.col0 * 2u;
    int issued = j0, open = 8;
    u32 dst = lds0;
    auto issue = [&]() {
      const u32 ox = __builtin_amdgcn_readlane(tab, issued);
      if (__builtin_expect(issued >= 8, 1)) dma8_lin(dst, vp, rl, ox, ox + dw);
      else {
        dma4(dst, vx0, vx1, vx0, vx1, rx, ox, 8192u);
        dma4(dst + 16384u, vw0, vw0, vw1, vw1, rw, w0 + (u32)issued * (64u * 8192u), 32768u);
      }
      ++issued; dst = dst == lds0 + (NST - 1) * 32768u ? lds0 : dst + 32768u;
    };
    for (int j = j0; j <= nkt; ++j) {
      if (__builtin_expect(issued == j + NST - 2 && issued < open, 1)) {
        issue();
        WAIT_VM(24);
        __builtin_amdgcn_s_barrier();
        continue;
      }
      const int cap = j + NST - (j > j0) < nkt ? j + NST - (j > j0) : nkt;
      if (issued < cap && issued == open) {
        const int got = wave_poll4(ws, rws, foff, care, (open - 8) >> 1, issued == j, call, lane);
        open = max(open, 8 + 2 * got);
      }
      while (issued < cap && issued < open) issue();
      const int after = issued - j - 1;
      if (after >= 4) WAIT_VM(32); else if (after == 3) WAIT_VM(24); else if (after == 2) WAIT_VM(16); else if (after == 1) WAIT_VM(8); else WAIT_VM(0);
      __builtin_amdgcn_s_barrier();
    }
    __builtin_amdgcn_s_barrier();
  }
}

__device__ __forceinline__ void computer(const Params& P, char* lds, char* ws, int lane, int wave, u32 call, u32 jobs) {
  const int a = P.rank;
  const int wr = wave >> 1, wc = wave & 1, fr = lane & 15, g = lane >> 4;
  const u32 xa0 = (u32)((wr * 64 + fr) * 128 + ((g ^ (fr >> 1)) * 16));
  const int tq = fr >> 2, tp = fr & 3;
  const u32 wa0 = 16384u + (u32)((8 * g + tq) * 256 + (((8 * wc + 2 * tp) ^ (tq | ((g & 1) << 3))) * 16) + (tp & 1) * 8);
  const u32x4 rl = mk_rsrc4(ws, WS_USED);
  int stamp = 1; (void)stamp;
  f32x4 fin[4][4];
  u32 fjob = 0;
  const u32 vo = (u32)lane * 16u;

  for (; (jobs & 0x3ffu) != 0; jobs >>= 10) {
    const Job J = decode_job(jobs & 0x3ffu, a);
    const bool cont = fjob != 0 && J.kind == JOB_O;
    f32x4 acc[4][4];
#pragma unroll
    for (int m = 0; m < 4; ++m)
#pragma unroll
      for (int n = 0; n < 4; ++n) acc[m][n] = cont ? fin[m][n] : (f32x4){0.f, 0.f, 0.f, 0.f};
    auto readf = [&](bf16x8 (&xf)[4], bf16x8 (&wf)[4], int slot, int ks) {
      LAS char* st = (LAS char*)lds + slot * 32768;
#pragma unroll
      for (int i = 0; i < 4; ++i) {
        const u32 o = (wa0 ^ (u32)((i >> 1) * 16 + (i & 1) * 8)) + (u32)(ks * 8192);
        const s16x4 lo = __builtin_amdgcn_ds_read_tr16_b64_v4i16((LAS s16x4*)(st + o));
        const s16x4 hi = __builtin_amdgcn_ds_read_tr16_b64_v4i16((LAS s16x4*)(st + o + 1024));
        wf[i] = __builtin_bit_cast(bf16x8, __builtin_shufflevector(lo, hi, 0, 1, 2, 3, 4, 5, 6, 7));
        xf[i] = *(LAS const bf16x8*)(st + ((xa0 ^ (u32)(ks * 64)) + (u32)(i * 2048)));
      }
    };
    auto mma = [&](const bf16x8 (&xf)[4], const bf16x8 (&wf)[4]) {
#pragma unroll
      for (int m = 0; m < 4; ++m)
#pragma unroll
        for (int n = 0; n < 4; ++n) acc[m][n] = __builtin_amdgcn_mfma_f32_16x16x32_bf16(wf[n], xf[m], acc[m][n], 0, 0, 0);
    };
#define INTERLEAVE(id) do { _Pragma("unroll") for (int i_ = 0; i_ < 6; ++i_) { __builtin_amdgcn_sched_group_barrier(0x8, 1, id); __builtin_amdgcn_sched_group_barrier(0x100, 2, id); } \
                            __builtin_amdgcn_sched_group_barrier(0x8, 10, id); __builtin_amdgcn_sched_barrier(0); } while (0)
    bf16x8 x0[4], w0[4], x1[4], w1[4];
    int slot = 0;
    BARRIER();
    readf(x0, w0, 0, 0);
    u64 bk = TAU;
    const int bk_at = J.kind == JOB_S && J.dO == 1 ? J.nkt - 2 : -1;
    for (int kt = cont ? 8 : 0; kt < J.nkt; ++kt) {
      if (kt == bk_at) book(bk, rl, OFF_TP);
      __builtin_amdgcn_s_waitcnt(0xC07F);
      __builtin_amdgcn_sched_barrier(0);
      readf(x1, w1, slot, 1); mma(x0, w0);
      INTERLEAVE(0);
      slot = slot == NST - 1 ? 0 : slot + 1;
      BARRIER();
      __builtin_amdgcn_sched_barrier(0);
      readf(x0, w0, slot, 0); mma(x1, w1);
      INTERLEAVE(1);
    }
    BARRIER();
    STAMP_AT(stamp++);

    if (J.kind == JOB_S) {
      const u32 slot = (u32)(box_base(J.dO) + box_idx(J.ja, J.p));
      u32x4 v[8];
#pragma unroll
      for (int jj = 0; jj < 8; ++jj) {
        const f32x4 u0 = acc[jj >> 1][(jj & 1) * 2], u1 = acc[jj >> 1][(jj & 1) * 2 + 1];
        v[jj].x = pack_bf16(u0[0], u0[1]); v[jj].y = pack_bf16(u0[2], u0[3]); v[jj].z = pack_bf16(u1[0], u1[1]); v[jj].w = pack_bf16(u1[2], u1[3]);
      }
      const u32 so = OFF_IN + slot * 32768u + (u32)wave * 8192u, fo = OFF_PF + slot * 512u + (u32)wave * 128u;
      if (J.dO >= 2) {
#pragma unroll
        for (int jj = 0; jj < 8; ++jj) st16<0>(v[jj], rl, vo, so + (u32)(jj * 1024));
        WAIT_VM(0);
        if (lane == 0) st_sys(P.ws[J.dO] + fo, call);
        STAMP_AT(stamp++);
      } else {
        asm volatile("s_waitcnt vmcnt(0)" : "+v"(bk) :: "memory");
        STAMP_AT(stamp++);
        const u32x4 rd = mk_rsrc4(P.ws[1], WS_USED);
        sleep_until(rfl((u32)(bk >> 4)));
#pragma unroll
        for (int jj = 0; jj < 8; ++jj) {
          st16<0>(v[jj], rd, vo, so + (u32)(jj * 1024));
          if (jj < 7) wait_vm<CAP - 1>();
        }
        WAIT_VM(4);
        if (lane == 0) st_sys(P.ws[1] + fo, call);
        WAIT_VM(1);
        if (lane == 0) st_sys(P.ws[1] + fo + 4u, call);
      }
      STAMP_AT(stamp++);
    } else {
      fjob = jobs & 0x3ffu;
#pragma unroll
      for (int m = 0; m < 4; ++m)
#pragma unroll
        for (int n = 0; n < 4; ++n) fin[m][n] = acc[m][n];
    }
  }
  if (!fjob) return;

  if (threadIdx.x < 2) ((LAS u32*)lds)[threadIdx.x] = 0;
  BARRIER();
  const Job J = decode_job(fjob, a);
  const __amdgpu_buffer_rsrc_t rws = mk_rsrc(ws, WS_USED);
  u32 n = 0;
  char* outp = P.out;
  asm volatile("" : "+s"(outp));
  const __amdgpu_buffer_rsrc_t ro = mk_rsrc(outp, OUT_BYTES);
  {
    float mx = 0.f;
    auto add = [&](int jj, u32x4 c) {
      f32x4& u0 = fin[jj >> 1][(jj & 1) * 2]; f32x4& u1 = fin[jj >> 1][(jj & 1) * 2 + 1];
      u0 += (f32x4){bf_lo(c.x), bf_hi(c.x), bf_lo(c.y), bf_hi(c.y)};
      u1 += (f32x4){bf_lo(c.z), bf_hi(c.z), bf_lo(c.w), bf_hi(c.w)};
    };
    auto done = [&](int jj) {
      f32x4& u0 = fin[jj >> 1][(jj & 1) * 2]; f32x4& u1 = fin[jj >> 1][(jj & 1) * 2 + 1];
#pragma unroll
      for (int e = 0; e < 4; ++e) mx = fmaxf(mx, fmaxf(u0[e], u1[e]));
      asm volatile("" : "+v"(u0), "+v"(u1), "+v"(mx));
    };
    if (J.kind == JOB_F) {
      const bool far = (J.ja >> 2) != 0;
      const int d0 = far ? 4 | ((a ^ J.sub) & 2) : 2;
      const u32 bi = far ? (u32)(J.p * 2 + (J.sub & 1)) : (u32)J.p;
      const u32 s0 = (u32)box_base(d0) + bi, s1 = (u32)box_base(d0 + 1) + bi;
      const __amdgpu_buffer_rsrc_t r0 = mk_rsrc(P.ws[d0], WS_USED), r1 = mk_rsrc(P.ws[d0 + 1], WS_USED);
      const u32 o = OFF_GO + (far ? 0u : 128u);
      wave_wait<SL_GO>(ws, o + 4u, lane == 0, call, lane);
      u32 t_go = rfl(__hip_atomic_load((GAS const u32*)(ws + o), __ATOMIC_RELAXED, __HIP_MEMORY_SCOPE_AGENT));
      const u32 round = (u32)(!far ? J.p >> 3 : J.p < 8 ? J.sub & 1 : J.p >> 2);
      const u32 seat = (u32)((!far || J.p < 8 ? J.p & 7 : (J.p & 3) | (J.sub & 1) << 2) + wave * 8);
      const u32 tok = OFF_TOK + (((u32)d0 * 2u + round) * 32u + seat) * 64u;
      const u32 step = far ? 4 * TAU_X : 4 * TAU_C;
      t_go += (round * 8u * step + seat * (step >> 5)) >> 4;
      sleep_until(t_go - 50u);
      wave_wait<4>(ws, lane == 2 ? tok - 32u * 64u : OFF_PF + (lane == 0 ? s0 : s1) * 512u + (u32)wave * 128u, lane < (round ? 3 : 2), call, lane);
      const u32 so0 = OFF_IN + s0 * 32768u + (u32)wave * 8192u, so1 = OFF_IN + s1 * 32768u + (u32)wave * 8192u;
      u32x4 v0[8], v1[8];
      u32 t_min = t_go;
#pragma unroll
      for (int jj = 0; jj < 8 + PD; ++jj) {
        if (jj < 8) {
          const u32 ti = t_go + (((u32)jj * step) >> 4);
          sleep_until((int)(ti - t_min) > 0 ? ti : t_min);
          t_min = (u32)__builtin_amdgcn_s_memrealtime() + ((step * CATCH) >> 8);
          v0[jj] = __builtin_amdgcn_raw_buffer_load_b128(r0, vo + (u32)((jj & 3) * 1024), so0 + (u32)((jj >> 2) * 4096), 17);
          v1[jj] = __builtin_amdgcn_raw_buffer_load_b128(r1, vo + (u32)((jj & 3) * 1024), so1 + (u32)((jj >> 2) * 4096), 17);
          if (jj == 6 && lane == 0) st_agent(ws + tok, call);
        }
        if (jj >= PD) { add(jj - PD, v0[jj - PD]); add(jj - PD, v1[jj - PD]); done(jj - PD); }
      }
    } else {
      const u32 s0 = (u32)(J.p * 3 + J.sub);
      const u32 so0 = OFF_IN + s0 * 32768u + (u32)wave * 8192u, fo = OFF_PF + s0 * 512u + (u32)wave * 128u;
#pragma unroll
      for (int hf = 0; hf < 2; ++hf) {
        wave_wait<SL_HDR>(ws, fo + (u32)hf * 4u, lane == 0, call, lane);
        u32x4 v0[4];
#pragma unroll
        for (int jj = 0; jj < 4; ++jj) v0[jj] = __builtin_amdgcn_raw_buffer_load_b128(rws, vo + (u32)(jj * 1024), so0 + (u32)(hf * 4096), 17);
#pragma unroll
        for (int jj = 0; jj < 4; ++jj) { add(hf * 4 + jj, v0[jj]); done(hf * 4 + jj); }
      }
    }
    TSTAMP(10);
    const u32 wm = wave_max(__builtin_bit_cast(u32, mx));
    if (lane == 0) {
      __hip_atomic_fetch_max((LAS u32*)lds, wm, __ATOMIC_RELAXED, __HIP_MEMORY_SCOPE_WORKGROUP);
      n = __hip_atomic_fetch_add((LAS u32*)lds + 1, 1u, __ATOMIC_RELAXED, __HIP_MEMORY_SCOPE_WORKGROUP);
      const u32 tm = __hip_atomic_load((LAS u32*)lds, __ATOMIC_RELAXED, __HIP_MEMORY_SCOPE_WORKGROUP);
      if (n == 3u)
        __hip_atomic_store((GAS u64*)(ws + OFF_LM + (u32)(J.p * 8 + (J.kind == JOB_F ? J.ja : J.sub)) * 128u), ((u64)call << 32) | tm, __ATOMIC_RELAXED, __HIP_MEMORY_SCOPE_AGENT);
    }
  }
  TSTAMP(11);
  if (g & 1) {
#pragma unroll
    for (int m = 0; m < 4; ++m) { f32x4 t = fin[m][0]; fin[m][0] = fin[m][1]; fin[m][1] = t; t = fin[m][2]; fin[m][2] = fin[m][3]; fin[m][3] = t; }
  }
  u32 oo[4];
#pragma unroll
  for (int m = 0; m < 4; ++m) { oo[m] = (u32)(J.sub * 128 + wr * 64 + m * 16 + fr) * 4096u + J.col0 + (u32)(wc * 64 + g * 16); asm volatile("" : "+v"(oo[m])); }
  Polls pq = {};
  if (rfl(n) == 3u) {
    const u32 gm = gather_max(ws, lane < NR ? OFF_AM + (u32)lane * 128u : 0x80000000u, call, lane, LZ_F, pq);
    if (lane == 0) ((LAS u32*)lds)[2] = gm;
  }
  BARRIER();
  const u32 gmax = rfl(((LAS u32*)lds)[2]);
  TSTAMP(12);
  const float inv = gmax ? 127.0f * __builtin_amdgcn_rcpf(as_f32(gmax)) : 0.f;
#pragma unroll
  for (int m = 0; m < 4; ++m) {
    u32x4 v;
#pragma unroll
    for (int n = 0; n < 4; ++n) {
      f32x4 y = fin[m][n] * inv;
      asm volatile("" : "+v"(y));
      u32 r = 0;
#pragma unroll
      for (int e = 0; e < 4; ++e) r = __builtin_amdgcn_cvt_pk_u8_f32(y[e], e, r);
      v[n] = r;
    }
    if (gmax == ~0u) v = (u32x4){0x80808080u, 0x80808080u, 0x80808080u, 0x80808080u};
    __builtin_amdgcn_raw_buffer_store_b128(v, ro, oo[m], 0, OUT_AUX);
  }
  pq.retire();
  TSTAMP(13);
}

__global__ void __launch_bounds__(512, 2) gemm_rs_q8(const Params P) {
  __shared__ __attribute__((aligned(16))) char lds[5 * 32768];
  const int lane = threadIdx.x & 63, wave = (int)rfl(threadIdx.x >> 6);
  char* ws = P.ws[0];
  if (__builtin_expect(blockIdx.x < NPUSH, 1)) {
    if (wave < PW) pusher(P, lds, ws, lane, wave);
    return;
  }
  if (blockIdx.x >= NPUSH + 8 && blockIdx.x < NPUSH + 24) {
    const bool second = blockIdx.x >= NPUSH + 16;
    const u32 o = (u32)(P.rank * 8 + (blockIdx.x & 7)) * 256u;
    const char* q = P.ws[wave] + (u32)lane * (2u << 20) + o + (320u << 10);
    bool on = lane < 12 && ((0x13u >> lane) & 1u) != second;
    if (second) {
      sleep_until((u32)__builtin_amdgcn_s_memrealtime() + TDELAY);
      if (wave == 0 && lane >= 12 && lane < 20) { on = true; q = (lane < 14 ? P.out : lane < 17 ? P.x : P.w) + (lane < 14 ? (u32)(lane - 12) * (OUT_BYTES - 32768u) : (u32)((lane - 14) % 3) * (X_BYTES / 2u - 16384u)) + o; }
    }
    if (on) { const u32 t = *(GAS const u32*)q; asm volatile("" :: "v"(t)); }
  }
  const u32 call = draw_call(ws, lane, wave);
  if (blockIdx.x == GRID - 1) {
    if (wave) return;
    coordinator(P, ws, lane, call);
    char* dst = P.ws[lane & 7] + OFF_AM + (u32)P.rank * 128u;
    asm volatile("" : "+v"(dst));
    Polls pq;
    u32 m = 0;
    for (u32 half = 0; half < 2; ++half) m = max(m, gather_max(ws, OFF_LM + (half * 64u + (u32)lane) * 128u, call, lane, LZ_C, pq));
    if (lane < NR) __hip_atomic_store((GAS u64*)dst, ((u64)call << 32) | m, __ATOMIC_RELAXED, __HIP_MEMORY_SCOPE_SYSTEM);
    pq.retire();
    return;
  }
  const u32 jobs = P.jobs[blockIdx.x];
  if (wave >= 4) loader(P, lds, ws, lane, wave, call, jobs);
  else computer(P, lds, ws, lane, wave, call, jobs);
}

#define LAG 12
#define R_FWD 2
#define R_EARLY 6
void build_schedule(Params& P) {
#define RAMP 4
#define SHIFT2 2
#define ORD 0
  static const int orders[5][32] = {
    {0, 1, 2, 16, 17, 5, 18, 6, 19, 7, 20, 8, 21, 22, 23, 9, 10, 11, 12, 24, 25, 26, 27, 4, 3, 28, 29, 30, 31, 13, 14, 15},
    {0, 1, 2, 16, 17, 5, 18, 6, 19, 7, 20, 8, 21, 22, 23, 24, 9, 25, 10, 11, 26, 12, 27, 4, 3, 28, 29, 30, 31, 13, 14, 15},
    {0, 1, 2, 16, 17, 5, 18, 6, 19, 7, 20, 8, 21, 22, 23, 24, 25, 9, 10, 26, 11, 12, 27, 4, 3, 28, 29, 30, 31, 13, 14, 15},
    {0, 1, 2, 16, 17, 18, 5, 19, 6, 20, 7, 21, 8, 22, 23, 24, 9, 25, 10, 26, 11, 12, 27, 4, 3, 28, 29, 30, 31, 13, 14, 15},
    {0, 1, 2, 16, 17, 5, 18, 6, 19, 7, 20, 8, 21, 22, 23, 24, 9, 25, 10, 11, 26, 12, 4, 27, 3, 28, 29, 30, 31, 13, 14, 15}};
  const int* order = orders[ORD];
  int pos[32];
  for (int i = 0; i < 32; ++i) pos[order[i]] = i;
  auto ready = [&](int t, int p) { return pos[t] > pos[16 + p] ? pos[t] : pos[16 + p]; };
  for (int d = 0; d < NR; ++d) {
    unsigned char list[MAXU]; int n = 0;
    for (int i = 0; i < MAXU; ++i) list[i] = 0;
    const int q = P.rank ^ d;
    auto relay = [&](int c) { return c >= 16 + R_END - 2 * RELAY && c < 16 + R_END; };
    if (d >= 2)
      for (int i = 0, nh1 = 0, lag = ((2 * (d - (d >= 4 ? 4 : 2)) + 1) * LAG + 10) / 20; i < 64; ++i) {
        auto put = [&](int k, int h) { const int c = order[k]; if (!(c < 16 && tile_dist(c) == d)) list[n++] = (unsigned char)(c * 2 + h); };
        if (i < 32) put(i, 0);
        for (; nh1 < 32 && nh1 + (nh1 * RAMP / 8 < lag ? nh1 * RAMP / 8 : lag) <= i; ++nh1)
          if (d < 4 || !relay(order[nh1])) put(nh1, 1);
        if (d >= 4 && i + R_EARLY < 32 && relay(order[i + R_EARLY]) && ((q >> 1) & 1) == (order[i + R_EARLY] & 1)) put(i + R_EARLY, 1);
        if (d < 4 && i >= R_FWD && i - R_FWD < 32 && relay(order[i - R_FWD]) && ((P.rank >> 1) & 1) == (order[i - R_FWD] & 1))
          for (int ds = 4; ds < 8; ++ds)
            if (((P.rank ^ ds) & 1) == (d & 1)) list[n++] = (unsigned char)(ds * 16 + order[i - R_FWD] - 16);
      }
    if (OWNLAST && d < 4) {
      unsigned char tmp[MAXU]; int m = 0;
      for (int pass = 0; pass < 2; ++pass)
        for (int i = 0; i < n; ++i) if ((list[i] >= 26 && list[i] < 32) == (pass == 1)) tmp[m++] = list[i];
      for (int i = 0; i < n; ++i) list[i] = tmp[i];
    }
    P.ucount[d] = (u32)n;
    for (int i = 0; i < MAXU / 4; ++i) P.units[d][i] = list[4 * i] | list[4 * i + 1] << 8 | list[4 * i + 2] << 16 | (u32)list[4 * i + 3] << 24;
  }
  for (int t = 0; t < 16; ++t) {
    P.relayed[t] = 0;
    for (int p = R_END - 2 * RELAY; p < R_END; ++p)
      if (((P.rank >> 1) & 1) != (p & 1) && t < 13 && pos[16 + p] + R_FWD > pos[t]) P.relayed[t] |= 1u << p;
  }
  static const int kord[2][12] = {{10, 8, 6, 4, 5, 7, 9, 11, 2, 0, 1, 3}, {10, 8, 6, 4, 2, 0, 1, 3, 5, 7, 9, 11}};
  for (int c = 0; c < 2; ++c)
    for (int sk = 0; sk < 8; ++sk) {
      u64 v = 0;
      for (int i = 0, n = 0; i < 12; ++i) if ((kord[c][i] >> 1) != sk) v |= (u64)kord[c][i] << (4 * n++);
      P.korder[c][sk] = v;
    }
  for (int b = 0; b < NPUSH; ++b)
    for (int w = 0; w < PW; ++w) {
      const int d = push_link(b, w);
      auto code = [&](int i) { return (P.units[d][i >> 4] >> ((i & 12) * 2)) & 0xffu; };
      const int i = push_seat(b, w);
      P.first[b][2 * w] = slice_src(code(i), (u32)i & 3u, P.rank ^ d, d) | code(i) >> 5;
      P.first[b][2 * w + 1] = code(i) | code(i + NW) << 8 | (u32)(P.rank ^ d) << 16;
    }
  static const int grp[16] = {0, 1, 0, 2, 1, 1, 0, 1, 0, 1, 0, 1, 0, 2, 2, 2};
  for (int i = 0; i < GRID; ++i) P.jobs[i] = 0;
  for (int x = 0; x < 8; ++x) {
    int ht[40], hp[40], n = 0;
    for (int r = 0; r < 32; ++r)
      for (int p = 0; p < 16; ++p)
        for (int t = 0; t < 16; ++t) {
          const int gsel = grp[t] == 2 ? (p >> 2) & 1 : grp[t];
          if (gsel == (x >> 2) && ((p + (t == 2 ? SHIFT2 : 0)) & 3) == (x & 3) && ready(t, p) == r) { ht[n] = t; hp[n] = p; ++n; }
        }
    auto enc = [&](int i) { return (u32)(((ht[i] >= 13 ? JOB_O : JOB_S) << 8) | (ht[i] << 4) | hp[i]); };
    const int nw = 32 - NPUSH / 8 - (x == 7);
    const int hi = x >> 2, lo = x & 3;
    const int fsub[10] = {4, 6, 5, 7, 3, 4 + hi, 6 + hi, 3, 5 - hi, 7 - hi};
    const int fpan[10] = {4 * hi + lo, 4 * hi + lo, 4 - 4 * hi + lo, 4 - 4 * hi + lo, x, 8 + lo, 8 + lo, 8 + x, 12 + lo, 12 + lo};
    int f = 0, extra = nw;
    for (int s = 0; s < nw; ++s) {
      u32 jb = enc(s);
      const bool host = extra < n && (ht[s] < 3 ? hp[s] >= PURE && hp[s] < 8 : f >= 4 && ht[s] < 13);
      if (host) { jb = (u32)((JOB_F << 8) | ((ht[extra] - 13) << 4) | hp[extra]) | jb << 10 | enc(extra) << 20; ++extra; }
      else if (ht[s] >= 3 && ht[s] < 13 && f < 10) {
        jb = jb << 10 | (u32)((JOB_F << 8) | (fsub[f] << 4) | fpan[f]); ++f;
      }
      P.jobs[NPUSH + s * 8 + x] = jb;
    }
  }
}

}

extern "C" void kernel_launch(int rank, int nranks, ncclComm_t comm, hipStream_t stream,
                              void* const* d_in, const int* in_sizes, int n_in,
                              void* d_out, int out_size,
                              const int* in_dtypes, int out_dtype,
                              void* const* d_out_all, void* const* d_ws_all, size_t ws_bytes) {
  (void)comm; (void)in_dtypes; (void)out_dtype; (void)d_out_all;
  if (nranks != NR || (unsigned)rank >= (unsigned)NR || n_in != 2 || ws_bytes < (24u << 20) || (u32)out_size != OUT_BYTES ||
      (u32)in_sizes[0] * 2u != X_BYTES || (u32)in_sizes[1] * 2u != W_BYTES) {
    fprintf(stderr, "unexpected problem shape\n");
    return;
  }
  Params P;
  P.x = (const char*)d_in[0]; P.w = (const char*)d_in[1]; P.out = (char*)d_out;
  for (int d = 0; d < NR; ++d) P.ws[d] = (char*)d_ws_all[rank ^ d];
  P.rank = rank; P.pad = 0;
  build_schedule(P);
  hipLaunchKernelGGL(gemm_rs_q8, dim3(GRID), dim3(512), 0, stream, P);
  HIP_CHECK(hipGetLastError());
}
```

```cpp
#include <hip/hip_runtime.h>
#include <rccl/rccl.h>
#include <cstdio>
#include <cstdint>

#define HIP_CHECK(expr) do { hipError_t _e = (expr); if (_e != hipSuccess) { \
    fprintf(stderr, "HIP error %s at %s:%d\n", hipGetErrorString(_e), __FILE__, __LINE__); \
    return; } } while (0)

namespace {

typedef unsigned u32;
typedef unsigned long long u64;
typedef u32 u32x4 __attribute__((ext_vector_type(4)));
typedef u32 u32x2 __attribute__((ext_vector_type(2)));
typedef float f32x4 __attribute__((ext_vector_type(4)));
typedef short s16x4 __attribute__((ext_vector_type(4)));
typedef __bf16 bf16x8 __attribute__((ext_vector_type(8)));
#define LAS __attribute__((address_space(3)))
#define GAS __attribute__((address_space(1)))

constexpr int NR = 8;
constexpr int GRID = 256;
constexpr int NPUSH = 32;
constexpr int PW = 3;
constexpr int NW = NPUSH * PW / 6;
constexpr int WIN = 4;
constexpr int RELAY = 4;
constexpr int MAXU = 72;
constexpr u64 DEADLINE_TICKS = 100000000ull;

constexpr u32 X_BYTES = 4096u * 512u * 2u, W_BYTES = 512u * 4096u * 2u, OUT_BYTES = 512u * 4096u;
constexpr u32 OFF_CNT = 0;
constexpr u32 OFF_TMO = 16u << 10;
constexpr u32 OFF_TK = (16u << 10) + 128u;
constexpr u32 OFF_TP = 18u << 10;
constexpr u32 OFF_T0 = 19u << 10;
constexpr u32 OFF_XF = 20u << 10;
constexpr u32 OFF_WF = 24u << 10;
constexpr u32 OFF_TR = 28u << 10;
constexpr u32 OFF_GO = 29u << 10;
constexpr u32 OFF_AM = 32u << 10;
constexpr u32 OFF_LM = 64u << 10;
constexpr u32 OFF_TOK = 128u << 10;
constexpr u32 OFF_PF = 192u << 10;
constexpr u32 OFF_XS = 2u << 20;
constexpr u32 OFF_WS = 8u << 20;
constexpr u32 OFF_IN = 14u << 20;
constexpr u32 WS_USED = OFF_IN + 208u * 32768u;
constexpr u32 TAU_X = 223, TAU_C = 227;
constexpr u32 TAU = 232;
constexpr u32 T_BOOK = 70;
constexpr u32 ANCH = 28, T_REST = 528;
constexpr u32 M_FAR = 360, M_NEAR = 590;
constexpr int T_FLY = 100;
constexpr u32 WDELAY = 150, TDELAY = 50;
constexpr u32 CATCH = 10;
constexpr int PD = 3;
constexpr int CAP = 6;
constexpr u32 PANEL_B = 6u * 65536u;

enum { JOB_NONE = 0, JOB_S = 1, JOB_O = 2, JOB_F = 3 };

struct Params {
  const char* x; const char* w; char* out;
  char* ws[NR];
  int rank; int pad;
  u32 jobs[GRID];
  u32 first[NPUSH][2 * PW];
  u32 units[NR][MAXU / 4];
  u64 korder[2][8];
  u32 relayed[16];
};

__host__ __device__ inline int tile_dist(int t) { return t < 3 ? 1 : t < 5 ? t - 1 : t < 13 ? 4 + ((t - 5) & 3) : 0; }
__host__ __device__ inline int tile_sub(int t, int a) { return t < 3 ? t : t < 5 ? 3 : t < 13 ? (a & 2) + ((t - 5) >> 2) : t - 13; }
__host__ __device__ inline int box_base(int d) { return d == 1 ? 0 : d < 4 ? 48 + 16 * (d - 2) : 80 + 32 * (d - 4); }
__host__ __device__ inline int box_idx(int t, int p) { return t < 3 ? p * 3 + t : t < 5 ? p : p * 2 + ((t - 5) >> 2); }

__host__ __device__ inline u32 link_units(int d) { return d < 4 ? 62u + 2u * RELAY : 60u - RELAY; }
__host__ __device__ inline int push_link(int block, int wave) { return 2 + ((block >> 3) * PW + wave + (block & 7)) % 6; }
__host__ __device__ inline int push_seat(int block, int wave) { return ((block >> 3) * PW + wave) / 6 * 8 + (block & 7); }
__host__ __device__ inline u32 k_half(int q, int d) { return (u32)(q ^ (d >= 4 ? (q >> 1) & (q ^ d) : 0)) & 1u; }
__host__ __device__ inline u32 slice_src(u32 code, u32 sl, int q, int d) {
  const u32 h = code & 1u, kh = k_half(q, d);
  if (code < 32) {
    const int t = code >> 1;
    return ((u32)((q ^ tile_dist(t)) * 512 + tile_sub(t, q) * 128) + (sl & 1u) * 64u) * 1024u + (kh * 256u + h * 128u + (sl >> 1) * 64u) * 2u;
  }
  return (kh * 256u + h * 128u + sl * 32u) * 8192u + ((u32)(q >> 2) * 2048u + ((code - 32u) >> 1) * 128u) * 2u;
}

__device__ __forceinline__ u32 rfl(u32 v) { return __builtin_amdgcn_readfirstlane(v); }
__device__ __forceinline__ __amdgpu_buffer_rsrc_t mk_rsrc(const void* p, u32 bytes) {
  return __builtin_amdgcn_make_buffer_rsrc((void*)p, 0, bytes, 0x00020000);
}
__device__ __forceinline__ u32x4 mk_rsrc4(const void* p, u32 bytes) {
  u64 a = (u64)p; u32x4 r; r.x = rfl((u32)a); r.y = rfl((u32)(a >> 32)) & 0xffffu; r.z = bytes; r.w = 0x00020000u; return r;
}
__device__ __forceinline__ u32 ld_sys(const char* p) { return __hip_atomic_load((GAS const u32*)p, __ATOMIC_RELAXED, __HIP_MEMORY_SCOPE_SYSTEM); }
__device__ __forceinline__ void st_sys(char* p, u32 v) { __hip_atomic_store((GAS u32*)p, v, __ATOMIC_RELAXED, __HIP_MEMORY_SCOPE_SYSTEM); }
__device__ __forceinline__ void st_agent(char* p, u32 v) { __hip_atomic_store((GAS u32*)p, v, __ATOMIC_RELAXED, __HIP_MEMORY_SCOPE_AGENT); }
#define WAIT_VM(n) asm volatile("s_waitcnt vmcnt(" #n ")" ::: "memory")
#define BARRIER() do { asm volatile("" ::: "memory"); __builtin_amdgcn_s_waitcnt(0xC07F);   __builtin_amdgcn_s_barrier(); asm volatile("" ::: "memory"); } while (0)


template <int SL> __device__ __forceinline__ void wave_wait(char* ws, u32 off, bool care, u32 call, int lane) {
  if (lane == 63 || !care) off = OFF_TMO;
  const u64 t0 = __builtin_amdgcn_s_memrealtime();
  for (;;) {
    const u32 v = ld_sys(ws + off);
    const bool ok = (lane == 63 || !care) ? true : (v >= call);
    if (__all(ok)) break;
    const bool dead = __any(lane == 63 && v == call);
    if (dead || __builtin_amdgcn_s_memrealtime() - t0 > DEADLINE_TICKS) {
      if (lane == 0) st_sys(ws + OFF_TMO, call);
      WAIT_VM(0);
      break;
    }
    __builtin_amdgcn_s_sleep(SL);
  }
  asm volatile("" ::: "memory");
}

__device__ __forceinline__ int wave_poll4(char* ws, __amdgpu_buffer_rsrc_t rws, u32 off, bool care, int have, bool block, u32 call, int lane) {
  if (!care) off = OFF_TMO;
  const u64 t0 = __builtin_amdgcn_s_memrealtime();
  for (;;) {
    const u32x4 v = __builtin_amdgcn_raw_buffer_load_b128(rws, off, 0, 17);
    asm volatile("" ::: "memory");
    const u32 m = (u32)__ballot(care && v.x >= call && v.y >= call && v.z >= call && v.w >= call);
    const int n = __builtin_ctz(~(m & (m >> 16)));
    if (n > have || !block) return n;
    const bool dead = __any(!care && v.x == call);
    if (dead || __builtin_amdgcn_s_memrealtime() - t0 > DEADLINE_TICKS) {
      if (lane == 0) st_sys(ws + OFF_TMO, call);
      WAIT_VM(0);
      return have + 1;
    }
    if (have == 0) __builtin_amdgcn_s_sleep(32); else __builtin_amdgcn_s_sleep(6);
  }
}
#define DMA_SC " sc0 sc1"

typedef __bf16 bf16x2 __attribute__((ext_vector_type(2)));
__device__ __forceinline__ u32 pack_bf16(float lo, float hi) { bf16x2 r; r.x = (__bf16)lo; r.y = (__bf16)hi; return __builtin_bit_cast(u32, r); }
__device__ __forceinline__ float as_f32(u32 v) { return __builtin_bit_cast(float, v); }
__device__ __forceinline__ float bf_lo(u32 v) { return __builtin_bit_cast(float, v << 16); }
__device__ __forceinline__ float bf_hi(u32 v) { return __builtin_bit_cast(float, v & 0xffff0000u); }
__device__ __forceinline__ u32 wave_max(u32 m) {
  m = max(m, (u32)__builtin_amdgcn_update_dpp(0, (int)m, 0x111, 0xf, 0xf, false));
  m = max(m, (u32)__builtin_amdgcn_update_dpp(0, (int)m, 0x112, 0xf, 0xf, false));
  m = max(m, (u32)__builtin_amdgcn_update_dpp(0, (int)m, 0x114, 0xf, 0xf, false));
  m = max(m, (u32)__builtin_amdgcn_update_dpp(0, (int)m, 0x118, 0xf, 0xf, false));
  const u32 a = max((u32)__builtin_amdgcn_readlane(m, 15), (u32)__builtin_amdgcn_readlane(m, 31));
  const u32 b = max((u32)__builtin_amdgcn_readlane(m, 47), (u32)__builtin_amdgcn_readlane(m, 63));
  return max(a, b);
}

__device__ __forceinline__ void dma4(u32 lds_dst, u32 v0, u32 v1, u32 v2, u32 v3, u32x4 rs, u32 so, u32 st) {
  u32 keep, t;
  asm volatile(
      "s_mov_b32 %0, m0\n\t"
      "s_mov_b32 m0, %2\n\ts_add_u32 %1, %8, %9\n\ts_nop 1\n\tbuffer_load_dwordx4 %3, %7, %8 offen" DMA_SC " lds\n\t"
      "s_add_u32 m0, %2, 0x400\n\ts_nop 0\n\tbuffer_load_dwordx4 %4, %7, %1 offen" DMA_SC " lds\n\t"
      "s_add_u32 m0, %2, 0x800\n\ts_add_u32 %1, %1, %9\n\tbuffer_load_dwordx4 %5, %7, %1 offen" DMA_SC " lds\n\t"
      "s_add_u32 m0, %2, 0xc00\n\ts_add_u32 %1, %1, %9\n\tbuffer_load_dwordx4 %6, %7, %1 offen" DMA_SC " lds\n\t"
      "s_mov_b32 m0, %0"
      : "=&s"(keep), "=&s"(t)
      : "s"(lds_dst), "v"(v0), "v"(v1), "v"(v2), "v"(v3), "s"(rs), "s"(so), "s"(st)
      : "memory", "scc");
}

__device__ __forceinline__ void dma8_lin(u32 lds_dst, u32 v, u32x4 rs, u32 sx, u32 sw) {
  u32 keep;
  asm volatile(
      "s_mov_b32 %0, m0\n\ts_mov_b32 m0, %1\n\ts_nop 2\n\t"
      "buffer_load_dwordx4 %2, %3, %4 offen" DMA_SC " lds\n\tbuffer_load_dwordx4 %2, %3, %4 offen offset:1024" DMA_SC " lds\n\t"
      "buffer_load_dwordx4 %2, %3, %4 offen offset:2048" DMA_SC " lds\n\tbuffer_load_dwordx4 %2, %3, %4 offen offset:3072" DMA_SC " lds\n\t"
      "s_add_u32 m0, %1, 0x4000\n\ts_nop 0\n\t"
      "buffer_load_dwordx4 %2, %3, %5 offen" DMA_SC " lds\n\tbuffer_load_dwordx4 %2, %3, %5 offen offset:1024" DMA_SC " lds\n\t"
      "buffer_load_dwordx4 %2, %3, %5 offen offset:2048" DMA_SC " lds\n\tbuffer_load_dwordx4 %2, %3, %5 offen offset:3072" DMA_SC " lds\n\t"
      "s_mov_b32 m0, %0"
      : "=&s"(keep) : "s"(lds_dst), "v"(v), "s"(rs), "s"(sx), "s"(sw) : "memory", "scc");
}

template <int N> __device__ __forceinline__ void wait_vm() { asm volatile("s_waitcnt vmcnt(%0)" :: "n"(N) : "memory"); }
template <int IMM> __device__ __forceinline__ void st16(u32x4 v, u32x4 rs, u32 voff, u32 soff) {
  asm volatile("buffer_store_dwordx4 %0, %1, %2, %3 offen offset:%4 sc0 sc1\n\ts_nop 1" :: "v"(v), "v"(voff), "s"(rs), "s"(soff), "n"(IMM) : "memory");
}
__device__ __forceinline__ void draw_async(u32& v, u32x4 rl, u32 off) {
  u64 ex; v = 1;
  asm volatile("s_mov_b64 %1, exec\n\ts_mov_b64 exec, 1\n\tbuffer_atomic_add %0, off, %2, %3 sc0\n\ts_mov_b64 exec, %1"
               : "+v"(v), "=&s"(ex) : "s"(rl), "s"(off) : "memory");
}
__device__ __forceinline__ void book(u64& tk, u32x4 rl, u32 off) {
  u64 ex; const u64 lo = (__builtin_amdgcn_s_memrealtime() + T_BOOK) * 16u;
  asm volatile("s_mov_b64 %1, exec\n\ts_mov_b64 exec, 1\n\tbuffer_atomic_umax_x2 %2, off, %3, %4\n\tbuffer_atomic_add_x2 %0, off, %3, %4 sc0\n\ts_mov_b64 exec, %1"
               : "+v"(tk), "=&s"(ex) : "v"(lo), "s"(rl), "s"(off) : "memory");
}
__device__ __forceinline__ void sleep_until(u32 due) {
  for (;;) {
    const u32 left = due - (u32)__builtin_amdgcn_s_memrealtime() - 1u;
    if (left >= 10000u) break;
    if (left > 200u) __builtin_amdgcn_s_sleep(48); else if (left > 40u) __builtin_amdgcn_s_sleep(8); else __builtin_amdgcn_s_sleep(1);
  }
}
struct Slice { u32 src, dst, flag, kind, dep; };
__device__ __forceinline__ u32 draw_call(char* ws, int lane, int wave) {
  u32 call = 0;
  if (lane == 0) call = __hip_atomic_fetch_add((GAS u32*)(ws + OFF_CNT + blockIdx.x * 64 + wave * 4), 1u, __ATOMIC_RELAXED, __HIP_MEMORY_SCOPE_AGENT) + 1u;
  return rfl(call);
}
__device__ __forceinline__ void pusher(const Params& P, char* lds, char* ws, int lane, int wave) {
  const u32 vx0 = (u32)(lane >> 3) * 1024u + (u32)(((lane & 7) ^ (lane >> 4)) * 16), vx1 = vx0 ^ 64u;
  const u32 vw0 = (u32)(lane >> 4) * 8192u + (u32)(((lane & 15) ^ (lane >> 4)) * 16), vw1 = vw0 ^ 128u;
  const int d = push_link(blockIdx.x, wave);
  u32 f0 = P.first[blockIdx.x][2 * wave], f1 = P.first[blockIdx.x][2 * wave + 1]; const char* xp = P.x; const char* wp = P.w;
  char* pw = P.ws[d];
  asm volatile("" : "+s"(f0), "+s"(f1), "+s"(xp), "+s"(wp), "+s"(pw), "+s"(ws));
  const u32x4 rx = mk_rsrc4(xp, X_BYTES), rw = mk_rsrc4(wp, W_BYTES), rl = mk_rsrc4(ws, WS_USED);
  const u32 vp = (u32)lane * 16u;
  const u32 ring0 = (u32)(size_t)(LAS char*)lds + (u32)wave * 16384u;
  auto load_all = [&](const Slice& c, u32 buf) {
    const u32 dst = ring0 + buf * 8192u;
    if (c.kind == 0) { dma4(dst, vx0, vx1, vx0, vx1, rx, c.src, 8192u); dma4(dst + 4096u, vx0, vx1, vx0, vx1, rx, c.src + 32768u, 8192u); }
    else if (c.kind == 1) { dma4(dst, vw0, vw0, vw1, vw1, rw, c.src, 32768u); dma4(dst + 4096u, vw0, vw0, vw1, vw1, rw, c.src + 131072u, 32768u); }
    else { dma4(dst, vp, vp, vp, vp, rl, c.src, 1024u); dma4(dst + 4096u, vp, vp, vp, vp, rl, c.src + 4096u, 1024u); }
  };
  { Slice c; c.src = f0 & ~1u; c.kind = f0 & 1u; c.dst = c.flag = c.dep = 0; load_all(c, 0); }
  u32 cnt; draw_async(cnt, rl, OFF_CNT + blockIdx.x * 64u + (u32)wave * 4u);
  const int q = (int)(f1 >> 16);
  const u32 ns = link_units(d) * 4u;
  auto decode = [&](u32 sidx, u32 code) -> Slice {
    Slice r;
    const u32 sl = sidx & 3u, h = code & 1u;
    if (code >= 64) {
      const u32 ds = code >> 4, pp = code & 15u;
      r.kind = 2;
      r.src = OFF_WS + pp * PANEL_B + (ds - 2u) * 65536u + 32768u + sl * 8192u;
      r.dep = OFF_WF + pp * 256u + (ds - 2u) * 32u + 16u + sl * 4u;
      r.dst = OFF_WS + pp * PANEL_B + ((ds ^ (u32)d) - 2u) * 65536u + 32768u + sl * 8192u;
      r.flag = OFF_WF + pp * 256u + ((ds ^ (u32)d) - 2u) * 32u + 16u + sl * 4u;
      return r;
    }
    r.kind = code >> 5; r.src = slice_src(code, sl, q, d); r.dep = OFF_TMO;
    r.dst = (code < 32 ? OFF_XS : OFF_WS) + ((code >> 1) & 15u) * PANEL_B + (u32)(d - 2) * 65536u + h * 32768u + sl * 8192u;
    r.flag = (code < 32 ? OFF_XF : OFF_WF) + ((code >> 1) & 15u) * 256u + (u32)(d - 2) * 32u + h * 16u + sl * 4u;
    return r;
  };
  const u32x4 rd = mk_rsrc4(pw, WS_USED);
  LAS char* ring = (LAS char*)lds + wave * 16384;
  u32 nxt_i = blockIdx.x;
  Slice cur = decode(blockIdx.x, f1 & 0xffu), nxt = decode(nxt_i, (f1 >> 8) & 0xffu);
  u32 buf = 0;
  asm volatile("s_waitcnt vmcnt(0)" : "+v"(cnt), "+s"(cur.dst), "+s"(nxt.src) :: "memory");
  const u32 call = rfl(cnt) + 1u;
  const u32 tk_base = (call - 1u) * (ns - NW);
  const u32 tk_off = OFF_TK + (u32)d * 128u;
  u32 pend = 0;
  auto publish = [&]() {
    if (lane == 0) {
      st_sys(pw + pend, call);
    }
  };
  auto send = [&](int k) {
    const u32x4 v = *(LAS const u32x4*)(ring + buf * 8192u + k * 1024 + lane * 16);
    st16<0>(v, rd, vp, cur.dst + (u32)k * 1024u);
  };
  for (;;) {
    u32 tk;
    send(0);
    load_all(nxt, buf ^ 1u);
    draw_async(tk, rl, tk_off);
    const u32 t_draw = (u32)__builtin_amdgcn_s_memrealtime();
    Slice nn = nxt;
    u32 dv = 0;
#pragma unroll
    for (int k = 0; k < 8; ++k) {
      if (k) send(k);
      if (k == WIN + 1) {
        asm volatile("" : "+v"(tk));
        nxt_i = rfl(tk) - tk_base;
        if (nxt_i == ns - ANCH && lane == 0)
          __hip_atomic_store((GAS u64*)(ws + OFF_T0 + (u32)d * 128u), ((u64)call << 32) | (t_draw + T_REST), __ATOMIC_RELAXED, __HIP_MEMORY_SCOPE_AGENT);
        nxt_i += 2u * NW;
        if (nxt_i < ns) nn = decode(nxt_i, (P.units[d][nxt_i >> 4] >> ((nxt_i & 12u) * 2u)) & 0xffu);
        if (nxt_i < ns && nn.kind == 2) asm volatile("buffer_load_dword %0, off, %1, %2 sc0 sc1" : "+v"(dv) : "s"(rl), "s"(nn.dep) : "memory");
      }
      if (k < WIN) wait_vm<WIN - 1 + 9>(); else wait_vm<WIN - 1>();
      if (k == WIN - 2 && pend) publish();
    }
    pend = cur.flag;
    cur = nxt; nxt = nn; buf ^= 1u;
    if (nxt_i >= ns) break;
    if (nxt.kind == 2) {
      asm volatile("s_waitcnt vmcnt(%1)" : "+v"(dv) : "n"(6 - WIN) : "memory");
      if (rfl(dv) < call) wave_wait<2>(ws, nxt.dep, lane == 0, call, lane);
    }
  }
#pragma unroll
  for (int k = 0; k < 8; ++k) {
    send(k);
    wait_vm<WIN - 1>();
    if (k == WIN - 2) publish();
  }
  pend = cur.flag;
  WAIT_VM(0);
  publish();
}

__device__ __forceinline__ void coordinator(const Params& P, char* ws, int lane, u32 call) {
  const u32 lo = (u32)(lane & 7) * 128u;
  const u32 off = lane == 63 ? OFF_TMO - 4u : lane < 16 ? (lane & 8 ? OFF_TR : OFF_T0) + lo : OFF_WF + (lane < 32 ? 14u : 12u) * 256u + (u32)((lane >> 2) - 6) * 32u + (u32)(lane & 3) * 4u - 4u;
  char* peer = P.ws[lane & 7];
  asm volatile("" : "+v"(peer));
  const u64 t0 = __builtin_amdgcn_s_memrealtime();
  if (lane >= 1 && lane < 8) st_sys(peer + OFF_TR + lo + 4u, call);
  auto rl = [&](u32 v, int i) { return (u32)__builtin_amdgcn_readlane(v, i); };
  bool seen = lane < 2 || lane == 8 || (lane >= 16 && lane < 24) || lane >= 48;
  u32 t = 0, sent = 0;
  for (;;) {
    u32 m1 = 0;
    if (!seen || lane == 63) m1 = ld_sys(ws + off + 4u);
    asm volatile("" : "+v"(m1));
    const u64 now = __builtin_amdgcn_s_memrealtime() - t0;
    const bool dead = rl(m1, 63) == call || now > DEADLINE_TICKS;
    if (dead && lane == 0) st_sys(ws + OFF_TMO, call);
    if (!seen && (m1 >= call || dead)) {
      t = lane < 8 ? ld_sys(ws + off) - (u32)t0 : lane < 16 ? (u32)max((int)(u32)now - T_FLY, 0) : (u32)now + (lane < 32 ? M_NEAR : M_FAR) - ((lane & 28) == 24 || (lane & 60) == 32 ? 57u : 0u);
      seen = true;
    }
    const u64 sm = __ballot(seen);
    const u32 rdy = ((sm & 0xffff0000fff0ull) == 0xffff0000fff0ull ? 1u : 0u) | ((sm & 0xff00ff0cull) == 0xff00ff0cull ? 2u : 0u);
    if (rdy & ~sent) {
      const u32 lag = wave_max(lane >> 3 == 1 ? t : 0u);
      u32 tq = t + (u32)__builtin_amdgcn_update_dpp(0, (int)t, 0xb1, 0xf, 0xf, false);
      tq = (tq + (u32)__builtin_amdgcn_update_dpp(0, (int)tq, 0x4e, 0xf, 0xf, false)) >> 2;
      const u32 tf = max(((rl(t, 4) + rl(t, 5) + rl(t, 6) + rl(t, 7)) >> 2) + lag, wave_max(lane >= 32 ? tq : 0u));
      const u32 tn = max(((rl(t, 2) + rl(t, 3)) >> 1) + lag, wave_max(lane >> 4 == 1 ? tq : 0u));
      if (lane < 2 && ((rdy >> lane) & 1u))
        __hip_atomic_store((GAS u64*)(ws + OFF_GO + lo), ((u64)call << 32) | ((lane ? tn : tf) + (u32)t0), __ATOMIC_RELAXED, __HIP_MEMORY_SCOPE_AGENT);
      sent = rdy;
      if (sent == 3u) return;
    }
    __builtin_amdgcn_s_sleep(1);
  }
}

constexpr int NST = 5;

struct Job { int kind, ja, p, dO, sub, nh, nkt; u32 col0; };
__device__ __forceinline__ Job decode_job(u32 code, int a) {
  Job j; j.kind = (code >> 8) & 3; j.ja = (code >> 4) & 15; j.p = code & 15;
  int chalf;
  if (j.kind == JOB_F) { j.dO = 0; j.sub = j.ja & 3; chalf = (a >> 2) ^ (j.ja >> 2); }
  else { j.dO = tile_dist(j.ja); j.sub = tile_sub(j.ja, a); chalf = a >> 2; }
  j.col0 = (u32)(chalf * 2048 + j.p * 128);
  j.nh = j.kind == JOB_F ? 0 : j.dO >= 2 ? 10 : 12;
  j.nkt = 8 + 2 * j.nh;
  return j;
}

struct Polls { u32x2 va, vb; u32 ta, tb;
  __device__ __forceinline__ void retire() { asm volatile("" :: "v"(va), "v"(vb), "v"(ta), "v"(tb)); } };
__device__ __forceinline__ u32 gather_max(char* ws, u32 off, u32 call, int lane, int lazy, Polls& q) {
  u32x2& va = q.va; u32x2& vb = q.vb; u32& ta = q.ta; u32& tb = q.tb;
  const __amdgpu_buffer_rsrc_t rws = mk_rsrc(ws, WS_USED);
  u32 gm = 0, tmo_v;
  const u32 toff = lane ? 0x80000000u : OFF_TMO;
  const u64 t0 = __builtin_amdgcn_s_memrealtime();
  va = __builtin_amdgcn_raw_buffer_load_b64(rws, off, 0, 17);
  ta = __builtin_amdgcn_raw_buffer_load_b32(rws, toff, 0, 17);
  __builtin_amdgcn_s_sleep(3);
  for (;;) {
    asm volatile("" ::: "memory");
    vb = __builtin_amdgcn_raw_buffer_load_b64(rws, off, 0, 17); tb = __builtin_amdgcn_raw_buffer_load_b32(rws, toff, 0, 17);
    asm volatile("" ::: "memory");
    if (va.y >= call) { gm = va.x; off = 0x80000000u; }
    tmo_v = rfl(ta);
    if (__all(off == 0x80000000u)) break;
    va = __builtin_amdgcn_raw_buffer_load_b64(rws, off, 0, 17); ta = __builtin_amdgcn_raw_buffer_load_b32(rws, toff, 0, 17);
    asm volatile("" ::: "memory");
    if (vb.y >= call) { gm = vb.x; off = 0x80000000u; }
    tmo_v = rfl(tb);
    const u64 missing = __ballot(off != 0x80000000u);
    if (!missing) break;
    if (tmo_v == call || __builtin_amdgcn_s_memrealtime() - t0 > DEADLINE_TICKS) {
      if (lane == 0) st_sys(ws + OFF_TMO, call);
      WAIT_VM(0);
      tmo_v = call;
      break;
    }
    if (__builtin_popcountll(missing) > lazy) __builtin_amdgcn_s_sleep(16);
  }
  gm = wave_max(gm);
  return tmo_v == call ? ~0u : gm;
}

__device__ __forceinline__ void loader(const Params& P, char* lds, char* ws, int lane, int wave, u32 call, u32 jobs) {
  const int a = P.rank;
  const __amdgpu_buffer_rsrc_t rws = mk_rsrc(ws, WS_USED);
  const u32 l4 = (u32)(wave - 4) * 4u;
  const u32 vx0 = (u32)(lane >> 3) * 1024u + (u32)(((lane & 7) ^ (lane >> 4)) * 16), vx1 = vx0 ^ 64u;
  const u32 vw0 = (u32)(lane >> 4) * 8192u + (u32)(((lane & 15) ^ (lane >> 4)) * 16), vw1 = vw0 ^ 128u;
  const u32 vp = (u32)lane * 16u;
  const u32x4 rx = mk_rsrc4(P.x, X_BYTES), rw = mk_rsrc4(P.w, W_BYTES), rl = mk_rsrc4(ws, WS_USED);
  const u32 lds0 = (u32)(size_t)(LAS char*)lds + l4 * 1024u;
  bool fin = false;
  if ((jobs >> 4) - (JOB_S << 4) >= 3u) sleep_until((u32)__builtin_amdgcn_s_memrealtime() + WDELAY);
  for (; (jobs & 0x3ffu) != 0; jobs >>= 10) {
    const Job J = decode_job(jobs & 0x3ffu, a);
    const int nh = J.nh, nkt = J.nkt;
    const u32 row0 = (u32)((a ^ J.dO) * 512 + J.sub * 128);
    const int j0 = fin && J.kind == JOB_O ? 8 : 0;
    fin = fin || J.kind != JOB_S;
    const u64 ko = P.korder[(P.relayed[J.ja] >> J.p) & 1u][J.dO >= 2 ? J.dO - 2 : 6];
    const int fl = lane & 15;
    const bool care = lane < 32 && fl < nh;
    const u32 foff = (lane < 16 ? OFF_XF + (u32)J.ja * 256u : OFF_WF + (u32)J.p * 256u) + ((u32)(ko >> (fl * 4)) & 15u) * 16u;
    const u32 tab = lane < 8 ? (row0 + l4 * 8u) * 1024u + (u32)lane * 128u
                             : OFF_XS + (u32)J.ja * PANEL_B + l4 * 1024u + (u32)(lane & 1) * 16384u + ((u32)(ko >> ((((lane - 8) >> 1) & 15) * 4)) & 15u) * 32768u;
    const u32 dw = OFF_WS - OFF_XS + (u32)(J.p - J.ja) * PANEL_B, w0 = l4 * (4u * 8192u) + J.col0 * 2u;
    int issued = j0, open = 8;
    u32 dst = lds0;
    auto issue = [&]() {
      const u32 ox = __builtin_amdgcn_readlane(tab, issued);
      if (__builtin_expect(issued >= 8, 1)) dma8_lin(dst, vp, rl, ox, ox + dw);
      else {
        dma4(dst, vx0, vx1, vx0, vx1, rx, ox, 8192u);
        dma4(dst + 16384u, vw0, vw0, vw1, vw1, rw, w0 + (u32)issued * (64u * 8192u), 32768u);
      }
      ++issued; dst = dst == lds0 + (NST - 1) * 32768u ? lds0 : dst + 32768u;
    };
    for (int j = j0; j <= nkt; ++j) {
      if (__builtin_expect(issued == j + NST - 2 && issued < open, 1)) {
        issue();
        WAIT_VM(24);
        __builtin_amdgcn_s_barrier();
        continue;
      }
      const int cap = j + NST - (j > j0) < nkt ? j + NST - (j > j0) : nkt;
      if (issued < cap && issued == open) {
        const int got = wave_poll4(ws, rws, foff, care, (open - 8) >> 1, issued == j, call, lane);
        open = max(open, 8 + 2 * got);
      }
      while (issued < cap && issued < open) issue();
      const int after = issued - j - 1;
      if (after >= 4) WAIT_VM(32); else if (after == 3) WAIT_VM(24); else if (after == 2) WAIT_VM(16); else if (after == 1) WAIT_VM(8); else WAIT_VM(0);
      __builtin_amdgcn_s_barrier();
    }
    __builtin_amdgcn_s_barrier();
  }
}

__device__ __forceinline__ void computer(const Params& P, char* lds, char* ws, int lane, int wave, u32 call, u32 jobs) {
  const int a = P.rank;
  const int wr = wave >> 1, wc = wave & 1, fr = lane & 15, g = lane >> 4;
  const u32 xa0 = (u32)((wr * 64 + fr) * 128 + ((g ^ (fr >> 1)) * 16));
  const int tq = fr >> 2, tp = fr & 3;
  const u32 wa0 = 16384u + (u32)((8 * g + tq) * 256 + (((8 * wc + 2 * tp) ^ (tq | ((g & 1) << 3))) * 16) + (tp & 1) * 8);
  const u32x4 rl = mk_rsrc4(ws, WS_USED);
  f32x4 fin[4][4];
  u32 fjob = 0;
  const u32 vo = (u32)lane * 16u;

  for (; (jobs & 0x3ffu) != 0; jobs >>= 10) {
    const Job J = decode_job(jobs & 0x3ffu, a);
    const bool cont = fjob != 0 && J.kind == JOB_O;
    f32x4 acc[4][4];
#pragma unroll
    for (int m = 0; m < 4; ++m)
#pragma unroll
      for (int n = 0; n < 4; ++n) acc[m][n] = cont ? fin[m][n] : (f32x4){0.f, 0.f, 0.f, 0.f};
    auto readf = [&](bf16x8 (&xf)[4], bf16x8 (&wf)[4], int slot, int ks) {
      LAS char* st = (LAS char*)lds + slot * 32768;
#pragma unroll
      for (int i = 0; i < 4; ++i) {
        const u32 o = (wa0 ^ (u32)((i >> 1) * 16 + (i & 1) * 8)) + (u32)(ks * 8192);
        const s16x4 lo = __builtin_amdgcn_ds_read_tr16_b64_v4i16((LAS s16x4*)(st + o));
        const s16x4 hi = __builtin_amdgcn_ds_read_tr16_b64_v4i16((LAS s16x4*)(st + o + 1024));
        wf[i] = __builtin_bit_cast(bf16x8, __builtin_shufflevector(lo, hi, 0, 1, 2, 3, 4, 5, 6, 7));
        xf[i] = *(LAS const bf16x8*)(st + ((xa0 ^ (u32)(ks * 64)) + (u32)(i * 2048)));
      }
    };
    auto mma = [&](const bf16x8 (&xf)[4], const bf16x8 (&wf)[4]) {
#pragma unroll
      for (int m = 0; m < 4; ++m)
#pragma unroll
        for (int n = 0; n < 4; ++n) acc[m][n] = __builtin_amdgcn_mfma_f32_16x16x32_bf16(wf[n], xf[m], acc[m][n], 0, 0, 0);
    };
#define INTERLEAVE(id) do { _Pragma("unroll") for (int i_ = 0; i_ < 6; ++i_) { __builtin_amdgcn_sched_group_barrier(0x8, 1, id); __builtin_amdgcn_sched_group_barrier(0x100, 2, id); } \
                            __builtin_amdgcn_sched_group_barrier(0x8, 10, id); __builtin_amdgcn_sched_barrier(0); } while (0)
    bf16x8 x0[4], w0[4], x1[4], w1[4];
    int slot = 0;
    BARRIER();
    readf(x0, w0, 0, 0);
    u64 bk = TAU;
    const int bk_at = J.kind == JOB_S && J.dO == 1 ? J.nkt - 2 : -1;
    for (int kt = cont ? 8 : 0; kt < J.nkt; ++kt) {
      if (kt == bk_at) book(bk, rl, OFF_TP);
      __builtin_amdgcn_s_waitcnt(0xC07F);
      __builtin_amdgcn_sched_barrier(0);
      readf(x1, w1, slot, 1); mma(x0, w0);
      INTERLEAVE(0);
      slot = slot == NST - 1 ? 0 : slot + 1;
      BARRIER();
      __builtin_amdgcn_sched_barrier(0);
      readf(x0, w0, slot, 0); mma(x1, w1);
      INTERLEAVE(1);
    }
    BARRIER();

    if (J.kind == JOB_S) {
      const u32 slot = (u32)(box_base(J.dO) + box_idx(J.ja, J.p));
      u32x4 v[8];
#pragma unroll
      for (int jj = 0; jj < 8; ++jj) {
        const f32x4 u0 = acc[jj >> 1][(jj & 1) * 2], u1 = acc[jj >> 1][(jj & 1) * 2 + 1];
        v[jj].x = pack_bf16(u0[0], u0[1]); v[jj].y = pack_bf16(u0[2], u0[3]); v[jj].z = pack_bf16(u1[0], u1[1]); v[jj].w = pack_bf16(u1[2], u1[3]);
      }
      const u32 so = OFF_IN + slot * 32768u + (u32)wave * 8192u, fo = OFF_PF + slot * 512u + (u32)wave * 128u;
      if (J.dO >= 2) {
#pragma unroll
        for (int jj = 0; jj < 8; ++jj) st16<0>(v[jj], rl, vo, so + (u32)(jj * 1024));
        WAIT_VM(0);
        if (lane == 0) st_sys(P.ws[J.dO] + fo, call);
      } else {
        asm volatile("s_waitcnt vmcnt(0)" : "+v"(bk) :: "memory");
        const u32x4 rd = mk_rsrc4(P.ws[1], WS_USED);
        sleep_until(rfl((u32)(bk >> 4)));
#pragma unroll
        for (int jj = 0; jj < 8; ++jj) {
          st16<0>(v[jj], rd, vo, so + (u32)(jj * 1024));
          if (jj < 7) wait_vm<CAP - 1>();
        }
        WAIT_VM(4);
        if (lane == 0) st_sys(P.ws[1] + fo, call);
        WAIT_VM(1);
        if (lane == 0) st_sys(P.ws[1] + fo + 4u, call);
      }
    } else {
      fjob = jobs & 0x3ffu;
#pragma unroll
      for (int m = 0; m < 4; ++m)
#pragma unroll
        for (int n = 0; n < 4; ++n) fin[m][n] = acc[m][n];
    }
  }
  if (!fjob) return;

  if (threadIdx.x < 2) ((LAS u32*)lds)[threadIdx.x] = 0;
  BARRIER();
  const Job J = decode_job(fjob, a);
  const __amdgpu_buffer_rsrc_t rws = mk_rsrc(ws, WS_USED);
  u32 n = 0;
  char* outp = P.out;
  asm volatile("" : "+s"(outp));
  const __amdgpu_buffer_rsrc_t ro = mk_rsrc(outp, OUT_BYTES);
  {
    float mx = 0.f;
    auto add = [&](int jj, u32x4 c) {
      f32x4& u0 = fin[jj >> 1][(jj & 1) * 2]; f32x4& u1 = fin[jj >> 1][(jj & 1) * 2 + 1];
      u0 += (f32x4){bf_lo(c.x), bf_hi(c.x), bf_lo(c.y), bf_hi(c.y)};
      u1 += (f32x4){bf_lo(c.z), bf_hi(c.z), bf_lo(c.w), bf_hi(c.w)};
    };
    auto done = [&](int jj) {
      f32x4& u0 = fin[jj >> 1][(jj & 1) * 2]; f32x4& u1 = fin[jj >> 1][(jj & 1) * 2 + 1];
#pragma unroll
      for (int e = 0; e < 4; ++e) mx = fmaxf(mx, fmaxf(u0[e], u1[e]));
      asm volatile("" : "+v"(u0), "+v"(u1), "+v"(mx));
    };
    if (J.kind == JOB_F) {
      const bool far = (J.ja >> 2) != 0;
      const int d0 = far ? 4 | ((a ^ J.sub) & 2) : 2;
      const u32 bi = far ? (u32)(J.p * 2 + (J.sub & 1)) : (u32)J.p;
      const u32 s0 = (u32)box_base(d0) + bi, s1 = (u32)box_base(d0 + 1) + bi;
      const __amdgpu_buffer_rsrc_t r0 = mk_rsrc(P.ws[d0], WS_USED), r1 = mk_rsrc(P.ws[d0 + 1], WS_USED);
      const u32 o = OFF_GO + (far ? 0u : 128u);
      wave_wait<16>(ws, o + 4u, lane == 0, call, lane);
      u32 t_go = rfl(__hip_atomic_load((GAS const u32*)(ws + o), __ATOMIC_RELAXED, __HIP_MEMORY_SCOPE_AGENT));
      const u32 round = (u32)(!far ? J.p >> 3 : J.p < 8 ? J.sub & 1 : J.p >> 2);
      const u32 seat = (u32)((!far || J.p < 8 ? J.p & 7 : (J.p & 3) | (J.sub & 1) << 2) + wave * 8);
      const u32 tok = OFF_TOK + (((u32)d0 * 2u + round) * 32u + seat) * 64u;
      const u32 step = far ? 4 * TAU_X : 4 * TAU_C;
      t_go += (round * 8u * step + seat * (step >> 5)) >> 4;
      sleep_until(t_go - 50u);
      wave_wait<4>(ws, lane == 2 ? tok - 32u * 64u : OFF_PF + (lane == 0 ? s0 : s1) * 512u + (u32)wave * 128u, lane < (round ? 3 : 2), call, lane);
      const u32 so0 = OFF_IN + s0 * 32768u + (u32)wave * 8192u, so1 = OFF_IN + s1 * 32768u + (u32)wave * 8192u;
      u32x4 v0[8], v1[8];
      u32 t_min = t_go;
#pragma unroll
      for (int jj = 0; jj < 8 + PD; ++jj) {
        if (jj < 8) {
          const u32 ti = t_go + (((u32)jj * step) >> 4);
          sleep_until((int)(ti - t_min) > 0 ? ti : t_min);
          t_min = (u32)__builtin_amdgcn_s_memrealtime() + ((step * CATCH) >> 8);
          v0[jj] = __builtin_amdgcn_raw_buffer_load_b128(r0, vo + (u32)((jj & 3) * 1024), so0 + (u32)((jj >> 2) * 4096), 17);
          v1[jj] = __builtin_amdgcn_raw_buffer_load_b128(r1, vo + (u32)((jj & 3) * 1024), so1 + (u32)((jj >> 2) * 4096), 17);
          if (jj == 6 && lane == 0) st_agent(ws + tok, call);
        }
        if (jj >= PD) { add(jj - PD, v0[jj - PD]); add(jj - PD, v1[jj - PD]); done(jj - PD); }
      }
    } else {
      const u32 s0 = (u32)(J.p * 3 + J.sub);
      const u32 so0 = OFF_IN + s0 * 32768u + (u32)wave * 8192u, fo = OFF_PF + s0 * 512u + (u32)wave * 128u;
#pragma unroll
      for (int hf = 0; hf < 2; ++hf) {
        wave_wait<4>(ws, fo + (u32)hf * 4u, lane == 0, call, lane);
        u32x4 v0[4];
#pragma unroll
        for (int jj = 0; jj < 4; ++jj) v0[jj] = __builtin_amdgcn_raw_buffer_load_b128(rws, vo + (u32)(jj * 1024), so0 + (u32)(hf * 4096), 17);
#pragma unroll
        for (int jj = 0; jj < 4; ++jj) { add(hf * 4 + jj, v0[jj]); done(hf * 4 + jj); }
      }
    }
    const u32 wm = wave_max(__builtin_bit_cast(u32, mx) & 0x7fffffffu);
    if (lane == 0) {
      __hip_atomic_fetch_max((LAS u32*)lds, wm, __ATOMIC_RELAXED, __HIP_MEMORY_SCOPE_WORKGROUP);
      asm volatile("" ::: "memory");
      n = __hip_atomic_fetch_add((LAS u32*)lds + 1, 1u, __ATOMIC_RELAXED, __HIP_MEMORY_SCOPE_WORKGROUP);
      asm volatile("" ::: "memory");
      const u32 tm = __hip_atomic_load((LAS u32*)lds, __ATOMIC_RELAXED, __HIP_MEMORY_SCOPE_WORKGROUP);
      if (n == 3u)
        __hip_atomic_store((GAS u64*)(ws + OFF_LM + (u32)(J.p * 8 + (J.kind == JOB_F ? J.ja : J.sub)) * 128u), ((u64)call << 32) | tm, __ATOMIC_RELAXED, __HIP_MEMORY_SCOPE_AGENT);
    }
  }
  if (g & 1) {
#pragma unroll
    for (int m = 0; m < 4; ++m) { f32x4 t = fin[m][0]; fin[m][0] = fin[m][1]; fin[m][1] = t; t = fin[m][2]; fin[m][2] = fin[m][3]; fin[m][3] = t; }
  }
  u32 oo[4];
#pragma unroll
  for (int m = 0; m < 4; ++m) { oo[m] = (u32)(J.sub * 128 + wr * 64 + m * 16 + fr) * 4096u + J.col0 + (u32)(wc * 64 + g * 16); asm volatile("" : "+v"(oo[m])); }
  Polls pq = {};
  if (rfl(n) == 3u) {
    const u32 gm = gather_max(ws, lane < NR ? OFF_AM + (u32)lane * 128u : 0x80000000u, call, lane, 8, pq);
    if (lane == 0) ((LAS u32*)lds)[2] = gm;
  }
  BARRIER();
  const u32 gmax = rfl(((LAS u32*)lds)[2]);
  const float inv = gmax ? 127.0f / as_f32(gmax) : 0.f;
#pragma unroll
  for (int m = 0; m < 4; ++m) {
    u32x4 v;
#pragma unroll
    for (int n = 0; n < 4; ++n) {
      f32x4 y = fin[m][n] * inv;
      asm volatile("" : "+v"(y));
      u32 r = 0;
#pragma unroll
      for (int e = 0; e < 4; ++e) r = __builtin_amdgcn_cvt_pk_u8_f32(y[e], e, r);
      v[n] = r;
    }
    if (gmax == ~0u) v = (u32x4){0x80808080u, 0x80808080u, 0x80808080u, 0x80808080u};
    __builtin_amdgcn_raw_buffer_store_b128(v, ro, oo[m], 0, 17);
  }
  pq.retire();
}

__global__ void __launch_bounds__(512, 2) gemm_rs_q8(const Params P) {
  __shared__ __attribute__((aligned(16))) char lds[5 * 32768];
  const int lane = threadIdx.x & 63, wave = (int)rfl(threadIdx.x >> 6);
  char* ws = P.ws[0];
  if (__builtin_expect(blockIdx.x < NPUSH, 1)) {
    if (wave < PW) pusher(P, lds, ws, lane, wave);
    return;
  }
  if (blockIdx.x >= NPUSH + 8 && blockIdx.x < NPUSH + 24) {
    const bool second = blockIdx.x >= NPUSH + 16;
    const u32 o = (u32)(P.rank * 8 + (blockIdx.x & 7)) * 256u;
    const char* q = P.ws[wave] + (u32)lane * (2u << 20) + o + (320u << 10);
    bool on = lane < 12 && ((0x13u >> lane) & 1u) != second;
    if (second) {
      sleep_until((u32)__builtin_amdgcn_s_memrealtime() + TDELAY);
      if (wave == 0 && lane >= 12 && lane < 20) { on = true; q = (lane < 14 ? P.out : lane < 17 ? P.x : P.w) + (lane < 14 ? (u32)(lane - 12) * (OUT_BYTES - 32768u) : (u32)((lane - 14) % 3) * (X_BYTES / 2u - 16384u)) + o; }
    }
    if (on) { const u32 t = ld_sys(q); asm volatile("" :: "v"(t)); }
  }
  const u32 call = draw_call(ws, lane, wave);
  if (blockIdx.x == GRID - 1) {
    if (wave) return;
    coordinator(P, ws, lane, call);
    char* dst = P.ws[lane & 7] + OFF_AM + (u32)P.rank * 128u;
    asm volatile("" : "+v"(dst));
    Polls pq;
    u32 m = 0;
    for (u32 half = 0; half < 2; ++half) m = max(m, gather_max(ws, OFF_LM + (half * 64u + (u32)lane) * 128u, call, lane, 40, pq));
    if (lane < NR) __hip_atomic_store((GAS u64*)dst, ((u64)call << 32) | m, __ATOMIC_RELAXED, __HIP_MEMORY_SCOPE_SYSTEM);
    pq.retire();
    return;
  }
  const u32 jobs = P.jobs[blockIdx.x];
  if (wave >= 4) loader(P, lds, ws, lane, wave, call, jobs);
  else computer(P, lds, ws, lane, wave, call, jobs);
}

constexpr int LAG = 12;
constexpr int RAMP = 4;
constexpr int R_FWD = 2;
constexpr int R_EARLY = 6;
void build_schedule(Params& P) {
  static const int order[32] = {0, 1, 2, 16, 17, 5, 18, 6, 19, 7, 20, 8, 21, 22, 23, 9, 10, 11, 12, 24, 25, 26, 27, 4, 3, 28, 29, 30, 31, 13, 14, 15};
  int pos[32];
  for (int i = 0; i < 32; ++i) pos[order[i]] = i;
  auto ready = [&](int t, int p) { return pos[t] > pos[16 + p] ? pos[t] : pos[16 + p]; };
  for (int d = 0; d < NR; ++d) {
    unsigned char list[MAXU]; int n = 0;
    for (int i = 0; i < MAXU; ++i) list[i] = 0;
    const int q = P.rank ^ d;
    auto relay = [&](int c) { return c >= 32 - 2 * RELAY; };
    if (d >= 2)
      for (int i = 0, nh1 = 0, lag = ((2 * (d - (d >= 4 ? 4 : 2)) + 1) * LAG + 10) / 20; i < 64; ++i) {
        auto put = [&](int k, int h) { const int c = order[k]; if (!(c < 16 && tile_dist(c) == d)) list[n++] = (unsigned char)(c * 2 + h); };
        if (i < 32) put(i, 0);
        for (; nh1 < 32 && nh1 + (nh1 * RAMP / 8 < lag ? nh1 * RAMP / 8 : lag) <= i; ++nh1)
          if (d < 4 || !relay(order[nh1])) put(nh1, 1);
        if (d >= 4 && i + R_EARLY < 32 && relay(order[i + R_EARLY]) && ((q >> 1) & 1) == (order[i + R_EARLY] & 1)) put(i + R_EARLY, 1);
        if (d < 4 && i >= R_FWD && i - R_FWD < 32 && relay(order[i - R_FWD]) && ((P.rank >> 1) & 1) == (order[i - R_FWD] & 1))
          for (int ds = 4; ds < 8; ++ds)
            if (((P.rank ^ ds) & 1) == (d & 1)) list[n++] = (unsigned char)(ds * 16 + order[i - R_FWD] - 16);
      }
    if (d < 4) {
      unsigned char tmp[MAXU]; int m = 0;
      for (int pass = 0; pass < 2; ++pass)
        for (int i = 0; i < n; ++i) if ((list[i] >= 26 && list[i] < 32) == (pass == 1)) tmp[m++] = list[i];
      for (int i = 0; i < n; ++i) list[i] = tmp[i];
    }
    for (int i = 0; i < MAXU / 4; ++i) P.units[d][i] = list[4 * i] | list[4 * i + 1] << 8 | list[4 * i + 2] << 16 | (u32)list[4 * i + 3] << 24;
  }
  for (int t = 0; t < 16; ++t) {
    P.relayed[t] = 0;
    for (int p = 16 - 2 * RELAY; p < 16; ++p)
      if (((P.rank >> 1) & 1) != (p & 1) && t < 13 && pos[16 + p] + R_FWD > pos[t]) P.relayed[t] |= 1u << p;
  }
  static const int kord[2][12] = {{10, 8, 6, 4, 5, 7, 9, 11, 2, 0, 1, 3}, {10, 8, 6, 4, 2, 0, 1, 3, 5, 7, 9, 11}};
  for (int c = 0; c < 2; ++c)
    for (int sk = 0; sk < 8; ++sk) {
      u64 v = 0;
      for (int i = 0, n = 0; i < 12; ++i) if ((kord[c][i] >> 1) != sk) v |= (u64)kord[c][i] << (4 * n++);
      P.korder[c][sk] = v;
    }
  for (int b = 0; b < NPUSH; ++b)
    for (int w = 0; w < PW; ++w) {
      const int d = push_link(b, w);
      auto code = [&](int i) { return (P.units[d][i >> 4] >> ((i & 12) * 2)) & 0xffu; };
      const int i = push_seat(b, w);
      P.first[b][2 * w] = slice_src(code(i), (u32)i & 3u, P.rank ^ d, d) | code(i) >> 5;
      P.first[b][2 * w + 1] = code(i) | code(i + NW) << 8 | (u32)(P.rank ^ d) << 16;
    }
  static const int grp[16] = {0, 1, 0, 2, 1, 1, 0, 1, 0, 1, 0, 1, 0, 2, 2, 2};
  for (int i = 0; i < GRID; ++i) P.jobs[i] = 0;
  for (int x = 0; x < 8; ++x) {
    int ht[40], hp[40], n = 0;
    for (int r = 0; r < 32; ++r)
      for (int p = 0; p < 16; ++p)
        for (int t = 0; t < 16; ++t) {
          const int gsel = grp[t] == 2 ? (p >> 2) & 1 : grp[t];
          if (gsel == (x >> 2) && ((p + (t == 2 ? 2 : 0)) & 3) == (x & 3) && ready(t, p) == r) { ht[n] = t; hp[n] = p; ++n; }
        }
    auto enc = [&](int i) { return (u32)(((ht[i] >= 13 ? JOB_O : JOB_S) << 8) | (ht[i] << 4) | hp[i]); };
    const int nw = 32 - NPUSH / 8 - (x == 7);
    const int hi = x >> 2, lo = x & 3;
    const int fsub[10] = {4, 6, 5, 7, 3, 4 + hi, 6 + hi, 3, 5 - hi, 7 - hi};
    const int fpan[10] = {4 * hi + lo, 4 * hi + lo, 4 - 4 * hi + lo, 4 - 4 * hi + lo, x, 8 + lo, 8 + lo, 8 + x, 12 + lo, 12 + lo};
    int f = 0, extra = nw;
    for (int s = 0; s < nw; ++s) {
      u32 jb = enc(s);
      const bool host = extra < n && (ht[s] < 3 ? hp[s] >= 4 && hp[s] < 8 : f >= 4 && ht[s] < 13);
      if (host) { jb = (u32)((JOB_F << 8) | ((ht[extra] - 13) << 4) | hp[extra]) | jb << 10 | enc(extra) << 20; ++extra; }
      else if (ht[s] >= 3 && ht[s] < 13 && f < 10) {
        jb = jb << 10 | (u32)((JOB_F << 8) | (fsub[f] << 4) | fpan[f]); ++f;
      }
      P.jobs[NPUSH + s * 8 + x] = jb;
    }
  }
}

}

extern "C" void kernel_launch(int rank, int nranks, ncclComm_t comm, hipStream_t stream,
                              void* const* d_in, const int* in_sizes, int n_in,
                              void* d_out, int out_size,
                              const int* in_dtypes, int out_dtype,
                              void* const* d_out_all, void* const* d_ws_all, size_t ws_bytes) {
  (void)comm; (void)in_dtypes; (void)out_dtype; (void)d_out_all;
  if (nranks != NR || (unsigned)rank >= (unsigned)NR || n_in != 2 || ws_bytes < (24u << 20) || (u32)out_size != OUT_BYTES ||
      (u32)in_sizes[0] * 2u != X_BYTES || (u32)in_sizes[1] * 2u != W_BYTES) {
    fprintf(stderr, "unexpected problem shape\n");
    return;
  }
  Params P;
  P.x = (const char*)d_in[0]; P.w = (const char*)d_in[1]; P.out = (char*)d_out;
  for (int d = 0; d < NR; ++d) P.ws[d] = (char*)d_ws_all[rank ^ d];
  P.rank = rank; P.pad = 0;
  build_schedule(P);
  hipLaunchKernelGGL(gemm_rs_q8, dim3(GRID), dim3(512), 0, stream, P);
  HIP_CHECK(hipGetLastError());
}
```

```cpp
#include <hip/hip_runtime.h>
#include <rccl/rccl.h>
#include <cstdio>
#include <cstdint>

#define HIP_CHECK(expr) do { hipError_t _e = (expr); if (_e != hipSuccess) { \
    fprintf(stderr, "HIP error %s at %s:%d\n", hipGetErrorString(_e), __FILE__, __LINE__); \
    return; } } while (0)

namespace {

typedef unsigned u32;
typedef unsigned long long u64;
typedef u32 u32x4 __attribute__((ext_vector_type(4)));
typedef u32 u32x2 __attribute__((ext_vector_type(2)));
typedef float f32x4 __attribute__((ext_vector_type(4)));
typedef short s16x4 __attribute__((ext_vector_type(4)));
typedef __bf16 bf16x8 __attribute__((ext_vector_type(8)));
#define LAS __attribute__((address_space(3)))
#define GAS __attribute__((address_space(1)))

constexpr int NR = 8;
constexpr int GRID = 256;
constexpr int NPUSH = 32;
constexpr int PW = 3;
constexpr int NW = NPUSH * PW / 6;
constexpr int WIN = 4;
constexpr int RELAY = 5;
constexpr int MAXU = 72;
constexpr u64 DEADLINE_TICKS = 100000000ull;

constexpr u32 X_BYTES = 4096u * 512u * 2u, W_BYTES = 512u * 4096u * 2u, OUT_BYTES = 512u * 4096u;
constexpr u32 OFF_CNT = 0;
constexpr u32 OFF_TMO = 16u << 10;
constexpr u32 OFF_TK = (16u << 10) + 128u;
constexpr u32 OFF_TP = 18u << 10;
constexpr u32 OFF_T0 = 19u << 10;
constexpr u32 OFF_XF = 20u << 10;
constexpr u32 OFF_WF = 24u << 10;
constexpr u32 OFF_TR = 28u << 10;
constexpr u32 OFF_GO = 29u << 10;
constexpr u32 OFF_AM = 32u << 10;
constexpr u32 OFF_LM = 64u << 10;
constexpr u32 OFF_TOK = 128u << 10;
constexpr u32 OFF_PF = 192u << 10;
constexpr u32 OFF_XS = 2u << 20;
constexpr u32 OFF_WS = 8u << 20;
constexpr u32 OFF_IN = 14u << 20;
constexpr u32 WS_USED = OFF_IN + 224u * 32768u;
constexpr u32 TAU_X = 224;
constexpr u32 TAU = 232, TAU_E = 260;
constexpr u32 T_BOOK = 70;
constexpr u32 ANCH = 28, T_REST = 528;
constexpr u32 M_FAR = 360, M_NEAR = 590;
constexpr int T_FLY = 100;
constexpr u32 WDELAY = 150, TDELAY = 50;
constexpr u32 CATCH = 10;
constexpr int PD = 3;
constexpr int CAP = 6;
constexpr u32 PANEL_B = 6u * 65536u;

enum { JOB_NONE = 0, JOB_S = 1, JOB_O = 2, JOB_F = 3 };

struct Params {
  const char* x; const char* w; char* out;
  char* ws[NR];
  int rank; int pad;
  u32 jobs[GRID];
  u32 first[NPUSH][2 * PW];
  u32 units[NR][MAXU / 4];
  u64 korder[2][8];
  u32 relayed[16];
};

__host__ __device__ inline int tile_dist(int t) { return t < 3 ? 1 : t < 5 ? t - 1 : t < 13 ? 4 + ((t - 5) & 3) : 0; }
__host__ __device__ inline int tile_sub(int t, int a) { return t < 3 ? t : t < 5 ? 3 : t < 13 ? (a & 2) + ((t - 5) >> 2) : t - 13; }
__host__ __device__ inline int box_base(int d) { return d == 1 ? 0 : d < 4 ? 48 + 16 * (d - 2) : 80 + 32 * (d - 4); }
__host__ __device__ inline int box_idx(int t, int p) { return t < 3 ? p * 3 + t : t < 5 ? p : p * 2 + ((t - 5) >> 2); }

__host__ __device__ inline u32 link_units(int d) { return d < 4 ? 60u + 2u * RELAY : 60u - RELAY; }
__host__ __device__ inline int push_link(int block, int wave) { return 2 + ((block >> 3) * PW + wave + (block & 7)) % 6; }
__host__ __device__ inline int push_seat(int block, int wave) { return ((block >> 3) * PW + wave) / 6 * 8 + (block & 7); }
__host__ __device__ inline u32 k_half(int q, int d) { return (u32)(q ^ (d >= 4 ? (q >> 1) & (q ^ d) : 0)) & 1u; }
__host__ __device__ inline u32 slice_src(u32 code, u32 sl, int q, int d) {
  const u32 h = code & 1u, kh = k_half(q, d);
  if (code < 32) {
    const int t = code >> 1;
    return ((u32)((q ^ tile_dist(t)) * 512 + tile_sub(t, q) * 128) + (sl & 1u) * 64u) * 1024u + (kh * 256u + h * 128u + (sl >> 1) * 64u) * 2u;
  }
  return (kh * 256u + h * 128u + sl * 32u) * 8192u + ((u32)(q >> 2) * 2048u + ((code - 32u) >> 1) * 128u) * 2u;
}

__device__ __forceinline__ u32 rfl(u32 v) { return __builtin_amdgcn_readfirstlane(v); }
__device__ __forceinline__ __amdgpu_buffer_rsrc_t mk_rsrc(const void* p, u32 bytes) {
  return __builtin_amdgcn_make_buffer_rsrc((void*)p, 0, bytes, 0x00020000);
}
__device__ __forceinline__ u32x4 mk_rsrc4(const void* p, u32 bytes) {
  u64 a = (u64)p; u32x4 r; r.x = rfl((u32)a); r.y = rfl((u32)(a >> 32)) & 0xffffu; r.z = bytes; r.w = 0x00020000u; return r;
}
__device__ __forceinline__ u32 ld_sys(const char* p) { return __hip_atomic_load((GAS const u32*)p, __ATOMIC_RELAXED, __HIP_MEMORY_SCOPE_SYSTEM); }
__device__ __forceinline__ void st_sys(char* p, u32 v) { __hip_atomic_store((GAS u32*)p, v, __ATOMIC_RELAXED, __HIP_MEMORY_SCOPE_SYSTEM); }
__device__ __forceinline__ void st_agent(char* p, u32 v) { __hip_atomic_store((GAS u32*)p, v, __ATOMIC_RELAXED, __HIP_MEMORY_SCOPE_AGENT); }
#define WAIT_VM(n) asm volatile("s_waitcnt vmcnt(" #n ")" ::: "memory")
#define BARRIER() do { asm volatile("" ::: "memory"); __builtin_amdgcn_s_waitcnt(0xC07F);   __builtin_amdgcn_s_barrier(); asm volatile("" ::: "memory"); } while (0)


template <int SL> __device__ __forceinline__ void wave_wait(char* ws, u32 off, bool care, u32 call, int lane) {
  if (lane == 63 || !care) off = OFF_TMO;
  const u64 t0 = __builtin_amdgcn_s_memrealtime();
  for (;;) {
    const u32 v = ld_sys(ws + off);
    const bool ok = (lane == 63 || !care) ? true : (v >= call);
    if (__all(ok)) break;
    const bool dead = __any(lane == 63 && v == call);
    if (dead || __builtin_amdgcn_s_memrealtime() - t0 > DEADLINE_TICKS) {
      if (lane == 0) st_sys(ws + OFF_TMO, call);
      WAIT_VM(0);
      break;
    }
    __builtin_amdgcn_s_sleep(SL);
  }
  asm volatile("" ::: "memory");
}

__device__ __forceinline__ int wave_poll4(char* ws, __amdgpu_buffer_rsrc_t rws, u32 off, bool care, int have, bool block, u32 call, int lane) {
  if (!care) off = OFF_TMO;
  const u64 t0 = __builtin_amdgcn_s_memrealtime();
  for (;;) {
    const u32x4 v = __builtin_amdgcn_raw_buffer_load_b128(rws, off, 0, 17);
    asm volatile("" ::: "memory");
    const u32 m = (u32)__ballot(care && v.x >= call && v.y >= call && v.z >= call && v.w >= call);
    const int n = __builtin_ctz(~(m & (m >> 16)));
    if (n > have || !block) return n;
    const bool dead = __any(!care && v.x == call);
    if (dead || __builtin_amdgcn_s_memrealtime() - t0 > DEADLINE_TICKS) {
      if (lane == 0) st_sys(ws + OFF_TMO, call);
      WAIT_VM(0);
      return have + 1;
    }
    if (have == 0) __builtin_amdgcn_s_sleep(32); else __builtin_amdgcn_s_sleep(6);
  }
}
#define DMA_SC " sc0 sc1"

typedef __bf16 bf16x2 __attribute__((ext_vector_type(2)));
__device__ __forceinline__ u32 pack_bf16(float lo, float hi) { bf16x2 r; r.x = (__bf16)lo; r.y = (__bf16)hi; return __builtin_bit_cast(u32, r); }
__device__ __forceinline__ float as_f32(u32 v) { return __builtin_bit_cast(float, v); }
__device__ __forceinline__ float bf_lo(u32 v) { return __builtin_bit_cast(float, v << 16); }
__device__ __forceinline__ float bf_hi(u32 v) { return __builtin_bit_cast(float, v & 0xffff0000u); }
__device__ __forceinline__ u32 wave_max(u32 m) {
  m = max(m, (u32)__builtin_amdgcn_update_dpp(0, (int)m, 0x111, 0xf, 0xf, false));
  m = max(m, (u32)__builtin_amdgcn_update_dpp(0, (int)m, 0x112, 0xf, 0xf, false));
  m = max(m, (u32)__builtin_amdgcn_update_dpp(0, (int)m, 0x114, 0xf, 0xf, false));
  m = max(m, (u32)__builtin_amdgcn_update_dpp(0, (int)m, 0x118, 0xf, 0xf, false));
  const u32 a = max((u32)__builtin_amdgcn_readlane(m, 15), (u32)__builtin_amdgcn_readlane(m, 31));
  const u32 b = max((u32)__builtin_amdgcn_readlane(m, 47), (u32)__builtin_amdgcn_readlane(m, 63));
  return max(a, b);
}

__device__ __forceinline__ void dma4(u32 lds_dst, u32 v0, u32 v1, u32 v2, u32 v3, u32x4 rs, u32 so, u32 st) {
  u32 keep, t;
  asm volatile(
      "s_mov_b32 %0, m0\n\t"
      "s_mov_b32 m0, %2\n\ts_add_u32 %1, %8, %9\n\ts_nop 1\n\tbuffer_load_dwordx4 %3, %7, %8 offen" DMA_SC " lds\n\t"
      "s_add_u32 m0, %2, 0x400\n\ts_nop 0\n\tbuffer_load_dwordx4 %4, %7, %1 offen" DMA_SC " lds\n\t"
      "s_add_u32 m0, %2, 0x800\n\ts_add_u32 %1, %1, %9\n\tbuffer_load_dwordx4 %5, %7, %1 offen" DMA_SC " lds\n\t"
      "s_add_u32 m0, %2, 0xc00\n\ts_add_u32 %1, %1, %9\n\tbuffer_load_dwordx4 %6, %7, %1 offen" DMA_SC " lds\n\t"
      "s_mov_b32 m0, %0"
      : "=&s"(keep), "=&s"(t)
      : "s"(lds_dst), "v"(v0), "v"(v1), "v"(v2), "v"(v3), "s"(rs), "s"(so), "s"(st)
      : "memory", "scc");
}

__device__ __forceinline__ void dma8_lin(u32 lds_dst, u32 v, u32x4 rs, u32 sx, u32 sw) {
  u32 keep;
  asm volatile(
      "s_mov_b32 %0, m0\n\ts_mov_b32 m0, %1\n\ts_nop 2\n\t"
      "buffer_load_dwordx4 %2, %3, %4 offen" DMA_SC " lds\n\tbuffer_load_dwordx4 %2, %3, %4 offen offset:1024" DMA_SC " lds\n\t"
      "buffer_load_dwordx4 %2, %3, %4 offen offset:2048" DMA_SC " lds\n\tbuffer_load_dwordx4 %2, %3, %4 offen offset:3072" DMA_SC " lds\n\t"
      "s_add_u32 m0, %1, 0x4000\n\ts_nop 0\n\t"
      "buffer_load_dwordx4 %2, %3, %5 offen" DMA_SC " lds\n\tbuffer_load_dwordx4 %2, %3, %5 offen offset:1024" DMA_SC " lds\n\t"
      "buffer_load_dwordx4 %2, %3, %5 offen offset:2048" DMA_SC " lds\n\tbuffer_load_dwordx4 %2, %3, %5 offen offset:3072" DMA_SC " lds\n\t"
      "s_mov_b32 m0, %0"
      : "=&s"(keep) : "s"(lds_dst), "v"(v), "s"(rs), "s"(sx), "s"(sw) : "memory", "scc");
}

template <int N> __device__ __forceinline__ void wait_vm() { asm volatile("s_waitcnt vmcnt(%0)" :: "n"(N) : "memory"); }
template <int IMM> __device__ __forceinline__ void st16(u32x4 v, u32x4 rs, u32 voff, u32 soff) {
  asm volatile("buffer_store_dwordx4 %0, %1, %2, %3 offen offset:%4 sc0 sc1\n\ts_nop 1" :: "v"(v), "v"(voff), "s"(rs), "s"(soff), "n"(IMM) : "memory");
}
__device__ __forceinline__ void draw_async(u32& v, u32x4 rl, u32 off) {
  u64 ex; v = 1;
  asm volatile("s_mov_b64 %1, exec\n\ts_mov_b64 exec, 1\n\tbuffer_atomic_add %0, off, %2, %3 sc0\n\ts_mov_b64 exec, %1"
               : "+v"(v), "=&s"(ex) : "s"(rl), "s"(off) : "memory");
}
__device__ __forceinline__ void book(u64& tk, u32x4 rl, u32 off) {
  u64 ex; const u64 lo = (__builtin_amdgcn_s_memrealtime() + T_BOOK) * 16u;
  asm volatile("s_mov_b64 %1, exec\n\ts_mov_b64 exec, 1\n\tbuffer_atomic_umax_x2 %2, off, %3, %4\n\tbuffer_atomic_add_x2 %0, off, %3, %4 sc0\n\ts_mov_b64 exec, %1"
               : "+v"(tk), "=&s"(ex) : "v"(lo), "s"(rl), "s"(off) : "memory");
}
__device__ __forceinline__ void sleep_until(u32 due) {
  for (;;) {
    const u32 left = due - (u32)__builtin_amdgcn_s_memrealtime() - 1u;
    if (left >= 10000u) break;
    if (left > 200u) __builtin_amdgcn_s_sleep(48); else if (left > 40u) __builtin_amdgcn_s_sleep(8); else __builtin_amdgcn_s_sleep(1);
  }
}
struct Slice { u32 src, dst, flag, kind, dep; };
__device__ __forceinline__ u32 draw_call(char* ws, int lane, int wave) {
  u32 call = 0;
  if (lane == 0) call = __hip_atomic_fetch_add((GAS u32*)(ws + OFF_CNT + blockIdx.x * 64 + wave * 4), 1u, __ATOMIC_RELAXED, __HIP_MEMORY_SCOPE_AGENT) + 1u;
  return rfl(call);
}
__device__ __forceinline__ void pusher(const Params& P, char* lds, char* ws, int lane, int wave) {
  const u32 vx0 = (u32)(lane >> 3) * 1024u + (u32)(((lane & 7) ^ (lane >> 4)) * 16), vx1 = vx0 ^ 64u;
  const u32 vw0 = (u32)(lane >> 4) * 8192u + (u32)(((lane & 15) ^ (lane >> 4)) * 16), vw1 = vw0 ^ 128u;
  const int d = push_link(blockIdx.x, wave);
  u32 f0 = P.first[blockIdx.x][2 * wave], f1 = P.first[blockIdx.x][2 * wave + 1]; const char* xp = P.x; const char* wp = P.w;
  char* pw = P.ws[d];
  asm volatile("" : "+s"(f0), "+s"(f1), "+s"(xp), "+s"(wp), "+s"(pw), "+s"(ws));
  const u32x4 rx = mk_rsrc4(xp, X_BYTES), rw = mk_rsrc4(wp, W_BYTES), rl = mk_rsrc4(ws, WS_USED);
  const u32 vp = (u32)lane * 16u;
  const u32 ring0 = (u32)(size_t)(LAS char*)lds + (u32)wave * 16384u;
  auto load_all = [&](const Slice& c, u32 buf) {
    const u32 dst = ring0 + buf * 8192u;
    if (c.kind == 0) { dma4(dst, vx0, vx1, vx0, vx1, rx, c.src, 8192u); dma4(dst + 4096u, vx0, vx1, vx0, vx1, rx, c.src + 32768u, 8192u); }
    else if (c.kind == 1) { dma4(dst, vw0, vw0, vw1, vw1, rw, c.src, 32768u); dma4(dst + 4096u, vw0, vw0, vw1, vw1, rw, c.src + 131072u, 32768u); }
    else { dma4(dst, vp, vp, vp, vp, rl, c.src, 1024u); dma4(dst + 4096u, vp, vp, vp, vp, rl, c.src + 4096u, 1024u); }
  };
  { Slice c; c.src = f0 & ~1u; c.kind = f0 & 1u; c.dst = c.flag = c.dep = 0; load_all(c, 0); }
  u32 cnt; draw_async(cnt, rl, OFF_CNT + blockIdx.x * 64u + (u32)wave * 4u);
  const int q = (int)(f1 >> 16);
  const u32 ns = link_units(d) * 4u;
  auto decode = [&](u32 sidx, u32 code) -> Slice {
    Slice r;
    const u32 sl = sidx & 3u, h = code & 1u;
    if (code >= 64) {
      const u32 ds = code >> 4, pp = code & 15u;
      r.kind = 2;
      r.src = OFF_WS + pp * PANEL_B + (ds - 2u) * 65536u + 32768u + sl * 8192u;
      r.dep = OFF_WF + pp * 256u + (ds - 2u) * 32u + 16u + sl * 4u;
      r.dst = OFF_WS + pp * PANEL_B + ((ds ^ (u32)d) - 2u) * 65536u + 32768u + sl * 8192u;
      r.flag = OFF_WF + pp * 256u + ((ds ^ (u32)d) - 2u) * 32u + 16u + sl * 4u;
      return r;
    }
    r.kind = code >> 5; r.src = slice_src(code, sl, q, d); r.dep = OFF_TMO;
    r.dst = (code < 32 ? OFF_XS : OFF_WS) + ((code >> 1) & 15u) * PANEL_B + (u32)(d - 2) * 65536u + h * 32768u + sl * 8192u;
    r.flag = (code < 32 ? OFF_XF : OFF_WF) + ((code >> 1) & 15u) * 256u + (u32)(d - 2) * 32u + h * 16u + sl * 4u;
    return r;
  };
  const u32x4 rd = mk_rsrc4(pw, WS_USED);
  LAS char* ring = (LAS char*)lds + wave * 16384;
  u32 nxt_i = blockIdx.x;
  Slice cur = decode(blockIdx.x, f1 & 0xffu), nxt = decode(nxt_i, (f1 >> 8) & 0xffu);
  u32 buf = 0;
  asm volatile("s_waitcnt vmcnt(1)" : "+s"(cur.dst), "+s"(nxt.src) :: "memory");
  u32 call = 0;
  const u32 tk_off = OFF_TK + (u32)d * 128u;
  u32 pend = 0;
  auto publish = [&]() {
    if (lane == 0) {
      st_sys(pw + pend, call);
    }
  };
  auto send = [&](int k) {
    const u32x4 v = *(LAS const u32x4*)(ring + buf * 8192u + k * 1024 + lane * 16);
    st16<0>(v, rd, vp, cur.dst + (u32)k * 1024u);
  };
  for (;;) {
    u32 tk;
    send(0);
    load_all(nxt, buf ^ 1u);
    draw_async(tk, rl, tk_off);
    const u32 t_draw = (u32)__builtin_amdgcn_s_memrealtime();
    Slice nn = nxt;
    u32 dv = 0;
#pragma unroll
    for (int k = 0; k < 8; ++k) {
      if (k) send(k);
      if (k == WIN + 1) {
        asm volatile("" : "+v"(tk), "+v"(cnt));
        call = rfl(cnt) + 1u;
        nxt_i = rfl(tk) - (call - 1u) * (ns - NW);
        if (nxt_i == ns - ANCH && lane == 0)
          __hip_atomic_store((GAS u64*)(ws + OFF_T0 + (u32)d * 128u), ((u64)call << 32) | (t_draw + T_REST), __ATOMIC_RELAXED, __HIP_MEMORY_SCOPE_AGENT);
        nxt_i += 2u * NW;
        if (nxt_i < ns) nn = decode(nxt_i, (P.units[d][nxt_i >> 4] >> ((nxt_i & 12u) * 2u)) & 0xffu);
        if (nxt_i < ns && nn.kind == 2) asm volatile("buffer_load_dword %0, off, %1, %2 sc0 sc1" : "+v"(dv) : "s"(rl), "s"(nn.dep) : "memory");
      }
      if (k < WIN) wait_vm<WIN - 1 + 9>(); else wait_vm<WIN - 1>();
      if (k == WIN - 2 && pend) publish();
    }
    pend = cur.flag;
    cur = nxt; nxt = nn; buf ^= 1u;
    if (nxt_i >= ns) break;
    if (nxt.kind == 2) {
      asm volatile("s_waitcnt vmcnt(%1)" : "+v"(dv) : "n"(6 - WIN) : "memory");
      if (rfl(dv) < call) wave_wait<2>(ws, nxt.dep, lane == 0, call, lane);
    }
  }
#pragma unroll
  for (int k = 0; k < 8; ++k) {
    send(k);
    wait_vm<WIN - 1>();
    if (k == WIN - 2) publish();
  }
  pend = cur.flag;
  WAIT_VM(0);
  publish();
}

__device__ __forceinline__ void coordinator(const Params& P, char* ws, int lane, u32 call) {
  const u32 lo = (u32)(lane & 7) * 128u;
  const u32 off = lane == 63 ? OFF_TMO - 4u : lane < 16 ? (lane & 8 ? OFF_TR : OFF_T0) + lo : OFF_WF + (lane < 32 ? 14u : 12u) * 256u + (u32)((lane >> 2) - 6) * 32u + (u32)(lane & 3) * 4u - 4u;
  char* peer = P.ws[lane & 7];
  asm volatile("" : "+v"(peer));
  const u64 t0 = __builtin_amdgcn_s_memrealtime();
  if (lane >= 1 && lane < 8) st_sys(peer + OFF_TR + lo + 4u, call);
  auto rl = [&](u32 v, int i) { return (u32)__builtin_amdgcn_readlane(v, i); };
  bool seen = lane < 2 || lane == 8 || (lane >= 16 && lane < 24) || lane >= 48;
  u32 t = 0, sent = 0;
  for (;;) {
    u32 m1 = 0;
    if (!seen || lane == 63) m1 = ld_sys(ws + off + 4u);
    asm volatile("" : "+v"(m1));
    const u64 now = __builtin_amdgcn_s_memrealtime() - t0;
    const bool dead = rl(m1, 63) == call || now > DEADLINE_TICKS;
    if (dead && lane == 0) st_sys(ws + OFF_TMO, call);
    if (!seen && (m1 >= call || dead)) {
      t = lane < 8 ? ld_sys(ws + off) - (u32)t0 : lane < 16 ? (u32)max((int)(u32)now - T_FLY, 0) : (u32)now + (lane < 32 ? M_NEAR : M_FAR) - ((lane & 28) == 24 || (lane & 60) == 32 ? 57u : 0u);
      seen = true;
    }
    const u64 sm = __ballot(seen);
    const u32 rdy = ((sm & 0xffff0000fff0ull) == 0xffff0000fff0ull ? 1u : 0u) | ((sm & 0xff00ff0cull) == 0xff00ff0cull ? 2u : 0u);
    if (rdy & ~sent) {
      const u32 lag = wave_max(lane >> 3 == 1 ? t : 0u);
      u32 tq = t + (u32)__builtin_amdgcn_update_dpp(0, (int)t, 0xb1, 0xf, 0xf, false);
      tq = (tq + (u32)__builtin_amdgcn_update_dpp(0, (int)tq, 0x4e, 0xf, 0xf, false)) >> 2;
      const u32 tf = max(((rl(t, 4) + rl(t, 5) + rl(t, 6) + rl(t, 7)) >> 2) + lag, wave_max(lane >= 32 ? tq : 0u));
      const u32 tn = max(((rl(t, 2) + rl(t, 3)) >> 1) + lag, wave_max(lane >> 4 == 1 ? tq : 0u));
      if (lane < 2 && ((rdy >> lane) & 1u))
        __hip_atomic_store((GAS u64*)(ws + OFF_GO + lo), ((u64)call << 32) | ((lane ? tn : tf) + (u32)t0), __ATOMIC_RELAXED, __HIP_MEMORY_SCOPE_AGENT);
      sent = rdy;
      if (sent == 3u) return;
    }
    __builtin_amdgcn_s_sleep(1);
  }
}

constexpr int NST = 5;

struct Job { int kind, ja, p, dO, sub, nh, nkt; u32 col0, slot; };
__device__ __forceinline__ Job decode_job(u32 code, int a) {
  Job j; j.kind = (code >> 8) & 3; j.ja = (code >> 4) & 15; j.p = code & 15;
  int chalf = a >> 2;
  if (j.kind != JOB_F) { j.dO = tile_dist(j.ja); j.sub = tile_sub(j.ja, a); j.nh = j.dO >= 4 ? 10 : j.dO >= 2 ? 8 : 12; j.slot = (u32)(box_base(j.dO) + box_idx(j.ja, j.p)); }
  else if (j.ja == 8) { j.kind = JOB_S; j.dO = 1; j.sub = 3; j.nh = 0; j.slot = 208u + (u32)j.p; }
  else { j.dO = 0; j.sub = j.ja & 3; chalf ^= j.ja >> 2; j.nh = 0; j.slot = 0; }
  j.col0 = (u32)(chalf * 2048 + j.p * 128);
  j.nkt = 8 + 2 * j.nh;
  return j;
}

struct Polls { u32x2 va, vb; u32 ta, tb;
  __device__ __forceinline__ void retire() { asm volatile("" :: "v"(va), "v"(vb), "v"(ta), "v"(tb)); } };
__device__ __forceinline__ u32 gather_max(char* ws, u32 off, u32 call, int lane, int lazy, Polls& q) {
  u32x2& va = q.va; u32x2& vb = q.vb; u32& ta = q.ta; u32& tb = q.tb;
  const __amdgpu_buffer_rsrc_t rws = mk_rsrc(ws, WS_USED);
  u32 gm = 0, tmo_v;
  const u32 toff = lane ? 0x80000000u : OFF_TMO;
  const u64 t0 = __builtin_amdgcn_s_memrealtime();
  va = __builtin_amdgcn_raw_buffer_load_b64(rws, off, 0, 17);
  ta = __builtin_amdgcn_raw_buffer_load_b32(rws, toff, 0, 17);
  __builtin_amdgcn_s_sleep(3);
  for (;;) {
    asm volatile("" ::: "memory");
    vb = __builtin_amdgcn_raw_buffer_load_b64(rws, off, 0, 17); tb = __builtin_amdgcn_raw_buffer_load_b32(rws, toff, 0, 17);
    asm volatile("" ::: "memory");
    if (va.y >= call) { gm = va.x; off = 0x80000000u; }
    tmo_v = rfl(ta);
    if (__all(off == 0x80000000u)) break;
    va = __builtin_amdgcn_raw_buffer_load_b64(rws, off, 0, 17); ta = __builtin_amdgcn_raw_buffer_load_b32(rws, toff, 0, 17);
    asm volatile("" ::: "memory");
    if (vb.y >= call) { gm = vb.x; off = 0x80000000u; }
    tmo_v = rfl(tb);
    const u64 missing = __ballot(off != 0x80000000u);
    if (!missing) break;
    if (tmo_v == call || __builtin_amdgcn_s_memrealtime() - t0 > DEADLINE_TICKS) {
      if (lane == 0) st_sys(ws + OFF_TMO, call);
      WAIT_VM(0);
      tmo_v = call;
      break;
    }
    if (__builtin_popcountll(missing) > lazy) __builtin_amdgcn_s_sleep(16);
  }
  gm = wave_max(gm);
  return tmo_v == call ? ~0u : gm;
}

__device__ __forceinline__ void loader(const Params& P, char* lds, char* ws, int lane, int wave, u32 call, u32 jobs) {
  const int a = P.rank;
  const __amdgpu_buffer_rsrc_t rws = mk_rsrc(ws, WS_USED);
  const u32 l4 = (u32)(wave - 4) * 4u;
  const u32 vx0 = (u32)(lane >> 3) * 1024u + (u32)(((lane & 7) ^ (lane >> 4)) * 16), vx1 = vx0 ^ 64u;
  const u32 vw0 = (u32)(lane >> 4) * 8192u + (u32)(((lane & 15) ^ (lane >> 4)) * 16), vw1 = vw0 ^ 128u;
  const u32 vp = (u32)lane * 16u;
  const u32x4 rx = mk_rsrc4(P.x, X_BYTES), rw = mk_rsrc4(P.w, W_BYTES), rl = mk_rsrc4(ws, WS_USED);
  const u32 lds0 = (u32)(size_t)(LAS char*)lds + l4 * 1024u;
  bool fin = false;
  if (decode_job(jobs & 0x3ffu, a).dO != 1) sleep_until((u32)__builtin_amdgcn_s_memrealtime() + WDELAY);
  for (; (jobs & 0x3ffu) != 0; jobs >>= 10) {
    const Job J = decode_job(jobs & 0x3ffu, a);
    const int nh = J.nh, nkt = J.nkt;
    const u32 row0 = (u32)((a ^ J.dO) * 512 + J.sub * 128);
    const int j0 = fin && J.kind == JOB_O ? 8 : 0;
    fin = fin || J.kind != JOB_S;
    const u64 ko = P.korder[(P.relayed[J.ja] >> J.p) & 1u][J.dO >= 2 ? J.dO - 2 : 6];
    const int fl = lane & 15;
    const bool care = lane < 32 && fl < nh;
    const u32 foff = (lane < 16 ? OFF_XF + (u32)J.ja * 256u : OFF_WF + (u32)J.p * 256u) + ((u32)(ko >> (fl * 4)) & 15u) * 16u;
    const u32 tab = lane < 8 ? (row0 + l4 * 8u) * 1024u + (u32)lane * 128u
                             : OFF_XS + (u32)J.ja * PANEL_B + l4 * 1024u + (u32)(lane & 1) * 16384u + ((u32)(ko >> ((((lane - 8) >> 1) & 15) * 4)) & 15u) * 32768u;
    const u32 dw = OFF_WS - OFF_XS + (u32)(J.p - J.ja) * PANEL_B, w0 = l4 * (4u * 8192u) + J.col0 * 2u;
    int issued = j0, open = 8;
    u32 dst = lds0;
    auto issue = [&]() {
      const u32 ox = __builtin_amdgcn_readlane(tab, issued);
      if (__builtin_expect(issued >= 8, 1)) dma8_lin(dst, vp, rl, ox, ox + dw);
      else {
        dma4(dst, vx0, vx1, vx0, vx1, rx, ox, 8192u);
        dma4(dst + 16384u, vw0, vw0, vw1, vw1, rw, w0 + (u32)issued * (64u * 8192u), 32768u);
      }
      ++issued; dst = dst == lds0 + (NST - 1) * 32768u ? lds0 : dst + 32768u;
    };
    for (int j = j0; j <= nkt; ++j) {
      if (__builtin_expect(issued == j + NST - 2 && issued < open, 1)) {
        issue();
        WAIT_VM(24);
        __builtin_amdgcn_s_barrier();
        continue;
      }
      const int cap = j + NST - (j > j0) < nkt ? j + NST - (j > j0) : nkt;
      if (issued < cap && issued == open) {
        const int got = wave_poll4(ws, rws, foff, care, (open - 8) >> 1, issued == j, call, lane);
        open = max(open, 8 + 2 * got);
      }
      while (issued < cap && issued < open) issue();
      const int after = issued - j - 1;
      if (after >= 4) WAIT_VM(32); else if (after == 3) WAIT_VM(24); else if (after == 2) WAIT_VM(16); else if (after == 1) WAIT_VM(8); else WAIT_VM(0);
      __builtin_amdgcn_s_barrier();
    }
    __builtin_amdgcn_s_barrier();
  }
}

__device__ __forceinline__ void computer(const Params& P, char* lds, char* ws, int lane, int wave, u32 call, u32 jobs) {
  const int a = P.rank;
  const int wr = wave >> 1, wc = wave & 1, fr = lane & 15, g = lane >> 4;
  const u32 xa0 = (u32)((wr * 64 + fr) * 128 + ((g ^ (fr >> 1)) * 16));
  const int tq = fr >> 2, tp = fr & 3;
  const u32 wa0 = 16384u + (u32)((8 * g + tq) * 256 + (((8 * wc + 2 * tp) ^ (tq | ((g & 1) << 3))) * 16) + (tp & 1) * 8);
  const u32x4 rl = mk_rsrc4(ws, WS_USED);
  f32x4 fin[4][4];
  u32 fjob = 0;
  const u32 vo = (u32)lane * 16u;

  for (; (jobs & 0x3ffu) != 0; jobs >>= 10) {
    const Job J = decode_job(jobs & 0x3ffu, a);
    const bool cont = fjob != 0 && J.kind == JOB_O;
    f32x4 acc[4][4];
#pragma unroll
    for (int m = 0; m < 4; ++m)
#pragma unroll
      for (int n = 0; n < 4; ++n) acc[m][n] = cont ? fin[m][n] : (f32x4){0.f, 0.f, 0.f, 0.f};
    auto readf = [&](bf16x8 (&xf)[4], bf16x8 (&wf)[4], int slot, int ks) {
      LAS char* st = (LAS char*)lds + slot * 32768;
#pragma unroll
      for (int i = 0; i < 4; ++i) {
        const u32 o = (wa0 ^ (u32)((i >> 1) * 16 + (i & 1) * 8)) + (u32)(ks * 8192);
        const s16x4 lo = __builtin_amdgcn_ds_read_tr16_b64_v4i16((LAS s16x4*)(st + o));
        const s16x4 hi = __builtin_amdgcn_ds_read_tr16_b64_v4i16((LAS s16x4*)(st + o + 1024));
        wf[i] = __builtin_bit_cast(bf16x8, __builtin_shufflevector(lo, hi, 0, 1, 2, 3, 4, 5, 6, 7));
        xf[i] = *(LAS const bf16x8*)(st + ((xa0 ^ (u32)(ks * 64)) + (u32)(i * 2048)));
      }
    };
    auto mma = [&](const bf16x8 (&xf)[4], const bf16x8 (&wf)[4]) {
#pragma unroll
      for (int m = 0; m < 4; ++m)
#pragma unroll
        for (int n = 0; n < 4; ++n) acc[m][n] = __builtin_amdgcn_mfma_f32_16x16x32_bf16(wf[n], xf[m], acc[m][n], 0, 0, 0);
    };
#define INTERLEAVE(id) do { _Pragma("unroll") for (int i_ = 0; i_ < 6; ++i_) { __builtin_amdgcn_sched_group_barrier(0x8, 1, id); __builtin_amdgcn_sched_group_barrier(0x100, 2, id); } \
                            __builtin_amdgcn_sched_group_barrier(0x8, 10, id); __builtin_amdgcn_sched_barrier(0); } while (0)
    bf16x8 x0[4], w0[4], x1[4], w1[4];
    int slot = 0;
    BARRIER();
    readf(x0, w0, 0, 0);
    u64 bk = J.nh ? TAU : TAU_E;
    const int bk_at = J.kind == JOB_S && J.dO == 1 ? J.nkt - 2 : -1;
    for (int kt = cont ? 8 : 0; kt < J.nkt; ++kt) {
      if (kt == bk_at) book(bk, rl, OFF_TP);
      __builtin_amdgcn_s_waitcnt(0xC07F);
      __builtin_amdgcn_sched_barrier(0);
      readf(x1, w1, slot, 1); mma(x0, w0);
      INTERLEAVE(0);
      slot = slot == NST - 1 ? 0 : slot + 1;
      BARRIER();
      __builtin_amdgcn_sched_barrier(0);
      readf(x0, w0, slot, 0); mma(x1, w1);
      INTERLEAVE(1);
    }
    BARRIER();

    if (J.kind == JOB_S) {
      const u32 slot = J.slot;
      u32x4 v[8];
#pragma unroll
      for (int jj = 0; jj < 8; ++jj) {
        const f32x4 u0 = acc[jj >> 1][(jj & 1) * 2], u1 = acc[jj >> 1][(jj & 1) * 2 + 1];
        v[jj].x = pack_bf16(u0[0], u0[1]); v[jj].y = pack_bf16(u0[2], u0[3]); v[jj].z = pack_bf16(u1[0], u1[1]); v[jj].w = pack_bf16(u1[2], u1[3]);
      }
      const u32 so = OFF_IN + slot * 32768u + (u32)wave * 8192u, fo = OFF_PF + slot * 512u + (u32)wave * 128u;
      if (J.dO >= 2) {
#pragma unroll
        for (int jj = 0; jj < 8; ++jj) st16<0>(v[jj], rl, vo, so + (u32)(jj * 1024));
        WAIT_VM(0);
        if (lane == 0) st_sys(P.ws[J.dO] + fo, call);
      } else {
        asm volatile("s_waitcnt vmcnt(0)" : "+v"(bk) :: "memory");
        const u32x4 rd = mk_rsrc4(P.ws[1], WS_USED);
        sleep_until(rfl((u32)(bk >> 4)));
#pragma unroll
        for (int jj = 0; jj < 8; ++jj) {
          st16<0>(v[jj], rd, vo, so + (u32)(jj * 1024));
          if (jj < 7) wait_vm<CAP - 1>();
        }
        WAIT_VM(4);
        if (lane == 0) st_sys(P.ws[1] + fo, call);
        WAIT_VM(1);
        if (lane == 0) st_sys(P.ws[1] + fo + 4u, call);
      }
    } else {
      fjob = jobs & 0x3ffu;
#pragma unroll
      for (int m = 0; m < 4; ++m)
#pragma unroll
        for (int n = 0; n < 4; ++n) fin[m][n] = acc[m][n];
    }
  }
  if (!fjob) return;

  if (threadIdx.x < 2) ((LAS u32*)lds)[threadIdx.x] = 0;
  BARRIER();
  const Job J = decode_job(fjob, a);
  const __amdgpu_buffer_rsrc_t rws = mk_rsrc(ws, WS_USED);
  u32 n = 0;
  char* outp = P.out;
  asm volatile("" : "+s"(outp));
  const __amdgpu_buffer_rsrc_t ro = mk_rsrc(outp, OUT_BYTES);
  {
    float mx = 0.f;
    auto add = [&](int jj, u32x4 c) {
      f32x4& u0 = fin[jj >> 1][(jj & 1) * 2]; f32x4& u1 = fin[jj >> 1][(jj & 1) * 2 + 1];
      u0 += (f32x4){bf_lo(c.x), bf_hi(c.x), bf_lo(c.y), bf_hi(c.y)};
      u1 += (f32x4){bf_lo(c.z), bf_hi(c.z), bf_lo(c.w), bf_hi(c.w)};
    };
    auto done = [&](int jj) {
      f32x4& u0 = fin[jj >> 1][(jj & 1) * 2]; f32x4& u1 = fin[jj >> 1][(jj & 1) * 2 + 1];
#pragma unroll
      for (int e = 0; e < 4; ++e) mx = fmaxf(mx, fmaxf(u0[e], u1[e]));
      asm volatile("" : "+v"(u0), "+v"(u1), "+v"(mx));
    };
    if (J.kind == JOB_O || J.ja == 3) {
      const u32 s0 = J.kind == JOB_O ? (u32)(J.p * 3 + J.sub) : 208u + (u32)J.p;
      const u32 so0 = OFF_IN + s0 * 32768u + (u32)wave * 8192u, fo = OFF_PF + s0 * 512u + (u32)wave * 128u;
#pragma unroll
      for (int hf = 0; hf < 2; ++hf) {
        wave_wait<4>(ws, fo + (u32)hf * 4u, lane == 0, call, lane);
        u32x4 v0[4];
#pragma unroll
        for (int jj = 0; jj < 4; ++jj) v0[jj] = __builtin_amdgcn_raw_buffer_load_b128(rws, vo + (u32)(jj * 1024), so0 + (u32)(hf * 4096), 17);
#pragma unroll
        for (int jj = 0; jj < 4; ++jj) { add(hf * 4 + jj, v0[jj]); done(hf * 4 + jj); }
      }
    }
    if (J.kind == JOB_F) {
      mx = 0.f;
      const bool far = (J.ja >> 2) != 0;
      const int d0 = far ? 4 | ((a ^ J.sub) & 2) : 2;
      const u32 bi = far ? (u32)(J.p * 2 + (J.sub & 1)) : (u32)J.p;
      const u32 s0 = (u32)box_base(d0) + bi, s1 = (u32)box_base(d0 + 1) + bi;
      const __amdgpu_buffer_rsrc_t r0 = mk_rsrc(P.ws[d0], WS_USED), r1 = mk_rsrc(P.ws[d0 + 1], WS_USED);
      const u32 o = OFF_GO + (far ? 0u : 128u);
      wave_wait<16>(ws, o + 4u, lane == 0, call, lane);
      u32 t_go = rfl(__hip_atomic_load((GAS const u32*)(ws + o), __ATOMIC_RELAXED, __HIP_MEMORY_SCOPE_AGENT));
      const u32 round = (u32)(!far ? J.p >> 3 : J.p < 8 ? J.sub & 1 : J.p >> 2);
      const u32 seat = (u32)((!far || J.p < 8 ? J.p & 7 : (J.p & 3) | (J.sub & 1) << 2) + wave * 8);
      const u32 tok = OFF_TOK + (((u32)d0 * 2u + round) * 32u + seat) * 64u;
      constexpr u32 step = 4 * TAU_X;
      t_go += (round * 8u * step + seat * (step >> 5)) >> 4;
      sleep_until(t_go - 50u);
      wave_wait<4>(ws, lane == 2 ? tok - 32u * 64u : OFF_PF + (lane == 0 ? s0 : s1) * 512u + (u32)wave * 128u, lane < (round ? 3 : 2), call, lane);
      const u32 so0 = OFF_IN + s0 * 32768u + (u32)wave * 8192u, so1 = OFF_IN + s1 * 32768u + (u32)wave * 8192u;
      u32x4 v0[8], v1[8];
      u32 t_min = t_go;
#pragma unroll
      for (int jj = 0; jj < 8 + PD; ++jj) {
        if (jj < 8) {
          const u32 ti = t_go + (((u32)jj * step) >> 4);
          sleep_until((int)(ti - t_min) > 0 ? ti : t_min);
          t_min = (u32)__builtin_amdgcn_s_memrealtime() + ((step * CATCH) >> 8);
          v0[jj] = __builtin_amdgcn_raw_buffer_load_b128(r0, vo + (u32)((jj & 3) * 1024), so0 + (u32)((jj >> 2) * 4096), 17);
          v1[jj] = __builtin_amdgcn_raw_buffer_load_b128(r1, vo + (u32)((jj & 3) * 1024), so1 + (u32)((jj >> 2) * 4096), 17);
          if (jj == 6 && lane == 0) st_agent(ws + tok, call);
        }
        if (jj >= PD) { add(jj - PD, v0[jj - PD]); add(jj - PD, v1[jj - PD]); done(jj - PD); }
      }
    }
    const u32 wm = wave_max(__builtin_bit_cast(u32, mx) & 0x7fffffffu);
    if (lane == 0) {
      __hip_atomic_fetch_max((LAS u32*)lds, wm, __ATOMIC_RELAXED, __HIP_MEMORY_SCOPE_WORKGROUP);
      asm volatile("" ::: "memory");
      n = __hip_atomic_fetch_add((LAS u32*)lds + 1, 1u, __ATOMIC_RELAXED, __HIP_MEMORY_SCOPE_WORKGROUP);
      asm volatile("" ::: "memory");
      const u32 tm = __hip_atomic_load((LAS u32*)lds, __ATOMIC_RELAXED, __HIP_MEMORY_SCOPE_WORKGROUP);
      if (n == 3u)
        __hip_atomic_store((GAS u64*)(ws + OFF_LM + (u32)(J.p * 8 + (J.kind == JOB_F ? J.ja : J.sub)) * 128u), ((u64)call << 32) | tm, __ATOMIC_RELAXED, __HIP_MEMORY_SCOPE_AGENT);
    }
  }
  if (g & 1) {
#pragma unroll
    for (int m = 0; m < 4; ++m) { f32x4 t = fin[m][0]; fin[m][0] = fin[m][1]; fin[m][1] = t; t = fin[m][2]; fin[m][2] = fin[m][3]; fin[m][3] = t; }
  }
  u32 oo[4];
#pragma unroll
  for (int m = 0; m < 4; ++m) { oo[m] = (u32)(J.sub * 128 + wr * 64 + m * 16 + fr) * 4096u + J.col0 + (u32)(wc * 64 + g * 16); asm volatile("" : "+v"(oo[m])); }
  Polls pq = {};
  if (rfl(n) == 3u) {
    const u32 gm = gather_max(ws, lane < NR ? OFF_AM + (u32)lane * 128u : 0x80000000u, call, lane, 8, pq);
    if (lane == 0) ((LAS u32*)lds)[2] = gm;
  }
  BARRIER();
  const u32 gmax = rfl(((LAS u32*)lds)[2]);
  const float inv = gmax ? 127.0f / as_f32(gmax) : 0.f;
#pragma unroll
  for (int m = 0; m < 4; ++m) {
    u32x4 v;
#pragma unroll
    for (int n = 0; n < 4; ++n) {
      f32x4 y = fin[m][n] * inv;
      asm volatile("" : "+v"(y));
      u32 r = 0;
#pragma unroll
      for (int e = 0; e < 4; ++e) r = __builtin_amdgcn_cvt_pk_u8_f32(y[e], e, r);
      v[n] = r;
    }
    if (gmax == ~0u) v = (u32x4){0x80808080u, 0x80808080u, 0x80808080u, 0x80808080u};
    __builtin_amdgcn_raw_buffer_store_b128(v, ro, oo[m], 0, 17);
  }
  pq.retire();
}

__global__ void __launch_bounds__(512, 2) gemm_rs_q8(const Params P) {
  __shared__ __attribute__((aligned(16))) char lds[5 * 32768];
  const int lane = threadIdx.x & 63, wave = (int)rfl(threadIdx.x >> 6);
  char* ws = P.ws[0];
  if (__builtin_expect(blockIdx.x < NPUSH, 1)) {
    if (wave < PW) pusher(P, lds, ws, lane, wave);
    return;
  }
  if (blockIdx.x >= NPUSH + 8 && blockIdx.x < NPUSH + 24) {
    const bool second = blockIdx.x >= NPUSH + 16;
    const u32 o = (u32)(P.rank * 8 + (blockIdx.x & 7)) * 256u;
    const char* q = P.ws[wave] + (u32)lane * (2u << 20) + o + (320u << 10);
    bool on = lane < 12 && ((0x13u >> lane) & 1u) != second;
    if (second) {
      sleep_until((u32)__builtin_amdgcn_s_memrealtime() + TDELAY);
      if (wave == 0 && lane >= 12 && lane < 20) { on = true; q = (lane < 14 ? P.out : lane < 17 ? P.x : P.w) + (lane < 14 ? (u32)(lane - 12) * (OUT_BYTES - 32768u) : (u32)((lane - 14) % 3) * (X_BYTES / 2u - 16384u)) + o; }
    }
    if (on) { const u32 t = ld_sys(q); asm volatile("" :: "v"(t)); }
  }
  const u32 call = draw_call(ws, lane, wave);
  if (blockIdx.x == GRID - 1) {
    if (wave) return;
    coordinator(P, ws, lane, call);
    char* dst = P.ws[lane & 7] + OFF_AM + (u32)P.rank * 128u;
    asm volatile("" : "+v"(dst));
    Polls pq;
    u32 m = 0;
    for (u32 half = 0; half < 2; ++half) m = max(m, gather_max(ws, OFF_LM + (half * 64u + (u32)lane) * 128u, call, lane, 40, pq));
    if (lane < NR) __hip_atomic_store((GAS u64*)dst, ((u64)call << 32) | m, __ATOMIC_RELAXED, __HIP_MEMORY_SCOPE_SYSTEM);
    pq.retire();
    return;
  }
  const u32 jobs = P.jobs[blockIdx.x];
  if (wave >= 4) loader(P, lds, ws, lane, wave, call, jobs);
  else computer(P, lds, ws, lane, wave, call, jobs);
}

constexpr int LAG = 12;
constexpr int RAMP = 4;
constexpr int R_FWD = 2;
constexpr int R_EARLY = 6;
void build_schedule(Params& P) {
  static const int order[32] = {0, 1, 2, 16, 17, 5, 18, 6, 19, 7, 20, 8, 21, 22, 23, 9, 10, 11, 12, 24, 25, 26, 27, 4, 3, 28, 29, 30, 31, 13, 14, 15};
  int pos[32];
  for (int i = 0; i < 32; ++i) pos[order[i]] = i;
  auto ready = [&](int t, int p) { return pos[t] > pos[16 + p] ? pos[t] : pos[16 + p]; };
  for (int d = 0; d < NR; ++d) {
    unsigned char list[MAXU]; int n = 0;
    for (int i = 0; i < MAXU; ++i) list[i] = 0;
    const int q = P.rank ^ d;
    auto relay = [&](int c) { return c >= 32 - 2 * RELAY; };
    if (d >= 2)
      for (int i = 0, nh1 = 0, lag = ((2 * (d - (d >= 4 ? 4 : 2)) + 1) * LAG + 10) / 20; i < 64; ++i) {
        auto put = [&](int k, int h) { const int c = order[k]; if (!(c < 16 && (tile_dist(c) | (d < 4)) == (d | (d < 4)))) list[n++] = (unsigned char)(c * 2 + h); };
        if (i < 32) put(i, 0);
        for (; nh1 < 32 && nh1 + (nh1 * RAMP / 8 < lag ? nh1 * RAMP / 8 : lag) <= i; ++nh1)
          if (d < 4 || !relay(order[nh1])) put(nh1, 1);
        if (d >= 4 && i + R_EARLY < 32 && relay(order[i + R_EARLY]) && ((q >> 1) & 1) == (order[i + R_EARLY] & 1)) put(i + R_EARLY, 1);
        if (d < 4 && i >= R_FWD && i - R_FWD < 32 && relay(order[i - R_FWD]) && ((P.rank >> 1) & 1) == (order[i - R_FWD] & 1))
          for (int ds = 4; ds < 8; ++ds)
            if (((P.rank ^ ds) & 1) == (d & 1)) list[n++] = (unsigned char)(ds * 16 + order[i - R_FWD] - 16);
      }
    if (d < 4) {
      unsigned char tmp[MAXU]; int m = 0;
      for (int pass = 0; pass < 2; ++pass)
        for (int i = 0; i < n; ++i) if ((list[i] >= 26 && list[i] < 32) == (pass == 1)) tmp[m++] = list[i];
      for (int i = 0; i < n; ++i) list[i] = tmp[i];
    }
    for (int i = 0; i < MAXU / 4; ++i) P.units[d][i] = list[4 * i] | list[4 * i + 1] << 8 | list[4 * i + 2] << 16 | (u32)list[4 * i + 3] << 24;
  }
  for (int t = 0; t < 16; ++t) {
    P.relayed[t] = 0;
    for (int p = 16 - 2 * RELAY; p < 16; ++p)
      if (((P.rank >> 1) & 1) != (p & 1) && t < 13 && pos[16 + p] + R_FWD > pos[t]) P.relayed[t] |= 1u << p;
  }
  static const int kord[2][12] = {{10, 8, 6, 4, 5, 7, 9, 11, 2, 0, 1, 3}, {10, 8, 6, 4, 2, 0, 1, 3, 5, 7, 9, 11}};
  for (int c = 0; c < 2; ++c)
    for (int sk = 0; sk < 8; ++sk) {
      u64 v = 0;
      for (int i = 0, n = 0; i < 12; ++i) if (sk < 2 ? kord[c][i] >= 4 : (kord[c][i] >> 1) != sk) v |= (u64)kord[c][i] << (4 * n++);
      P.korder[c][sk] = v;
    }
  for (int b = 0; b < NPUSH; ++b)
    for (int w = 0; w < PW; ++w) {
      const int d = push_link(b, w);
      auto code = [&](int i) { return (P.units[d][i >> 4] >> ((i & 12) * 2)) & 0xffu; };
      const int i = push_seat(b, w);
      P.first[b][2 * w] = slice_src(code(i), (u32)i & 3u, P.rank ^ d, d) | code(i) >> 5;
      P.first[b][2 * w + 1] = code(i) | code(i + NW) << 8 | (u32)(P.rank ^ d) << 16;
    }
  static const int grp[16] = {0, 1, 0, 2, 1, 1, 0, 1, 0, 1, 0, 1, 0, 2, 2, 2};
  for (int i = 0; i < GRID; ++i) P.jobs[i] = 0;
  for (int x = 0; x < 8; ++x) {
    int ht[40], hp[40], n = 0;
    for (int r = 0; r < 32; ++r)
      for (int p = 0; p < 16; ++p)
        for (int t = 0; t < 16; ++t) {
          const int gsel = grp[t] == 2 ? (p >> 2) & 1 : grp[t];
          if (gsel == (x >> 2) && ((p + (t == 2 ? 2 : 0)) & 3) == (x & 3) && ready(t, p) == r) { ht[n] = t; hp[n] = p; ++n; }
        }
    auto enc = [&](int i) { return (u32)(((ht[i] >= 13 ? JOB_O : JOB_S) << 8) | (ht[i] << 4) | hp[i]); };
    const int nw = 32 - NPUSH / 8 - (x == 7);
    const int hi = x >> 2, lo = x & 3;
    const int fsub[10] = {4, 6, 5, 7, 3, 4 + hi, 6 + hi, 3, 5 - hi, 7 - hi};
    const int fpan[10] = {4 * hi + lo, 4 * hi + lo, 4 - 4 * hi + lo, 4 - 4 * hi + lo, x, 8 + lo, 8 + lo, 8 + x, 12 + lo, 12 + lo};
    int f = 0, e = 0, extra = nw;
    for (int s = 0; s < nw; ++s) {
      u32 jb = enc(s);
      const bool host = extra < n && (ht[s] < 3 ? hp[s] >= 4 && hp[s] < 8 : f >= 4 && ht[s] < 13);
      if (host) { jb = (u32)((JOB_F << 8) | ((ht[extra] - 13) << 4) | hp[extra]) | jb << 10 | enc(extra) << 20; ++extra; }
      else if (ht[s] >= 3 && ht[s] < 13 && f < 10) {
        jb = jb << 10 | (u32)((JOB_F << 8) | (fsub[f] << 4) | fpan[f]); ++f;
      }
      else if (hp[s] >= 12 && ht[s] >= 3 && e < 2) { jb = jb << 10 | (u32)((JOB_F << 8) | (8 << 4) | (x + 8 * e)); ++e; }
      P.jobs[NPUSH + s * 8 + x] = jb;
    }
  }
}

}

extern "C" void kernel_launch(int rank, int nranks, ncclComm_t comm, hipStream_t stream,
                              void* const* d_in, const int* in_sizes, int n_in,
                              void* d_out, int out_size,
                              const int* in_dtypes, int out_dtype,
                              void* const* d_out_all, void* const* d_ws_all, size_t ws_bytes) {
  (void)comm; (void)in_dtypes; (void)out_dtype; (void)d_out_all;
  if (nranks != NR || (unsigned)rank >= (unsigned)NR || n_in != 2 || ws_bytes < (24u << 20) || (u32)out_size != OUT_BYTES ||
      (u32)in_sizes[0] * 2u != X_BYTES || (u32)in_sizes[1] * 2u != W_BYTES) {
    fprintf(stderr, "unexpected problem shape\n");
    return;
  }
  Params P;
  P.x = (const char*)d_in[0]; P.w = (const char*)d_in[1]; P.out = (char*)d_out;
  for (int d = 0; d < NR; ++d) P.ws[d] = (char*)d_ws_all[rank ^ d];
  P.rank = rank; P.pad = 0;
  build_schedule(P);
  hipLaunchKernelGGL(gemm_rs_q8, dim3(GRID), dim3(512), 0, stream, P);
  HIP_CHECK(hipGetLastError());
}
```

```cpp
#include <hip/hip_runtime.h>
#include <rccl/rccl.h>
#include <cstdio>
#include <cstdint>

#define HIP_CHECK(expr) do { hipError_t _e = (expr); if (_e != hipSuccess) { \
    fprintf(stderr, "HIP error %s at %s:%d\n", hipGetErrorString(_e), __FILE__, __LINE__); \
    return; } } while (0)

namespace {

typedef unsigned u32;
typedef unsigned long long u64;
typedef u32 u32x4 __attribute__((ext_vector_type(4)));
typedef u32 u32x2 __attribute__((ext_vector_type(2)));
typedef float f32x4 __attribute__((ext_vector_type(4)));
typedef short s16x4 __attribute__((ext_vector_type(4)));
typedef __bf16 bf16x8 __attribute__((ext_vector_type(8)));
#define LAS __attribute__((address_space(3)))
#define GAS __attribute__((address_space(1)))

constexpr int NR = 8;
constexpr int GRID = 256;
constexpr int NPUSH = 32;
constexpr int PW = 3;
constexpr int NW = NPUSH * PW / 6;
constexpr int WIN = 4;
constexpr int RELAY = 5;
constexpr int MAXU = 72;
constexpr u64 DEADLINE_TICKS = 100000000ull;

constexpr u32 X_BYTES = 4096u * 512u * 2u, W_BYTES = 512u * 4096u * 2u, OUT_BYTES = 512u * 4096u;
constexpr u32 OFF_CNT = 0;
constexpr u32 OFF_TMO = 16u << 10;
constexpr u32 OFF_TK = (16u << 10) + 128u;
constexpr u32 OFF_TP = 18u << 10;
constexpr u32 OFF_T0 = 19u << 10;
constexpr u32 OFF_XF = 20u << 10;
constexpr u32 OFF_WF = 24u << 10;
constexpr u32 OFF_GO = 29u << 10;
constexpr u32 OFF_AM = 32u << 10;
constexpr u32 OFF_LM = 64u << 10;
constexpr u32 OFF_TOK = 128u << 10;
constexpr u32 OFF_PF = 192u << 10;
constexpr u32 OFF_XS = 2u << 20;
constexpr u32 OFF_WS = 8u << 20;
constexpr u32 OFF_IN = 14u << 20;
constexpr u32 WS_USED = OFF_IN + 224u * 32768u;
constexpr u32 TAU_X = 224;
constexpr u32 TAU = 232, TAU_E = 260;
constexpr u32 T_BOOK = 70;
constexpr u32 ANCH = 28, T_REST = 528;
constexpr u32 M_FAR = 390, M_NEAR = 600;
constexpr u32 WDELAY = 150, TDELAY = 50;
constexpr u32 CATCH = 10;
constexpr int PD = 3;
constexpr int CAP = 6;
constexpr u32 PANEL_B = 6u * 65536u;

enum { JOB_NONE = 0, JOB_S = 1, JOB_O = 2, JOB_F = 3 };

struct Params {
  const char* x; const char* w; char* out;
  char* ws[NR];
  int rank; int pad;
  u32 jobs[GRID];
  u32 first[NPUSH][2 * PW];
  u32 units[NR][MAXU / 4];
  u64 korder[2][8];
  u32 relayed[16];
};

__host__ __device__ inline int tile_dist(int t) { return t < 3 ? 1 : t < 5 ? t - 1 : t < 13 ? 4 + ((t - 5) & 3) : 0; }
__host__ __device__ inline int tile_sub(int t, int a) { return t < 3 ? t : t < 5 ? 3 : t < 13 ? (a & 2) + ((t - 5) >> 2) : t - 13; }
__host__ __device__ inline int box_base(int d) { return d == 1 ? 0 : d < 4 ? 48 + 16 * (d - 2) : 80 + 32 * (d - 4); }
__host__ __device__ inline int box_idx(int t, int p) { return t < 3 ? p * 3 + t : t < 5 ? p : p * 2 + ((t - 5) >> 2); }

__host__ __device__ inline u32 link_units(int d) { return d < 4 ? 60u + 2u * RELAY : 60u - RELAY; }
__host__ __device__ inline int push_link(int block, int wave) { return 2 + ((block >> 3) * PW + wave + (block & 7)) % 6; }
__host__ __device__ inline int push_seat(int block, int wave) { return ((block >> 3) * PW + wave) / 6 * 8 + (block & 7); }
__host__ __device__ inline u32 k_half(int q, int d) { return (u32)(q ^ (d >= 4 ? (q >> 1) & (q ^ d) : 0)) & 1u; }
__host__ __device__ inline u32 slice_src(u32 code, u32 sl, int q, int d) {
  const u32 h = code & 1u, kh = k_half(q, d);
  if (code < 32) {
    const int t = code >> 1;
    return ((u32)((q ^ tile_dist(t)) * 512 + tile_sub(t, q) * 128) + (sl & 1u) * 64u) * 1024u + (kh * 256u + h * 128u + (sl >> 1) * 64u) * 2u;
  }
  return (kh * 256u + h * 128u + sl * 32u) * 8192u + ((u32)(q >> 2) * 2048u + ((code - 32u) >> 1) * 128u) * 2u;
}

__device__ __forceinline__ u32 rfl(u32 v) { return __builtin_amdgcn_readfirstlane(v); }
__device__ __forceinline__ __amdgpu_buffer_rsrc_t mk_rsrc(const void* p, u32 bytes) {
  return __builtin_amdgcn_make_buffer_rsrc((void*)p, 0, bytes, 0x00020000);
}
__device__ __forceinline__ u32x4 mk_rsrc4(const void* p, u32 bytes) {
  u64 a = (u64)p; u32x4 r; r.x = rfl((u32)a); r.y = rfl((u32)(a >> 32)) & 0xffffu; r.z = bytes; r.w = 0x00020000u; return r;
}
__device__ __forceinline__ u32 ld_sys(const char* p) { return __hip_atomic_load((GAS const u32*)p, __ATOMIC_RELAXED, __HIP_MEMORY_SCOPE_SYSTEM); }
__device__ __forceinline__ void st_sys(char* p, u32 v) { __hip_atomic_store((GAS u32*)p, v, __ATOMIC_RELAXED, __HIP_MEMORY_SCOPE_SYSTEM); }
__device__ __forceinline__ void st_agent(char* p, u32 v) { __hip_atomic_store((GAS u32*)p, v, __ATOMIC_RELAXED, __HIP_MEMORY_SCOPE_AGENT); }
#define WAIT_VM(n) asm volatile("s_waitcnt vmcnt(" #n ")" ::: "memory")
#define BARRIER() do { asm volatile("" ::: "memory"); __builtin_amdgcn_s_waitcnt(0xC07F);   __builtin_amdgcn_s_barrier(); asm volatile("" ::: "memory"); } while (0)


template <int SL> __device__ __forceinline__ void wave_wait(char* ws, u32 off, bool care, u32 call, int lane) {
  if (lane == 63 || !care) off = OFF_TMO;
  const u64 t0 = __builtin_amdgcn_s_memrealtime();
  for (;;) {
    const u32 v = ld_sys(ws + off);
    const bool ok = (lane == 63 || !care) ? true : (v >= call);
    if (__all(ok)) break;
    const bool dead = __any(lane == 63 && v == call);
    if (dead || __builtin_amdgcn_s_memrealtime() - t0 > DEADLINE_TICKS) {
      if (lane == 0) st_sys(ws + OFF_TMO, call);
      WAIT_VM(0);
      break;
    }
    __builtin_amdgcn_s_sleep(SL);
  }
  asm volatile("" ::: "memory");
}

__device__ __forceinline__ int wave_poll4(char* ws, __amdgpu_buffer_rsrc_t rws, u32 off, bool care, int have, bool block, u32 call, int lane) {
  if (!care) off = OFF_TMO;
  const u64 t0 = __builtin_amdgcn_s_memrealtime();
  for (;;) {
    const u32x4 v = __builtin_amdgcn_raw_buffer_load_b128(rws, off, 0, 17);
    asm volatile("" ::: "memory");
    const u32 m = (u32)__ballot(care && v.x >= call && v.y >= call && v.z >= call && v.w >= call);
    const int n = __builtin_ctz(~(m & (m >> 16)));
    if (n > have || !block) return n;
    const bool dead = __any(!care && v.x == call);
    if (dead || __builtin_amdgcn_s_memrealtime() - t0 > DEADLINE_TICKS) {
      if (lane == 0) st_sys(ws + OFF_TMO, call);
      WAIT_VM(0);
      return have + 1;
    }
    if (have == 0) __builtin_amdgcn_s_sleep(32); else __builtin_amdgcn_s_sleep(6);
  }
}
#define DMA_SC " sc0 sc1"

typedef __bf16 bf16x2 __attribute__((ext_vector_type(2)));
__device__ __forceinline__ u32 pack_bf16(float lo, float hi) { bf16x2 r; r.x = (__bf16)lo; r.y = (__bf16)hi; return __builtin_bit_cast(u32, r); }
__device__ __forceinline__ float as_f32(u32 v) { return __builtin_bit_cast(float, v); }
__device__ __forceinline__ float bf_lo(u32 v) { return __builtin_bit_cast(float, v << 16); }
__device__ __forceinline__ float bf_hi(u32 v) { return __builtin_bit_cast(float, v & 0xffff0000u); }
__device__ __forceinline__ u32 wave_max(u32 m) {
  m = max(m, (u32)__builtin_amdgcn_update_dpp(0, (int)m, 0x111, 0xf, 0xf, false));
  m = max(m, (u32)__builtin_amdgcn_update_dpp(0, (int)m, 0x112, 0xf, 0xf, false));
  m = max(m, (u32)__builtin_amdgcn_update_dpp(0, (int)m, 0x114, 0xf, 0xf, false));
  m = max(m, (u32)__builtin_amdgcn_update_dpp(0, (int)m, 0x118, 0xf, 0xf, false));
  const u32 a = max((u32)__builtin_amdgcn_readlane(m, 15), (u32)__builtin_amdgcn_readlane(m, 31));
  const u32 b = max((u32)__builtin_amdgcn_readlane(m, 47), (u32)__builtin_amdgcn_readlane(m, 63));
  return max(a, b);
}

__device__ __forceinline__ void dma4(u32 lds_dst, u32 v0, u32 v1, u32 v2, u32 v3, u32x4 rs, u32 so, u32 st) {
  u32 keep, t;
  asm volatile(
      "s_mov_b32 %0, m0\n\t"
      "s_mov_b32 m0, %2\n\ts_add_u32 %1, %8, %9\n\ts_nop 1\n\tbuffer_load_dwordx4 %3, %7, %8 offen" DMA_SC " lds\n\t"
      "s_add_u32 m0, %2, 0x400\n\ts_nop 0\n\tbuffer_load_dwordx4 %4, %7, %1 offen" DMA_SC " lds\n\t"
      "s_add_u32 m0, %2, 0x800\n\ts_add_u32 %1, %1, %9\n\tbuffer_load_dwordx4 %5, %7, %1 offen" DMA_SC " lds\n\t"
      "s_add_u32 m0, %2, 0xc00\n\ts_add_u32 %1, %1, %9\n\tbuffer_load_dwordx4 %6, %7, %1 offen" DMA_SC " lds\n\t"
      "s_mov_b32 m0, %0"
      : "=&s"(keep), "=&s"(t)
      : "s"(lds_dst), "v"(v0), "v"(v1), "v"(v2), "v"(v3), "s"(rs), "s"(so), "s"(st)
      : "memory", "scc");
}

__device__ __forceinline__ void dma8_lin(u32 lds_dst, u32 v, u32x4 rs, u32 sx, u32 sw) {
  u32 keep;
  asm volatile(
      "s_mov_b32 %0, m0\n\ts_mov_b32 m0, %1\n\ts_nop 2\n\t"
      "buffer_load_dwordx4 %2, %3, %4 offen" DMA_SC " lds\n\tbuffer_load_dwordx4 %2, %3, %4 offen offset:1024" DMA_SC " lds\n\t"
      "buffer_load_dwordx4 %2, %3, %4 offen offset:2048" DMA_SC " lds\n\tbuffer_load_dwordx4 %2, %3, %4 offen offset:3072" DMA_SC " lds\n\t"
      "s_add_u32 m0, %1, 0x4000\n\ts_nop 0\n\t"
      "buffer_load_dwordx4 %2, %3, %5 offen" DMA_SC " lds\n\tbuffer_load_dwordx4 %2, %3, %5 offen offset:1024" DMA_SC " lds\n\t"
      "buffer_load_dwordx4 %2, %3, %5 offen offset:2048" DMA_SC " lds\n\tbuffer_load_dwordx4 %2, %3, %5 offen offset:3072" DMA_SC " lds\n\t"
      "s_mov_b32 m0, %0"
      : "=&s"(keep) : "s"(lds_dst), "v"(v), "s"(rs), "s"(sx), "s"(sw) : "memory", "scc");
}

template <int N> __device__ __forceinline__ void wait_vm() { asm volatile("s_waitcnt vmcnt(%0)" :: "n"(N) : "memory"); }
template <int IMM> __device__ __forceinline__ void st16(u32x4 v, u32x4 rs, u32 voff, u32 soff) {
  asm volatile("buffer_store_dwordx4 %0, %1, %2, %3 offen offset:%4 sc0 sc1\n\ts_nop 1" :: "v"(v), "v"(voff), "s"(rs), "s"(soff), "n"(IMM) : "memory");
}
__device__ __forceinline__ void draw_async(u32& v, u32x4 rl, u32 off) {
  u64 ex; v = 1;
  asm volatile("s_mov_b64 %1, exec\n\ts_mov_b64 exec, 1\n\tbuffer_atomic_add %0, off, %2, %3 sc0\n\ts_mov_b64 exec, %1"
               : "+v"(v), "=&s"(ex) : "s"(rl), "s"(off) : "memory");
}
__device__ __forceinline__ void book(u64& tk, u32x4 rl, u32 off) {
  u64 ex; const u64 lo = (__builtin_amdgcn_s_memrealtime() + T_BOOK) * 16u;
  asm volatile("s_mov_b64 %1, exec\n\ts_mov_b64 exec, 1\n\tbuffer_atomic_umax_x2 %2, off, %3, %4\n\tbuffer_atomic_add_x2 %0, off, %3, %4 sc0\n\ts_mov_b64 exec, %1"
               : "+v"(tk), "=&s"(ex) : "v"(lo), "s"(rl), "s"(off) : "memory");
}
__device__ __forceinline__ void sleep_until(u32 due) {
  for (;;) {
    const u32 left = due - (u32)__builtin_amdgcn_s_memrealtime() - 1u;
    if (left >= 10000u) break;
    if (left > 200u) __builtin_amdgcn_s_sleep(48); else if (left > 40u) __builtin_amdgcn_s_sleep(8); else __builtin_amdgcn_s_sleep(1);
  }
}
struct Slice { u32 src, dst, flag, kind, dep; };
__device__ __forceinline__ u32 draw_call(char* ws, int lane, int wave) {
  u32 call = 0;
  if (lane == 0) call = __hip_atomic_fetch_add((GAS u32*)(ws + OFF_CNT + blockIdx.x * 64 + wave * 4), 1u, __ATOMIC_RELAXED, __HIP_MEMORY_SCOPE_AGENT) + 1u;
  return rfl(call);
}
__device__ __forceinline__ void pusher(const Params& P, char* lds, char* ws, int lane, int wave) {
  const u32 vx0 = (u32)(lane >> 3) * 1024u + (u32)(((lane & 7) ^ (lane >> 4)) * 16), vx1 = vx0 ^ 64u;
  const u32 vw0 = (u32)(lane >> 4) * 8192u + (u32)(((lane & 15) ^ (lane >> 4)) * 16), vw1 = vw0 ^ 128u;
  const int d = push_link(blockIdx.x, wave);
  u32 f0 = P.first[blockIdx.x][2 * wave], f1 = P.first[blockIdx.x][2 * wave + 1]; const char* xp = P.x; const char* wp = P.w;
  char* pw = P.ws[d];
  asm volatile("" : "+s"(f0), "+s"(f1), "+s"(xp), "+s"(wp), "+s"(pw), "+s"(ws));
  const u32x4 rx = mk_rsrc4(xp, X_BYTES), rw = mk_rsrc4(wp, W_BYTES), rl = mk_rsrc4(ws, WS_USED);
  const u32 vp = (u32)lane * 16u;
  const u32 ring0 = (u32)(size_t)(LAS char*)lds + (u32)wave * 16384u;
  auto load_all = [&](const Slice& c, u32 buf) {
    const u32 dst = ring0 + buf * 8192u;
    if (c.kind == 0) { dma4(dst, vx0, vx1, vx0, vx1, rx, c.src, 8192u); dma4(dst + 4096u, vx0, vx1, vx0, vx1, rx, c.src + 32768u, 8192u); }
    else if (c.kind == 1) { dma4(dst, vw0, vw0, vw1, vw1, rw, c.src, 32768u); dma4(dst + 4096u, vw0, vw0, vw1, vw1, rw, c.src + 131072u, 32768u); }
    else { dma4(dst, vp, vp, vp, vp, rl, c.src, 1024u); dma4(dst + 4096u, vp, vp, vp, vp, rl, c.src + 4096u, 1024u); }
  };
  { Slice c; c.src = f0 & ~1u; c.kind = f0 & 1u; c.dst = c.flag = c.dep = 0; load_all(c, 0); }
  u32 cnt; draw_async(cnt, rl, OFF_CNT + blockIdx.x * 64u + (u32)wave * 4u);
  const int q = (int)(f1 >> 16);
  const u32 ns = link_units(d) * 4u;
  auto decode = [&](u32 sidx, u32 code) -> Slice {
    Slice r;
    const u32 sl = sidx & 3u, h = code & 1u;
    if (code >= 64) {
      const u32 ds = code >> 4, pp = code & 15u;
      r.kind = 2;
      r.src = OFF_WS + pp * PANEL_B + (ds - 2u) * 65536u + 32768u + sl * 8192u;
      r.dep = OFF_WF + pp * 256u + (ds - 2u) * 32u + 16u + sl * 4u;
      r.dst = OFF_WS + pp * PANEL_B + ((ds ^ (u32)d) - 2u) * 65536u + 32768u + sl * 8192u;
      r.flag = OFF_WF + pp * 256u + ((ds ^ (u32)d) - 2u) * 32u + 16u + sl * 4u;
      return r;
    }
    r.kind = code >> 5; r.src = slice_src(code, sl, q, d); r.dep = OFF_TMO;
    r.dst = (code < 32 ? OFF_XS : OFF_WS) + ((code >> 1) & 15u) * PANEL_B + (u32)(d - 2) * 65536u + h * 32768u + sl * 8192u;
    r.flag = (code < 32 ? OFF_XF : OFF_WF) + ((code >> 1) & 15u) * 256u + (u32)(d - 2) * 32u + h * 16u + sl * 4u;
    return r;
  };
  const u32x4 rd = mk_rsrc4(pw, WS_USED);
  LAS char* ring = (LAS char*)lds + wave * 16384;
  u32 nxt_i = blockIdx.x;
  Slice cur = decode(blockIdx.x, f1 & 0xffu), nxt = decode(nxt_i, (f1 >> 8) & 0xffu);
  u32 buf = 0;
  asm volatile("s_waitcnt vmcnt(1)" : "+s"(cur.dst), "+s"(nxt.src) :: "memory");
  u32 call = 0;
  const u32 tk_off = OFF_TK + (u32)d * 128u;
  u32 pend = 0;
  auto publish = [&]() {
    if (lane == 0) {
      st_sys(pw + pend, call);
    }
  };
  auto send = [&](int k) {
    const u32x4 v = *(LAS const u32x4*)(ring + buf * 8192u + k * 1024 + lane * 16);
    st16<0>(v, rd, vp, cur.dst + (u32)k * 1024u);
  };
  for (;;) {
    u32 tk;
    send(0);
    load_all(nxt, buf ^ 1u);
    draw_async(tk, rl, tk_off);
    const u32 t_draw = (u32)__builtin_amdgcn_s_memrealtime();
    Slice nn = nxt;
    u32 dv = 0;
#pragma unroll
    for (int k = 0; k < 8; ++k) {
      if (k) send(k);
      if (k == WIN + 1) {
        asm volatile("" : "+v"(tk), "+v"(cnt));
        call = rfl(cnt) + 1u;
        nxt_i = rfl(tk) - (call - 1u) * (ns - NW);
        if (nxt_i == ns - ANCH && lane == 0)
          __hip_atomic_store((GAS u64*)(ws + OFF_T0 + (u32)d * 128u), ((u64)call << 32) | (t_draw + T_REST), __ATOMIC_RELAXED, __HIP_MEMORY_SCOPE_AGENT);
        nxt_i += 2u * NW;
        if (nxt_i < ns) nn = decode(nxt_i, (P.units[d][nxt_i >> 4] >> ((nxt_i & 12u) * 2u)) & 0xffu);
        if (nxt_i < ns && nn.kind == 2) asm volatile("buffer_load_dword %0, off, %1, %2 sc0 sc1" : "+v"(dv) : "s"(rl), "s"(nn.dep) : "memory");
      }
      if (k < WIN) wait_vm<WIN - 1 + 9>(); else wait_vm<WIN - 1>();
      if (k == WIN - 2 && pend) publish();
    }
    pend = cur.flag;
    cur = nxt; nxt = nn; buf ^= 1u;
    if (nxt_i >= ns) break;
    if (nxt.kind == 2) {
      asm volatile("s_waitcnt vmcnt(%1)" : "+v"(dv) : "n"(6 - WIN) : "memory");
      if (rfl(dv) < call) wave_wait<2>(ws, nxt.dep, lane == 0, call, lane);
    }
  }
#pragma unroll
  for (int k = 0; k < 8; ++k) {
    send(k);
    wait_vm<WIN - 1>();
    if (k == WIN - 2) publish();
  }
  pend = cur.flag;
  WAIT_VM(0);
  publish();
}

__device__ __forceinline__ void coordinator(char* ws, int lane, u32 call) {
  const u32 off = lane == 63 ? OFF_TMO : lane < 8 ? OFF_T0 + (u32)lane * 128u + 4u
                                                  : OFF_WF + (lane < 32 ? 14u : 12u) * 256u + (u32)((lane >> 2) - 6) * 32u + (u32)(lane & 3) * 4u;
  const u64 t0 = __builtin_amdgcn_s_memrealtime();
  auto rl = [&](u32 v, int i) { return (u32)__builtin_amdgcn_readlane(v, i); };
  bool seen = lane < 2 || (lane >= 8 && lane < 24) || lane >= 48;
  u32 t = 0, sent = 0;
  for (;;) {
    u32 m = 0;
    if (!seen || lane == 63) m = ld_sys(ws + off);
    asm volatile("" : "+v"(m));
    const u64 now = __builtin_amdgcn_s_memrealtime() - t0;
    const bool dead = rl(m, 63) == call || now > DEADLINE_TICKS;
    if (dead && lane == 0) st_sys(ws + OFF_TMO, call);
    if (!seen && (m >= call || dead)) {
      t = lane < 8 ? ld_sys(ws + off - 4u) - (u32)t0 : (u32)now + (lane < 32 ? M_NEAR : M_FAR) - (lane >> 2 == 6 || lane >> 2 == 8 ? 57u : 0u);
      seen = true;
    }
    const u64 sm = __ballot(seen);
    const u32 rdy = ((sm & 0xffff000000f0ull) == 0xffff000000f0ull ? 1u : 0u) | ((sm & 0xff00000cull) == 0xff00000cull ? 2u : 0u);
    if (rdy & ~sent) {
      u32 tq = t + (u32)__builtin_amdgcn_update_dpp(0, (int)t, 0xb1, 0xf, 0xf, false);
      const u32 tn = rl(tq, 2) >> 1;
      tq = (tq + (u32)__builtin_amdgcn_update_dpp(0, (int)tq, 0x4e, 0xf, 0xf, false)) >> 2;
      const u32 gn = max(tn, max(rl(tq, 24), rl(tq, 28))), gf = max(rl(tq, 4), max(max(rl(tq, 32), rl(tq, 36)), max(rl(tq, 40), rl(tq, 44))));
      const u32 go = lane ? gn : gf;
      if (lane < 2 && ((rdy >> lane) & 1u))
        __hip_atomic_store((GAS u64*)(ws + OFF_GO + (u32)lane * 128u), ((u64)call << 32) | (go + (u32)t0), __ATOMIC_RELAXED, __HIP_MEMORY_SCOPE_AGENT);
      sent = rdy;
      if (sent == 3u) return;
    }
    __builtin_amdgcn_s_sleep(1);
  }
}

constexpr int NST = 5;

struct Job { int kind, ja, p, dO, sub, nh, nkt; u32 col0, slot; };
__device__ __forceinline__ Job decode_job(u32 code, int a) {
  Job j; j.kind = (code >> 8) & 3; j.ja = (code >> 4) & 15; j.p = code & 15;
  int chalf = a >> 2;
  if (j.kind != JOB_F) { j.dO = tile_dist(j.ja); j.sub = tile_sub(j.ja, a); j.nh = j.dO >= 4 ? 10 : j.dO >= 2 ? 8 : 12; j.slot = (u32)(box_base(j.dO) + box_idx(j.ja, j.p)); }
  else if (j.ja == 8) { j.kind = JOB_S; j.dO = 1; j.sub = 3; j.nh = 0; j.slot = 208u + (u32)j.p; }
  else { j.dO = 0; j.sub = j.ja & 3; chalf ^= j.ja >> 2; j.nh = 0; j.slot = 0; }
  j.col0 = (u32)(chalf * 2048 + j.p * 128);
  j.nkt = 8 + 2 * j.nh;
  return j;
}

struct Polls { u32x2 va, vb; u32 ta, tb;
  __device__ __forceinline__ void retire() { asm volatile("" :: "v"(va), "v"(vb), "v"(ta), "v"(tb)); } };
__device__ __forceinline__ u32 gather_max(char* ws, u32 off, u32 call, int lane, int lazy, Polls& q) {
  u32x2& va = q.va; u32x2& vb = q.vb; u32& ta = q.ta; u32& tb = q.tb;
  const __amdgpu_buffer_rsrc_t rws = mk_rsrc(ws, WS_USED);
  u32 gm = 0, tmo_v;
  const u32 toff = lane ? 0x80000000u : OFF_TMO;
  const u64 t0 = __builtin_amdgcn_s_memrealtime();
  va = __builtin_amdgcn_raw_buffer_load_b64(rws, off, 0, 17);
  ta = __builtin_amdgcn_raw_buffer_load_b32(rws, toff, 0, 17);
  __builtin_amdgcn_s_sleep(3);
  for (;;) {
    asm volatile("" ::: "memory");
    vb = __builtin_amdgcn_raw_buffer_load_b64(rws, off, 0, 17); tb = __builtin_amdgcn_raw_buffer_load_b32(rws, toff, 0, 17);
    asm volatile("" ::: "memory");
    if (va.y >= call) { gm = va.x; off = 0x80000000u; }
    tmo_v = rfl(ta);
    if (__all(off == 0x80000000u)) break;
    va = __builtin_amdgcn_raw_buffer_load_b64(rws, off, 0, 17); ta = __builtin_amdgcn_raw_buffer_load_b32(rws, toff, 0, 17);
    asm volatile("" ::: "memory");
    if (vb.y >= call) { gm = vb.x; off = 0x80000000u; }
    tmo_v = rfl(tb);
    const u64 missing = __ballot(off != 0x80000000u);
    if (!missing) break;
    if (tmo_v == call || __builtin_amdgcn_s_memrealtime() - t0 > DEADLINE_TICKS) {
      if (lane == 0) st_sys(ws + OFF_TMO, call);
      WAIT_VM(0);
      tmo_v = call;
      break;
    }
    if (__builtin_popcountll(missing) > lazy) __builtin_amdgcn_s_sleep(16);
  }
  gm = wave_max(gm);
  return tmo_v == call ? ~0u : gm;
}

__device__ __forceinline__ void loader(const Params& P, char* lds, char* ws, int lane, int wave, u32 call, u32 jobs) {
  const int a = P.rank;
  const __amdgpu_buffer_rsrc_t rws = mk_rsrc(ws, WS_USED);
  const u32 l4 = (u32)(wave - 4) * 4u;
  const u32 vx0 = (u32)(lane >> 3) * 1024u + (u32)(((lane & 7) ^ (lane >> 4)) * 16), vx1 = vx0 ^ 64u;
  const u32 vw0 = (u32)(lane >> 4) * 8192u + (u32)(((lane & 15) ^ (lane >> 4)) * 16), vw1 = vw0 ^ 128u;
  const u32 vp = (u32)lane * 16u;
  const u32x4 rx = mk_rsrc4(P.x, X_BYTES), rw = mk_rsrc4(P.w, W_BYTES), rl = mk_rsrc4(ws, WS_USED);
  const u32 lds0 = (u32)(size_t)(LAS char*)lds + l4 * 1024u;
  bool fin = false;
  if (decode_job(jobs & 0x3ffu, a).dO != 1) sleep_until((u32)__builtin_amdgcn_s_memrealtime() + WDELAY);
  for (; (jobs & 0x3ffu) != 0; jobs >>= 10) {
    const Job J = decode_job(jobs & 0x3ffu, a);
    const int nh = J.nh, nkt = J.nkt;
    const u32 row0 = (u32)((a ^ J.dO) * 512 + J.sub * 128);
    const int j0 = fin && J.kind == JOB_O ? 8 : 0;
    fin = fin || J.kind != JOB_S;
    const u64 ko = P.korder[(P.relayed[J.ja] >> J.p) & 1u][J.dO >= 2 ? J.dO - 2 : 6];
    const int fl = lane & 15;
    const bool care = lane < 32 && fl < nh;
    const u32 foff = (lane < 16 ? OFF_XF + (u32)J.ja * 256u : OFF_WF + (u32)J.p * 256u) + ((u32)(ko >> (fl * 4)) & 15u) * 16u;
    const u32 tab = lane < 8 ? (row0 + l4 * 8u) * 1024u + (u32)lane * 128u
                             : OFF_XS + (u32)J.ja * PANEL_B + l4 * 1024u + (u32)(lane & 1) * 16384u + ((u32)(ko >> ((((lane - 8) >> 1) & 15) * 4)) & 15u) * 32768u;
    const u32 dw = OFF_WS - OFF_XS + (u32)(J.p - J.ja) * PANEL_B, w0 = l4 * (4u * 8192u) + J.col0 * 2u;
    int issued = j0, open = 8;
    u32 dst = lds0;
    auto issue = [&]() {
      const u32 ox = __builtin_amdgcn_readlane(tab, issued);
      if (__builtin_expect(issued >= 8, 1)) dma8_lin(dst, vp, rl, ox, ox + dw);
      else {
        dma4(dst, vx0, vx1, vx0, vx1, rx, ox, 8192u);
        dma4(dst + 16384u, vw0, vw0, vw1, vw1, rw, w0 + (u32)issued * (64u * 8192u), 32768u);
      }
      ++issued; dst = dst == lds0 + (NST - 1) * 32768u ? lds0 : dst + 32768u;
    };
    for (int j = j0; j <= nkt; ++j) {
      if (__builtin_expect(issued == j + NST - 2 && issued < open, 1)) {
        issue();
        WAIT_VM(24);
        __builtin_amdgcn_s_barrier();
        continue;
      }
      const int cap = j + NST - (j > j0) < nkt ? j + NST - (j > j0) : nkt;
      if (issued < cap && issued == open) {
        const int got = wave_poll4(ws, rws, foff, care, (open - 8) >> 1, issued == j, call, lane);
        open = max(open, 8 + 2 * got);
      }
      while (issued < cap && issued < open) issue();
      const int after = issued - j - 1;
      if (after >= 4) WAIT_VM(32); else if (after == 3) WAIT_VM(24); else if (after == 2) WAIT_VM(16); else if (after == 1) WAIT_VM(8); else WAIT_VM(0);
      __builtin_amdgcn_s_barrier();
    }
    __builtin_amdgcn_s_barrier();
  }
}

__device__ __forceinline__ void computer(const Params& P, char* lds, char* ws, int lane, int wave, u32 call, u32 jobs) {
  const int a = P.rank;
  const int wr = wave >> 1, wc = wave & 1, fr = lane & 15, g = lane >> 4;
  const u32 xa0 = (u32)((wr * 64 + fr) * 128 + ((g ^ (fr >> 1)) * 16));
  const int tq = fr >> 2, tp = fr & 3;
  const u32 wa0 = 16384u + (u32)((8 * g + tq) * 256 + (((8 * wc + 2 * tp) ^ (tq | ((g & 1) << 3))) * 16) + (tp & 1) * 8);
  const u32x4 rl = mk_rsrc4(ws, WS_USED);
  f32x4 fin[4][4];
  u32 fjob = 0;
  const u32 vo = (u32)lane * 16u;

  for (; (jobs & 0x3ffu) != 0; jobs >>= 10) {
    const Job J = decode_job(jobs & 0x3ffu, a);
    const bool cont = fjob != 0 && J.kind == JOB_O;
    f32x4 acc[4][4];
#pragma unroll
    for (int m = 0; m < 4; ++m)
#pragma unroll
      for (int n = 0; n < 4; ++n) acc[m][n] = cont ? fin[m][n] : (f32x4){0.f, 0.f, 0.f, 0.f};
    auto readf = [&](bf16x8 (&xf)[4], bf16x8 (&wf)[4], int slot, int ks) {
      LAS char* st = (LAS char*)lds + slot * 32768;
#pragma unroll
      for (int i = 0; i < 4; ++i) {
        const u32 o = (wa0 ^ (u32)((i >> 1) * 16 + (i & 1) * 8)) + (u32)(ks * 8192);
        const s16x4 lo = __builtin_amdgcn_ds_read_tr16_b64_v4i16((LAS s16x4*)(st + o));
        const s16x4 hi = __builtin_amdgcn_ds_read_tr16_b64_v4i16((LAS s16x4*)(st + o + 1024));
        wf[i] = __builtin_bit_cast(bf16x8, __builtin_shufflevector(lo, hi, 0, 1, 2, 3, 4, 5, 6, 7));
        xf[i] = *(LAS const bf16x8*)(st + ((xa0 ^ (u32)(ks * 64)) + (u32)(i * 2048)));
      }
    };
    auto mma = [&](const bf16x8 (&xf)[4], const bf16x8 (&wf)[4]) {
#pragma unroll
      for (int m = 0; m < 4; ++m)
#pragma unroll
        for (int n = 0; n < 4; ++n) acc[m][n] = __builtin_amdgcn_mfma_f32_16x16x32_bf16(wf[n], xf[m], acc[m][n], 0, 0, 0);
    };
#define INTERLEAVE(id) do { _Pragma("unroll") for (int i_ = 0; i_ < 6; ++i_) { __builtin_amdgcn_sched_group_barrier(0x8, 1, id); __builtin_amdgcn_sched_group_barrier(0x100, 2, id); } \
                            __builtin_amdgcn_sched_group_barrier(0x8, 10, id); __builtin_amdgcn_sched_barrier(0); } while (0)
    bf16x8 x0[4], w0[4], x1[4], w1[4];
    int slot = 0;
    BARRIER();
    readf(x0, w0, 0, 0);
    u64 bk = J.nh ? TAU : TAU_E;
    const int bk_at = J.kind == JOB_S && J.dO == 1 ? J.nkt - 2 : -1;
    for (int kt = cont ? 8 : 0; kt < J.nkt; ++kt) {
      if (kt == bk_at) book(bk, rl, OFF_TP);
      __builtin_amdgcn_s_waitcnt(0xC07F);
      __builtin_amdgcn_sched_barrier(0);
      readf(x1, w1, slot, 1); mma(x0, w0);
      INTERLEAVE(0);
      slot = slot == NST - 1 ? 0 : slot + 1;
      BARRIER();
      __builtin_amdgcn_sched_barrier(0);
      readf(x0, w0, slot, 0); mma(x1, w1);
      INTERLEAVE(1);
    }
    BARRIER();

    if (J.kind == JOB_S) {
      const u32 slot = J.slot;
      u32x4 v[8];
#pragma unroll
      for (int jj = 0; jj < 8; ++jj) {
        const f32x4 u0 = acc[jj >> 1][(jj & 1) * 2], u1 = acc[jj >> 1][(jj & 1) * 2 + 1];
        v[jj].x = pack_bf16(u0[0], u0[1]); v[jj].y = pack_bf16(u0[2], u0[3]); v[jj].z = pack_bf16(u1[0], u1[1]); v[jj].w = pack_bf16(u1[2], u1[3]);
      }
      const u32 so = OFF_IN + slot * 32768u + (u32)wave * 8192u, fo = OFF_PF + slot * 512u + (u32)wave * 128u;
      if (J.dO >= 2) {
#pragma unroll
        for (int jj = 0; jj < 8; ++jj) st16<0>(v[jj], rl, vo, so + (u32)(jj * 1024));
        WAIT_VM(0);
        if (lane == 0) st_sys(P.ws[J.dO] + fo, call);
      } else {
        asm volatile("s_waitcnt vmcnt(0)" : "+v"(bk) :: "memory");
        const u32x4 rd = mk_rsrc4(P.ws[1], WS_USED);
        sleep_until(rfl((u32)(bk >> 4)));
#pragma unroll
        for (int jj = 0; jj < 8; ++jj) {
          st16<0>(v[jj], rd, vo, so + (u32)(jj * 1024));
          if (jj < 7) wait_vm<CAP - 1>();
        }
        WAIT_VM(4);
        if (lane == 0) st_sys(P.ws[1] + fo, call);
        WAIT_VM(1);
        if (lane == 0) st_sys(P.ws[1] + fo + 4u, call);
      }
    } else {
      fjob = jobs & 0x3ffu;
#pragma unroll
      for (int m = 0; m < 4; ++m)
#pragma unroll
        for (int n = 0; n < 4; ++n) fin[m][n] = acc[m][n];
    }
  }
  if (!fjob) return;

  if (threadIdx.x < 2) ((LAS u32*)lds)[threadIdx.x] = 0;
  BARRIER();
  const Job J = decode_job(fjob, a);
  const __amdgpu_buffer_rsrc_t rws = mk_rsrc(ws, WS_USED);
  u32 n = 0;
  char* outp = P.out;
  asm volatile("" : "+s"(outp));
  const __amdgpu_buffer_rsrc_t ro = mk_rsrc(outp, OUT_BYTES);
  {
    float mx = 0.f;
    auto add = [&](int jj, u32x4 c) {
      f32x4& u0 = fin[jj >> 1][(jj & 1) * 2]; f32x4& u1 = fin[jj >> 1][(jj & 1) * 2 + 1];
      u0 += (f32x4){bf_lo(c.x), bf_hi(c.x), bf_lo(c.y), bf_hi(c.y)};
      u1 += (f32x4){bf_lo(c.z), bf_hi(c.z), bf_lo(c.w), bf_hi(c.w)};
    };
    auto done = [&](int jj) {
      f32x4& u0 = fin[jj >> 1][(jj & 1) * 2]; f32x4& u1 = fin[jj >> 1][(jj & 1) * 2 + 1];
#pragma unroll
      for (int e = 0; e < 4; ++e) mx = fmaxf(mx, fmaxf(u0[e], u1[e]));
      asm volatile("" : "+v"(u0), "+v"(u1), "+v"(mx));
    };
    if (J.kind == JOB_O || J.ja == 3) {
      const u32 s0 = J.kind == JOB_O ? (u32)(J.p * 3 + J.sub) : 208u + (u32)J.p;
      const u32 so0 = OFF_IN + s0 * 32768u + (u32)wave * 8192u, fo = OFF_PF + s0 * 512u + (u32)wave * 128u;
#pragma unroll
      for (int hf = 0; hf < 2; ++hf) {
        wave_wait<4>(ws, fo + (u32)hf * 4u, lane == 0, call, lane);
        u32x4 v0[4];
#pragma unroll
        for (int jj = 0; jj < 4; ++jj) v0[jj] = __builtin_amdgcn_raw_buffer_load_b128(rws, vo + (u32)(jj * 1024), so0 + (u32)(hf * 4096), 17);
#pragma unroll
        for (int jj = 0; jj < 4; ++jj) { add(hf * 4 + jj, v0[jj]); done(hf * 4 + jj); }
      }
    }
    if (J.kind == JOB_F) {
      mx = 0.f;
      const bool far = (J.ja >> 2) != 0;
      const int d0 = far ? 4 | ((a ^ J.sub) & 2) : 2;
      const u32 bi = far ? (u32)(J.p * 2 + (J.sub & 1)) : (u32)J.p;
      const u32 s0 = (u32)box_base(d0) + bi, s1 = (u32)box_base(d0 + 1) + bi;
      const __amdgpu_buffer_rsrc_t r0 = mk_rsrc(P.ws[d0], WS_USED), r1 = mk_rsrc(P.ws[d0 + 1], WS_USED);
      const u32 o = OFF_GO + (far ? 0u : 128u);
      wave_wait<16>(ws, o + 4u, lane == 0, call, lane);
      u32 t_go = rfl(__hip_atomic_load((GAS const u32*)(ws + o), __ATOMIC_RELAXED, __HIP_MEMORY_SCOPE_AGENT));
      const u32 round = (u32)(!far ? J.p >> 3 : J.p < 8 ? J.sub & 1 : J.p >> 2);
      const u32 seat = (u32)((!far || J.p < 8 ? J.p & 7 : (J.p & 3) | (J.sub & 1) << 2) + wave * 8);
      const u32 tok = OFF_TOK + (((u32)d0 * 2u + round) * 32u + seat) * 64u;
      constexpr u32 step = 4 * TAU_X;
      t_go += (round * 8u * step + seat * (step >> 5)) >> 4;
      sleep_until(t_go - 50u);
      wave_wait<4>(ws, lane == 2 ? tok - 32u * 64u : OFF_PF + (lane == 0 ? s0 : s1) * 512u + (u32)wave * 128u, lane < (round ? 3 : 2), call, lane);
      const u32 so0 = OFF_IN + s0 * 32768u + (u32)wave * 8192u, so1 = OFF_IN + s1 * 32768u + (u32)wave * 8192u;
      u32x4 v0[8], v1[8];
      u32 t_min = t_go;
#pragma unroll
      for (int jj = 0; jj < 8 + PD; ++jj) {
        if (jj < 8) {
          const u32 ti = t_go + (((u32)jj * step) >> 4);
          sleep_until((int)(ti - t_min) > 0 ? ti : t_min);
          t_min = (u32)__builtin_amdgcn_s_memrealtime() + ((step * CATCH) >> 8);
          v0[jj] = __builtin_amdgcn_raw_buffer_load_b128(r0, vo + (u32)((jj & 3) * 1024), so0 + (u32)((jj >> 2) * 4096), 17);
          v1[jj] = __builtin_amdgcn_raw_buffer_load_b128(r1, vo + (u32)((jj & 3) * 1024), so1 + (u32)((jj >> 2) * 4096), 17);
          if (jj == 6 && lane == 0) st_agent(ws + tok, call);
        }
        if (jj >= PD) { add(jj - PD, v0[jj - PD]); add(jj - PD, v1[jj - PD]); done(jj - PD); }
      }
    }
    const u32 wm = wave_max(__builtin_bit_cast(u32, mx) & 0x7fffffffu);
    if (lane == 0) {
      __hip_atomic_fetch_max((LAS u32*)lds, wm, __ATOMIC_RELAXED, __HIP_MEMORY_SCOPE_WORKGROUP);
      asm volatile("" ::: "memory");
      n = __hip_atomic_fetch_add((LAS u32*)lds + 1, 1u, __ATOMIC_RELAXED, __HIP_MEMORY_SCOPE_WORKGROUP);
      asm volatile("" ::: "memory");
      const u32 tm = __hip_atomic_load((LAS u32*)lds, __ATOMIC_RELAXED, __HIP_MEMORY_SCOPE_WORKGROUP);
      if (n == 3u)
        __hip_atomic_store((GAS u64*)(ws + OFF_LM + (u32)(J.p * 8 + (J.kind == JOB_F ? J.ja : J.sub)) * 128u), ((u64)call << 32) | tm, __ATOMIC_RELAXED, __HIP_MEMORY_SCOPE_AGENT);
    }
  }
  if (g & 1) {
#pragma unroll
    for (int m = 0; m < 4; ++m) { f32x4 t = fin[m][0]; fin[m][0] = fin[m][1]; fin[m][1] = t; t = fin[m][2]; fin[m][2] = fin[m][3]; fin[m][3] = t; }
  }
  u32 oo[4];
#pragma unroll
  for (int m = 0; m < 4; ++m) { oo[m] = (u32)(J.sub * 128 + wr * 64 + m * 16 + fr) * 4096u + J.col0 + (u32)(wc * 64 + g * 16); asm volatile("" : "+v"(oo[m])); }
  Polls pq = {};
  if (rfl(n) == 3u) {
    const u32 gm = gather_max(ws, lane < NR ? OFF_AM + (u32)lane * 128u : 0x80000000u, call, lane, 8, pq);
    if (lane == 0) ((LAS u32*)lds)[2] = gm;
  }
  BARRIER();
  const u32 gmax = rfl(((LAS u32*)lds)[2]);
  const float inv = gmax ? 127.0f / as_f32(gmax) : 0.f;
#pragma unroll
  for (int m = 0; m < 4; ++m) {
    u32x4 v;
#pragma unroll
    for (int n = 0; n < 4; ++n) {
      f32x4 y = fin[m][n] * inv;
      asm volatile("" : "+v"(y));
      u32 r = 0;
#pragma unroll
      for (int e = 0; e < 4; ++e) r = __builtin_amdgcn_cvt_pk_u8_f32(y[e], e, r);
      v[n] = r;
    }
    if (gmax == ~0u) v = (u32x4){0x80808080u, 0x80808080u, 0x80808080u, 0x80808080u};
    __builtin_amdgcn_raw_buffer_store_b128(v, ro, oo[m], 0, 17);
  }
  pq.retire();
}

__global__ void __launch_bounds__(512, 2) gemm_rs_q8(const Params P) {
  __shared__ __attribute__((aligned(16))) char lds[5 * 32768];
  const int lane = threadIdx.x & 63, wave = (int)rfl(threadIdx.x >> 6);
  char* ws = P.ws[0];
  if (__builtin_expect(blockIdx.x < NPUSH, 1)) {
    if (wave < PW) pusher(P, lds, ws, lane, wave);
    return;
  }
  if (blockIdx.x >= NPUSH + 8 && blockIdx.x < NPUSH + 24) {
    const bool second = blockIdx.x >= NPUSH + 16;
    const u32 o = (u32)(P.rank * 8 + (blockIdx.x & 7)) * 256u;
    const char* q = P.ws[wave] + (u32)lane * (2u << 20) + o + (320u << 10);
    bool on = lane < 12 && ((0x13u >> lane) & 1u) != second;
    if (second) {
      sleep_until((u32)__builtin_amdgcn_s_memrealtime() + TDELAY);
      if (wave == 0 && lane >= 12 && lane < 20) { on = true; q = (lane < 14 ? P.out : lane < 17 ? P.x : P.w) + (lane < 14 ? (u32)(lane - 12) * (OUT_BYTES - 32768u) : (u32)((lane - 14) % 3) * (X_BYTES / 2u - 16384u)) + o; }
    }
    if (on) { const u32 t = ld_sys(q); asm volatile("" :: "v"(t)); }
  }
  const u32 call = draw_call(ws, lane, wave);
  if (blockIdx.x == GRID - 1) {
    if (wave) return;
    coordinator(ws, lane, call);
    char* dst = P.ws[lane & 7] + OFF_AM + (u32)P.rank * 128u;
    asm volatile("" : "+v"(dst));
    Polls pq;
    u32 m = 0;
    for (u32 half = 0; half < 2; ++half) m = max(m, gather_max(ws, OFF_LM + (half * 64u + (u32)lane) * 128u, call, lane, 40, pq));
    if (lane < NR) __hip_atomic_store((GAS u64*)dst, ((u64)call << 32) | m, __ATOMIC_RELAXED, __HIP_MEMORY_SCOPE_SYSTEM);
    pq.retire();
    return;
  }
  const u32 jobs = P.jobs[blockIdx.x];
  if (wave >= 4) loader(P, lds, ws, lane, wave, call, jobs);
  else computer(P, lds, ws, lane, wave, call, jobs);
}

constexpr int LAG = 12;
constexpr int RAMP = 4;
constexpr int R_FWD = 2;
constexpr int R_EARLY = 6;
void build_schedule(Params& P) {
  static const int order[32] = {0, 1, 2, 16, 17, 5, 18, 6, 19, 7, 20, 8, 21, 22, 23, 9, 10, 11, 12, 24, 25, 26, 27, 4, 3, 28, 29, 30, 31, 13, 14, 15};
  int pos[32];
  for (int i = 0; i < 32; ++i) pos[order[i]] = i;
  auto ready = [&](int t, int p) { return pos[t] > pos[16 + p] ? pos[t] : pos[16 + p]; };
  for (int d = 0; d < NR; ++d) {
    unsigned char list[MAXU]; int n = 0;
    for (int i = 0; i < MAXU; ++i) list[i] = 0;
    const int q = P.rank ^ d;
    auto relay = [&](int c) { return c >= 32 - 2 * RELAY; };
    if (d >= 2)
      for (int i = 0, nh1 = 0, lag = ((2 * (d - (d >= 4 ? 4 : 2)) + 1) * LAG + 10) / 20; i < 64; ++i) {
        auto put = [&](int k, int h) { const int c = order[k]; if (!(c < 16 && (tile_dist(c) | (d < 4)) == (d | (d < 4)))) list[n++] = (unsigned char)(c * 2 + h); };
        if (i < 32) put(i, 0);
        for (; nh1 < 32 && nh1 + (nh1 * RAMP / 8 < lag ? nh1 * RAMP / 8 : lag) <= i; ++nh1)
          if (d < 4 || !relay(order[nh1])) put(nh1, 1);
        if (d >= 4 && i + R_EARLY < 32 && relay(order[i + R_EARLY]) && ((q >> 1) & 1) == (order[i + R_EARLY] & 1)) put(i + R_EARLY, 1);
        if (d < 4 && i >= R_FWD && i - R_FWD < 32 && relay(order[i - R_FWD]) && ((P.rank >> 1) & 1) == (order[i - R_FWD] & 1))
          for (int ds = 4; ds < 8; ++ds)
            if (((P.rank ^ ds) & 1) == (d & 1)) list[n++] = (unsigned char)(ds * 16 + order[i - R_FWD] - 16);
      }
    if (d < 4) {
      unsigned char tmp[MAXU]; int m = 0;
      for (int pass = 0; pass < 2; ++pass)
        for (int i = 0; i < n; ++i) if ((list[i] >= 26 && list[i] < 32) == (pass == 1)) tmp[m++] = list[i];
      for (int i = 0; i < n; ++i) list[i] = tmp[i];
    }
    for (int i = 0; i < MAXU / 4; ++i) P.units[d][i] = list[4 * i] | list[4 * i + 1] << 8 | list[4 * i + 2] << 16 | (u32)list[4 * i + 3] << 24;
  }
  for (int t = 0; t < 16; ++t) {
    P.relayed[t] = 0;
    for (int p = 16 - 2 * RELAY; p < 16; ++p)
      if (((P.rank >> 1) & 1) != (p & 1) && t < 13 && pos[16 + p] + R_FWD > pos[t]) P.relayed[t] |= 1u << p;
  }
  static const int kord[2][12] = {{10, 8, 6, 4, 5, 7, 9, 11, 2, 0, 1, 3}, {10, 8, 6, 4, 2, 0, 1, 3, 5, 7, 9, 11}};
  for (int c = 0; c < 2; ++c)
    for (int sk = 0; sk < 8; ++sk) {
      u64 v = 0;
      for (int i = 0, n = 0; i < 12; ++i) if (sk < 2 ? kord[c][i] >= 4 : (kord[c][i] >> 1) != sk) v |= (u64)kord[c][i] << (4 * n++);
      P.korder[c][sk] = v;
    }
  for (int b = 0; b < NPUSH; ++b)
    for (int w = 0; w < PW; ++w) {
      const int d = push_link(b, w);
      auto code = [&](int i) { return (P.units[d][i >> 4] >> ((i & 12) * 2)) & 0xffu; };
      const int i = push_seat(b, w);
      P.first[b][2 * w] = slice_src(code(i), (u32)i & 3u, P.rank ^ d, d) | code(i) >> 5;
      P.first[b][2 * w + 1] = code(i) | code(i + NW) << 8 | (u32)(P.rank ^ d) << 16;
    }
  static const int grp[16] = {0, 1, 0, 2, 1, 1, 0, 1, 0, 1, 0, 1, 0, 2, 2, 2};
  for (int i = 0; i < GRID; ++i) P.jobs[i] = 0;
  for (int x = 0; x < 8; ++x) {
    int ht[40], hp[40], n = 0;
    for (int r = 0; r < 32; ++r)
      for (int p = 0; p < 16; ++p)
        for (int t = 0; t < 16; ++t) {
          const int gsel = grp[t] == 2 ? (p >> 2) & 1 : grp[t];
          if (gsel == (x >> 2) && ((p + (t == 2 ? 2 : 0)) & 3) == (x & 3) && ready(t, p) == r) { ht[n] = t; hp[n] = p; ++n; }
        }
    auto enc = [&](int i) { return (u32)(((ht[i] >= 13 ? JOB_O : JOB_S) << 8) | (ht[i] << 4) | hp[i]); };
    const int nw = 32 - NPUSH / 8 - (x == 7);
    const int hi = x >> 2, lo = x & 3;
    const int fsub[10] = {4, 6, 5, 7, 3, 4 + hi, 6 + hi, 3, 5 - hi, 7 - hi};
    const int fpan[10] = {4 * hi + lo, 4 * hi + lo, 4 - 4 * hi + lo, 4 - 4 * hi + lo, x, 8 + lo, 8 + lo, 8 + x, 12 + lo, 12 + lo};
    int f = 0, e = 0, extra = nw;
    for (int s = 0; s < nw; ++s) {
      u32 jb = enc(s);
      const bool host = extra < n && (ht[s] < 3 ? hp[s] >= 4 && hp[s] < 8 : f >= 4 && ht[s] < 13);
      if (host) { jb = (u32)((JOB_F << 8) | ((ht[extra] - 13) << 4) | hp[extra]) | jb << 10 | enc(extra) << 20; ++extra; }
      else if (ht[s] >= 3 && ht[s] < 13 && f < 10) {
        jb = jb << 10 | (u32)((JOB_F << 8) | (fsub[f] << 4) | fpan[f]); ++f;
      }
      else if (hp[s] >= 12 && ht[s] >= 3 && e < 2) { jb = jb << 10 | (u32)((JOB_F << 8) | (8 << 4) | (x + 8 * e)); ++e; }
      P.jobs[NPUSH + s * 8 + x] = jb;
    }
  }
}

}

extern "C" void kernel_launch(int rank, int nranks, ncclComm_t comm, hipStream_t stream,
                              void* const* d_in, const int* in_sizes, int n_in,
                              void* d_out, int out_size,
                              const int* in_dtypes, int out_dtype,
                              void* const* d_out_all, void* const* d_ws_all, size_t ws_bytes) {
  (void)comm; (void)in_dtypes; (void)out_dtype; (void)d_out_all;
  if (nranks != NR || (unsigned)rank >= (unsigned)NR || n_in != 2 || ws_bytes < (24u << 20) || (u32)out_size != OUT_BYTES ||
      (u32)in_sizes[0] * 2u != X_BYTES || (u32)in_sizes[1] * 2u != W_BYTES) {
    fprintf(stderr, "unexpected problem shape\n");
    return;
  }
  Params P;
  P.x = (const char*)d_in[0]; P.w = (const char*)d_in[1]; P.out = (char*)d_out;
  for (int d = 0; d < NR; ++d) P.ws[d] = (char*)d_ws_all[rank ^ d];
  P.rank = rank; P.pad = 0;
  build_schedule(P);
  hipLaunchKernelGGL(gemm_rs_q8, dim3(GRID), dim3(512), 0, stream, P);
  HIP_CHECK(hipGetLastError());
}
```
